```python
import numpy as np

kernel_name = "gpu_gemm_ar_m4096_k4096_n4096_w8a8_dequant"


ABSMAX_RTOL = 0.001

COMPUTE_DTYPE = "int8"
EPILOGUE = "dequant"
IN_QUANT = "int8"
OUT_QUANT = None
IN_DTYPES = {"x": "int8", "w_mat": "int8",
             "scale_x": "float32", "scale_w": "float32"}
WORLD_SIZE = 8

def setup_inputs():

    x = np.random.randint(-127, 128, size=(4096, 4096)).astype(np.int8)
    w_mat = np.random.randint(-127, 128, size=(4096, 4096)).astype(np.int8)
    scale_x = (0.02 * (1.0 + np.random.rand(1))).astype(np.float32)
    scale_w = (0.02 * (1.0 + np.random.rand(1))).astype(np.float32)
    return {"x": x, "w_mat": w_mat, "scale_x": scale_x, "scale_w": scale_w}

def reference(x, w_mat, scale_x, scale_w):
    acc = x.astype(np.float64) @ w_mat.astype(np.float64)
    y = (acc.astype(np.float32) * (scale_x * scale_w)).astype(np.float32)
    return y
```

```cpp
#include <hip/hip_runtime.h>
#include <rccl/rccl.h>
#include <cstdio>
#include <cstdint>
#include <utility>

#define HIP_CHECK(expr) do { hipError_t _e = (expr); if (_e != hipSuccess) { \
    fprintf(stderr, "HIP error %s at %s:%d\n", hipGetErrorString(_e), __FILE__, __LINE__); \
    return; } } while (0)

#define LAS __attribute__((address_space(3)))
#define GAS __attribute__((address_space(1)))
typedef int i32x4 __attribute__((ext_vector_type(4)));
typedef unsigned u32x4 __attribute__((ext_vector_type(4)));
typedef float f32x4 __attribute__((ext_vector_type(4)));

namespace {
constexpr int NR = 8;
constexpr int DIM = 4096;
constexpr int KSH = DIM / NR;
constexpr int NKS = KSH / 64;
constexpr int NWG = 256;
constexpr int NSTEP = 2 * NKS;
constexpr int NITEM = NR * NKS;
constexpr size_t WS_FLAGS = 0;
constexpr size_t WS_FLAGS_BYTES = (size_t)NR * NSTEP * NWG * 4;
constexpr size_t WS_CTR = WS_FLAGS_BYTES;
constexpr size_t WS_TMO = WS_CTR + (size_t)NWG * 128;
constexpr size_t WS_AGG = WS_TMO + 128;
constexpr size_t WS_STAGE = 1u << 20;
#define SKEW_KB 0
constexpr size_t STEP_BYTES = (size_t)(NWG + SKEW_KB) * 1024;
constexpr size_t SENDER_BYTES = NSTEP * STEP_BYTES;
constexpr size_t PARITY_BYTES = NR * SENDER_BYTES;
constexpr size_t WS_NEED = WS_STAGE + 2 * PARITY_BYTES;
constexpr int NSLOT = 4, SLOT_BYTES = 32768;
constexpr int L_RING = 0;
constexpr int L_WST = NSLOT * SLOT_BYTES;
constexpr int L_CTL = L_WST + NKS * 1024;
constexpr int L_FULL = L_CTL;
constexpr int L_FREE = L_CTL + 64;
constexpr int L_QCNT = L_CTL + 128;
constexpr int L_ABORT = L_CTL + 132;
constexpr int L_CALL = L_CTL + 136;
constexpr int L_WRDY = L_CTL + 140;
constexpr int L_QUEUE = L_CTL + 256;
constexpr int LDS_BYTES = L_CTL + 256 + NITEM * 4;
constexpr unsigned long long DEADLINE_TICKS = 50000000ull;
#define PDEPTH_ 1
constexpr int PDEPTH = PDEPTH_;

struct Params {
    const signed char* x; const signed char* w; const float* sx; const float* sw;
    float* out; char* ws[NR]; int rank;
};

#define STAMP0(idx) do { } while (0)
#define STAMP1(idx) do { } while (0)
#define STAMPV1(idx, v) do { } while (0)
#define STAMP_OUT() do { } while (0)
#define AGG_MASK 0x3fff
#define TAU16 1056
#define CLS_MODE 1
constexpr int NST = NSTEP * 7;
__host__ __device__ constexpr bool agg_step(int s) { return (AGG_MASK >> s) & 1; }
__host__ __device__ constexpr int npost(int n) {
    const int o = n - PDEPTH - 1;
    if (o < 0) return 0;
    return agg_step(o / 7) ? (o % 7 == 6 ? 1 : 0) : 1;
}
__host__ __device__ constexpr int nyoung(int n) {
    int c = PDEPTH;
    for (int i = n - PDEPTH; i <= n; ++i) c += npost(i);
    return c;
}
template <class F, int... I> __device__ __forceinline__ void static_for_impl(F&& f, std::integer_sequence<int, I...>) { (f(std::integral_constant<int, I>{}), ...); }
template <int N, class F> __device__ __forceinline__ void static_for(F&& f) { static_for_impl(f, std::make_integer_sequence<int, N>{}); }
template <int N> __device__ __forceinline__ void wait_vm() { asm volatile("s_waitcnt vmcnt(%0)" :: "n"(N) : "memory"); }
__device__ __forceinline__ void wait_lgkm0() { asm volatile("s_waitcnt lgkmcnt(0)" ::: "memory"); }
__device__ __forceinline__ unsigned long long rt() { return __builtin_amdgcn_s_memrealtime(); }
__device__ __forceinline__ void st16_sys(const char* base, unsigned off, u32x4 v) {
    asm volatile("global_store_dwordx4 %0, %1, %2 sc0 sc1\n\ts_nop 1" :: "v"(off), "v"(v), "s"(base) : "memory");
}
__device__ __forceinline__ void st4_sys(const char* base, unsigned off, unsigned v) {
    asm volatile("global_store_dword %0, %1, %2 sc0 sc1\n\ts_nop 1" :: "v"(off), "v"(v), "s"(base) : "memory");
}
__device__ __forceinline__ void lds_st(unsigned addr, unsigned v) { asm volatile("ds_write_b32 %0, %1" :: "v"(addr), "v"(v) : "memory"); }
__device__ __forceinline__ unsigned lds_ld(unsigned addr) {
    unsigned v; asm volatile("ds_read_b32 %0, %1\n\ts_waitcnt lgkmcnt(0)" : "=v"(v) : "v"(addr) : "memory");
    return __builtin_amdgcn_readfirstlane(v);
}
__device__ __forceinline__ void dma4k(const char* base, unsigned off, unsigned lds_dst) {
    unsigned keep;
    asm volatile("s_mov_b32 %0, m0\n\ts_mov_b32 m0, %3\n\ts_nop 0\n\t"
                 "global_load_lds_dwordx4 %1, %2 sc0 sc1\n\t"
                 "global_load_lds_dwordx4 %1, %2 offset:1024 sc0 sc1\n\t"
                 "global_load_lds_dwordx4 %1, %2 offset:2048 sc0 sc1\n\t"
                 "global_load_lds_dwordx4 %1, %2 offset:3072 sc0 sc1\n\t"
                 "s_mov_b32 m0, %0"
                 : "=&s"(keep) : "v"(off), "s"(base), "s"(lds_dst) : "memory");
}
}

__global__ void __launch_bounds__(768, 3) gemm_ar_fused(Params P) {
    __shared__ __attribute__((aligned(16))) unsigned char lds_raw[LDS_BYTES];
    LAS unsigned char* lds = (LAS unsigned char*)lds_raw;
    const unsigned lbase = (unsigned)(size_t)lds;
    const int tid = threadIdx.x, lane = tid & 63;
    const int wave = __builtin_amdgcn_readfirstlane(tid >> 6);
    const int b = blockIdx.x;
    const int rank = P.rank;
    const int c = ((b & 7) << 5) | (b >> 3);
    char* myws = P.ws[rank];
    const int ti = ((b & 7) >> 1) * 4 + (b >> 6), tj = (b & 1) * 8 + ((b >> 3) & 7);

    if (wave < 9) STAMP1(0);
    GAS unsigned* ctr = (GAS unsigned*)(myws + WS_CTR + (size_t)b * 128);
    GAS unsigned* tmo = (GAS unsigned*)(myws + WS_TMO);

    if (wave < 8) {
        const int ks = wave;
        const int cc = lane & 15, g = lane >> 4;
        const GAS unsigned* wp = (const GAS unsigned*)(P.w + (size_t)(ks * 64 + g * 16) * DIM + (c >> 2) * 64 + cc * 4);
        unsigned d[16];
#pragma unroll
        for (int t = 0; t < 16; ++t) d[t] = wp[(size_t)t * (DIM / 4)];
        const u32x4 xv = *(const GAS u32x4*)(P.x + (size_t)(c * 16 + cc) * KSH + ks * 64 + g * 16);
        __builtin_amdgcn_s_barrier();
        const unsigned call = lds_ld(lbase + L_CALL);
        const unsigned q = c & 3;
        const unsigned sel_lo = 0x0c0c0000u | ((4u + q) << 8) | q;
        const unsigned sel_hi = 0x00000c0cu | ((4u + q) << 24) | (q << 16);
        u32x4 wv;
#pragma unroll
        for (int e = 0; e < 4; ++e)
            wv[e] = __builtin_amdgcn_perm(d[4 * e + 1], d[4 * e], sel_lo) | __builtin_amdgcn_perm(d[4 * e + 3], d[4 * e + 2], sel_hi);
        *(LAS u32x4*)(lds + L_WST + ks * 1024 + lane * 16) = wv;
        asm volatile("" ::: "memory");
        if (lane == 0) __hip_atomic_fetch_add((LAS unsigned*)(lds + L_WRDY), 1u, __ATOMIC_RELAXED, __HIP_MEMORY_SCOPE_WORKGROUP);
        const unsigned soff = (unsigned)(WS_STAGE + (call & 1u) * PARITY_BYTES + (size_t)rank * SENDER_BYTES + (size_t)(2 * ks) * STEP_BYTES) + c * 1024 + lane * 16;
        st16_sys(myws, soff, xv);
        st16_sys(myws, soff + (unsigned)STEP_BYTES, wv);
        STAMP1(1);
        wait_vm<0>();
        if (lane < 2) st4_sys(myws, (unsigned)(WS_FLAGS + ((rank * NSTEP + 2 * ks + lane) * NWG + c) * 4), call);

        const int wr = wave >> 2, wc = wave & 3;
        i32x4 acc[8][4];
#pragma unroll
        for (int m = 0; m < 8; ++m)
#pragma unroll
            for (int n = 0; n < 4; ++n) acc[m][n] = (i32x4){0, 0, 0, 0};
        const unsigned aoff = wr * 4096 + lane * 16, boff = 16384 + wc * 4096 + lane * 16;
        volatile LAS unsigned* full = (volatile LAS unsigned*)(lds + L_FULL);
        for (int it = 0; it < NITEM; ++it) {
            const int slot = it & (NSLOT - 1);
            const unsigned need = it + 1;
            {
                unsigned sp = 0; unsigned long long t0 = 0;
                for (;;) {
                    const unsigned fa = full[slot * 2], fb = full[slot * 2 + 1];
                    if (__builtin_amdgcn_readfirstlane(fa < fb ? fa : fb) >= need) break;
                    __builtin_amdgcn_s_sleep(1);
                    if ((++sp & 1023u) == 0u) { if (t0 == 0) t0 = rt(); else if (rt() - t0 > 2 * DEADLINE_TICKS) { lds_st(lbase + L_ABORT, 1u); break; } }
                }
            }
            STAMP0(it);
            asm volatile("" ::: "memory");
            const LAS unsigned char* sb = lds + L_RING + slot * SLOT_BYTES;
            i32x4 bf[4];
#pragma unroll
            for (int n = 0; n < 4; ++n) bf[n] = *(const LAS i32x4*)(sb + boff + n * 1024);
#pragma unroll
            for (int m = 0; m < 8; ++m) {
                const i32x4 af = *(const LAS i32x4*)(sb + aoff + (m >> 2) * 8192 + (m & 3) * 1024);
                if (m == 7) {
                    asm volatile("" ::: "memory");
                    if (lane == 0) __hip_atomic_fetch_add((LAS unsigned*)(lds + L_FREE) + slot, 1u, __ATOMIC_RELAXED, __HIP_MEMORY_SCOPE_WORKGROUP);
                    asm volatile("" ::: "memory");
                }
#pragma unroll
                for (int n = 0; n < 4; ++n) acc[m][n] = __builtin_amdgcn_mfma_i32_16x16x64_i8(af, bf[n], acc[m][n], 0, 0, 0);
            }
        }
        STAMP1(2);
        float scale = P.sx[0] * P.sw[0];
        const bool bad = lds_ld(lbase + L_ABORT) != 0u || __hip_atomic_load(tmo, __ATOMIC_RELAXED, __HIP_MEMORY_SCOPE_AGENT) == call;
        if (bad) scale = __builtin_nanf("");
        float* orow = P.out + (size_t)(ti * 256 + wr * 64 + (lane >> 4) * 4) * DIM + tj * 256 + wc * 64 + (lane & 15) * 4;
#pragma unroll
        for (int m = 0; m < 8; ++m)
#pragma unroll
            for (int j = 0; j < 4; ++j) {
                f32x4 o = {(float)acc[m][0][j] * scale, (float)acc[m][1][j] * scale, (float)acc[m][2][j] * scale, (float)acc[m][3][j] * scale};
                *(GAS f32x4*)(orow + (size_t)((m >> 2) * 128 + (m & 3) * 16 + j) * DIM) = o;
            }
        STAMP1(3);
        STAMP1(4);
    } else if (wave == 8) {
        u32x4 xv[NKS];
#pragma unroll
        for (int ks = 0; ks < NKS; ++ks) xv[ks] = *(const GAS u32x4*)(P.x + (size_t)(c * 16 + (lane & 15)) * KSH + ks * 64 + (lane >> 4) * 16);
        const int cls = CLS_MODE == 0 ? (c * 7) >> 8 : (b * 7) >> 8;
        __builtin_amdgcn_s_setprio(3);
        const char* peer[7];
#pragma unroll
        for (int r = 0; r < 7; ++r) { int o = cls + r; o = 1 + (o >= 7 ? o - 7 : o); peer[r] = P.ws[(rank + o) & (NR - 1)]; }
        const char* fl_lane = peer[0];
#pragma unroll
        for (int r = 1; r < 7; ++r) fl_lane = (lane >> 2) == r ? peer[r] : fl_lane;
        fl_lane += WS_FLAGS + (size_t)(rank * NSTEP * NWG + (c & ~15)) * 4 + (lane & 3) * 16;
        __builtin_amdgcn_s_barrier();
        const unsigned call = lds_ld(lbase + L_CALL);
        const unsigned doff = (unsigned)(WS_STAGE + (call & 1u) * PARITY_BYTES + (size_t)rank * SENDER_BYTES) + c * 1024 + lane * 16;
        const unsigned foff = (unsigned)(WS_FLAGS + (rank * NSTEP * NWG + c) * 4);
        const unsigned aoff = (unsigned)(WS_AGG + (c >> 4) * 128);
        const u32x4 callv = {call, call, call, call};
        asm volatile("s_waitcnt vmcnt(0)" : "+v"(xv[0]), "+v"(xv[1]), "+v"(xv[2]), "+v"(xv[3]), "+v"(xv[4]), "+v"(xv[5]), "+v"(xv[6]), "+v"(xv[7]) :: "memory");
        STAMP1(1);
        const unsigned long long tp0 = rt();
        unsigned tick = 0;
        u32x4 v = xv[0];
        static_for<NST + 2 * PDEPTH + 3>([&](auto nc) __attribute__((always_inline)) {
            constexpr int n = decltype(nc)::value;
            constexpr int s = n / 7, r = n % 7;
            if constexpr (n < NST) {
                if constexpr (r == 0) {
                    STAMP0(s);
                    if constexpr (s == 1) { unsigned sp = 0; while (lds_ld(lbase + L_WRDY) < 8u && ++sp < (1u << 24)) __builtin_amdgcn_s_sleep(1); }
                    if constexpr (s & 1) { v = *(const LAS u32x4*)(lds + L_WST + (s >> 1) * 1024 + lane * 16); asm volatile("s_waitcnt lgkmcnt(0)" : "+v"(v) :: "memory"); }
                    else v = xv[s >> 1];
                }
                if constexpr (TAU16 > 0) { while (rt() - tp0 < (unsigned long long)((n * TAU16) >> 4)) __builtin_amdgcn_s_sleep(1); }
                st16_sys(peer[r], doff + (unsigned)(s * STEP_BYTES), v);
            }
            constexpr int o = n - PDEPTH - 1;
            if constexpr (o >= 0 && o < NST) {
                if constexpr (!agg_step(o / 7)) { if (lane == 0) st4_sys(peer[o % 7], foff + (o / 7) * NWG * 4, call); }
                else if constexpr (o % 7 == 6) {
                    unsigned long long keep;
                    asm volatile("s_mov_b64 %1, exec\n\ts_mov_b64 exec, 1\n\tglobal_atomic_add %0, %2, %3, %4 sc0 sc1\n\ts_mov_b64 exec, %1"
                                 : "=&v"(tick), "=&s"(keep) : "v"(aoff + (unsigned)((o / 7) * 16 * 128)), "v"(1u), "s"(myws) : "memory");
                }
            }
            constexpr int o2 = n - 2 * PDEPTH - 3;
            if constexpr (o2 >= 0 && o2 < NST && agg_step(o2 / 7) && o2 % 7 == 6) {
                asm volatile("" : "+v"(tick));
                if ((__builtin_amdgcn_readfirstlane(tick) & 15u) == 15u) {
                    if (lane < 28) asm volatile("global_store_dwordx4 %0, %1, off sc0 sc1\n\ts_nop 1" :: "v"(fl_lane + (o2 / 7) * NWG * 4), "v"(callv) : "memory");
                }
            }
            if constexpr (n < NST - 1) { if constexpr (n >= PDEPTH) wait_vm<nyoung(n)>(); } else wait_vm<0>();
            if constexpr (n == NST - 1) STAMP1(2);
        });
        STAMP1(3);
    } else if (wave == 9) {
        const unsigned call = __builtin_amdgcn_readfirstlane(__hip_atomic_load(ctr, __ATOMIC_RELAXED, __HIP_MEMORY_SCOPE_AGENT)) + 1u;
        for (int u = lane; u < (LDS_BYTES - L_CTL) / 4; u += 64) lds_st(lbase + L_CTL + u * 4, u == (L_CALL - L_CTL) / 4 ? call : 0u);
        wait_lgkm0();
        __builtin_amdgcn_s_barrier();
        if (lane == 0) __hip_atomic_store(ctr, call, __ATOMIC_RELAXED, __HIP_MEMORY_SCOPE_AGENT);
        const int p = lane >> 3, sub = lane & 7;
        const unsigned fl0 = (unsigned)(WS_FLAGS + ((p * NSTEP + (sub >> 2)) * NWG + ((sub >> 2) ? tj : ti) * 16 + (sub & 3) * 4) * 4);
        int nks = 0;
        int queued = 0;
        bool dead = false;
        unsigned long long t0 = rt();
        while (queued < NITEM) {
            const int kc = nks < NKS ? nks : NKS - 1;
            u32x4 f;
            asm volatile("global_load_dwordx4 %0, %1, %2 sc0 sc1\n\ts_waitcnt vmcnt(0)" : "=v"(f) : "v"(fl0 + (unsigned)kc * (2 * NWG * 4)), "s"(myws) : "memory");
            const bool ok = dead || (f[0] >= call && f[1] >= call && f[2] >= call && f[3] >= call);
            const unsigned long long bal = __ballot(ok);
            const bool rdy = ((bal >> (p * 8)) & 0xffull) == 0xffull && nks < NKS;
            const unsigned long long lead = __ballot(rdy && sub == 0);
            if (lead) {
                const int pos = queued + __builtin_popcountll(lead & ((1ull << lane) - 1ull));
                if (rdy && sub == 0) lds_st(lbase + L_QUEUE + pos * 4, (unsigned)(p * NKS + nks));
                if (rdy) ++nks;
                queued += __builtin_popcountll(lead);
                lds_st(lbase + L_QCNT, (unsigned)queued);
                t0 = rt();
            } else {
                __builtin_amdgcn_s_sleep(4);
                if (__builtin_amdgcn_readfirstlane(__hip_atomic_load(tmo, __ATOMIC_RELAXED, __HIP_MEMORY_SCOPE_AGENT)) == call || rt() - t0 > DEADLINE_TICKS) {
                    if (lane == 0) __hip_atomic_store(tmo, call, __ATOMIC_RELAXED, __HIP_MEMORY_SCOPE_AGENT);
                    lds_st(lbase + L_ABORT, 1u);
                    dead = true;
                }
            }
        }
        wait_vm<0>();
    } else {
        const int hb = wave - 10;
        __builtin_amdgcn_s_barrier();
        const unsigned call = lds_ld(lbase + L_CALL);
        const unsigned goff = (unsigned)(WS_STAGE + (call & 1u) * PARITY_BYTES + hb * STEP_BYTES) + (hb ? tj : ti) * 16384 + lane * 16;
        int issued = 0, pub = 0;
        unsigned sp = 0; unsigned long long t0 = 0;
        while (pub < NITEM) {
            bool can = issued < NITEM && issued - pub < 3;
            if (can) can = lds_ld(lbase + L_QCNT) > (unsigned)issued;
            if (can && issued >= NSLOT) can = lds_ld(lbase + L_FREE + (issued & (NSLOT - 1)) * 4) >= 8u * (unsigned)(issued / NSLOT);
            if (can) {
                const unsigned item = lds_ld(lbase + L_QUEUE + issued * 4) & (NITEM - 1);
                const unsigned src = goff + item * (unsigned)(2 * STEP_BYTES);
                const unsigned dst = lbase + L_RING + (issued & (NSLOT - 1)) * SLOT_BYTES + hb * 16384;
#pragma unroll
                for (int k = 0; k < 4; ++k) dma4k(myws, src + k * 4096, dst + k * 4096);
                STAMP0(issued);
                ++issued; sp = 0; t0 = 0;
            } else if (pub < issued) {
                const int fly = issued - pub;
                if (fly == 1) wait_vm<0>(); else if (fly == 2) wait_vm<16>(); else wait_vm<32>();
                lds_st(lbase + L_FULL + ((pub & (NSLOT - 1)) * 2 + hb) * 4, (unsigned)(pub + 1));
                STAMP1(pub);
                ++pub;
            } else {
                __builtin_amdgcn_s_sleep(1);
                if ((++sp & 1023u) == 0u) { if (t0 == 0) t0 = rt(); else if (rt() - t0 > 2 * DEADLINE_TICKS) { lds_st(lbase + L_ABORT, 1u); break; } }
            }
        }
        wait_vm<0>();
    }
    STAMP_OUT();
}

extern "C" void kernel_launch(int rank, int nranks, ncclComm_t comm,
                              hipStream_t stream,
                              void* const* d_in, const int* in_sizes, int n_in,
                              void* d_out, int out_size,
                              const int* in_dtypes, int out_dtype,
                              void* const* d_out_all,
                              void* const* d_ws_all, size_t ws_bytes) {
    (void)comm; (void)in_dtypes; (void)out_dtype; (void)d_out_all;
    if (nranks != NR || n_in != 4 || (unsigned)rank >= (unsigned)NR || ws_bytes < WS_NEED || out_size != DIM * DIM ||
        in_sizes[0] != DIM * KSH || in_sizes[1] != KSH * DIM) {
        fprintf(stderr, "kernel_launch: unexpected problem shape\n");
        return;
    }
    Params P;
    P.x = (const signed char*)d_in[0]; P.w = (const signed char*)d_in[1];
    P.sx = (const float*)d_in[2]; P.sw = (const float*)d_in[3];
    P.out = (float*)d_out; P.rank = rank;
    for (int p = 0; p < NR; ++p) P.ws[p] = (char*)d_ws_all[p];
    hipLaunchKernelGGL(gemm_ar_fused, dim3(NWG), dim3(768), 0, stream, P);
    HIP_CHECK(hipGetLastError());
}
```

```cpp
#include <hip/hip_runtime.h>
#include <rccl/rccl.h>
#include <cstdio>
#include <cstdint>
#include <utility>

#define HIP_CHECK(expr) do { hipError_t _e = (expr); if (_e != hipSuccess) { \
    fprintf(stderr, "HIP error %s at %s:%d\n", hipGetErrorString(_e), __FILE__, __LINE__); \
    return; } } while (0)

#define LAS __attribute__((address_space(3)))
#define GAS __attribute__((address_space(1)))
typedef int i32x4 __attribute__((ext_vector_type(4)));
typedef unsigned u32x4 __attribute__((ext_vector_type(4)));
typedef unsigned u32x2 __attribute__((ext_vector_type(2)));
typedef unsigned u32x8 __attribute__((ext_vector_type(8)));
typedef float f32x4 __attribute__((ext_vector_type(4)));

#define LATE 1
#define PDEPTH_ 1
#define NFINE 2
#define TAU16 1020
#define T0TICKS 70
#define DRIP 2
#define LPRIO 2
#define CSLEEP 1
#define LSLEEP 1
#define CPRIO_HI 0
#define DGAP 0
#define PH2 112
#define TAU2 TAU16
#define PH2W0 1
#define PROT 32
#define DRIPFL 1
#define FINFL 1
#define EPW 2
#define PSLEEP 1
#define PROW 256
#define L0T 125
#define WARMSTEP 99
#define NSA 5
#define NSB 4

namespace {
constexpr int NR = 8;
constexpr int DIM = 4096;
constexpr int KSH = DIM / NR;
constexpr int NKS = KSH / 64;
constexpr int NWG = 256;
constexpr int NSTEP = 16;
constexpr int NSPLIT = 2 * LATE;
constexpr int NFULL = NKS - NSPLIT;
constexpr int IPS = NFULL + 2 * NSPLIT;
constexpr int NITEM = NR * IPS;
constexpr int NUP = NR * NKS;
struct StepDesc { int kind, k; };
constexpr StepDesc step_desc(int s) {
    if (s < 2 * NFULL) return {s & 1, s >> 1};
    s -= 2 * NFULL;
    if (s < 3 * LATE) { const int g = s / 3, r = s % 3; return r == 0 ? StepDesc{2, NFULL + 2 * g} : StepDesc{1, NFULL + 2 * g + r - 1}; }
    s -= 3 * LATE;
    return {3, NFULL + 2 * s};
}
constexpr int find_step(int kind, int k) { for (int s = 0; s < NSTEP; ++s) if (step_desc(s).kind == kind && step_desc(s).k == k) return s; return -1; }
constexpr int nth_step(bool w, int n) { for (int s = 0; s < NSTEP; ++s) if ((step_desc(s).kind == 1) == w && n-- == 0) return s; return -1; }
constexpr unsigned item_desc(int i) {
    if (i < NFULL) return (unsigned)(find_step(0, i) | find_step(1, i) << 4);
    i -= NFULL;
    const int lower = i >= NSPLIT, k = NFULL + (lower ? i - NSPLIT : i);
    return (unsigned)(find_step(2 + lower, NFULL + ((k - NFULL) & ~1)) | find_step(1, k) << 4 | ((k - NFULL) & 1) << 8 | (1 + lower) << 9 | (lower && !((k - NFULL) & 1)) << 11);
}
constexpr unsigned long long step_tab(int ab) { unsigned long long t = 0; for (int i = 0; i < IPS; ++i) t |= (unsigned long long)((item_desc(i) >> (4 * ab)) & 15u) << (4 * i); return t; }
constexpr unsigned long long item_tab(int first) { unsigned long long t = 0; for (int i = 0; i < 5; ++i) if (first + i < IPS) t |= (unsigned long long)item_desc(first + i) << (12 * i); return t; }
constexpr unsigned long long xstep_tab() {
    unsigned long long t = 0; for (int n = 0; n < 8; ++n) t |= (unsigned long long)(nth_step(false, n) | step_desc(nth_step(false, n)).kind << 4) << (8 * n); return t; }
constexpr unsigned xk_tab() { unsigned t = 0; for (int n = 0; n < 8; ++n) t |= (unsigned)step_desc(nth_step(false, n)).k << (4 * n); return t; }
constexpr unsigned wstep_tab() { unsigned t = 0; for (int k = 0; k < 8; ++k) t |= (unsigned)find_step(1, k) << (4 * k); return t; }
constexpr int x_index(int s) { int n = 0; for (int i = 0; i < s; ++i) n += step_desc(i).kind != 1; return n; }
static_assert(IPS <= 15 && step_desc(0).kind == 0, "schedule tables");

constexpr size_t WS_PROG = 0;
constexpr size_t WS_FLAGS_BYTES = 32 * PROW;
constexpr size_t WS_CTR = WS_FLAGS_BYTES;
constexpr size_t WS_TMO = WS_CTR + (size_t)NWG * 128;
constexpr size_t WS_AGG = WS_TMO + 128;
constexpr size_t WS_STAGE = 1u << 20;
constexpr size_t STEP_BYTES = (size_t)NWG * 1024;
constexpr size_t SENDER_BYTES = NSTEP * STEP_BYTES;
constexpr size_t PARITY_BYTES = NR * SENDER_BYTES;
constexpr size_t WS_NEED = WS_STAGE + 2 * PARITY_BYTES;
static_assert(WS_AGG + (NSTEP + 1) * 16 * 128 <= WS_STAGE, "control region");
constexpr int L_RA = 0;
constexpr int L_RB = NSA * 16384;
constexpr int L_WST = L_RB + NSB * 16384;
constexpr int L_CTL = L_WST + NKS * 1024;
constexpr int L_PUB = L_CTL;
constexpr int L_REL = L_CTL + 64;
constexpr int L_QCNT = L_CTL + 128;
constexpr int L_ABORT = L_CTL + 132;
constexpr int L_CALL = L_CTL + 136;
constexpr int L_WRDY = L_CTL + 140;
constexpr int L_LOC = L_CTL + 144;
constexpr int L_QUEUE = L_CTL + 256;
constexpr int LDS_BYTES = L_QUEUE + 128 * 4;
static_assert(LDS_BYTES <= 163840 && NITEM < 128, "LDS");
constexpr unsigned long long DEADLINE_TICKS = 50000000ull;
constexpr int PDEPTH = PDEPTH_;
constexpr int NST = NSTEP * 7;

struct Params {
    const signed char* x; const signed char* w; const float* sx; const float* sw;
    float* out; char* ws[NR]; int rank;
};

#define STAMP0(idx) do { } while (0)
#define STAMP1(idx) do { } while (0)
#define STAMP_OUT() do { } while (0)

constexpr unsigned step_len16(int s) { return 7u * (s < PH2 ? TAU16 : TAU2); }
constexpr unsigned step_t(int s) { unsigned t = 0; for (int i = 0; i < s; ++i) t += step_len16(i); return t >> 4; }
constexpr int sched_t(int n) { return (n <= PH2 ? n * TAU16 : PH2 * TAU16 + (n - PH2) * TAU2) >> 4; }
constexpr bool agg_step(int s) { return s < NSTEP - NFINE; }
constexpr int npost(int n) {
    const int o = n - PDEPTH;
    if (o < 0 || o >= NST) return 0;
    return agg_step(o / 7) ? (o % 7 == 6 ? 1 : 0) : 1;
}
constexpr int nyoung(int n) {
    int c = 0;
    for (int i = n - PDEPTH + 1; i <= n; ++i) c += i < NST;
    for (int i = n - PDEPTH; i < n; ++i) c += npost(i);
    return c;
}
template <class F, int... I> __device__ __forceinline__ void static_for_impl(F&& f, std::integer_sequence<int, I...>) { (f(std::integral_constant<int, I>{}), ...); }
template <int N, class F> __device__ __forceinline__ void static_for(F&& f) { static_for_impl(f, std::make_integer_sequence<int, N>{}); }
template <int N> __device__ __forceinline__ void wait_vm() { asm volatile("s_waitcnt vmcnt(%0)" :: "n"(N) : "memory"); }
__device__ __forceinline__ void wait_vm_x4(int n4) {
    switch (n4) { case 0: wait_vm<0>(); break; case 1: wait_vm<4>(); break; case 2: wait_vm<8>(); break; case 3: wait_vm<12>(); break;
                  case 4: wait_vm<16>(); break; case 5: wait_vm<20>(); break; case 6: wait_vm<24>(); break; case 7: wait_vm<28>(); break;
                  default: wait_vm<32>(); break; }
}
__device__ __forceinline__ unsigned vm_outstanding() {
    const unsigned r = __builtin_amdgcn_s_getreg(7 | (31 << 11));
    return (r & 15u) | ((r >> 18) & 48u);
}
__device__ __forceinline__ void wait_lgkm0() { asm volatile("s_waitcnt lgkmcnt(0)" ::: "memory"); }
__device__ __forceinline__ unsigned long long rt() { return __builtin_amdgcn_s_memrealtime(); }
__device__ __forceinline__ void st16_sys(const char* base, unsigned off, u32x4 v) {
    asm volatile("global_store_dwordx4 %0, %1, %2 sc0 sc1\n\ts_nop 1" :: "v"(off), "v"(v), "s"(base) : "memory");
}
__device__ __forceinline__ void st4_sys(const char* base, unsigned off, unsigned v) {
    asm volatile("global_store_dword %0, %1, %2 sc0 sc1\n\ts_nop 1" :: "v"(off), "v"(v), "s"(base) : "memory");
}
__device__ __forceinline__ void ld16_async(u32x4& v, const GAS u32x4* p) { asm volatile("global_load_dwordx4 %0, %1, off" : "=v"(v) : "v"(p) : "memory"); }
__device__ __forceinline__ void ld4_wait(unsigned& v, const char* p) { asm volatile("global_load_dword %0, %1, off\n\ts_waitcnt vmcnt(0)" : "=v"(v) : "v"(p) : "memory"); }
template <class T> __device__ __forceinline__ void pin(T& v) { asm volatile("" : "+v"(v)); }
__device__ __forceinline__ void st4_sys_lane(const char* p, unsigned v) { asm volatile("global_store_dword %0, %1, off sc0 sc1\n\ts_nop 1" :: "v"(p), "v"(v) : "memory"); }
__device__ __forceinline__ void st16_sys_lane(const char* p, u32x4 v) { asm volatile("global_store_dwordx4 %0, %1, off sc0 sc1\n\ts_nop 1" :: "v"(p), "v"(v) : "memory"); }
__device__ __forceinline__ void ticket_async(unsigned& tick, const char* base, unsigned off) {
    unsigned long long keep;
    asm volatile("s_mov_b64 %1, exec\n\ts_mov_b64 exec, 1\n\tglobal_atomic_add %0, %2, %3, %4 sc0 sc1\n\ts_mov_b64 exec, %1"
                 : "=&v"(tick), "=&s"(keep) : "v"(off), "v"(1u), "s"(base) : "memory");
}
__device__ __forceinline__ void lds_st(unsigned addr, unsigned v) { asm volatile("ds_write_b32 %0, %1" :: "v"(addr), "v"(v) : "memory"); }
__device__ __forceinline__ unsigned lds_ldv(unsigned addr) {
    unsigned v; asm volatile("ds_read_b32 %0, %1\n\ts_waitcnt lgkmcnt(0)" : "=v"(v) : "v"(addr) : "memory");
    return v;
}
__device__ __forceinline__ unsigned lds_ld(unsigned addr) { return __builtin_amdgcn_readfirstlane(lds_ldv(addr)); }
__device__ __forceinline__ void dma4k(const char* base, unsigned off, unsigned lds_dst) {
    unsigned keep;
    asm volatile("s_mov_b32 %0, m0\n\ts_mov_b32 m0, %3\n\ts_nop 0\n\t"
                 "global_load_lds_dwordx4 %1, %2 sc0 sc1\n\t"
                 "global_load_lds_dwordx4 %1, %2 offset:1024 sc0 sc1\n\t"
                 "global_load_lds_dwordx4 %1, %2 offset:2048 sc0 sc1\n\t"
                 "global_load_lds_dwordx4 %1, %2 offset:3072 sc0 sc1\n\t"
                 "s_mov_b32 m0, %0"
                 : "=&s"(keep) : "v"(off), "s"(base), "s"(lds_dst) : "memory");
}
__device__ __forceinline__ const GAS u32x4* x_src(const signed char* x, int kind, int k, int c, int lane) {
    const int row16 = kind == 0 ? c : (c & ~15) + (kind - 2) * 8 + (c & 7);
    const int ks = kind == 0 ? k : k + ((c >> 3) & 1);
    return (const GAS u32x4*)(x + (size_t)(row16 * 16 + (lane & 15)) * KSH + ks * 64 + (lane >> 4) * 16);
}
}

#define STAMPS_ON 0
#define CONSUMER_ASM R"ASM(
.macro MM m, n, a
  v_mfma_i32_16x16x64_i8 v[16*(\m)+4*(\n):16*(\m)+4*(\n)+3], v[144+4*(\a):147+4*(\a)], v[128+4*(\n):131+4*(\n)], v[16*(\m)+4*(\n):16*(\m)+4*(\n)+3]
.endm
.macro MM4 m, a
  MM \m, 0, \a
  MM \m, 1, \a
  MM \m, 2, \a
  MM \m, 3, \a
.endm
.macro MC m, n, a
  MM \m, \n, \a
  MM \m+1, \n, \a+1
.endm
.macro LDA a, off
  ds_read_b128 v[144+4*(\a):147+4*(\a)], v161 offset:\off
.endm
.macro LDB n
  ds_read_b128 v[128+4*(\n):131+4*(\n)], v162 offset:1024*(\n)
.endm
.macro STAMPI
.if %12
  s_sub_u32 s48, s53, %7-64
  s_and_b32 s48, s48, 63
  s_memrealtime s[56:57]
  s_waitcnt lgkmcnt(0)
  s_mov_b32 m0, s48
  s_nop 0
  v_writelane_b32 v160, s56, m0
.endif
.endm
.macro STAMPK k, ins=s_memrealtime
.if %12
  \ins s[56:57]
  s_waitcnt lgkmcnt(0)
  v_mov_b32 v144, s56
  v_mov_b32 v163, 256+4*(\k)
  global_store_dword v163, v144, %6
  s_nop 1
.endif
.endm
.macro ADV_A
  s_add_u32 s41, s41, 1
  s_cmp_eq_u32 s41, %9
  s_cselect_b32 s44, s50, s52
  s_cselect_b32 s41, 0, s41
.endm
.macro ADV_B
  s_add_u32 s42, s42, 1
  s_cmp_eq_u32 s42, %10
  s_cselect_b32 s40, s51, s52
  s_cselect_b32 s42, 0, s42
.endm
// waits for the next item of this wave's half, passing over (and releasing) the items of the other half
.macro SPIN back
  s_mov_b32 s47, 0
1:
  s_cmp_ge_u32 s43, 0x400
  s_cbranch_scc0 2f
  s_mov_b64 exec, 1
  ds_write2_b32 v165, v166, v166 offset0:0 offset1:8
  s_mov_b64 exec, -1
  ADV_A
  ADV_B
  v_add_u32 v164, 4, v164
  v_add_u32 v161, s44, v161
  v_add_u32 v162, s40, v162
  v_add_u32 v166, 1, v166
  s_add_u32 s53, s53, 1
  s_mov_b32 s47, 0
  s_branch 3f
2:
  s_sleep %13
3:
  ds_read_b32 v163, v164 offset:256
  s_add_u32 s47, s47, 1
  s_waitcnt lgkmcnt(0)
  v_readfirstlane_b32 s43, v163
  s_and_b32 s43, s43, %11
  s_cmp_eq_u32 s43, 0x300
  s_cbranch_scc1 \back
  s_cmp_lt_u32 s47, 0x1000000
  s_cbranch_scc1 1b
  v_mov_b32 v163, %3
  v_mov_b32 v144, 1
  ds_write_b32 v163, v144 offset:132
  s_branch FIN_%=
.endm
.macro OUT_ST u, t, fl
  v_cvt_f32_i32 v[\t], v[16*((\u)/4)+((\u)&3)]
  v_cvt_f32_i32 v[\t+1], v[16*((\u)/4)+((\u)&3)+4]
  v_cvt_f32_i32 v[\t+2], v[16*((\u)/4)+((\u)&3)+8]
  v_cvt_f32_i32 v[\t+3], v[16*((\u)/4)+((\u)&3)+12]
  s_add_u32 s54, s58, (((\u)/4)*16+((\u)&3))*16384
  s_addc_u32 s55, s59, 0
  v_mul_f32 v[\t], s49, v[\t]
  v_mul_f32 v[\t+1], s49, v[\t+1]
  v_mul_f32 v[\t+2], s49, v[\t+2]
  v_mul_f32 v[\t+3], s49, v[\t+3]
.if \fl == 0
  global_store_dwordx4 v167, v[\t:\t+3], s[54:55]
.elseif \fl == 1
  global_store_dwordx4 v167, v[\t:\t+3], s[54:55] sc0 sc1
.else
  global_store_dwordx4 v167, v[\t:\t+3], s[54:55] nt
.endif
.endm

  v_mbcnt_lo_u32_b32 v160, -1, 0
  v_mbcnt_hi_u32_b32 v160, -1, v160
  v_lshrrev_b32 v167, 4, v160
  v_lshlrev_b32 v167, 16, v167
  v_and_b32 v166, 15, v160
  v_lshl_add_u32 v167, v166, 4, v167
  v_lshlrev_b32 v160, 4, v160
  v_add_u32 v161, %0, v160
  v_add_u32 v162, %1, v160
  v_mov_b32 v164, %3
  v_mov_b32 v165, %2
  v_mov_b32 v166, 1
  v_mov_b32 v160, 0
  s_mov_b64 s[58:59], %4
  s_mov_b32 s49, %5
  s_mov_b32 s41, 0
  s_mov_b32 s42, 0
  s_mov_b32 s43, 0
  s_mov_b32 s45, %8
  s_mov_b32 s50, -(%9-1)*16384
  s_mov_b32 s51, -(%10-1)*16384
  s_mov_b32 s52, 16384
  s_mov_b32 s53, 0
.set zi, 0
.rept 128
  v_mov_b32 v[zi], 0
.set zi, zi+1
.endr
  SPIN START_%=
SPINF_%=:
  s_cmp_eq_u32 s45, 0
  s_cbranch_scc1 TAIL_%=
  SPIN BACK_%=
START_%=:
  STAMPK 6, s_memtime
  STAMPK 8
  LDA 0, 0
  LDA 1, 1024
  LDB 0
  LDB 1
  LDB 2
  LDB 3
  ds_write_b32 v165, v166 offset:32
LOOP_%=:
  LDA 2, 2048
  LDA 3, 3072
  s_waitcnt lgkmcnt(6)
  MC 0, 0, 0
  s_waitcnt lgkmcnt(5)
  MC 0, 1, 0
  s_waitcnt lgkmcnt(4)
  MC 0, 2, 0
  s_waitcnt lgkmcnt(3)
  MC 0, 3, 0
  LDA 0, 4096
  LDA 1, 5120
  s_waitcnt lgkmcnt(3)
  MM 2, 0, 2
  s_add_u32 s53, s53, 1
  MM 2, 1, 2
  s_sub_u32 s45, s45, 1
  MM 2, 2, 2
  v_add_u32 v164, 4, v164
  MM 2, 3, 2
  s_waitcnt lgkmcnt(2)
  MM4 3, 3
  ds_read_b32 v163, v164 offset:256
  LDA 2, 6144
  LDA 3, 7168
  s_mov_b64 exec, 1
  ds_write_b32 v165, v166
  s_mov_b64 exec, -1
  s_waitcnt lgkmcnt(5)
  MM 4, 0, 0
  ADV_A
  MM 4, 1, 0
  v_add_u32 v161, s44, v161
  MM 4, 2, 0
  ADV_B
  MM 4, 3, 0
  v_add_u32 v162, s40, v162
  s_waitcnt lgkmcnt(4)
  MM 5, 0, 1
  v_add_u32 v166, 1, v166
  MM 5, 1, 1
  s_waitcnt lgkmcnt(3)
  v_readfirstlane_b32 s43, v163
  MM 5, 2, 1
  s_and_b32 s43, s43, %11
  s_cmp_eq_u32 s43, 0x300
  MM 5, 3, 1
  s_cbranch_scc0 SPINF_%=
BACK_%=:
  STAMPI
  LDA 0, 0
  LDA 1, 1024
  s_waitcnt lgkmcnt(3)
  MC 6, 0, 2
  LDB 0
  MC 6, 1, 2
  LDB 1
  MC 6, 2, 2
  LDB 2
  MM 6, 3, 2
  MM 7, 3, 3
  LDB 3
  s_mov_b64 exec, 1
  ds_write_b32 v165, v166 offset:32
  s_mov_b64 exec, -1
  s_branch LOOP_%=
TAIL_%=:
  s_waitcnt lgkmcnt(0)
  MM4 6, 2
  MM4 7, 3
FIN_%=:
  v_mov_b32 v166, 0x7fffffff
  v_mov_b32 v163, %3
  ds_write2_b32 v165, v166, v166 offset0:0 offset1:8
  ds_read_b32 v166, v163 offset:132
  STAMPK 2
  STAMPK 7, s_memtime
  s_waitcnt lgkmcnt(0)
  v_readfirstlane_b32 s48, v166
  s_cmp_eq_u32 s48, 0
  s_cselect_b32 s49, s49, 0x7fc00000
  s_cmp_eq_u32 %11, 0x700
  s_cbranch_scc1 EARLY_%=
.set du, 0
.rept 32
  OUT_ST du, 128+4*(du&7), %16
.set du, du+1
.endr
  s_branch END_%=
EARLY_%=:
  s_setprio 0
.set du, 0
.rept 32
  OUT_ST du, 128+4*(du&7), %15
  s_waitcnt vmcnt(%14)
.set du, du+1
.endr
END_%=:
.if %12
  STAMPK 3
  s_waitcnt vmcnt(0)
  STAMPK 4
  v_mbcnt_lo_u32_b32 v166, -1, 0
  v_mbcnt_hi_u32_b32 v166, -1, v166
  v_lshlrev_b32 v166, 2, v166
  global_store_dword v166, v160, %6
.endif
.purgem MM
.purgem MM4
.purgem MC
.purgem LDA
.purgem LDB
.purgem STAMPI
.purgem STAMPK
.purgem SPIN
.purgem ADV_A
.purgem ADV_B
.purgem OUT_ST
)ASM"

#define LOADER_ASM R"ASM(
.macro LSTAMP vr, idx
.if %14
  s_memrealtime s[56:57]
  s_sub_u32 s58, \idx, %13-64
  s_and_b32 s58, s58, 63
  s_mov_b32 s59, m0
  s_mov_b32 m0, s58
  s_waitcnt lgkmcnt(0)
  v_writelane_b32 \vr, s56, m0
  s_mov_b32 m0, s59
.endif
.endm
.macro PUBCHK
  s_getreg_b32 s47, hwreg(HW_REG_IB_STS)
  s_and_b32 s48, s47, 15
  s_bfe_u32 s47, s47, 0x20016
  s_lshl4_add_u32 s47, s47, s48
  s_sub_u32 s47, s40, s47
  s_cmp_ge_u32 s47, s41
  s_cbranch_scc0 7f
  LSTAMP v108, s44
  s_mov_b64 exec, 1
  ds_add_u32 v105, v109 offset:256
  s_mov_b64 exec, -1
  s_add_u32 s44, s44, 1
  s_mov_b32 s41, s42
  s_mov_b32 s42, s43
  s_mov_b32 s43, s49
  v_add_u32 v105, 4, v105
7:
.endm
.macro DMA4 va
  global_load_lds_dwordx4 \va, %4 sc0 sc1
  global_load_lds_dwordx4 \va, %4 offset:1024 sc0 sc1
  global_load_lds_dwordx4 \va, %4 offset:2048 sc0 sc1
  global_load_lds_dwordx4 \va, %4 offset:3072 sc0 sc1
  s_add_u32 s40, s40, 4
  s_add_u32 m0, m0, 4096
  v_add_u32 \va, 4096, \va
.endm
  s_mov_b32 s40, 0
  s_mov_b32 s49, 0x7fffffff
  s_mov_b32 s41, s49
  s_mov_b32 s42, s49
  s_mov_b32 s43, s49
  s_mov_b32 s44, 0
  s_mov_b32 s45, 0
  s_mov_b32 s46, %5
  s_mov_b32 s60, %6
  s_mov_b32 s50, 0
  v_mov_b32 v100, %1
  v_mov_b32 v101, %10
  v_mov_b32 v105, %10
  v_mov_b32 v109, %11
  v_mov_b32 v107, 0
  v_mov_b32 v108, 0
  s_waitcnt vmcnt(0)
LLOOP_%=:
  ds_read_b32 v102, %0
  ds_read_b32 v103, v101 offset:256
  PUBCHK
  s_waitcnt lgkmcnt(0)
  v_cmp_ge_i32 vcc, v102, v100
  s_nop 0
  s_cmp_eq_u64 vcc, -1
  s_cselect_b32 s47, s43, 0
  s_cmp_eq_u32 s47, s49
  s_cbranch_scc0 LIDLE_%=
LGO_%=:
  v_readfirstlane_b32 s51, v103
  s_and_b32 s48, s51, 15
  s_lshl_b32 s47, s48, 2
  s_and_b32 s61, s51, 0x70
  s_lshr_b64 s[52:53], %9, s47
  s_and_b32 s52, s52, 15
  s_or_b32 s52, s52, s61
  s_lshl_b32 s52, s52, 18
  v_add_u32 v110, s52, %3
  s_cmp_ge_u32 s48, %16
  s_cselect_b32 s51, 1, 0
  s_cmp_eq_u32 %11, 0x200
  s_cbranch_scc1 LB_%=
  s_lshr_b64 s[52:53], %8, s47
  s_and_b32 s52, s52, 15
  s_or_b32 s52, s52, s61
  s_lshl_b32 s52, s52, 5
  s_and_b32 s47, s48, s51
  s_or_b32 s52, s52, s47
  s_lshl_b32 s52, s52, 13
  v_add_u32 v104, s52, %2
  s_lshr_b32 s52, s48, 3
  s_lshl_b32 s52, s52, 13
  s_add_u32 m0, s46, s52
  s_add_u32 s46, s46, 16384
  DMA4 v104
  PUBCHK
  DMA4 v104
  s_cmp_eq_u32 s51, 1
  s_cbranch_scc0 8f
  s_mov_b32 m0, s60
  PUBCHK
  DMA4 v110
  s_branch LDONE_%=
8:
  PUBCHK
  DMA4 v104
  PUBCHK
  DMA4 v104
  s_branch LDONE_%=
LB_%=:
  s_lshl_b32 s52, s51, 12
  s_add_u32 m0, s60, s52
  v_add_u32 v110, s52, v110
  s_cmp_eq_u32 s51, 1
  s_cbranch_scc1 9f
  DMA4 v110
  PUBCHK
9:
  DMA4 v110
  PUBCHK
  DMA4 v110
  PUBCHK
  DMA4 v110
LDONE_%=:
  LSTAMP v107, s45
  s_cmp_eq_u32 s42, s49
  s_cselect_b32 s43, s49, s40
  s_cselect_b32 s47, s40, s42
  s_cmp_eq_u32 s41, s49
  s_cselect_b32 s42, s49, s47
  s_cselect_b32 s41, s40, s41
  s_cmp_eq_u32 s46, %6
  s_cselect_b32 s46, %5, s46
  s_add_u32 s60, s60, 16384
  s_cmp_eq_u32 s60, %7
  s_cselect_b32 s60, %6, s60
  s_add_u32 s45, s45, 1
  v_add_u32 v100, 1, v100
  v_add_u32 v101, 4, v101
  s_mov_b32 s50, 0
  s_cmp_lt_u32 s45, %13
  s_cbranch_scc1 LLOOP_%=
LDRAIN_%=:
  PUBCHK
  s_cmp_lt_u32 s44, %13
  s_cbranch_scc1 LDRAIN_%=
  s_branch LEND_%=
LIDLE_%=:
  s_add_u32 s50, s50, 1
  s_cmp_eq_u32 s41, s49
  s_cbranch_scc0 LLOOP_%=
LWAIT_%=:
  s_sleep %15
  ds_read_b32 v102, %0
  ds_read_b32 v103, v101 offset:256
  s_add_u32 s50, s50, 1
  s_waitcnt lgkmcnt(0)
  v_cmp_ge_i32 vcc, v102, v100
  s_cmp_eq_u64 vcc, -1
  s_cbranch_scc1 LGO_%=
  s_cmp_lt_u32 s50, 0x1000000
  s_cbranch_scc1 LWAIT_%=
  v_mov_b32 v106, 1
  v_mov_b32 v101, %10
  ds_write_b32 v101, v106 offset:132
LEND_%=:
  s_waitcnt vmcnt(0) lgkmcnt(0)
.if %14
  v_mbcnt_lo_u32_b32 v106, -1, 0
  v_mbcnt_hi_u32_b32 v106, -1, v106
  v_lshlrev_b32 v106, 2, v106
  global_store_dword v106, v107, %12
  global_store_dword v106, v108, %12 offset:256
.endif
.purgem LSTAMP
.purgem PUBCHK
.purgem DMA4
)ASM"
#define CLOBBER_S "s40", "s41", "s42", "s43", "s44", "s45", "s46", "s47", "s48", "s49", "s50", "s51", "s52", "s53", "s54", "s55", "s56", "s57", "s58", "s59", "s60", "s61", "s62", "s63"
#define V10(t) "v" #t "0", "v" #t "1", "v" #t "2", "v" #t "3", "v" #t "4", "v" #t "5", "v" #t "6", "v" #t "7", "v" #t "8", "v" #t "9"
#define CLOBBER_V V10(), V10(1), V10(2), V10(3), V10(4), V10(5), V10(6), V10(7), V10(8), V10(9), V10(10), V10(11), V10(12), V10(13), V10(14), V10(15), \
    "v160", "v161", "v162", "v163", "v164", "v165", "v166", "v167"

__global__ void __launch_bounds__(768, 3) gemm_ar_fused(Params P) {
    __shared__ __attribute__((aligned(16))) unsigned char lds_raw[LDS_BYTES];
    LAS unsigned char* lds = (LAS unsigned char*)lds_raw;
    const unsigned lbase = (unsigned)(size_t)lds;
    const int tid = threadIdx.x, lane = tid & 63;
    const int wave = __builtin_amdgcn_readfirstlane(tid >> 6);
    const int b = blockIdx.x;
    const int rank = P.rank;
    const int c = ((b >> 5) << 5) | ((b & 7) << 2) | ((b >> 3) & 3);
    char* myws = P.ws[rank];
    const int ti = ((b & 7) >> 1) * 4 + (b >> 6), tj = (b & 1) * 8 + ((b >> 3) & 7);
    if (wave < 9) STAMP1(0);
    GAS unsigned* tmo = (GAS unsigned*)(myws + WS_TMO);

    if (wave < 8) {
        __builtin_amdgcn_s_barrier();
        const unsigned call = lds_ld(lbase + L_CALL);
        {
            const int ks = wave;
            const int cc = lane & 15, g = lane >> 4;
            const GAS unsigned* wp = (const GAS unsigned*)(P.w + (size_t)(ks * 64 + g * 16) * DIM + (c >> 2) * 64 + cc * 4);
            unsigned d[16];
#pragma unroll
            for (int t = 0; t < 16; ++t) d[t] = wp[(size_t)t * (DIM / 4)];
            const unsigned xs = (unsigned)(xstep_tab() >> (8 * wave)) & 0xffu;
            const int sx = xs & 15, sw = (wstep_tab() >> (4 * wave)) & 15;
            const u32x4 xv = *x_src(P.x, xs >> 4, (xk_tab() >> (4 * wave)) & 15, c, lane);
            const unsigned q = c & 3;
            const unsigned sel_lo = 0x0c0c0000u | ((4u + q) << 8) | q;
            const unsigned sel_hi = 0x00000c0cu | ((4u + q) << 24) | (q << 16);
            u32x4 wv;
#pragma unroll
            for (int e = 0; e < 4; ++e)
                wv[e] = __builtin_amdgcn_perm(d[4 * e + 1], d[4 * e], sel_lo) | __builtin_amdgcn_perm(d[4 * e + 3], d[4 * e + 2], sel_hi);
            *(LAS u32x4*)(lds + L_WST + ks * 1024 + lane * 16) = wv;
            asm volatile("" ::: "memory");
            if (lane == 0) __hip_atomic_fetch_add((LAS unsigned*)(lds + L_WRDY), 1u, __ATOMIC_RELAXED, __HIP_MEMORY_SCOPE_WORKGROUP);
            const unsigned soff = (unsigned)(WS_STAGE + (call & 1u) * PARITY_BYTES + (size_t)rank * SENDER_BYTES) + c * 1024 + lane * 16;
            st16_sys(myws, soff + sx * (unsigned)STEP_BYTES, xv);
            st16_sys(myws, soff + sw * (unsigned)STEP_BYTES, wv);
            STAMP1(1);
            wait_vm<0>();
            if (lane == 0) {
                if (__hip_atomic_fetch_add((LAS unsigned*)(lds + L_LOC), 1u, __ATOMIC_RELAXED, __HIP_MEMORY_SCOPE_WORKGROUP) == 7u) {
                    GAS unsigned* tk = (GAS unsigned*)(myws + WS_AGG + (NSTEP * 16 + (c >> 4)) * 128);
                    if ((__hip_atomic_fetch_add(tk, 1u, __ATOMIC_RELAXED, __HIP_MEMORY_SCOPE_AGENT) & 15u) == 15u)
                    {
                        __hip_atomic_store((GAS unsigned*)(myws + WS_PROG + (c >> 4) * PROW + rank * 4), call * 32u + NSTEP, __ATOMIC_RELAXED, __HIP_MEMORY_SCOPE_SYSTEM);
                        __hip_atomic_store((GAS unsigned*)(myws + WS_PROG + (16 + (c >> 4)) * PROW + rank * 4), call * 32u + NSTEP, __ATOMIC_RELAXED, __HIP_MEMORY_SCOPE_SYSTEM);
                    }
                }
            }
        }
        const int wr = wave >> 2, wc = wave & 3;
        const unsigned scale = __builtin_amdgcn_readfirstlane(__float_as_uint(P.sx[0] * P.sw[0]));
        const char* obase = (const char*)(P.out + (size_t)(ti * 256 + wr * 128) * DIM + tj * 256 + wc * 64);
        char* dbg = myws;
        if (CPRIO_HI && wave >= 4) __builtin_amdgcn_s_setprio(CPRIO_HI);
        asm volatile(CONSUMER_ASM
            :: "s"(lbase + L_RA + wr * 8192), "s"(lbase + L_RB + wc * 4096), "s"(lbase + L_REL + wave * 4), "s"(lbase + L_CTL),
               "s"(obase), "s"(scale), "s"(dbg),
               "n"(NITEM), "n"(NUP), "n"(NSA), "n"(NSB), "s"(0x300u | (0x400u << wr)), "n"(STAMPS_ON), "n"(CSLEEP), "n"(LATE ? EPW : 63), "n"(DRIPFL), "n"(FINFL)
            : "memory", "scc", "vcc", CLOBBER_S, CLOBBER_V);
        return;
    } else if (wave == 8) {
        const unsigned tp0 = (unsigned)rt() + T0TICKS;
        __builtin_amdgcn_s_setprio(3);
        GAS unsigned* ctr = (GAS unsigned*)(myws + WS_CTR + (size_t)b * 128);
        unsigned callv; u32x4 xv[8];
        asm volatile("global_load_dword %0, %1, off sc1" : "=v"(callv) : "v"(ctr) : "memory");
        static_for<8>([&](auto nc) __attribute__((always_inline)) {
            constexpr int n = decltype(nc)::value; constexpr StepDesc sd = step_desc(nth_step(false, n));
            ld16_async(xv[n], x_src(P.x, sd.kind, sd.k, c, lane));
        });
        const char* peer[7];
#pragma unroll
        for (int r = 0; r < 7; ++r) peer[r] = P.ws[(rank + 1 + r) & (NR - 1)];
        const char* fl_lane = peer[0];
#pragma unroll
        for (int r = 1; r < 7; ++r) fl_lane = lane == r ? peer[r] : fl_lane;
        fl_lane += WS_PROG + (c >> 4) * PROW + rank * 4;
        const unsigned phase = (unsigned)((c + PROT * rank) & 255);
        asm volatile("s_waitcnt vmcnt(7)" : "+v"(callv), "+v"(xv[0]) :: "memory");
        const unsigned call = __builtin_amdgcn_readfirstlane(callv) + 1u;
        const unsigned doff = (unsigned)(WS_STAGE + (call & 1u) * PARITY_BYTES + (size_t)rank * SENDER_BYTES) + c * 1024 + lane * 16;
        const unsigned aoff = (unsigned)(WS_AGG + (c >> 4) * 128);
        unsigned tack = 0;
        STAMP1(1);
        lds_st(lbase + L_CALL, call); wait_lgkm0(); __builtin_amdgcn_s_barrier();
        static_for<NSTEP>([&](auto sc) __attribute__((always_inline)) {
            constexpr int s = decltype(sc)::value;
            u32x4 v;
            if constexpr (s == 1) {
                pin(xv[1]); pin(xv[2]); pin(xv[3]); pin(xv[4]); pin(xv[5]); pin(xv[6]); pin(xv[7]);
                unsigned sp = 0; while (lds_ld(lbase + L_WRDY) < 8u && ++sp < (1u << 24)) __builtin_amdgcn_s_sleep(1);
            }
            if constexpr (step_desc(s).kind == 1) { v = *(const LAS u32x4*)(lds + L_WST + step_desc(s).k * 1024 + lane * 16); pin(v); }
            else v = xv[x_index(s)];
            {
                unsigned tgo = tp0 + step_t(s) + (phase * step_len16(s) >> 12);
                if constexpr (L0T > 0 && s > 0) { const unsigned t2 = tack + (step_len16(s) >> 4) - L0T; if ((int)(t2 - tgo) > 0) tgo = t2; }
                int rem;
                while ((rem = (int)(tgo - (unsigned)rt())) > 0) { if (rem > 40) __builtin_amdgcn_s_sleep(8); else __builtin_amdgcn_s_sleep(1); }
            }
            STAMP0(s);
#pragma unroll
            for (int r = 0; r < 7; ++r) st16_sys(peer[(r + s) % 7], doff + (unsigned)(s * STEP_BYTES), v);
            wait_vm<0>();
            STAMP0(16 + s);
            unsigned tick;
            ticket_async(tick, myws, aoff + (unsigned)(s * 16 * 128));
            if constexpr (L0T > 0) tack = (unsigned)rt();
            wait_vm<0>();
            pin(tick);
            if ((__builtin_amdgcn_readfirstlane(tick) & 15u) == 15u) {
                if (lane < 7) st4_sys_lane(fl_lane + (step_desc(s).kind == 1 ? 16 * PROW : 0), call * 32u + s + 1);
            }
            STAMP0(32 + s);
            if constexpr (s == WARMSTEP) {
                unsigned dummy;
                ld4_wait(dummy, (const char*)(P.out + (size_t)(ti * 256 + lane * 4) * DIM + tj * 256));
            }
        });
        STAMP1(2);
        if (lane == 0) __hip_atomic_store(ctr, call, __ATOMIC_RELAXED, __HIP_MEMORY_SCOPE_AGENT);
        wait_vm<0>();
        STAMP1(3);
    } else if (wave == 9) {
        for (int u = lane; u < (LDS_BYTES - L_CTL) / 4; u += 64) if (u != (L_CALL - L_CTL) / 4) lds_st(lbase + L_CTL + u * 4, 0u);
        wait_lgkm0();
        __builtin_amdgcn_s_barrier();
        const unsigned call = lds_ld(lbase + L_CALL);
        __builtin_amdgcn_s_setprio(2);
        const int p = lane >> 1, sub = lane & 1;
        const char* px = myws + WS_PROG + ti * PROW;
        const char* pw = myws + WS_PROG + (16 + tj) * PROW;
        unsigned long long tab = sub ? step_tab(1) : step_tab(0);
        unsigned ent = p * 16 + sub;
        unsigned base = lane < 16 ? call * 32u : 0xfffffff0u;
        bool dead = false;
        int queued = 0;
        unsigned spins = 0, lastsum = 0xffffffffu;
        unsigned long long t0 = 0;
        while (queued < NITEM) {
            u32x8 fx, fw;
            asm volatile("s_load_dwordx8 %0, %2, 0x0 glc\n\ts_load_dwordx8 %1, %3, 0x0 glc\n\ts_waitcnt lgkmcnt(0)" : "=&s"(fx), "=&s"(fw) : "s"(px), "s"(pw) : "memory");
            unsigned sum = 0;
#pragma unroll
            for (int i = 0; i < 8; ++i) sum += fx[i] + fw[i];
            if (sum != lastsum) {
                lastsum = sum; spins = 0;
                unsigned val = 0;
                asm volatile("v_writelane_b32 %0, %1, 0\n\tv_writelane_b32 %0, %9, 1\n\tv_writelane_b32 %0, %2, 2\n\tv_writelane_b32 %0, %10, 3\n\t"
                             "v_writelane_b32 %0, %3, 4\n\tv_writelane_b32 %0, %11, 5\n\tv_writelane_b32 %0, %4, 6\n\tv_writelane_b32 %0, %12, 7\n\t"
                             "v_writelane_b32 %0, %5, 8\n\tv_writelane_b32 %0, %13, 9\n\tv_writelane_b32 %0, %6, 10\n\tv_writelane_b32 %0, %14, 11\n\t"
                             "v_writelane_b32 %0, %7, 12\n\tv_writelane_b32 %0, %15, 13\n\tv_writelane_b32 %0, %8, 14\n\tv_writelane_b32 %0, %16, 15"
                             : "+v"(val) : "s"(fx[0]), "s"(fx[1]), "s"(fx[2]), "s"(fx[3]), "s"(fx[4]), "s"(fx[5]), "s"(fx[6]), "s"(fx[7]),
                               "s"(fw[0]), "s"(fw[1]), "s"(fw[2]), "s"(fw[3]), "s"(fw[4]), "s"(fw[5]), "s"(fw[6]), "s"(fw[7]));
                if (dead) val = 0xffffffe0u;
                for (;;) {
                    unsigned bal = (unsigned)__ballot(val > base + ((unsigned)tab & 15u));
                    bal &= bal >> 1; bal &= 0x5555u;
                    if (!bal) break;
                    const bool rdy = (bal >> (p * 2)) & 1u;
                    const bool two = ((ent - sub) & 15u) >= (unsigned)NKS;
                    const bool emit = rdy && lane < 16 && (sub == 0 || two);
                    const unsigned lead = (unsigned)__ballot(emit);
                    const int pos = queued + __builtin_popcount(lead & ((1u << (lane & 31)) - 1u));
                    const int nq = queued + __builtin_popcount(lead);
                    if (emit) lds_st(lbase + L_QUEUE + pos * 4, ent | (two ? 0x400u : ((ent & 15u) >= (unsigned)NFULL ? 0x800u : 0u)));
                    if (lane == 0) lds_st(lbase + L_QCNT, (unsigned)nq);
                    const unsigned adv = (rdy && lane < 16) ? (two ? 2u : 1u) : 0u;
                    ent += adv; tab >>= 4 * adv;
                    if (((ent - sub) & 15u) >= (unsigned)IPS) base = 0xfffffff0u;
                    queued = nq;
                }
            } else {
                __builtin_amdgcn_s_sleep(PSLEEP);
                if ((++spins & 1023u) == 0u) {
                    if (spins == 1024u) t0 = rt();
                    if (__builtin_amdgcn_readfirstlane(__hip_atomic_load(tmo, __ATOMIC_RELAXED, __HIP_MEMORY_SCOPE_AGENT)) == call || rt() - t0 > DEADLINE_TICKS) {
                        if (lane == 0) __hip_atomic_store(tmo, call, __ATOMIC_RELAXED, __HIP_MEMORY_SCOPE_AGENT);
                        lds_st(lbase + L_ABORT, 1u);
                        dead = true; lastsum = ~sum;
                    }
                }
            }
        }
        wait_vm<0>();
    } else {
        const int hb = wave - 10;
        __builtin_amdgcn_s_barrier();
        const unsigned call = lds_ld(lbase + L_CALL);
        __builtin_amdgcn_s_setprio(LPRIO);
        const unsigned goff = (unsigned)(WS_STAGE + (call & 1u) * PARITY_BYTES) + lane * 16;
        char* dbg = myws;
        const unsigned paddr = lane < 16 ? lbase + L_REL + lane * 4 : lbase + L_QCNT;
        const int thr = lane < 8 ? (hb ? -0x40000000 : 1 - NSA) : (lane < 16 ? 1 - NSB : (lane == 16 ? 1 : -0x40000000));
        asm volatile(LOADER_ASM
            :: "v"(paddr), "v"(thr), "v"(goff + ti * 16384), "v"(goff + tj * 16384), "s"(myws),
               "s"(lbase + L_RA), "s"(lbase + L_RB), "s"(lbase + L_RB + NSB * 16384), "s"(step_tab(0)), "s"(step_tab(1)),
               "s"(lbase + L_CTL), "s"(256u << hb), "s"(dbg), "n"(NITEM), "n"(STAMPS_ON), "n"(LSLEEP), "n"(NFULL)
            : "memory", "scc", "vcc", CLOBBER_S, "v100", "v101", "v102", "v103", "v104", "v105", "v106", "v107", "v108", "v109", "v110");
        return;
    }
    STAMP_OUT();
}

extern "C" void kernel_launch(int rank, int nranks, ncclComm_t comm,
                              hipStream_t stream,
                              void* const* d_in, const int* in_sizes, int n_in,
                              void* d_out, int out_size,
                              const int* in_dtypes, int out_dtype,
                              void* const* d_out_all,
                              void* const* d_ws_all, size_t ws_bytes) {
    (void)comm; (void)in_dtypes; (void)out_dtype; (void)d_out_all;
    if (nranks != NR || n_in != 4 || (unsigned)rank >= (unsigned)NR || ws_bytes < WS_NEED || out_size != DIM * DIM ||
        in_sizes[0] != DIM * KSH || in_sizes[1] != KSH * DIM) {
        fprintf(stderr, "kernel_launch: unexpected problem shape\n");
        return;
    }
    Params P;
    P.x = (const signed char*)d_in[0]; P.w = (const signed char*)d_in[1];
    P.sx = (const float*)d_in[2]; P.sw = (const float*)d_in[3];
    P.out = (float*)d_out; P.rank = rank;
    for (int p = 0; p < NR; ++p) P.ws[p] = (char*)d_ws_all[p];
    hipLaunchKernelGGL(gemm_ar_fused, dim3(NWG), dim3(768), 0, stream, P);
    HIP_CHECK(hipGetLastError());
}
```

```cpp
#include <hip/hip_runtime.h>
#include <rccl/rccl.h>
#include <cstdio>
#include <cstdint>
#include <utility>

#define HIP_CHECK(expr) do { hipError_t _e = (expr); if (_e != hipSuccess) { \
    fprintf(stderr, "HIP error %s at %s:%d\n", hipGetErrorString(_e), __FILE__, __LINE__); \
    return; } } while (0)

#define LAS __attribute__((address_space(3)))
#define GAS __attribute__((address_space(1)))
typedef int i32x4 __attribute__((ext_vector_type(4)));
typedef unsigned u32x4 __attribute__((ext_vector_type(4)));
typedef unsigned u32x2 __attribute__((ext_vector_type(2)));
typedef unsigned u32x8 __attribute__((ext_vector_type(8)));
typedef float f32x4 __attribute__((ext_vector_type(4)));

#define LATE 1
#define PDEPTH_ 1
#define NFINE 2
#define TAU16 1024
#define T0TICKS 70
#define DRIP 2
#define LPRIO 2
#define CSLEEP 1
#define LSLEEP 1
#define CPRIO_HI 0
#define DGAP 0
#define PH2 112
#define TAU2 TAU16
#define PH2W0 1
#define PROT 32
#define DRIPFL 1
#define FINFL 1
#define EPW 2
#define PSLEEP 1
#define PROW 256
#define L0T 110
#define WARMSTEP 99
#define GOPH 128
#define ALPHA8 2
#define TLBWARM 1
#define PEERROT 1
#define TOUCH 0
#define NSA 5
#define NSB 4

namespace {
constexpr int NR = 8;
constexpr int DIM = 4096;
constexpr int KSH = DIM / NR;
constexpr int NKS = KSH / 64;
constexpr int NWG = 256;
constexpr int NSTEP = 16;
constexpr int NSPLIT = 2 * LATE;
constexpr int NFULL = NKS - NSPLIT;
constexpr int IPS = NFULL + 2 * NSPLIT;
constexpr int NITEM = NR * IPS;
constexpr int NUP = NR * NKS;
struct StepDesc { int kind, k; };
constexpr StepDesc step_desc(int s) {
    if (s < 2 * NFULL) return {s & 1, s >> 1};
    s -= 2 * NFULL;
    if (s < 3 * LATE) { const int g = s / 3, r = s % 3; return r == 0 ? StepDesc{2, NFULL + 2 * g} : StepDesc{1, NFULL + 2 * g + r - 1}; }
    s -= 3 * LATE;
    return {3, NFULL + 2 * s};
}
constexpr int find_step(int kind, int k) { for (int s = 0; s < NSTEP; ++s) if (step_desc(s).kind == kind && step_desc(s).k == k) return s; return -1; }
constexpr int nth_step(bool w, int n) { for (int s = 0; s < NSTEP; ++s) if ((step_desc(s).kind == 1) == w && n-- == 0) return s; return -1; }
constexpr unsigned item_desc(int i) {
    if (i < NFULL) return (unsigned)(find_step(0, i) | find_step(1, i) << 4);
    i -= NFULL;
    const int lower = i >= NSPLIT, k = NFULL + (lower ? i - NSPLIT : i);
    return (unsigned)(find_step(2 + lower, NFULL + ((k - NFULL) & ~1)) | find_step(1, k) << 4 | ((k - NFULL) & 1) << 8 | (1 + lower) << 9 | (lower && !((k - NFULL) & 1)) << 11);
}
constexpr unsigned long long step_tab(int ab) { unsigned long long t = 0; for (int i = 0; i < IPS; ++i) t |= (unsigned long long)((item_desc(i) >> (4 * ab)) & 15u) << (4 * i); return t; }
constexpr unsigned long long item_tab(int first) { unsigned long long t = 0; for (int i = 0; i < 5; ++i) if (first + i < IPS) t |= (unsigned long long)item_desc(first + i) << (12 * i); return t; }
constexpr unsigned long long xstep_tab() {
    unsigned long long t = 0; for (int n = 0; n < 8; ++n) t |= (unsigned long long)(nth_step(false, n) | step_desc(nth_step(false, n)).kind << 4) << (8 * n); return t; }
constexpr unsigned xk_tab() { unsigned t = 0; for (int n = 0; n < 8; ++n) t |= (unsigned)step_desc(nth_step(false, n)).k << (4 * n); return t; }
constexpr unsigned wstep_tab() { unsigned t = 0; for (int k = 0; k < 8; ++k) t |= (unsigned)find_step(1, k) << (4 * k); return t; }
constexpr int x_index(int s) { int n = 0; for (int i = 0; i < s; ++i) n += step_desc(i).kind != 1; return n; }
static_assert(IPS <= 15 && step_desc(0).kind == 0, "schedule tables");

constexpr size_t WS_PROG = 0;
constexpr size_t WS_FLAGS_BYTES = 32 * PROW;
constexpr size_t WS_CTR = WS_FLAGS_BYTES;
constexpr size_t WS_TMO = WS_CTR + (size_t)NWG * 128;
constexpr size_t WS_AGG = WS_TMO + 128;
constexpr size_t WS_STAGE = 4u << 20;
constexpr size_t STEP_BYTES = (size_t)NWG * 1024;
constexpr size_t SENDER_BYTES = NSTEP * STEP_BYTES;
constexpr size_t PARITY_BYTES = NR * SENDER_BYTES;
constexpr size_t WS_NEED = WS_STAGE + 2 * PARITY_BYTES;
static_assert(WS_AGG + (NSTEP + 1) * 16 * 128 <= WS_STAGE, "control region");
constexpr int L_RA = 0;
constexpr int L_RB = NSA * 16384;
constexpr int L_WST = L_RB + NSB * 16384;
constexpr int L_CTL = L_WST + NKS * 1024;
constexpr int L_PUB = L_CTL;
constexpr int L_REL = L_CTL + 64;
constexpr int L_QCNT = L_CTL + 128;
constexpr int L_ABORT = L_CTL + 132;
constexpr int L_CALL = L_CTL + 136;
constexpr int L_WRDY = L_CTL + 140;
constexpr int L_LOC = L_CTL + 144;
constexpr int L_GO = L_CTL + 148;
constexpr int L_QUEUE = L_CTL + 256;
constexpr int LDS_BYTES = L_QUEUE + 128 * 4;
static_assert(LDS_BYTES <= 163840 && NITEM < 128, "LDS");
constexpr unsigned long long DEADLINE_TICKS = 50000000ull;
constexpr int PDEPTH = PDEPTH_;
constexpr int NST = NSTEP * 7;

struct Params {
    const signed char* x; const signed char* w; const float* sx; const float* sw;
    float* out; char* ws[NR]; int rank;
};

#define STAMP0(idx) do { } while (0)
#define STAMP1(idx) do { } while (0)
#define STAMP_OUT() do { } while (0)

constexpr unsigned step_len16(int s) { return 7u * (s < PH2 ? TAU16 : TAU2); }
constexpr unsigned step_t(int s) { unsigned t = 0; for (int i = 0; i < s; ++i) t += step_len16(i); return t >> 4; }
constexpr int sched_t(int n) { return (n <= PH2 ? n * TAU16 : PH2 * TAU16 + (n - PH2) * TAU2) >> 4; }
constexpr bool agg_step(int s) { return s < NSTEP - NFINE; }
constexpr int npost(int n) {
    const int o = n - PDEPTH;
    if (o < 0 || o >= NST) return 0;
    return agg_step(o / 7) ? (o % 7 == 6 ? 1 : 0) : 1;
}
constexpr int nyoung(int n) {
    int c = 0;
    for (int i = n - PDEPTH + 1; i <= n; ++i) c += i < NST;
    for (int i = n - PDEPTH; i < n; ++i) c += npost(i);
    return c;
}
template <class F, int... I> __device__ __forceinline__ void static_for_impl(F&& f, std::integer_sequence<int, I...>) { (f(std::integral_constant<int, I>{}), ...); }
template <int N, class F> __device__ __forceinline__ void static_for(F&& f) { static_for_impl(f, std::make_integer_sequence<int, N>{}); }
template <int N> __device__ __forceinline__ void wait_vm() { asm volatile("s_waitcnt vmcnt(%0)" :: "n"(N) : "memory"); }
__device__ __forceinline__ void wait_vm_x4(int n4) {
    switch (n4) { case 0: wait_vm<0>(); break; case 1: wait_vm<4>(); break; case 2: wait_vm<8>(); break; case 3: wait_vm<12>(); break;
                  case 4: wait_vm<16>(); break; case 5: wait_vm<20>(); break; case 6: wait_vm<24>(); break; case 7: wait_vm<28>(); break;
                  default: wait_vm<32>(); break; }
}
__device__ __forceinline__ unsigned vm_outstanding() {
    const unsigned r = __builtin_amdgcn_s_getreg(7 | (31 << 11));
    return (r & 15u) | ((r >> 18) & 48u);
}
__device__ __forceinline__ void wait_lgkm0() { asm volatile("s_waitcnt lgkmcnt(0)" ::: "memory"); }
__device__ __forceinline__ unsigned long long rt() { return __builtin_amdgcn_s_memrealtime(); }
__device__ __forceinline__ void st16_sys(const char* base, unsigned off, u32x4 v) {
    asm volatile("global_store_dwordx4 %0, %1, %2 sc0 sc1\n\ts_nop 1" :: "v"(off), "v"(v), "s"(base) : "memory");
}
__device__ __forceinline__ void st4_sys(const char* base, unsigned off, unsigned v) {
    asm volatile("global_store_dword %0, %1, %2 sc0 sc1\n\ts_nop 1" :: "v"(off), "v"(v), "s"(base) : "memory");
}
__device__ __forceinline__ void ld16_async(u32x4& v, const GAS u32x4* p) { asm volatile("global_load_dwordx4 %0, %1, off" : "=v"(v) : "v"(p) : "memory"); }
__device__ __forceinline__ void ld4_wait(unsigned& v, const char* p) { asm volatile("global_load_dword %0, %1, off\n\ts_waitcnt vmcnt(0)" : "=v"(v) : "v"(p) : "memory"); }
__device__ __forceinline__ void stouch(const char* p) { unsigned d; asm volatile("s_load_dword %0, %1, 0x0 glc\n\ts_waitcnt lgkmcnt(0)" : "=s"(d) : "s"(p) : "memory"); }
template <class T> __device__ __forceinline__ void pin(T& v) { asm volatile("" : "+v"(v)); }
__device__ __forceinline__ void st4_sys_lane(const char* p, unsigned v) { asm volatile("global_store_dword %0, %1, off sc0 sc1\n\ts_nop 1" :: "v"(p), "v"(v) : "memory"); }
__device__ __forceinline__ void st16_sys_lane(const char* p, u32x4 v) { asm volatile("global_store_dwordx4 %0, %1, off sc0 sc1\n\ts_nop 1" :: "v"(p), "v"(v) : "memory"); }
__device__ __forceinline__ void ticket_async(unsigned& tick, const char* base, unsigned off) {
    unsigned long long keep;
    asm volatile("s_mov_b64 %1, exec\n\ts_mov_b64 exec, 1\n\tglobal_atomic_add %0, %2, %3, %4 sc0 sc1\n\ts_mov_b64 exec, %1"
                 : "=&v"(tick), "=&s"(keep) : "v"(off), "v"(1u), "s"(base) : "memory");
}
__device__ __forceinline__ void lds_st(unsigned addr, unsigned v) { asm volatile("ds_write_b32 %0, %1" :: "v"(addr), "v"(v) : "memory"); }
__device__ __forceinline__ unsigned lds_ldv(unsigned addr) {
    unsigned v; asm volatile("ds_read_b32 %0, %1\n\ts_waitcnt lgkmcnt(0)" : "=v"(v) : "v"(addr) : "memory");
    return v;
}
__device__ __forceinline__ unsigned lds_ld(unsigned addr) { return __builtin_amdgcn_readfirstlane(lds_ldv(addr)); }
__device__ __forceinline__ void dma4k(const char* base, unsigned off, unsigned lds_dst) {
    unsigned keep;
    asm volatile("s_mov_b32 %0, m0\n\ts_mov_b32 m0, %3\n\ts_nop 0\n\t"
                 "global_load_lds_dwordx4 %1, %2 sc0 sc1\n\t"
                 "global_load_lds_dwordx4 %1, %2 offset:1024 sc0 sc1\n\t"
                 "global_load_lds_dwordx4 %1, %2 offset:2048 sc0 sc1\n\t"
                 "global_load_lds_dwordx4 %1, %2 offset:3072 sc0 sc1\n\t"
                 "s_mov_b32 m0, %0"
                 : "=&s"(keep) : "v"(off), "s"(base), "s"(lds_dst) : "memory");
}
__device__ __forceinline__ const GAS u32x4* x_src(const signed char* x, int kind, int k, int c, int lane) {
    const int row16 = kind == 0 ? c : (c & ~15) + (kind - 2) * 8 + (c & 7);
    const int ks = kind == 0 ? k : k + ((c >> 3) & 1);
    return (const GAS u32x4*)(x + (size_t)(row16 * 16 + (lane & 15)) * KSH + ks * 64 + (lane >> 4) * 16);
}
}

#define STAMPS_ON 0
#define CONSUMER_ASM R"ASM(
.macro MM m, n, a
  v_mfma_i32_16x16x64_i8 v[16*(\m)+4*(\n):16*(\m)+4*(\n)+3], v[144+4*(\a):147+4*(\a)], v[128+4*(\n):131+4*(\n)], v[16*(\m)+4*(\n):16*(\m)+4*(\n)+3]
.endm
.macro MM4 m, a
  MM \m, 0, \a
  MM \m, 1, \a
  MM \m, 2, \a
  MM \m, 3, \a
.endm
.macro MC m, n, a
  MM \m, \n, \a
  MM \m+1, \n, \a+1
.endm
.macro LDA a, off
  ds_read_b128 v[144+4*(\a):147+4*(\a)], v161 offset:\off
.endm
.macro LDB n
  ds_read_b128 v[128+4*(\n):131+4*(\n)], v162 offset:1024*(\n)
.endm
.macro STAMPI
.if %12
  s_sub_u32 s48, s53, %7-64
  s_and_b32 s48, s48, 63
  s_memrealtime s[56:57]
  s_waitcnt lgkmcnt(0)
  s_mov_b32 m0, s48
  s_nop 0
  v_writelane_b32 v160, s56, m0
.endif
.endm
.macro STAMPK k, ins=s_memrealtime
.if %12
  \ins s[56:57]
  s_waitcnt lgkmcnt(0)
  v_mov_b32 v144, s56
  v_mov_b32 v163, 256+4*(\k)
  global_store_dword v163, v144, %6
  s_nop 1
.endif
.endm
.macro ADV_A
  s_add_u32 s41, s41, 1
  s_cmp_eq_u32 s41, %9
  s_cselect_b32 s44, s50, s52
  s_cselect_b32 s41, 0, s41
.endm
.macro ADV_B
  s_add_u32 s42, s42, 1
  s_cmp_eq_u32 s42, %10
  s_cselect_b32 s40, s51, s52
  s_cselect_b32 s42, 0, s42
.endm
// waits for the next item of this wave's half, passing over (and releasing) the items of the other half
.macro SPIN back
  s_mov_b32 s47, 0
1:
  s_cmp_ge_u32 s43, 0x400
  s_cbranch_scc0 2f
  s_mov_b64 exec, 1
  ds_write2_b32 v165, v166, v166 offset0:0 offset1:8
  s_mov_b64 exec, -1
  ADV_A
  ADV_B
  v_add_u32 v164, 4, v164
  v_add_u32 v161, s44, v161
  v_add_u32 v162, s40, v162
  v_add_u32 v166, 1, v166
  s_add_u32 s53, s53, 1
  s_mov_b32 s47, 0
  s_branch 3f
2:
  s_sleep %13
3:
  ds_read_b32 v163, v164 offset:256
  s_add_u32 s47, s47, 1
  s_waitcnt lgkmcnt(0)
  v_readfirstlane_b32 s43, v163
  s_and_b32 s43, s43, %11
  s_cmp_eq_u32 s43, 0x300
  s_cbranch_scc1 \back
  s_cmp_lt_u32 s47, 0x1000000
  s_cbranch_scc1 1b
  v_mov_b32 v163, %3
  v_mov_b32 v144, 1
  ds_write_b32 v163, v144 offset:132
  s_branch FIN_%=
.endm
.macro OUT_ST u, t, fl
  v_cvt_f32_i32 v[\t], v[16*((\u)/4)+((\u)&3)]
  v_cvt_f32_i32 v[\t+1], v[16*((\u)/4)+((\u)&3)+4]
  v_cvt_f32_i32 v[\t+2], v[16*((\u)/4)+((\u)&3)+8]
  v_cvt_f32_i32 v[\t+3], v[16*((\u)/4)+((\u)&3)+12]
  s_add_u32 s54, s58, (((\u)/4)*16+((\u)&3))*16384
  s_addc_u32 s55, s59, 0
  v_mul_f32 v[\t], s49, v[\t]
  v_mul_f32 v[\t+1], s49, v[\t+1]
  v_mul_f32 v[\t+2], s49, v[\t+2]
  v_mul_f32 v[\t+3], s49, v[\t+3]
.if \fl == 0
  global_store_dwordx4 v167, v[\t:\t+3], s[54:55]
.elseif \fl == 1
  global_store_dwordx4 v167, v[\t:\t+3], s[54:55] sc0 sc1
.else
  global_store_dwordx4 v167, v[\t:\t+3], s[54:55] nt
.endif
.endm

  v_mbcnt_lo_u32_b32 v160, -1, 0
  v_mbcnt_hi_u32_b32 v160, -1, v160
  v_lshrrev_b32 v167, 4, v160
  v_lshlrev_b32 v167, 16, v167
  v_and_b32 v166, 15, v160
  v_lshl_add_u32 v167, v166, 4, v167
  v_lshlrev_b32 v160, 4, v160
  v_add_u32 v161, %0, v160
  v_add_u32 v162, %1, v160
  v_mov_b32 v164, %3
  v_mov_b32 v165, %2
  v_mov_b32 v166, 1
  v_mov_b32 v160, 0
  s_mov_b64 s[58:59], %4
  s_mov_b32 s49, %5
  s_mov_b32 s41, 0
  s_mov_b32 s42, 0
  s_mov_b32 s43, 0
  s_mov_b32 s45, %8
  s_mov_b32 s50, -(%9-1)*16384
  s_mov_b32 s51, -(%10-1)*16384
  s_mov_b32 s52, 16384
  s_mov_b32 s53, 0
.set zi, 0
.rept 128
  v_mov_b32 v[zi], 0
.set zi, zi+1
.endr
  SPIN START_%=
SPINF_%=:
  s_cmp_eq_u32 s45, 0
  s_cbranch_scc1 TAIL_%=
  SPIN BACK_%=
START_%=:
  STAMPK 6, s_memtime
  STAMPK 8
  LDA 0, 0
  LDA 1, 1024
  LDB 0
  LDB 1
  LDB 2
  LDB 3
  ds_write_b32 v165, v166 offset:32
LOOP_%=:
  LDA 2, 2048
  LDA 3, 3072
  s_waitcnt lgkmcnt(6)
  MC 0, 0, 0
  s_waitcnt lgkmcnt(5)
  MC 0, 1, 0
  s_waitcnt lgkmcnt(4)
  MC 0, 2, 0
  s_waitcnt lgkmcnt(3)
  MC 0, 3, 0
  LDA 0, 4096
  LDA 1, 5120
  s_waitcnt lgkmcnt(3)
  MM 2, 0, 2
  s_add_u32 s53, s53, 1
  MM 2, 1, 2
  s_sub_u32 s45, s45, 1
  MM 2, 2, 2
  v_add_u32 v164, 4, v164
  MM 2, 3, 2
  s_waitcnt lgkmcnt(2)
  MM4 3, 3
  ds_read_b32 v163, v164 offset:256
  LDA 2, 6144
  LDA 3, 7168
  s_mov_b64 exec, 1
  ds_write_b32 v165, v166
  s_mov_b64 exec, -1
  s_waitcnt lgkmcnt(5)
  MM 4, 0, 0
  ADV_A
  MM 4, 1, 0
  v_add_u32 v161, s44, v161
  MM 4, 2, 0
  ADV_B
  MM 4, 3, 0
  v_add_u32 v162, s40, v162
  s_waitcnt lgkmcnt(4)
  MM 5, 0, 1
  v_add_u32 v166, 1, v166
  MM 5, 1, 1
  s_waitcnt lgkmcnt(3)
  v_readfirstlane_b32 s43, v163
  MM 5, 2, 1
  s_and_b32 s43, s43, %11
  s_cmp_eq_u32 s43, 0x300
  MM 5, 3, 1
  s_cbranch_scc0 SPINF_%=
BACK_%=:
  STAMPI
  LDA 0, 0
  LDA 1, 1024
  s_waitcnt lgkmcnt(3)
  MC 6, 0, 2
  LDB 0
  MC 6, 1, 2
  LDB 1
  MC 6, 2, 2
  LDB 2
  MM 6, 3, 2
  MM 7, 3, 3
  LDB 3
  s_mov_b64 exec, 1
  ds_write_b32 v165, v166 offset:32
  s_mov_b64 exec, -1
  s_branch LOOP_%=
TAIL_%=:
  s_waitcnt lgkmcnt(0)
  MM4 6, 2
  MM4 7, 3
FIN_%=:
  v_mov_b32 v166, 0x7fffffff
  v_mov_b32 v163, %3
  ds_write2_b32 v165, v166, v166 offset0:0 offset1:8
  ds_read_b32 v166, v163 offset:132
  STAMPK 2
  STAMPK 7, s_memtime
  s_waitcnt lgkmcnt(0)
  v_readfirstlane_b32 s48, v166
  s_cmp_eq_u32 s48, 0
  s_cselect_b32 s49, s49, 0x7fc00000
  s_cmp_eq_u32 %11, 0x700
  s_cbranch_scc1 EARLY_%=
.set du, 0
.rept 32
  OUT_ST du, 128+4*(du&7), %16
.set du, du+1
.endr
  s_branch END_%=
EARLY_%=:
  s_setprio 0
.set du, 0
.rept 32
  OUT_ST du, 128+4*(du&7), %15
  s_waitcnt vmcnt(%14)
.set du, du+1
.endr
END_%=:
.if %12
  STAMPK 3
  s_waitcnt vmcnt(0)
  STAMPK 4
  v_mbcnt_lo_u32_b32 v166, -1, 0
  v_mbcnt_hi_u32_b32 v166, -1, v166
  v_lshlrev_b32 v166, 2, v166
  global_store_dword v166, v160, %6
.endif
.purgem MM
.purgem MM4
.purgem MC
.purgem LDA
.purgem LDB
.purgem STAMPI
.purgem STAMPK
.purgem SPIN
.purgem ADV_A
.purgem ADV_B
.purgem OUT_ST
)ASM"

#define LOADER_ASM R"ASM(
.macro LSTAMP vr, idx
.if %14
  s_memrealtime s[56:57]
  s_sub_u32 s58, \idx, %13-64
  s_and_b32 s58, s58, 63
  s_mov_b32 s59, m0
  s_mov_b32 m0, s58
  s_waitcnt lgkmcnt(0)
  v_writelane_b32 \vr, s56, m0
  s_mov_b32 m0, s59
.endif
.endm
.macro PUBCHK
  s_getreg_b32 s47, hwreg(HW_REG_IB_STS)
  s_and_b32 s48, s47, 15
  s_bfe_u32 s47, s47, 0x20016
  s_lshl4_add_u32 s47, s47, s48
  s_sub_u32 s47, s40, s47
  s_cmp_ge_u32 s47, s41
  s_cbranch_scc0 7f
  LSTAMP v108, s44
  s_mov_b64 exec, 1
  ds_add_u32 v105, v109 offset:256
  s_mov_b64 exec, -1
  s_add_u32 s44, s44, 1
  s_mov_b32 s41, s42
  s_mov_b32 s42, s43
  s_mov_b32 s43, s49
  v_add_u32 v105, 4, v105
7:
.endm
.macro DMA4 va
  global_load_lds_dwordx4 \va, %4 sc0 sc1
  global_load_lds_dwordx4 \va, %4 offset:1024 sc0 sc1
  global_load_lds_dwordx4 \va, %4 offset:2048 sc0 sc1
  global_load_lds_dwordx4 \va, %4 offset:3072 sc0 sc1
  s_add_u32 s40, s40, 4
  s_add_u32 m0, m0, 4096
  v_add_u32 \va, 4096, \va
.endm
  s_mov_b32 s40, 0
  s_mov_b32 s49, 0x7fffffff
  s_mov_b32 s41, s49
  s_mov_b32 s42, s49
  s_mov_b32 s43, s49
  s_mov_b32 s44, 0
  s_mov_b32 s45, 0
  s_mov_b32 s46, %5
  s_mov_b32 s60, %6
  s_mov_b32 s50, 0
  v_mov_b32 v100, %1
  v_mov_b32 v101, %10
  v_mov_b32 v105, %10
  v_mov_b32 v109, %11
  v_mov_b32 v107, 0
  v_mov_b32 v108, 0
  s_waitcnt vmcnt(0)
LLOOP_%=:
  ds_read_b32 v102, %0
  ds_read_b32 v103, v101 offset:256
  PUBCHK
  s_waitcnt lgkmcnt(0)
  v_cmp_ge_i32 vcc, v102, v100
  s_nop 0
  s_cmp_eq_u64 vcc, -1
  s_cselect_b32 s47, s43, 0
  s_cmp_eq_u32 s47, s49
  s_cbranch_scc0 LIDLE_%=
LGO_%=:
  v_readfirstlane_b32 s51, v103
  s_and_b32 s48, s51, 15
  s_lshl_b32 s47, s48, 2
  s_and_b32 s61, s51, 0x70
  s_lshr_b64 s[52:53], %9, s47
  s_and_b32 s52, s52, 15
  s_or_b32 s52, s52, s61
  s_lshl_b32 s52, s52, 18
  v_add_u32 v110, s52, %3
  s_cmp_ge_u32 s48, %16
  s_cselect_b32 s51, 1, 0
  s_cmp_eq_u32 %11, 0x200
  s_cbranch_scc1 LB_%=
  s_lshr_b64 s[52:53], %8, s47
  s_and_b32 s52, s52, 15
  s_or_b32 s52, s52, s61
  s_lshl_b32 s52, s52, 5
  s_and_b32 s47, s48, s51
  s_or_b32 s52, s52, s47
  s_lshl_b32 s52, s52, 13
  v_add_u32 v104, s52, %2
  s_lshr_b32 s52, s48, 3
  s_lshl_b32 s52, s52, 13
  s_add_u32 m0, s46, s52
  s_add_u32 s46, s46, 16384
  DMA4 v104
  PUBCHK
  DMA4 v104
  s_cmp_eq_u32 s51, 1
  s_cbranch_scc0 8f
  s_mov_b32 m0, s60
  PUBCHK
  DMA4 v110
  s_branch LDONE_%=
8:
  PUBCHK
  DMA4 v104
  PUBCHK
  DMA4 v104
  s_branch LDONE_%=
LB_%=:
  s_lshl_b32 s52, s51, 12
  s_add_u32 m0, s60, s52
  v_add_u32 v110, s52, v110
  s_cmp_eq_u32 s51, 1
  s_cbranch_scc1 9f
  DMA4 v110
  PUBCHK
9:
  DMA4 v110
  PUBCHK
  DMA4 v110
  PUBCHK
  DMA4 v110
LDONE_%=:
  LSTAMP v107, s45
  s_cmp_eq_u32 s42, s49
  s_cselect_b32 s43, s49, s40
  s_cselect_b32 s47, s40, s42
  s_cmp_eq_u32 s41, s49
  s_cselect_b32 s42, s49, s47
  s_cselect_b32 s41, s40, s41
  s_cmp_eq_u32 s46, %6
  s_cselect_b32 s46, %5, s46
  s_add_u32 s60, s60, 16384
  s_cmp_eq_u32 s60, %7
  s_cselect_b32 s60, %6, s60
  s_add_u32 s45, s45, 1
  v_add_u32 v100, 1, v100
  v_add_u32 v101, 4, v101
  s_mov_b32 s50, 0
  s_cmp_lt_u32 s45, %13
  s_cbranch_scc1 LLOOP_%=
LDRAIN_%=:
  PUBCHK
  s_cmp_lt_u32 s44, %13
  s_cbranch_scc1 LDRAIN_%=
  s_branch LEND_%=
LIDLE_%=:
  s_add_u32 s50, s50, 1
  s_cmp_eq_u32 s41, s49
  s_cbranch_scc0 LLOOP_%=
LWAIT_%=:
  s_sleep %15
  ds_read_b32 v102, %0
  ds_read_b32 v103, v101 offset:256
  s_add_u32 s50, s50, 1
  s_waitcnt lgkmcnt(0)
  v_cmp_ge_i32 vcc, v102, v100
  s_cmp_eq_u64 vcc, -1
  s_cbranch_scc1 LGO_%=
  s_cmp_lt_u32 s50, 0x1000000
  s_cbranch_scc1 LWAIT_%=
  v_mov_b32 v106, 1
  v_mov_b32 v101, %10
  ds_write_b32 v101, v106 offset:132
LEND_%=:
  s_waitcnt vmcnt(0) lgkmcnt(0)
.if %14
  v_mbcnt_lo_u32_b32 v106, -1, 0
  v_mbcnt_hi_u32_b32 v106, -1, v106
  v_lshlrev_b32 v106, 2, v106
  global_store_dword v106, v107, %12
  global_store_dword v106, v108, %12 offset:256
.endif
.purgem LSTAMP
.purgem PUBCHK
.purgem DMA4
)ASM"
#define CLOBBER_S "s40", "s41", "s42", "s43", "s44", "s45", "s46", "s47", "s48", "s49", "s50", "s51", "s52", "s53", "s54", "s55", "s56", "s57", "s58", "s59", "s60", "s61", "s62", "s63"
#define V10(t) "v" #t "0", "v" #t "1", "v" #t "2", "v" #t "3", "v" #t "4", "v" #t "5", "v" #t "6", "v" #t "7", "v" #t "8", "v" #t "9"
#define CLOBBER_V V10(), V10(1), V10(2), V10(3), V10(4), V10(5), V10(6), V10(7), V10(8), V10(9), V10(10), V10(11), V10(12), V10(13), V10(14), V10(15), \
    "v160", "v161", "v162", "v163", "v164", "v165", "v166", "v167"

__global__ void __launch_bounds__(768, 3) gemm_ar_fused(Params P) {
    __shared__ __attribute__((aligned(16))) unsigned char lds_raw[LDS_BYTES];
    LAS unsigned char* lds = (LAS unsigned char*)lds_raw;
    const unsigned lbase = (unsigned)(size_t)lds;
    const int tid = threadIdx.x, lane = tid & 63;
    const int wave = __builtin_amdgcn_readfirstlane(tid >> 6);
    const int b = blockIdx.x;
    const int rank = P.rank;
    const int c = ((b >> 5) << 5) | ((b & 7) << 2) | ((b >> 3) & 3);
    char* myws = P.ws[0];
    const int ti = ((b & 7) >> 1) * 4 + (b >> 6), tj = (b & 1) * 8 + ((b >> 3) & 7);
    if (wave < 9) STAMP1(0);
    GAS unsigned* tmo = (GAS unsigned*)(myws + WS_TMO);

    if (wave < 8) {
        unsigned warm = 0;
        if (TLBWARM && wave == 0) {
            const unsigned ph = (unsigned)((c + PROT * rank) & 255), kk = ((ph >> 5) - 4u) * 4u + (ph & 3u);
            if (ph >= 128u && kk < 14u) {
                const char* pw = P.ws[1 + kk % 7u] + WS_STAGE + (kk / 7u) * PARITY_BYTES + (size_t)rank * SENDER_BYTES;
                asm volatile("s_load_dword %0, %1, 0x0 glc" : "=s"(warm) : "s"(pw) : "memory");
            }
        }
        __builtin_amdgcn_s_barrier();
        const unsigned call = lds_ld(lbase + L_CALL);
        asm volatile("" :: "s"(warm));
        if (((c + PROT * rank) & 255) < GOPH) { unsigned sp = 0; while (lds_ld(lbase + L_GO) == 0u && ++sp < (1u << 22)) __builtin_amdgcn_s_sleep(2); }
        {
            const int ks = wave;
            const int cc = lane & 15, g = lane >> 4;
            const GAS unsigned* wp = (const GAS unsigned*)(P.w + (size_t)(ks * 64 + g * 16) * DIM + (c >> 2) * 64 + cc * 4);
            unsigned d[16];
#pragma unroll
            for (int t = 0; t < 16; ++t) d[t] = wp[(size_t)t * (DIM / 4)];
            const unsigned xs = (unsigned)(xstep_tab() >> (8 * wave)) & 0xffu;
            const int sx = xs & 15, sw = (wstep_tab() >> (4 * wave)) & 15;
            const u32x4 xv = *x_src(P.x, xs >> 4, (xk_tab() >> (4 * wave)) & 15, c, lane);
            const unsigned q = c & 3;
            const unsigned sel_lo = 0x0c0c0000u | ((4u + q) << 8) | q;
            const unsigned sel_hi = 0x00000c0cu | ((4u + q) << 24) | (q << 16);
            u32x4 wv;
#pragma unroll
            for (int e = 0; e < 4; ++e)
                wv[e] = __builtin_amdgcn_perm(d[4 * e + 1], d[4 * e], sel_lo) | __builtin_amdgcn_perm(d[4 * e + 3], d[4 * e + 2], sel_hi);
            *(LAS u32x4*)(lds + L_WST + ks * 1024 + lane * 16) = wv;
            asm volatile("" ::: "memory");
            if (lane == 0) __hip_atomic_fetch_add((LAS unsigned*)(lds + L_WRDY), 1u, __ATOMIC_RELAXED, __HIP_MEMORY_SCOPE_WORKGROUP);
            const unsigned soff = (unsigned)(WS_STAGE + (call & 1u) * PARITY_BYTES + (size_t)rank * SENDER_BYTES) + c * 1024 + lane * 16;
            st16_sys(myws, soff + sx * (unsigned)STEP_BYTES, xv);
            st16_sys(myws, soff + sw * (unsigned)STEP_BYTES, wv);
            STAMP1(1);
            wait_vm<0>();
            if (lane == 0) {
                if (__hip_atomic_fetch_add((LAS unsigned*)(lds + L_LOC), 1u, __ATOMIC_RELAXED, __HIP_MEMORY_SCOPE_WORKGROUP) == 7u) {
                    GAS unsigned* tk = (GAS unsigned*)(myws + WS_AGG + (NSTEP * 16 + (c >> 4)) * 128);
                    if ((__hip_atomic_fetch_add(tk, 1u, __ATOMIC_RELAXED, __HIP_MEMORY_SCOPE_AGENT) & 15u) == 15u)
                    {
                        __hip_atomic_store((GAS unsigned*)(myws + WS_PROG + (c >> 4) * PROW + rank * 4), call * 32u + NSTEP, __ATOMIC_RELAXED, __HIP_MEMORY_SCOPE_SYSTEM);
                        __hip_atomic_store((GAS unsigned*)(myws + WS_PROG + (16 + (c >> 4)) * PROW + rank * 4), call * 32u + NSTEP, __ATOMIC_RELAXED, __HIP_MEMORY_SCOPE_SYSTEM);
                    }
                }
            }
        }
        const int wr = wave >> 2, wc = wave & 3;
        const unsigned scale = __builtin_amdgcn_readfirstlane(__float_as_uint(P.sx[0] * P.sw[0]));
        const char* obase = (const char*)(P.out + (size_t)(ti * 256 + wr * 128) * DIM + tj * 256 + wc * 64);
        char* dbg = myws;
        if (CPRIO_HI && wave >= 4) __builtin_amdgcn_s_setprio(CPRIO_HI);
        asm volatile(CONSUMER_ASM
            :: "s"(lbase + L_RA + wr * 8192), "s"(lbase + L_RB + wc * 4096), "s"(lbase + L_REL + wave * 4), "s"(lbase + L_CTL),
               "s"(obase), "s"(scale), "s"(dbg),
               "n"(NITEM), "n"(NUP), "n"(NSA), "n"(NSB), "s"(0x300u | (0x400u << wr)), "n"(STAMPS_ON), "n"(CSLEEP), "n"(LATE ? EPW : 63), "n"(DRIPFL), "n"(FINFL)
            : "memory", "scc", "vcc", CLOBBER_S, CLOBBER_V);
        return;
    } else if (wave == 8) {
        const unsigned tp0 = (unsigned)rt() + T0TICKS;
        __builtin_amdgcn_s_setprio(3);
        GAS unsigned* ctr = (GAS unsigned*)(myws + WS_CTR + (size_t)b * 128);
        unsigned callv; u32x4 xv[8];
        asm volatile("global_load_dword %0, %1, off sc1" : "=v"(callv) : "v"(ctr) : "memory");
        static_for<8>([&](auto nc) __attribute__((always_inline)) {
            constexpr int n = decltype(nc)::value; constexpr StepDesc sd = step_desc(nth_step(false, n));
            ld16_async(xv[n], x_src(P.x, sd.kind, sd.k, c, lane));
        });
        const char* peer[7];
#pragma unroll
        for (int r = 0; r < 7; ++r) { int o = PEERROT ? r + (int)((unsigned)(b >> 3) % 7u) : r; o = o >= 7 ? o - 7 : o; peer[r] = P.ws[1 + o]; }
        const char* fl_lane = peer[0];
#pragma unroll
        for (int r = 1; r < 7; ++r) fl_lane = lane == r ? peer[r] : fl_lane;
        fl_lane += WS_PROG + (c >> 4) * PROW + rank * 4;
        const unsigned phase = (unsigned)((c + PROT * rank) & 255);
        asm volatile("s_waitcnt vmcnt(7)" : "+v"(callv), "+v"(xv[0]) :: "memory");
        const unsigned call = __builtin_amdgcn_readfirstlane(callv) + 1u;
        const unsigned doff = (unsigned)(WS_STAGE + (call & 1u) * PARITY_BYTES + (size_t)rank * SENDER_BYTES) + c * 1024 + lane * 16;
        const unsigned aoff = (unsigned)(WS_AGG + (c >> 4) * 128);
        const size_t sbase = WS_STAGE + (call & 1u) * PARITY_BYTES + (size_t)rank * SENDER_BYTES;
        unsigned tack = 0;
        STAMP1(1);
        lds_st(lbase + L_CALL, call); wait_lgkm0(); __builtin_amdgcn_s_barrier();
        static_for<NSTEP>([&](auto sc) __attribute__((always_inline)) {
            constexpr int s = decltype(sc)::value;
            u32x4 v;
            if constexpr (s == 1) {
                pin(xv[1]); pin(xv[2]); pin(xv[3]); pin(xv[4]); pin(xv[5]); pin(xv[6]); pin(xv[7]);
                unsigned sp = 0; while (lds_ld(lbase + L_WRDY) < 8u && ++sp < (1u << 24)) __builtin_amdgcn_s_sleep(1);
            }
            if constexpr (step_desc(s).kind == 1) { v = *(const LAS u32x4*)(lds + L_WST + step_desc(s).k * 1024 + lane * 16); pin(v); }
            else v = xv[x_index(s)];
            {
                unsigned tgo = tp0 + step_t(s) + (phase * step_len16(s) >> 12);
                if constexpr (L0T > 0 && s > 0) { const int late = (int)(tack + (step_len16(s) >> 4) - L0T - tgo); if (late > 0) tgo += (unsigned)(late * (8 - ALPHA8) >> 3); }
                int rem;
                while ((rem = (int)(tgo - (unsigned)rt())) > 0) { if (rem > 40) __builtin_amdgcn_s_sleep(8); else __builtin_amdgcn_s_sleep(1); }
            }
            STAMP0(s);
#pragma unroll
            for (int r = 0; r < 7; ++r) st16_sys(peer[(r + s) % 7], doff + (unsigned)(s * STEP_BYTES), v);
            wait_vm<0>();
            STAMP0(16 + s);
            unsigned tick;
            ticket_async(tick, myws, aoff + (unsigned)(s * 16 * 128));
            if constexpr (L0T > 0) tack = (unsigned)rt();
            if constexpr (s == 0) lds_st(lbase + L_GO, 1u);
            wait_vm<0>();
            pin(tick);
            if ((__builtin_amdgcn_readfirstlane(tick) & 15u) == 15u) {
                if (lane < 7) st4_sys_lane(fl_lane + (step_desc(s).kind == 1 ? 16 * PROW : 0), call * 32u + s + 1);
            }
            STAMP0(32 + s);
            if constexpr (TOUCH && s + 2 < NSTEP) {
                if (phase < 64u) stouch(peer[0] + sbase + (size_t)(s + 3) * STEP_BYTES - 64);
                else if (phase < 128u) stouch(myws + WS_STAGE + (call & 1u) * PARITY_BYTES + (size_t)((b >> 3) & 7) * SENDER_BYTES + (size_t)(s + 3) * STEP_BYTES - 64);
            }
            if constexpr (TOUCH && s >= 8 && s < 12) stouch((const char*)(P.out + (size_t)(ti * 256 + (s - 8) * 85) * DIM + tj * 256));
        });
        STAMP1(2);
        if (lane == 0) __hip_atomic_store(ctr, call, __ATOMIC_RELAXED, __HIP_MEMORY_SCOPE_AGENT);
        wait_vm<0>();
        STAMP1(3);
    } else if (wave == 9) {
        for (int u = lane; u < (LDS_BYTES - L_CTL) / 4; u += 64) if (u != (L_CALL - L_CTL) / 4) lds_st(lbase + L_CTL + u * 4, 0u);
        wait_lgkm0();
        __builtin_amdgcn_s_barrier();
        const unsigned call = lds_ld(lbase + L_CALL);
        __builtin_amdgcn_s_setprio(2);
        const int p = lane >> 1, sub = lane & 1;
        const char* px = myws + WS_PROG + ti * PROW;
        const char* pw = myws + WS_PROG + (16 + tj) * PROW;
        unsigned long long tab = sub ? step_tab(1) : step_tab(0);
        unsigned ent = p * 16 + sub;
        unsigned base = lane < 16 ? call * 32u : 0xfffffff0u;
        bool dead = false;
        int queued = 0;
        unsigned spins = 0, lastsum = 0xffffffffu;
        unsigned long long t0 = 0;
        while (queued < NITEM) {
            u32x8 fx, fw;
            asm volatile("s_load_dwordx8 %0, %2, 0x0 glc\n\ts_load_dwordx8 %1, %3, 0x0 glc\n\ts_waitcnt lgkmcnt(0)" : "=&s"(fx), "=&s"(fw) : "s"(px), "s"(pw) : "memory");
            unsigned sum = 0;
#pragma unroll
            for (int i = 0; i < 8; ++i) sum += fx[i] + fw[i];
            if (sum != lastsum) {
                lastsum = sum; spins = 0;
                unsigned val = 0;
                asm volatile("v_writelane_b32 %0, %1, 0\n\tv_writelane_b32 %0, %9, 1\n\tv_writelane_b32 %0, %2, 2\n\tv_writelane_b32 %0, %10, 3\n\t"
                             "v_writelane_b32 %0, %3, 4\n\tv_writelane_b32 %0, %11, 5\n\tv_writelane_b32 %0, %4, 6\n\tv_writelane_b32 %0, %12, 7\n\t"
                             "v_writelane_b32 %0, %5, 8\n\tv_writelane_b32 %0, %13, 9\n\tv_writelane_b32 %0, %6, 10\n\tv_writelane_b32 %0, %14, 11\n\t"
                             "v_writelane_b32 %0, %7, 12\n\tv_writelane_b32 %0, %15, 13\n\tv_writelane_b32 %0, %8, 14\n\tv_writelane_b32 %0, %16, 15"
                             : "+v"(val) : "s"(fx[0]), "s"(fx[1]), "s"(fx[2]), "s"(fx[3]), "s"(fx[4]), "s"(fx[5]), "s"(fx[6]), "s"(fx[7]),
                               "s"(fw[0]), "s"(fw[1]), "s"(fw[2]), "s"(fw[3]), "s"(fw[4]), "s"(fw[5]), "s"(fw[6]), "s"(fw[7]));
                if (dead) val = 0xffffffe0u;
                for (;;) {
                    unsigned bal = (unsigned)__ballot(val > base + ((unsigned)tab & 15u));
                    bal &= bal >> 1; bal &= 0x5555u;
                    if (!bal) break;
                    const bool rdy = (bal >> (p * 2)) & 1u;
                    const bool two = ((ent - sub) & 15u) >= (unsigned)NKS;
                    const bool emit = rdy && lane < 16 && (sub == 0 || two);
                    const unsigned lead = (unsigned)__ballot(emit);
                    const int pos = queued + __builtin_popcount(lead & ((1u << (lane & 31)) - 1u));
                    const int nq = queued + __builtin_popcount(lead);
                    if (emit) lds_st(lbase + L_QUEUE + pos * 4, ent | (two ? 0x400u : ((ent & 15u) >= (unsigned)NFULL ? 0x800u : 0u)));
                    if (lane == 0) lds_st(lbase + L_QCNT, (unsigned)nq);
                    const unsigned adv = (rdy && lane < 16) ? (two ? 2u : 1u) : 0u;
                    ent += adv; tab >>= 4 * adv;
                    if (((ent - sub) & 15u) >= (unsigned)IPS) base = 0xfffffff0u;
                    queued = nq;
                }
            } else {
                __builtin_amdgcn_s_sleep(PSLEEP);
                if ((++spins & 1023u) == 0u) {
                    if (spins == 1024u) t0 = rt();
                    if (__builtin_amdgcn_readfirstlane(__hip_atomic_load(tmo, __ATOMIC_RELAXED, __HIP_MEMORY_SCOPE_AGENT)) == call || rt() - t0 > DEADLINE_TICKS) {
                        if (lane == 0) __hip_atomic_store(tmo, call, __ATOMIC_RELAXED, __HIP_MEMORY_SCOPE_AGENT);
                        lds_st(lbase + L_ABORT, 1u);
                        dead = true; lastsum = ~sum;
                    }
                }
            }
        }
        wait_vm<0>();
    } else {
        const int hb = wave - 10;
        __builtin_amdgcn_s_barrier();
        const unsigned call = lds_ld(lbase + L_CALL);
        __builtin_amdgcn_s_setprio(LPRIO);
        const unsigned goff = (unsigned)(WS_STAGE + (call & 1u) * PARITY_BYTES) + lane * 16;
        char* dbg = myws;
        const unsigned paddr = lane < 16 ? lbase + L_REL + lane * 4 : lbase + L_QCNT;
        const int thr = lane < 8 ? (hb ? -0x40000000 : 1 - NSA) : (lane < 16 ? 1 - NSB : (lane == 16 ? 1 : -0x40000000));
        asm volatile(LOADER_ASM
            :: "v"(paddr), "v"(thr), "v"(goff + ti * 16384), "v"(goff + tj * 16384), "s"(myws),
               "s"(lbase + L_RA), "s"(lbase + L_RB), "s"(lbase + L_RB + NSB * 16384), "s"(step_tab(0)), "s"(step_tab(1)),
               "s"(lbase + L_CTL), "s"(256u << hb), "s"(dbg), "n"(NITEM), "n"(STAMPS_ON), "n"(LSLEEP), "n"(NFULL)
            : "memory", "scc", "vcc", CLOBBER_S, "v100", "v101", "v102", "v103", "v104", "v105", "v106", "v107", "v108", "v109", "v110");
        return;
    }
    STAMP_OUT();
}

extern "C" void kernel_launch(int rank, int nranks, ncclComm_t comm,
                              hipStream_t stream,
                              void* const* d_in, const int* in_sizes, int n_in,
                              void* d_out, int out_size,
                              const int* in_dtypes, int out_dtype,
                              void* const* d_out_all,
                              void* const* d_ws_all, size_t ws_bytes) {
    (void)comm; (void)in_dtypes; (void)out_dtype; (void)d_out_all;
    if (nranks != NR || n_in != 4 || (unsigned)rank >= (unsigned)NR || ws_bytes < WS_NEED || out_size != DIM * DIM ||
        in_sizes[0] != DIM * KSH || in_sizes[1] != KSH * DIM) {
        fprintf(stderr, "kernel_launch: unexpected problem shape\n");
        return;
    }
    Params P;
    P.x = (const signed char*)d_in[0]; P.w = (const signed char*)d_in[1];
    P.sx = (const float*)d_in[2]; P.sw = (const float*)d_in[3];
    P.out = (float*)d_out; P.rank = rank;
    for (int o = 0; o < NR; ++o) P.ws[o] = (char*)d_ws_all[(rank + o) % NR];
    hipLaunchKernelGGL(gemm_ar_fused, dim3(NWG), dim3(768), 0, stream, P);
    HIP_CHECK(hipGetLastError());
}
```

```cpp
#include <hip/hip_runtime.h>
#include <rccl/rccl.h>
#include <cstdio>
#include <cstdint>
#include <utility>

#define HIP_CHECK(expr) do { hipError_t _e = (expr); if (_e != hipSuccess) { \
    fprintf(stderr, "HIP error %s at %s:%d\n", hipGetErrorString(_e), __FILE__, __LINE__); \
    return; } } while (0)

#define LAS __attribute__((address_space(3)))
#define GAS __attribute__((address_space(1)))
typedef int i32x4 __attribute__((ext_vector_type(4)));
typedef unsigned u32x4 __attribute__((ext_vector_type(4)));
typedef unsigned u32x2 __attribute__((ext_vector_type(2)));
typedef unsigned u32x8 __attribute__((ext_vector_type(8)));
typedef float f32x4 __attribute__((ext_vector_type(4)));

#define LATE 1
#define PDEPTH_ 1
#define NFINE 2
#define TAU16 1024
#define T0TICKS 70
#define DRIP 2
#define LPRIO 2
#define CSLEEP 1
#define LSLEEP 1
#define CPRIO_HI 0
#define DGAP 0
#define PH2 112
#define TAU2 TAU16
#define PH2W0 1
#define PROT 32
#define DRIPFL 1
#define FINFL 9
#define EPW 2
#define PSLEEP 1
#define PROW 256
#define L0T 110
#define WARMSTEP 99
#define GOPH 128
#define ALPHA8 2
#define TLBWARM 1
#define PEERROT 1
#define TOUCH 0
#define ADAPT0 0
#define TICKET 0
#define PPRIO 2
#define MREP 2
#define LATEBAR 1
#define NSA 5
#define NSB 4

namespace {
constexpr int NR = 8;
constexpr int DIM = 4096;
constexpr int KSH = DIM / NR;
constexpr int NKS = KSH / 64;
constexpr int NWG = 256;
constexpr int NSTEP = 16;
constexpr int NFULL = 6;
constexpr int IPS = 12;
constexpr int NITEM = NR * IPS;
constexpr int NUP = NR * NKS;
struct StepDesc { int w, k, half; };
constexpr StepDesc step_desc(int s) {
    if (s < 2 * NFULL) return {s & 1, s >> 1, -1};
    return s == 12 ? StepDesc{0, 6, 0} : s == 13 ? StepDesc{1, 6, 0} : s == 14 ? StepDesc{1, 6, 1} : StepDesc{0, 6, 1};
}
constexpr int nth_step(int w, int n) { for (int s = 0; s < NSTEP; ++s) if (step_desc(s).w == w && n-- == 0) return s; return -1; }
constexpr int op_index(int s) { int n = 0; for (int i = 0; i < s; ++i) n += step_desc(i).w == step_desc(s).w; return n; }
constexpr unsigned nth_tab(int w) { unsigned t = 0; for (int n = 0; n < 8; ++n) t |= (unsigned)nth_step(w, n) << (4 * n); return t; }
constexpr unsigned long long step_tab(int w) {
    unsigned long long t = 0;
    for (int i = 0; i < IPS; ++i) t |= (unsigned long long)(i < NFULL ? 2 * i + w : (w ? (i < 8 ? 13 : 14) : (i < 10 ? 12 : 15))) << (4 * i);
    return t;
}

constexpr size_t WS_PROG = 0;
constexpr size_t WS_FLAGS_BYTES = 32 * MREP * PROW;
constexpr size_t WS_CTR = WS_FLAGS_BYTES;
constexpr size_t WS_TMO = WS_CTR + (size_t)NWG * 128;
constexpr size_t WS_AGG = WS_TMO + 128;
constexpr size_t WS_STAGE = 4u << 20;
constexpr size_t STEP_BYTES = (size_t)NWG * 1024;
constexpr size_t SENDER_BYTES = NSTEP * STEP_BYTES;
constexpr size_t PARITY_BYTES = NR * SENDER_BYTES;
constexpr size_t WS_NEED = WS_STAGE + 2 * PARITY_BYTES;
static_assert(WS_AGG + (NSTEP + 1) * 16 * 128 <= WS_STAGE, "control region");
constexpr int L_RA = 0;
constexpr int L_RB = NSA * 16384;
constexpr int L_WST = L_RB + NSB * 16384;
constexpr int L_CTL = L_WST + NKS * 1024;
constexpr int L_PUB = L_CTL;
constexpr int L_REL = L_CTL + 64;
constexpr int L_QCNT = L_CTL + 128;
constexpr int L_ABORT = L_CTL + 132;
constexpr int L_CALL = L_CTL + 136;
constexpr int L_WRDY = L_CTL + 140;
constexpr int L_LOC = L_CTL + 144;
constexpr int L_GO = L_CTL + 148;
constexpr int L_QUEUE = L_CTL + 256;
constexpr int L_LDQ = L_QUEUE + 128 * 4;
constexpr int L_DUMMY = L_LDQ + 128 * 4;
constexpr int LDS_BYTES = L_DUMMY + 320;
static_assert(LDS_BYTES <= 163840 && NITEM < 128, "LDS");
constexpr unsigned long long DEADLINE_TICKS = 50000000ull;
constexpr int PDEPTH = PDEPTH_;
constexpr int NST = NSTEP * 7;

struct Params {
    const signed char* x; const signed char* w; const float* sx; const float* sw;
    float* out; char* ws[NR]; int rank;
};

#define STAMP0(idx) do { } while (0)
#define STAMP1(idx) do { } while (0)
#define STAMP_OUT() do { } while (0)

constexpr unsigned step_len16(int s) { return 7u * (s < PH2 ? TAU16 : TAU2); }
constexpr unsigned step_t(int s) { unsigned t = 0; for (int i = 0; i < s; ++i) t += step_len16(i); return t >> 4; }
constexpr int sched_t(int n) { return (n <= PH2 ? n * TAU16 : PH2 * TAU16 + (n - PH2) * TAU2) >> 4; }
constexpr bool agg_step(int s) { return s < NSTEP - NFINE; }
constexpr int npost(int n) {
    const int o = n - PDEPTH;
    if (o < 0 || o >= NST) return 0;
    return agg_step(o / 7) ? (o % 7 == 6 ? 1 : 0) : 1;
}
constexpr int nyoung(int n) {
    int c = 0;
    for (int i = n - PDEPTH + 1; i <= n; ++i) c += i < NST;
    for (int i = n - PDEPTH; i < n; ++i) c += npost(i);
    return c;
}
template <class F, int... I> __device__ __forceinline__ void static_for_impl(F&& f, std::integer_sequence<int, I...>) { (f(std::integral_constant<int, I>{}), ...); }
template <int N, class F> __device__ __forceinline__ void static_for(F&& f) { static_for_impl(f, std::make_integer_sequence<int, N>{}); }
template <int N> __device__ __forceinline__ void wait_vm() { asm volatile("s_waitcnt vmcnt(%0)" :: "n"(N) : "memory"); }
__device__ __forceinline__ void wait_vm_x4(int n4) {
    switch (n4) { case 0: wait_vm<0>(); break; case 1: wait_vm<4>(); break; case 2: wait_vm<8>(); break; case 3: wait_vm<12>(); break;
                  case 4: wait_vm<16>(); break; case 5: wait_vm<20>(); break; case 6: wait_vm<24>(); break; case 7: wait_vm<28>(); break;
                  default: wait_vm<32>(); break; }
}
__device__ __forceinline__ unsigned vm_outstanding() {
    const unsigned r = __builtin_amdgcn_s_getreg(7 | (31 << 11));
    return (r & 15u) | ((r >> 18) & 48u);
}
__device__ __forceinline__ void wait_lgkm0() { asm volatile("s_waitcnt lgkmcnt(0)" ::: "memory"); }
__device__ __forceinline__ unsigned long long rt() { return __builtin_amdgcn_s_memrealtime(); }
__device__ __forceinline__ void st16_sys(const char* base, unsigned off, u32x4 v) {
    asm volatile("global_store_dwordx4 %0, %1, %2 sc0 sc1\n\ts_nop 1" :: "v"(off), "v"(v), "s"(base) : "memory");
}
__device__ __forceinline__ void st4_sys(const char* base, unsigned off, unsigned v) {
    asm volatile("global_store_dword %0, %1, %2 sc0 sc1\n\ts_nop 1" :: "v"(off), "v"(v), "s"(base) : "memory");
}
__device__ __forceinline__ void ld16_async(u32x4& v, const GAS u32x4* p) { asm volatile("global_load_dwordx4 %0, %1, off" : "=v"(v) : "v"(p) : "memory"); }
__device__ __forceinline__ void ld4_wait(unsigned& v, const char* p) { asm volatile("global_load_dword %0, %1, off\n\ts_waitcnt vmcnt(0)" : "=v"(v) : "v"(p) : "memory"); }
__device__ __forceinline__ void stouch(const char* p) { unsigned d; asm volatile("s_load_dword %0, %1, 0x0 glc\n\ts_waitcnt lgkmcnt(0)" : "=s"(d) : "s"(p) : "memory"); }
typedef unsigned u32x16 __attribute__((ext_vector_type(16)));
constexpr unsigned FWD_SPINS = 1u << 21;
__device__ __forceinline__ unsigned piece_min(const char* p) {
    u32x16 f;
    asm volatile("s_load_dwordx16 %0, %1, 0x0 glc\n\ts_waitcnt lgkmcnt(0)" : "=s"(f) : "s"(p) : "memory");
    unsigned m = f[0];
#pragma unroll
    for (int i = 1; i < 15; ++i) m = f[i] < m ? f[i] : m;
    return m;
}
template <class T> __device__ __forceinline__ void pin(T& v) { asm volatile("" : "+v"(v)); }
__device__ __forceinline__ void st4_sys_lane(const char* p, unsigned v) { asm volatile("global_store_dword %0, %1, off sc0 sc1\n\ts_nop 1" :: "v"(p), "v"(v) : "memory"); }
__device__ __forceinline__ void st16_sys_lane(const char* p, u32x4 v) { asm volatile("global_store_dwordx4 %0, %1, off sc0 sc1\n\ts_nop 1" :: "v"(p), "v"(v) : "memory"); }
__device__ __forceinline__ void ticket_async(unsigned& tick, const char* base, unsigned off) {
    unsigned long long keep;
    asm volatile("s_mov_b64 %1, exec\n\ts_mov_b64 exec, 1\n\tglobal_atomic_add %0, %2, %3, %4 sc0 sc1\n\ts_mov_b64 exec, %1"
                 : "=&v"(tick), "=&s"(keep) : "v"(off), "v"(1u), "s"(base) : "memory");
}
__device__ __forceinline__ void lds_st(unsigned addr, unsigned v) { asm volatile("ds_write_b32 %0, %1" :: "v"(addr), "v"(v) : "memory"); }
__device__ __forceinline__ unsigned lds_ldv(unsigned addr) {
    unsigned v; asm volatile("ds_read_b32 %0, %1\n\ts_waitcnt lgkmcnt(0)" : "=v"(v) : "v"(addr) : "memory");
    return v;
}
__device__ __forceinline__ unsigned lds_ld(unsigned addr) { return __builtin_amdgcn_readfirstlane(lds_ldv(addr)); }
__device__ __forceinline__ void dma4k(const char* base, unsigned off, unsigned lds_dst) {
    unsigned keep;
    asm volatile("s_mov_b32 %0, m0\n\ts_mov_b32 m0, %3\n\ts_nop 0\n\t"
                 "global_load_lds_dwordx4 %1, %2 sc0 sc1\n\t"
                 "global_load_lds_dwordx4 %1, %2 offset:1024 sc0 sc1\n\t"
                 "global_load_lds_dwordx4 %1, %2 offset:2048 sc0 sc1\n\t"
                 "global_load_lds_dwordx4 %1, %2 offset:3072 sc0 sc1\n\t"
                 "s_mov_b32 m0, %0"
                 : "=&s"(keep) : "v"(off), "s"(base), "s"(lds_dst) : "memory");
}
__device__ __forceinline__ int frag_blk(int n, int c) { return n < NFULL ? c : (c & ~15) + (n - NFULL) * 8 + (c & 7); }
__device__ __forceinline__ int frag_ks(int n, int c) { return n < NFULL ? n : NFULL + ((c >> 3) & 1); }
__device__ __forceinline__ const GAS u32x4* x_src(const signed char* x, int n, int c, int lane) {
    return (const GAS u32x4*)(x + (size_t)(frag_blk(n, c) * 16 + (lane & 15)) * KSH + frag_ks(n, c) * 64 + (lane >> 4) * 16);
}
}

#define STAMPS_ON 0
#define CONSUMER_ASM R"ASM(
.macro MM m, n, a
  v_mfma_i32_16x16x64_i8 v[16*(\m)+4*(\n):16*(\m)+4*(\n)+3], v[144+4*(\a):147+4*(\a)], v[128+4*(\n):131+4*(\n)], v[16*(\m)+4*(\n):16*(\m)+4*(\n)+3]
.endm
.macro MM4 m, a
  MM \m, 0, \a
  MM \m, 1, \a
  MM \m, 2, \a
  MM \m, 3, \a
.endm
.macro MC m, n, a
  MM \m, \n, \a
  MM \m+1, \n, \a+1
.endm
.macro LDA a, off
  ds_read_b128 v[144+4*(\a):147+4*(\a)], v161 offset:\off
.endm
.macro LDB n
  ds_read_b128 v[128+4*(\n):131+4*(\n)], v162 offset:1024*(\n)
.endm
.macro STAMPI
.if %12 == 1
  s_sub_u32 s48, s53, %7-64
  s_and_b32 s48, s48, 63
  s_memrealtime s[56:57]
  s_waitcnt lgkmcnt(0)
  s_mov_b32 m0, s48
  s_nop 0
  v_writelane_b32 v160, s56, m0
.endif
.endm
.macro STAMPK k, ins=s_memrealtime
.if %12
  \ins s[56:57]
  s_waitcnt lgkmcnt(0)
  v_mov_b32 v144, s56
  v_mov_b32 v163, 256+4*(\k)
  global_store_dword v163, v144, %6
  s_nop 1
.endif
.endm
.macro ADV_A
  s_add_u32 s41, s41, 1
  s_cmp_eq_u32 s41, %9
  s_cselect_b32 s44, s50, s52
  s_cselect_b32 s41, 0, s41
.endm
.macro ADV_B
  s_add_u32 s42, s42, 1
  s_cmp_eq_u32 s42, %10
  s_cselect_b32 s40, s51, s52
  s_cselect_b32 s42, 0, s42
.endm
// waits for the next item of this wave's half, passing over (and releasing) the items of the other half
.macro SPIN back
  s_mov_b32 s47, 0
1:
  s_cmp_ge_u32 s43, 0x400
  s_cbranch_scc0 2f
  ds_write2_b32 v165, v166, v166 offset0:0 offset1:8
  ADV_A
  ADV_B
  v_add_u32 v164, 4, v164
  v_add_u32 v161, s44, v161
  v_add_u32 v162, s40, v162
  v_add_u32 v166, 1, v166
  s_add_u32 s53, s53, 1
  s_mov_b32 s47, 0
  s_branch 3f
2:
  s_sleep %13
3:
  ds_read_b32 v163, v164 offset:256
  s_add_u32 s47, s47, 1
  s_waitcnt lgkmcnt(0)
  v_readfirstlane_b32 s43, v163
  s_and_b32 s43, s43, %11
  s_cmp_eq_u32 s43, 0x300
  s_cbranch_scc1 \back
  s_cmp_lt_u32 s47, 0x1000000
  s_cbranch_scc1 1b
  v_mov_b32 v163, %3
  v_mov_b32 v144, 1
  ds_write_b32 v163, v144 offset:132
  s_branch FIN_%=
.endm
.macro OUT_ST u, t, fl
  v_cvt_f32_i32 v[\t], v[16*((\u)/4)+((\u)&3)]
  v_cvt_f32_i32 v[\t+1], v[16*((\u)/4)+((\u)&3)+4]
  v_cvt_f32_i32 v[\t+2], v[16*((\u)/4)+((\u)&3)+8]
  v_cvt_f32_i32 v[\t+3], v[16*((\u)/4)+((\u)&3)+12]
  s_add_u32 s54, s58, (((\u)/4)*16+((\u)&3))*16384
  s_addc_u32 s55, s59, 0
  v_mul_f32 v[\t], s49, v[\t]
  v_mul_f32 v[\t+1], s49, v[\t+1]
  v_mul_f32 v[\t+2], s49, v[\t+2]
  v_mul_f32 v[\t+3], s49, v[\t+3]
.if \fl == 0
  global_store_dwordx4 v167, v[\t:\t+3], s[54:55]
.elseif \fl == 1
  global_store_dwordx4 v167, v[\t:\t+3], s[54:55] sc0 sc1
.else
  global_store_dwordx4 v167, v[\t:\t+3], s[54:55] nt
.endif
.endm

  v_mbcnt_lo_u32_b32 v160, -1, 0
  v_mbcnt_hi_u32_b32 v160, -1, v160
  v_lshrrev_b32 v167, 4, v160
  v_lshlrev_b32 v167, 16, v167
  v_and_b32 v166, 15, v160
  v_lshl_add_u32 v167, v166, 4, v167
  v_lshlrev_b32 v160, 4, v160
  v_add_u32 v161, %0, v160
  v_add_u32 v162, %1, v160
  v_mov_b32 v164, %3
  v_lshrrev_b32 v165, 2, v160
  v_add_u32 v165, %17, v165
  v_writelane_b32 v165, %2, 0
  v_mov_b32 v166, 1
  v_mov_b32 v160, 0
  s_mov_b64 s[58:59], %4
  s_mov_b32 s49, %5
  s_mov_b32 s41, 0
  s_mov_b32 s42, 0
  s_mov_b32 s43, 0
  s_mov_b32 s45, %8
  s_mov_b32 s50, -(%9-1)*16384
  s_mov_b32 s51, -(%10-1)*16384
  s_mov_b32 s52, 16384
  s_mov_b32 s53, 0
.set zi, 0
.rept 128
  v_mov_b32 v[zi], 0
.set zi, zi+1
.endr
  SPIN START_%=
SPINF_%=:
  s_cmp_eq_u32 s45, 0
  s_cbranch_scc1 TAIL_%=
  SPIN BACK_%=
START_%=:
  STAMPK 6, s_memtime
  STAMPK 8
  LDA 0, 0
  LDA 1, 1024
  LDB 0
  LDB 1
  LDB 2
  LDB 3
  ds_write_b32 v165, v166 offset:32
LOOP_%=:
  s_waitcnt lgkmcnt(4)
  MM 0, 0, 0
  LDA 2, 2048
  MM 1, 0, 1
  LDA 3, 3072
  s_waitcnt lgkmcnt(5)
  MM 0, 1, 0
  s_add_u32 s53, s53, 1
  MM 1, 1, 1
  s_waitcnt lgkmcnt(4)
  MM 0, 2, 0
  s_sub_u32 s45, s45, 1
  MM 1, 2, 1
  s_waitcnt lgkmcnt(3)
  MM 0, 3, 0
  v_add_u32 v164, 4, v164
  MM 1, 3, 1
  s_waitcnt lgkmcnt(1)
  MM 2, 0, 2
  LDA 0, 4096
  MM 2, 1, 2
  LDA 1, 5120
  MM 2, 2, 2
  s_add_u32 s41, s41, 1
  s_cmp_eq_u32 s41, %9
  MM 2, 3, 2
  s_waitcnt lgkmcnt(2)
  MM 3, 0, 3
  LDA 2, 6144
  MM 3, 1, 3
  s_cselect_b32 s44, s50, s52
  s_cselect_b32 s41, 0, s41
  MM 3, 2, 3
  s_add_u32 s42, s42, 1
  s_cmp_eq_u32 s42, %10
  MM 3, 3, 3
  s_waitcnt lgkmcnt(2)
  MM 4, 0, 0
  LDA 3, 7168
  MM 4, 1, 0
  ds_read_b32 v163, v164 offset:256
  MM 4, 2, 0
  ds_write_b32 v165, v166
  MM 4, 3, 0
  s_cselect_b32 s40, s51, s52
  s_waitcnt lgkmcnt(4)
  MM 5, 0, 1
  s_cselect_b32 s42, 0, s42
  v_add_u32 v161, s44, v161
  MM 5, 1, 1
  v_add_u32 v162, s40, v162
  v_add_u32 v166, 1, v166
  MM 5, 2, 1
  s_waitcnt lgkmcnt(1)
  v_readfirstlane_b32 s43, v163
  MM 5, 3, 1
  s_and_b32 s43, s43, %11
  s_cmp_eq_u32 s43, 0x300
  s_cbranch_scc0 SPINF_%=
BACK_%=:
  STAMPI
.if %12 == 2
  s_cmp_eq_u32 s45, 12
  s_cbranch_scc0 5f
  STAMPK 9
  STAMPK 10, s_memtime
5:
.endif
  MM 6, 0, 2
  LDA 0, 0
  MM 7, 0, 3
  LDA 1, 1024
  MM 6, 1, 2
  LDB 0
  MM 7, 1, 3
  LDB 1
  MM 6, 2, 2
  MM 7, 2, 3
  LDB 2
  MM 6, 3, 2
  MM 7, 3, 3
  LDB 3
  ds_write_b32 v165, v166 offset:32
  s_branch LOOP_%=
TAIL_%=:
  s_waitcnt lgkmcnt(0)
  MM4 6, 2
  MM4 7, 3
FIN_%=:
  v_mov_b32 v166, 0x7fffffff
  v_mov_b32 v163, %3
  ds_write2_b32 v165, v166, v166 offset0:0 offset1:8
  ds_read_b32 v166, v163 offset:132
  STAMPK 2
  STAMPK 7, s_memtime
  s_waitcnt lgkmcnt(0)
  v_readfirstlane_b32 s48, v166
  s_cmp_eq_u32 s48, 0
  s_cselect_b32 s49, s49, 0x7fc00000
.if %16 == 9
  s_cmp_lt_u32 %11, 0x1000
  s_cselect_b64 vcc, -1, 0
.else
  s_cmp_lt_u32 %11, 0x1000
  s_cbranch_scc1 EARLY_%=
.set du, 0
.rept 32
  OUT_ST du, 128+4*(du&7), %16
.set du, du+1
.endr
  s_branch END_%=
EARLY_%=:
  s_mov_b64 vcc, -1
.endif
  s_setprio 0
  v_mov_b32 v163, %3
.if %14 == 62
6:
  s_sleep 4
  ds_read_b32 v166, v163 offset:128
  s_waitcnt lgkmcnt(0)
  v_cmp_eq_u32 vcc, 0, v166
  s_cbranch_vccnz 6b
.endif
.set du, 0
.rept 32
  OUT_ST du, 128+4*(du&7), %15
  s_cbranch_vccz 4f
  ds_read_b32 v166, v163 offset:128
  s_waitcnt vmcnt(%14) lgkmcnt(0)
  v_cmp_eq_u32 vcc, 0, v166
4:
.set du, du+1
.endr
END_%=:
.if %12
  STAMPK 3
  s_waitcnt vmcnt(0)
  STAMPK 4
  v_mbcnt_lo_u32_b32 v166, -1, 0
  v_mbcnt_hi_u32_b32 v166, -1, v166
  v_lshlrev_b32 v166, 2, v166
  global_store_dword v166, v160, %6
.endif
.purgem MM
.purgem MM4
.purgem MC
.purgem LDA
.purgem LDB
.purgem STAMPI
.purgem STAMPK
.purgem SPIN
.purgem ADV_A
.purgem ADV_B
.purgem OUT_ST
)ASM"

#define LOADER_ASM R"ASM(
.macro LSTAMP vr, idx
.if %12 == 1
  s_memrealtime s[56:57]
  s_sub_u32 s58, \idx, %11-64
  s_and_b32 s58, s58, 63
  s_mov_b32 s59, m0
  s_mov_b32 m0, s58
  s_waitcnt lgkmcnt(0)
  v_writelane_b32 \vr, s56, m0
  s_mov_b32 m0, s59
.endif
.endm
.macro PUBLISH
  LSTAMP v108, s44
  ds_add_u32 v105, v109 offset:256
  s_add_u32 s44, s44, 1
  s_mov_b32 s41, s42
  s_mov_b32 s42, s43
  s_mov_b32 s43, s49
  v_add_u32 v105, v112, v105
.endm
.macro LANDED
  s_getreg_b32 s47, hwreg(HW_REG_IB_STS)
  s_and_b32 s48, s47, 15
  s_bfe_u32 s47, s47, 0x20016
  s_lshl4_add_u32 s47, s47, s48
  s_sub_u32 s47, s40, s47
  s_cmp_ge_u32 s47, s41
.endm
.macro PUBCHK
  LANDED
  s_cbranch_scc0 7f
  PUBLISH
7:
.endm
.macro DMA4 va
  global_load_lds_dwordx4 \va, %4 sc0 sc1
  global_load_lds_dwordx4 \va, %4 offset:1024 sc0 sc1
  global_load_lds_dwordx4 \va, %4 offset:2048 sc0 sc1
  global_load_lds_dwordx4 \va, %4 offset:3072 sc0 sc1
  s_add_u32 s40, s40, 4
  s_add_u32 m0, m0, 4096
  v_add_u32 \va, 4096, \va
.endm
.macro BODY hb
  ds_read_b32 v102, v101
LLOOP\hb\()_%=:
  PUBCHK
  s_waitcnt lgkmcnt(0)
  v_cmp_ge_i32 vcc, v102, v100
  v_readlane_b32 s51, v102, 16
  s_cmp_eq_u64 vcc, -1
  s_cselect_b32 s47, s43, 0
  s_cmp_eq_u32 s47, s49
  s_cbranch_scc0 LIDLE\hb\()_%=
LGO\hb\()_%=:
  s_bfe_u32 s52, s51, 0xc000c
  s_lshl_b32 s52, s52, 13
  v_add_u32 v110, s52, %3
  s_bfe_u32 s53, s51, 0x20018
.if \hb == 0
  s_and_b32 s52, s51, 0xfff
  s_lshl_b32 s52, s52, 13
  v_add_u32 v104, s52, %2
  s_cmp_eq_u32 s53, 3
  s_cselect_b32 s52, 8192, 0
  s_add_u32 m0, s46, s52
  s_add_u32 s46, s46, 16384
  DMA4 v104
  v_add_u32 v100, v103, v100
  v_add_u32 v101, v111, v101
  s_cmp_eq_u32 s46, %6
  s_cselect_b32 s46, %5, s46
  PUBCHK
  s_cmp_eq_u32 s53, 0
  s_cbranch_scc0 8f
  DMA4 v104
  PUBCHK
  DMA4 v104
  PUBCHK
8:
  ds_read_b32 v102, v101
  DMA4 v104
  s_cmp_eq_u32 s53, 3
  s_cbranch_scc0 9f
  s_mov_b32 m0, s60
  v_subrev_u32 v110, 0x40000, v110
  PUBCHK
  DMA4 v110
9:
.else
  s_cmp_ge_u32 s53, 2
  s_cselect_b32 s52, 8192, 0
  s_add_u32 m0, s60, s52
  v_add_u32 v100, v103, v100
  DMA4 v110
  v_add_u32 v101, v111, v101
  PUBCHK
  s_cmp_eq_u32 s53, 0
  s_cbranch_scc0 8f
  DMA4 v110
  PUBCHK
  DMA4 v110
  PUBCHK
8:
  ds_read_b32 v102, v101
  DMA4 v110
  s_cmp_eq_u32 s53, 3
  s_cbranch_scc0 9f
  s_add_u32 m0, s60, 4096
  v_subrev_u32 v110, 0x41000, v110
  PUBCHK
  DMA4 v110
9:
.endif
  LSTAMP v107, s45
  s_cmp_eq_u32 s42, s49
  s_cselect_b32 s43, s49, s40
  s_cselect_b32 s47, s40, s42
  s_cmp_eq_u32 s41, s49
  s_cselect_b32 s42, s49, s47
  s_cselect_b32 s41, s40, s41
  s_add_u32 s60, s60, 16384
  s_cmp_eq_u32 s60, %7
  s_cselect_b32 s60, %6, s60
  s_add_u32 s45, s45, 1
  s_mov_b32 s50, 0
  s_cmp_lt_u32 s45, %11
  s_cbranch_scc1 LLOOP\hb\()_%=
  s_branch LDRAIN_%=
LIDLE\hb\()_%=:
  ds_read_b32 v102, v101
  s_cmp_eq_u32 s41, s49
  s_cbranch_scc0 LLOOP\hb\()_%=
LWAIT\hb\()_%=:
  s_add_u32 s50, s50, 1
  s_waitcnt lgkmcnt(0)
  v_cmp_ge_i32 vcc, v102, v100
  v_readlane_b32 s51, v102, 16
  s_cmp_eq_u64 vcc, -1
  s_cbranch_scc1 LGO\hb\()_%=
  s_sleep %13
  ds_read_b32 v102, v101
  s_cmp_lt_u32 s50, 0x1000000
  s_cbranch_scc1 LWAIT\hb\()_%=
  v_mov_b32 v106, 1
  v_mov_b32 v101, %8
  ds_write_b32 v101, v106 offset:132
  s_branch LEND_%=
.endm
  s_mov_b32 s40, 0
  s_mov_b32 s49, 0x7fffffff
  s_mov_b32 s41, s49
  s_mov_b32 s42, s49
  s_mov_b32 s43, s49
  s_mov_b32 s44, 0
  s_mov_b32 s45, 0
  s_mov_b32 s46, %5
  s_mov_b32 s60, %6
  s_mov_b32 s50, 0
  v_mov_b32 v100, %1
  v_mov_b32 v101, %0
  v_mov_b32 v103, 1
  v_writelane_b32 v103, 0, 16
  v_mov_b32 v111, 0
  v_writelane_b32 v111, 4, 16
  v_mov_b32 v105, %14
  v_writelane_b32 v105, %8, 0
  v_mov_b32 v112, 0
  v_writelane_b32 v112, 4, 0
  v_mov_b32 v109, %9
  v_mov_b32 v107, 0
  v_mov_b32 v108, 0
  s_waitcnt vmcnt(0)
  s_cmp_eq_u32 %9, 0x200
  s_cbranch_scc1 LONE_%=
  BODY 0
LONE_%=:
  BODY 1
LDRAIN_%=:
  PUBCHK
  s_cmp_lt_u32 s44, %11
  s_cbranch_scc1 LDRAIN_%=
LEND_%=:
  v_mov_b32 v106, 1
  v_mov_b32 v101, %8
  ds_write_b32 v101, v106 offset:128
  s_waitcnt vmcnt(0) lgkmcnt(0)
.if %12
  v_mbcnt_lo_u32_b32 v106, -1, 0
  v_mbcnt_hi_u32_b32 v106, -1, v106
  v_lshlrev_b32 v106, 2, v106
  global_store_dword v106, v107, %10
  global_store_dword v106, v108, %10 offset:256
.endif
.purgem LSTAMP
.purgem PUBLISH
.purgem LANDED
.purgem PUBCHK
.purgem DMA4
.purgem BODY
)ASM"
#define CLOBBER_S "s40", "s41", "s42", "s43", "s44", "s45", "s46", "s47", "s48", "s49", "s50", "s51", "s52", "s53", "s54", "s55", "s56", "s57", "s58", "s59", "s60", "s61", "s62", "s63"
#define V10(t) "v" #t "0", "v" #t "1", "v" #t "2", "v" #t "3", "v" #t "4", "v" #t "5", "v" #t "6", "v" #t "7", "v" #t "8", "v" #t "9"
#define CLOBBER_V V10(), V10(1), V10(2), V10(3), V10(4), V10(5), V10(6), V10(7), V10(8), V10(9), V10(10), V10(11), V10(12), V10(13), V10(14), V10(15), \
    "v160", "v161", "v162", "v163", "v164", "v165", "v166", "v167"

__global__ void __launch_bounds__(768, 3) gemm_ar_fused(Params P) {
    __shared__ __attribute__((aligned(16))) unsigned char lds_raw[LDS_BYTES];
    LAS unsigned char* lds = (LAS unsigned char*)lds_raw;
    const unsigned lbase = (unsigned)(size_t)lds;
    const int tid = threadIdx.x, lane = tid & 63;
    const int wave = __builtin_amdgcn_readfirstlane(tid >> 6);
    const int b = blockIdx.x;
    const int rank = P.rank;
    const int c = ((b >> 5) << 5) | ((b & 7) << 2) | ((b >> 3) & 3);
    char* myws = P.ws[0];
    const int ti = ((b & 7) >> 1) * 4 + (b >> 6), tj = (b & 1) * 8 + ((b >> 3) & 7);
    if (wave < 9) STAMP1(0);
    GAS unsigned* tmo = (GAS unsigned*)(myws + WS_TMO);

    if (wave < 8) {
        unsigned warm = 0;
        if (TLBWARM && wave == 0) {
            const unsigned ph = (unsigned)((c + PROT * rank) & 255), kk = ((ph >> 5) - 4u) * 4u + (ph & 3u);
            if (ph >= 128u && kk < 14u) {
                const char* pw = P.ws[1 + kk % 7u] + WS_STAGE + (kk / 7u) * PARITY_BYTES + (size_t)rank * SENDER_BYTES;
                asm volatile("s_load_dword %0, %1, 0x0 glc" : "=s"(warm) : "s"(pw) : "memory");
            }
        }
        __builtin_amdgcn_s_barrier();
        const unsigned call = lds_ld(lbase + L_CALL);
        asm volatile("" :: "s"(warm));
        if (((c + PROT * rank) & 255) < GOPH) { unsigned sp = 0; while (lds_ld(lbase + L_GO) == 0u && ++sp < (1u << 22)) __builtin_amdgcn_s_sleep(2); }
        {
            const int ks = wave;
            const int cc = lane & 15, g = lane >> 4;
            const GAS unsigned* wp = (const GAS unsigned*)(P.w + (size_t)(frag_ks(ks, c) * 64 + g * 16) * DIM + (frag_blk(ks, c) >> 2) * 64 + cc * 4);
            unsigned d[16];
#pragma unroll
            for (int t = 0; t < 16; ++t) d[t] = wp[(size_t)t * (DIM / 4)];
            const int sx = (nth_tab(0) >> (4 * wave)) & 15, sw = (nth_tab(1) >> (4 * wave)) & 15;
            const u32x4 xv = *x_src(P.x, ks, c, lane);
            const unsigned q = c & 3;
            const unsigned sel_lo = 0x0c0c0000u | ((4u + q) << 8) | q;
            const unsigned sel_hi = 0x00000c0cu | ((4u + q) << 24) | (q << 16);
            u32x4 wv;
#pragma unroll
            for (int e = 0; e < 4; ++e)
                wv[e] = __builtin_amdgcn_perm(d[4 * e + 1], d[4 * e], sel_lo) | __builtin_amdgcn_perm(d[4 * e + 3], d[4 * e + 2], sel_hi);
            *(LAS u32x4*)(lds + L_WST + ks * 1024 + lane * 16) = wv;
            asm volatile("" ::: "memory");
            if (lane == 0) __hip_atomic_fetch_add((LAS unsigned*)(lds + L_WRDY), 1u, __ATOMIC_RELAXED, __HIP_MEMORY_SCOPE_WORKGROUP);
            const unsigned soff = (unsigned)(WS_STAGE + (call & 1u) * PARITY_BYTES + (size_t)rank * SENDER_BYTES) + c * 1024 + lane * 16;
            st16_sys(myws, soff + sx * (unsigned)STEP_BYTES, xv);
            st16_sys(myws, soff + sw * (unsigned)STEP_BYTES, wv);
            STAMP1(1);
            wait_vm<0>();
            if (lane == 0) {
                if (__hip_atomic_fetch_add((LAS unsigned*)(lds + L_LOC), 1u, __ATOMIC_RELAXED, __HIP_MEMORY_SCOPE_WORKGROUP) == 7u) {
                    GAS unsigned* tk = (GAS unsigned*)(myws + WS_AGG + (NSTEP * 16 + (c >> 4)) * 128);
                    if ((__hip_atomic_fetch_add(tk, 1u, __ATOMIC_RELAXED, __HIP_MEMORY_SCOPE_AGENT) & 15u) == 15u)
                        for (int k = 0; k < 2 * MREP; ++k)
                            __hip_atomic_store((GAS unsigned*)(myws + WS_PROG + (k * 16 + (c >> 4)) * PROW + rank * 4), call * 32u + NSTEP, __ATOMIC_RELAXED, __HIP_MEMORY_SCOPE_SYSTEM);
                }
            }
        }
        const int wr = wave >> 2, wc = wave & 3;
        const unsigned scale = __builtin_amdgcn_readfirstlane(__float_as_uint(P.sx[0] * P.sw[0]));
        const char* obase = (const char*)(P.out + (size_t)(ti * 256 + wr * 128) * DIM + tj * 256 + wc * 64);
        char* dbg = myws;
        if (CPRIO_HI && wave >= 4) __builtin_amdgcn_s_setprio(CPRIO_HI);
        asm volatile(CONSUMER_ASM
            :: "s"(lbase + L_RA + wr * 8192), "s"(lbase + L_RB + wc * 4096), "s"(lbase + L_REL + wave * 4), "s"(lbase + L_CTL),
               "s"(obase), "s"(scale), "s"(dbg),
               "n"(NITEM), "n"(NUP), "n"(NSA), "n"(NSB), "s"(0x300u | (0x400u << (wr ? 2 : wc >> 1))), "n"(STAMPS_ON), "n"(CSLEEP), "n"(EPW), "n"(DRIPFL), "n"(FINFL), "s"(lbase + L_DUMMY)
            : "memory", "scc", "vcc", CLOBBER_S, CLOBBER_V);
        return;
    } else if (wave == 8) {
        const unsigned tp0 = (unsigned)rt() + T0TICKS;
        __builtin_amdgcn_s_setprio(3);
        GAS unsigned* ctr = (GAS unsigned*)(myws + WS_CTR + (size_t)b * 128);
        unsigned callv; u32x4 xv[8];
        asm volatile("global_load_dword %0, %1, off sc1" : "=v"(callv) : "v"(ctr) : "memory");
        static_for<8>([&](auto nc) __attribute__((always_inline)) {
            constexpr int n = decltype(nc)::value;
            ld16_async(xv[n], x_src(P.x, n, c, lane));
        });
        const char* peer[7];
#pragma unroll
        for (int r = 0; r < 7; ++r) { int o = PEERROT ? r + (int)((unsigned)(b >> 3) % 7u) : r; o = o >= 7 ? o - 7 : o; peer[r] = P.ws[1 + o]; }
        const char* fl_lane = peer[0];
#pragma unroll
        for (int r = 1; r < 7; ++r) fl_lane = (lane & 7) == r ? peer[r] : fl_lane;
        fl_lane += WS_PROG + ((c >> 4) + (lane >> 3) * 32) * PROW + rank * 4;
        const unsigned phase = (unsigned)((c + PROT * rank) & 255);
        asm volatile("s_waitcnt vmcnt(7)" : "+v"(callv), "+v"(xv[0]) :: "memory");
        const unsigned call = __builtin_amdgcn_readfirstlane(callv) + 1u;
        const unsigned doff = (unsigned)(WS_STAGE + (call & 1u) * PARITY_BYTES + (size_t)rank * SENDER_BYTES) + c * 1024 + lane * 16;
        const unsigned aoff = (unsigned)(WS_AGG + (c >> 4) * 128);
        const size_t sbase = WS_STAGE + (call & 1u) * PARITY_BYTES + (size_t)rank * SENDER_BYTES;
        unsigned tack = 0, fwd_dead = 0;
        const int late0 = (int)((unsigned)rt() - tp0);
        STAMP1(1);
        lds_st(lbase + L_CALL, call); wait_lgkm0();
        const bool latebar = LATEBAR && phase < GOPH;
        if (!latebar) __builtin_amdgcn_s_barrier();
        static_for<NSTEP>([&](auto sc) __attribute__((always_inline)) {
            constexpr int s = decltype(sc)::value;
            u32x4 v;
            if constexpr (s == 1) {
                pin(xv[1]); pin(xv[2]); pin(xv[3]); pin(xv[4]); pin(xv[5]); pin(xv[6]); pin(xv[7]);
                unsigned sp = 0; while (lds_ld(lbase + L_WRDY) < 8u && ++sp < (1u << 24)) __builtin_amdgcn_s_sleep(1);
            }
            if constexpr (step_desc(s).w) { v = *(const LAS u32x4*)(lds + L_WST + op_index(s) * 1024 + lane * 16); pin(v); }
            else v = xv[op_index(s)];
            {
                unsigned tgo = tp0 + step_t(s) + (phase * step_len16(s) >> 12);
                if constexpr (ADAPT0 && s == 0) { if (late0 > 0) tgo += (unsigned)late0; }
                if constexpr (L0T > 0 && s > 0) { const int late = (int)(tack + (step_len16(s) >> 4) - L0T - tgo); if (late > 0) tgo += (unsigned)(late * (8 - ALPHA8) >> 3); }
                int rem;
                while ((rem = (int)(tgo - (unsigned)rt())) > 0) { if (rem > 40) __builtin_amdgcn_s_sleep(8); else __builtin_amdgcn_s_sleep(1); }
            }
            STAMP0(s);
#pragma unroll
            for (int r = 0; r < 7; ++r) st16_sys(peer[(r + s) % 7], doff + (unsigned)(s * STEP_BYTES), v);
            if constexpr (s == 0) { if (latebar) __builtin_amdgcn_s_barrier(); }
            wait_vm<0>();
            STAMP0(16 + s);
            if constexpr (L0T > 0) tack = (unsigned)rt();
            if constexpr (s == 0) lds_st(lbase + L_GO, 1u);
            if ((c & 15) != 15) {
                if (lane == 0) st4_sys(myws, aoff + (c & 15) * 4, call * 32u + s + 1);
            } else {
                unsigned sp = fwd_dead;
                while (sp < FWD_SPINS && piece_min(myws + aoff) < call * 32u + s + 1) ++sp;
                if (sp >= FWD_SPINS) fwd_dead = FWD_SPINS;
                else { if ((lane & 7) < 7 && lane < 8 * MREP) st4_sys_lane(fl_lane + (step_desc(s).w ? 16 * PROW : 0), call * 32u + s + 1); }
            }
            STAMP0(32 + s);
            if constexpr (TOUCH && s + 2 < NSTEP) {
                if (phase < 64u) stouch(peer[0] + sbase + (size_t)(s + 3) * STEP_BYTES - 64);
                else if (phase < 128u) stouch(myws + WS_STAGE + (call & 1u) * PARITY_BYTES + (size_t)((b >> 3) & 7) * SENDER_BYTES + (size_t)(s + 3) * STEP_BYTES - 64);
            }
            if constexpr (TOUCH && s >= 8 && s < 12) stouch((const char*)(P.out + (size_t)(ti * 256 + (s - 8) * 85) * DIM + tj * 256));
        });
        STAMP1(2);
        if (lane == 0) __hip_atomic_store(ctr, call, __ATOMIC_RELAXED, __HIP_MEMORY_SCOPE_AGENT);
        wait_vm<0>();
        STAMP1(3);
    } else if (wave == 9) {
        for (int u = lane; u < (LDS_BYTES - L_CTL) / 4; u += 64) if (u != (L_CALL - L_CTL) / 4) lds_st(lbase + L_CTL + u * 4, 0u);
        wait_lgkm0();
        __builtin_amdgcn_s_barrier();
        const unsigned call = lds_ld(lbase + L_CALL);
        __builtin_amdgcn_s_setprio(PPRIO);
        const int p = lane >> 1, sub = lane & 1;
        const char* px = myws + WS_PROG + (ti + (tj & (MREP - 1)) * 32) * PROW;
        const char* pw = myws + WS_PROG + (16 + tj + (ti & (MREP - 1)) * 32) * PROW;
        unsigned long long tab = sub ? step_tab(1) : step_tab(0);
        unsigned ent = p * 16 + sub;
        unsigned base = lane < 16 ? call * 32u : 0xfffffff0u;
        bool dead = false;
        int queued = 0;
        unsigned spins = 0, lastsum = 0xffffffffu;
        unsigned long long t0 = 0;
        while (queued < NITEM) {
            u32x8 fx, fw;
            asm volatile("s_load_dwordx8 %0, %2, 0x0 glc\n\ts_load_dwordx8 %1, %3, 0x0 glc\n\ts_waitcnt lgkmcnt(0)" : "=&s"(fx), "=&s"(fw) : "s"(px), "s"(pw) : "memory");
            unsigned sum = 0;
#pragma unroll
            for (int i = 0; i < 8; ++i) sum += fx[i] + fw[i];
            if (sum != lastsum) {
                lastsum = sum; spins = 0;
                unsigned val = 0;
                asm volatile("v_writelane_b32 %0, %1, 0\n\tv_writelane_b32 %0, %9, 1\n\tv_writelane_b32 %0, %2, 2\n\tv_writelane_b32 %0, %10, 3\n\t"
                             "v_writelane_b32 %0, %3, 4\n\tv_writelane_b32 %0, %11, 5\n\tv_writelane_b32 %0, %4, 6\n\tv_writelane_b32 %0, %12, 7\n\t"
                             "v_writelane_b32 %0, %5, 8\n\tv_writelane_b32 %0, %13, 9\n\tv_writelane_b32 %0, %6, 10\n\tv_writelane_b32 %0, %14, 11\n\t"
                             "v_writelane_b32 %0, %7, 12\n\tv_writelane_b32 %0, %15, 13\n\tv_writelane_b32 %0, %8, 14\n\tv_writelane_b32 %0, %16, 15"
                             : "+v"(val) : "s"(fx[0]), "s"(fx[1]), "s"(fx[2]), "s"(fx[3]), "s"(fx[4]), "s"(fx[5]), "s"(fx[6]), "s"(fx[7]),
                               "s"(fw[0]), "s"(fw[1]), "s"(fw[2]), "s"(fw[3]), "s"(fw[4]), "s"(fw[5]), "s"(fw[6]), "s"(fw[7]));
                if (dead) val = 0xffffffe0u;
                for (;;) {
                    unsigned bal = (unsigned)__ballot(val > base + ((unsigned)tab & 15u));
                    bal &= bal >> 1; bal &= 0x5555u;
                    if (!bal) break;
                    const bool rdy = (bal >> (p * 2)) & 1u;
                    const bool two = ((ent - sub) & 15u) >= (unsigned)NFULL;
                    const bool emit = rdy && lane < 16 && (sub == 0 || two);
                    const unsigned lead = (unsigned)__ballot(emit);
                    const int pos = queued + __builtin_popcount(lead & ((1u << (lane & 31)) - 1u));
                    const int nq = queued + __builtin_popcount(lead);
                    if (emit) {
                        const unsigned it = ent & 15u, ty = two ? (it - (NFULL - 2)) >> 1 : 0u, hf = two ? it & 1u : 0u;
                        const unsigned sa = (unsigned)(step_tab(0) >> (4 * it)) & 15u, sb = (unsigned)(step_tab(1) >> (4 * it)) & 15u;
                        lds_st(lbase + L_QUEUE + pos * 4, two ? 0x1c00u ^ (0x200u << ty) : 0u);
                        lds_st(lbase + L_LDQ + pos * 4, 0x4000000u | ty << 24 | (((p * 16 + sb) << 5 | hf) << 12) | ((p * 16 + sa) << 5 | hf));
                    }
                    const unsigned adv = (rdy && lane < 16) ? (two ? 2u : 1u) : 0u;
                    ent += adv; tab >>= 4 * adv;
                    if (((ent - sub) & 15u) >= (unsigned)IPS) base = 0xfffffff0u;
                    queued = nq;
                }
            } else {
                __builtin_amdgcn_s_sleep(PSLEEP);
                if ((++spins & 1023u) == 0u) {
                    if (spins == 1024u) t0 = rt();
                    if (__builtin_amdgcn_readfirstlane(__hip_atomic_load(tmo, __ATOMIC_RELAXED, __HIP_MEMORY_SCOPE_AGENT)) == call || rt() - t0 > DEADLINE_TICKS) {
                        if (lane == 0) __hip_atomic_store(tmo, call, __ATOMIC_RELAXED, __HIP_MEMORY_SCOPE_AGENT);
                        lds_st(lbase + L_ABORT, 1u);
                        dead = true; lastsum = ~sum;
                    }
                }
            }
        }
        wait_vm<0>();
    } else {
        const int hb = wave - 10;
        __builtin_amdgcn_s_barrier();
        const unsigned call = lds_ld(lbase + L_CALL);
        __builtin_amdgcn_s_setprio(LPRIO);
        const unsigned goff = (unsigned)(WS_STAGE + (call & 1u) * PARITY_BYTES) + lane * 16;
        char* dbg = myws;
        const unsigned paddr = lane < 16 ? lbase + L_REL + lane * 4 : lbase + L_LDQ;
        const int thr = lane < 8 ? (hb ? -0x40000000 : 1 - NSA) : (lane < 16 ? 1 - NSB : (lane == 16 ? 1 : -0x40000000));
        asm volatile(LOADER_ASM
            :: "v"(paddr), "v"(thr), "v"(goff + ti * 16384), "v"(goff + tj * 16384), "s"(myws),
               "s"(lbase + L_RA), "s"(lbase + L_RB), "s"(lbase + L_RB + NSB * 16384),
               "s"(lbase + L_CTL), "s"(256u << hb), "s"(dbg), "n"(NITEM), "n"(STAMPS_ON), "n"(LSLEEP), "v"(lbase + L_DUMMY - 256 + lane * 4)
            : "memory", "scc", "vcc", CLOBBER_S, "v100", "v101", "v102", "v103", "v104", "v105", "v106", "v107", "v108", "v109", "v110", "v111", "v112");
        return;
    }
    STAMP_OUT();
}

extern "C" void kernel_launch(int rank, int nranks, ncclComm_t comm,
                              hipStream_t stream,
                              void* const* d_in, const int* in_sizes, int n_in,
                              void* d_out, int out_size,
                              const int* in_dtypes, int out_dtype,
                              void* const* d_out_all,
                              void* const* d_ws_all, size_t ws_bytes) {
    (void)comm; (void)in_dtypes; (void)out_dtype; (void)d_out_all;
    if (nranks != NR || n_in != 4 || (unsigned)rank >= (unsigned)NR || ws_bytes < WS_NEED || out_size != DIM * DIM ||
        in_sizes[0] != DIM * KSH || in_sizes[1] != KSH * DIM) {
        fprintf(stderr, "kernel_launch: unexpected problem shape\n");
        return;
    }
    Params P;
    P.x = (const signed char*)d_in[0]; P.w = (const signed char*)d_in[1];
    P.sx = (const float*)d_in[2]; P.sw = (const float*)d_in[3];
    P.out = (float*)d_out; P.rank = rank;
    for (int o = 0; o < NR; ++o) P.ws[o] = (char*)d_ws_all[(rank + o) % NR];
    hipLaunchKernelGGL(gemm_ar_fused, dim3(NWG), dim3(768), 0, stream, P);
    HIP_CHECK(hipGetLastError());
}
```

```cpp
#include <hip/hip_runtime.h>
#include <rccl/rccl.h>
#include <cstdio>
#include <cstdint>
#include <utility>

#define HIP_CHECK(expr) do { hipError_t _e = (expr); if (_e != hipSuccess) { \
    fprintf(stderr, "HIP error %s at %s:%d\n", hipGetErrorString(_e), __FILE__, __LINE__); \
    return; } } while (0)

#define LAS __attribute__((address_space(3)))
#define GAS __attribute__((address_space(1)))
typedef int i32x4 __attribute__((ext_vector_type(4)));
typedef unsigned u32x4 __attribute__((ext_vector_type(4)));
typedef unsigned u32x2 __attribute__((ext_vector_type(2)));
typedef unsigned u32x8 __attribute__((ext_vector_type(8)));
typedef float f32x4 __attribute__((ext_vector_type(4)));

#define LATE 1
#define PDEPTH_ 1
#define NFINE 2
#define TAU16 1024
#define T0TICKS 70
#define DRIP 2
#define LPRIO 2
#define CSLEEP 1
#define LSLEEP 1
#define CPRIO_HI 0
#define DGAP 0
#define PH2 112
#define TAU2 TAU16
#define PH2W0 1
#define PROT 32
#define DRIPFL 1
#define FINFL 9
#define EPW 2
#define PSLEEP 1
#define PROW 256
#define L0T 110
#define WARMSTEP 99
#define GOPH 128
#define ALPHA8 2
#define TLBWARM 1
#define PEERROT 1
#define TOUCH 0
#define ADAPT0 0
#define TICKET 0
#define PPRIO 2
#define MREP 2
#define LATEBAR 1
#define CMAP 1
#define ORIGIN 0
#define LAT0 90
#define WROT 2
#define PREW 22
#define PREWLD 0
#define PS0 2
#define PS1 8
#define PGAP 0
#define NSA 5
#define NSB 4

namespace {
constexpr int NR = 8;
constexpr int DIM = 4096;
constexpr int KSH = DIM / NR;
constexpr int NKS = KSH / 64;
constexpr int NWG = 256;
constexpr int NSTEP = 16;
constexpr int NFULL = 6;
constexpr int IPS = 12;
constexpr int NITEM = NR * IPS;
constexpr int NUP = NR * NKS;
struct StepDesc { int w, k, half; };
constexpr StepDesc step_desc(int s) {
    if (s < 2 * NFULL) return {s & 1, s >> 1, -1};
    return s == 12 ? StepDesc{0, 6, 0} : s == 13 ? StepDesc{1, 6, 0} : s == 14 ? StepDesc{1, 6, 1} : StepDesc{0, 6, 1};
}
constexpr int nth_step(int w, int n) { for (int s = 0; s < NSTEP; ++s) if (step_desc(s).w == w && n-- == 0) return s; return -1; }
constexpr int op_index(int s) { int n = 0; for (int i = 0; i < s; ++i) n += step_desc(i).w == step_desc(s).w; return n; }
constexpr unsigned nth_tab(int w) { unsigned t = 0; for (int n = 0; n < 8; ++n) t |= (unsigned)nth_step(w, n) << (4 * n); return t; }
constexpr unsigned long long step_tab(int w) {
    unsigned long long t = 0;
    for (int i = 0; i < IPS; ++i) t |= (unsigned long long)(i < NFULL ? 2 * i + w : (w ? (i < 8 ? 13 : 14) : (i < 10 ? 12 : 15))) << (4 * i);
    return t;
}

constexpr size_t WS_PROG = 0;
constexpr size_t WS_FLAGS_BYTES = 32 * MREP * PROW;
constexpr size_t WS_CTR = WS_FLAGS_BYTES;
constexpr size_t WS_TMO = WS_CTR + (size_t)NWG * 128;
constexpr size_t WS_ORG = WS_TMO + 128;
constexpr size_t WS_AGG = WS_ORG + 128;
constexpr size_t WS_STAGE = 4u << 20;
constexpr size_t STEP_BYTES = (size_t)NWG * 1024;
constexpr size_t SENDER_BYTES = NSTEP * STEP_BYTES;
constexpr size_t PARITY_BYTES = NR * SENDER_BYTES;
constexpr size_t WS_NEED = WS_STAGE + 2 * PARITY_BYTES;
static_assert(WS_AGG + (NSTEP + 1) * 16 * 128 <= WS_STAGE, "control region");
constexpr int L_RA = 0;
constexpr int L_RB = NSA * 16384;
constexpr int L_WST = L_RB + NSB * 16384;
constexpr int L_CTL = L_WST + NKS * 1024;
constexpr int L_PUB = L_CTL;
constexpr int L_REL = L_CTL + 64;
constexpr int L_QCNT = L_CTL + 128;
constexpr int L_ABORT = L_CTL + 132;
constexpr int L_CALL = L_CTL + 136;
constexpr int L_WRDY = L_CTL + 140;
constexpr int L_LOC = L_CTL + 144;
constexpr int L_GO = L_CTL + 148;
constexpr int L_PREW = L_CTL + 152;
static_assert(L_PREW == L_ABORT + 20 && (PS1 - PS0) * PREW <= 256, "pre-write");
constexpr int L_QUEUE = L_CTL + 256;
constexpr int L_LDQ = L_QUEUE + 128 * 4;
constexpr int L_DUMMY = L_LDQ + 128 * 4;
constexpr int LDS_BYTES = L_DUMMY + 320;
static_assert(LDS_BYTES <= 163840 && NITEM < 128, "LDS");
constexpr unsigned long long DEADLINE_TICKS = 50000000ull;
constexpr int PDEPTH = PDEPTH_;
constexpr int NST = NSTEP * 7;

struct Params {
    const signed char* x; const signed char* w; const float* sx; const float* sw;
    float* out; char* ws[NR]; int rank;
};

#define STAMP0(idx) do { } while (0)
#define STAMP1(idx) do { } while (0)
#define STAMP_OUT() do { } while (0)

constexpr unsigned step_len16(int s) { return 7u * (s < PH2 ? TAU16 : TAU2); }
constexpr unsigned step_t(int s) { unsigned t = 0; for (int i = 0; i < s; ++i) t += step_len16(i); return t >> 4; }
constexpr int sched_t(int n) { return (n <= PH2 ? n * TAU16 : PH2 * TAU16 + (n - PH2) * TAU2) >> 4; }
constexpr bool agg_step(int s) { return s < NSTEP - NFINE; }
constexpr int npost(int n) {
    const int o = n - PDEPTH;
    if (o < 0 || o >= NST) return 0;
    return agg_step(o / 7) ? (o % 7 == 6 ? 1 : 0) : 1;
}
constexpr int nyoung(int n) {
    int c = 0;
    for (int i = n - PDEPTH + 1; i <= n; ++i) c += i < NST;
    for (int i = n - PDEPTH; i < n; ++i) c += npost(i);
    return c;
}
template <class F, int... I> __device__ __forceinline__ void static_for_impl(F&& f, std::integer_sequence<int, I...>) { (f(std::integral_constant<int, I>{}), ...); }
template <int N, class F> __device__ __forceinline__ void static_for(F&& f) { static_for_impl(f, std::make_integer_sequence<int, N>{}); }
template <int N> __device__ __forceinline__ void wait_vm() { asm volatile("s_waitcnt vmcnt(%0)" :: "n"(N) : "memory"); }
__device__ __forceinline__ void wait_vm_x4(int n4) {
    switch (n4) { case 0: wait_vm<0>(); break; case 1: wait_vm<4>(); break; case 2: wait_vm<8>(); break; case 3: wait_vm<12>(); break;
                  case 4: wait_vm<16>(); break; case 5: wait_vm<20>(); break; case 6: wait_vm<24>(); break; case 7: wait_vm<28>(); break;
                  default: wait_vm<32>(); break; }
}
__device__ __forceinline__ unsigned vm_outstanding() {
    const unsigned r = __builtin_amdgcn_s_getreg(7 | (31 << 11));
    return (r & 15u) | ((r >> 18) & 48u);
}
__device__ __forceinline__ void wait_lgkm0() { asm volatile("s_waitcnt lgkmcnt(0)" ::: "memory"); }
__device__ __forceinline__ unsigned long long rt() { return __builtin_amdgcn_s_memrealtime(); }
__device__ __forceinline__ void st16_sys(const char* base, unsigned off, u32x4 v) {
    asm volatile("global_store_dwordx4 %0, %1, %2 sc0 sc1\n\ts_nop 1" :: "v"(off), "v"(v), "s"(base) : "memory");
}
__device__ __forceinline__ void st4_sys(const char* base, unsigned off, unsigned v) {
    asm volatile("global_store_dword %0, %1, %2 sc0 sc1\n\ts_nop 1" :: "v"(off), "v"(v), "s"(base) : "memory");
}
__device__ __forceinline__ void ld16_async(u32x4& v, const GAS u32x4* p) { asm volatile("global_load_dwordx4 %0, %1, off" : "=v"(v) : "v"(p) : "memory"); }
__device__ __forceinline__ void ld4_wait(unsigned& v, const char* p) { asm volatile("global_load_dword %0, %1, off\n\ts_waitcnt vmcnt(0)" : "=v"(v) : "v"(p) : "memory"); }
__device__ __forceinline__ void stouch(const char* p) { unsigned d; asm volatile("s_load_dword %0, %1, 0x0 glc\n\ts_waitcnt lgkmcnt(0)" : "=s"(d) : "s"(p) : "memory"); }
typedef unsigned u32x16 __attribute__((ext_vector_type(16)));
constexpr unsigned FWD_SPINS = 1u << 21;
__device__ __forceinline__ unsigned piece_min(const char* p) {
    u32x16 f;
    asm volatile("s_load_dwordx16 %0, %1, 0x0 glc\n\ts_waitcnt lgkmcnt(0)" : "=s"(f) : "s"(p) : "memory");
    unsigned m = f[0];
#pragma unroll
    for (int i = 1; i < 15; ++i) m = f[i] < m ? f[i] : m;
    return m;
}
__device__ __forceinline__ unsigned long long sld2_glc(const char* p) { unsigned long long d; asm volatile("s_load_dwordx2 %0, %1, 0x0 glc\n\ts_waitcnt lgkmcnt(0)" : "=s"(d) : "s"(p) : "memory"); return d; }
__device__ __forceinline__ void ld16nt_async(u32x4& v, const char* base, unsigned off) { asm volatile("global_load_dwordx4 %0, %1, %2 nt" : "=v"(v) : "v"(off), "s"(base) : "memory"); }
template <class T> __device__ __forceinline__ void pin(T& v) { asm volatile("" : "+v"(v)); }
__device__ __forceinline__ void st4_sys_lane(const char* p, unsigned v) { asm volatile("global_store_dword %0, %1, off sc0 sc1\n\ts_nop 1" :: "v"(p), "v"(v) : "memory"); }
__device__ __forceinline__ void st16_sys_lane(const char* p, u32x4 v) { asm volatile("global_store_dwordx4 %0, %1, off sc0 sc1\n\ts_nop 1" :: "v"(p), "v"(v) : "memory"); }
__device__ __forceinline__ void ticket_async(unsigned& tick, const char* base, unsigned off) {
    unsigned long long keep;
    asm volatile("s_mov_b64 %1, exec\n\ts_mov_b64 exec, 1\n\tglobal_atomic_add %0, %2, %3, %4 sc0 sc1\n\ts_mov_b64 exec, %1"
                 : "=&v"(tick), "=&s"(keep) : "v"(off), "v"(1u), "s"(base) : "memory");
}
__device__ __forceinline__ void lds_st(unsigned addr, unsigned v) { asm volatile("ds_write_b32 %0, %1" :: "v"(addr), "v"(v) : "memory"); }
__device__ __forceinline__ unsigned lds_ldv(unsigned addr) {
    unsigned v; asm volatile("ds_read_b32 %0, %1\n\ts_waitcnt lgkmcnt(0)" : "=v"(v) : "v"(addr) : "memory");
    return v;
}
__device__ __forceinline__ unsigned lds_ld(unsigned addr) { return __builtin_amdgcn_readfirstlane(lds_ldv(addr)); }
__device__ __forceinline__ void dma4k(const char* base, unsigned off, unsigned lds_dst) {
    unsigned keep;
    asm volatile("s_mov_b32 %0, m0\n\ts_mov_b32 m0, %3\n\ts_nop 0\n\t"
                 "global_load_lds_dwordx4 %1, %2 sc0 sc1\n\t"
                 "global_load_lds_dwordx4 %1, %2 offset:1024 sc0 sc1\n\t"
                 "global_load_lds_dwordx4 %1, %2 offset:2048 sc0 sc1\n\t"
                 "global_load_lds_dwordx4 %1, %2 offset:3072 sc0 sc1\n\t"
                 "s_mov_b32 m0, %0"
                 : "=&s"(keep) : "v"(off), "s"(base), "s"(lds_dst) : "memory");
}
__device__ __forceinline__ int frag_blk(int n, int c) { return n < NFULL ? c : (c & ~15) + (n - NFULL) * 8 + (c & 7); }
__device__ __forceinline__ int frag_ks(int n, int c) { return n < NFULL ? n : NFULL + ((c >> 3) & 1); }
__device__ __forceinline__ const GAS u32x4* x_src(const signed char* x, int n, int c, int lane) {
    return (const GAS u32x4*)(x + (size_t)(frag_blk(n, c) * 16 + (lane & 15)) * KSH + frag_ks(n, c) * 64 + (lane >> 4) * 16);
}
}

#define PREW_ON 1
#define STAMPS_ON 0
#define CONSUMER_ASM R"ASM(
.macro MM m, n, a
  v_mfma_i32_16x16x64_i8 v[16*(\m)+4*(\n):16*(\m)+4*(\n)+3], v[144+4*(\a):147+4*(\a)], v[128+4*(\n):131+4*(\n)], v[16*(\m)+4*(\n):16*(\m)+4*(\n)+3]
.endm
.macro MM4 m, a
  MM \m, 0, \a
  MM \m, 1, \a
  MM \m, 2, \a
  MM \m, 3, \a
.endm
.macro MC m, n, a
  MM \m, \n, \a
  MM \m+1, \n, \a+1
.endm
.macro LDA a, off
  ds_read_b128 v[144+4*(\a):147+4*(\a)], v161 offset:\off
.endm
.macro LDB n
  ds_read_b128 v[128+4*(\n):131+4*(\n)], v162 offset:1024*(\n)
.endm
.macro STAMPI
.if %12 == 1
  s_sub_u32 s48, s53, %7-64
  s_and_b32 s48, s48, 63
  s_memrealtime s[56:57]
  s_waitcnt lgkmcnt(0)
  s_mov_b32 m0, s48
  s_nop 0
  v_writelane_b32 v160, s56, m0
.endif
.endm
.macro STAMPK k, ins=s_memrealtime
.if %12
  \ins s[56:57]
  s_waitcnt lgkmcnt(0)
  v_mov_b32 v144, s56
  v_mov_b32 v163, 256+4*(\k)
  global_store_dword v163, v144, %6
  s_nop 1
.endif
.endm
.macro ADV_A
  s_add_u32 s41, s41, 1
  s_cmp_eq_u32 s41, %9
  s_cselect_b32 s44, s50, s52
  s_cselect_b32 s41, 0, s41
.endm
.macro ADV_B
  s_add_u32 s42, s42, 1
  s_cmp_eq_u32 s42, %10
  s_cselect_b32 s40, s51, s52
  s_cselect_b32 s42, 0, s42
.endm
// waits for the next item of this wave's half, passing over (and releasing) the items of the other half
.macro SPIN back
  s_mov_b32 s47, 0
1:
  s_cmp_ge_u32 s43, 0x400
  s_cbranch_scc0 2f
  ds_write2_b32 v165, v166, v166 offset0:0 offset1:8
  ADV_A
  ADV_B
  v_add_u32 v164, 4, v164
  v_add_u32 v161, s44, v161
  v_add_u32 v162, s40, v162
  v_add_u32 v166, 1, v166
  s_add_u32 s53, s53, 1
  s_mov_b32 s47, 0
  s_branch 3f
2:
  s_sleep %13
3:
  ds_read_b32 v163, v164 offset:256
  s_add_u32 s47, s47, 1
  s_waitcnt lgkmcnt(0)
  v_readfirstlane_b32 s43, v163
  s_and_b32 s43, s43, %11
  s_cmp_eq_u32 s43, 0x300
  s_cbranch_scc1 \back
  s_cmp_lt_u32 s47, 0x1000000
  s_cbranch_scc1 1b
  v_mov_b32 v163, %3
  v_mov_b32 v144, 1
  ds_write_b32 v163, v144 offset:132
  s_branch FIN_%=
.endm
.macro OUT_ST u, t, fl
  v_cvt_f32_i32 v[\t], v[16*((\u)/4)+((\u)&3)]
  v_cvt_f32_i32 v[\t+1], v[16*((\u)/4)+((\u)&3)+4]
  v_cvt_f32_i32 v[\t+2], v[16*((\u)/4)+((\u)&3)+8]
  v_cvt_f32_i32 v[\t+3], v[16*((\u)/4)+((\u)&3)+12]
  s_add_u32 s54, s58, (((\u)/4)*16+((\u)&3))*16384
  s_addc_u32 s55, s59, 0
  v_mul_f32 v[\t], s49, v[\t]
  v_mul_f32 v[\t+1], s49, v[\t+1]
  v_mul_f32 v[\t+2], s49, v[\t+2]
  v_mul_f32 v[\t+3], s49, v[\t+3]
.if \fl == 0
  global_store_dwordx4 v167, v[\t:\t+3], s[54:55]
.elseif \fl == 1
  global_store_dwordx4 v167, v[\t:\t+3], s[54:55] sc0 sc1
.else
  global_store_dwordx4 v167, v[\t:\t+3], s[54:55] nt
.endif
.endm

  v_mbcnt_lo_u32_b32 v160, -1, 0
  v_mbcnt_hi_u32_b32 v160, -1, v160
  v_lshrrev_b32 v167, 4, v160
  v_lshlrev_b32 v167, 16, v167
  v_and_b32 v166, 15, v160
  v_lshl_add_u32 v167, v166, 4, v167
  v_lshlrev_b32 v160, 4, v160
  v_add_u32 v161, %0, v160
  v_add_u32 v162, %1, v160
  v_mov_b32 v164, %3
  v_lshrrev_b32 v165, 2, v160
  v_add_u32 v165, %17, v165
  v_writelane_b32 v165, %2, 0
  v_mov_b32 v166, 1
  v_mov_b32 v160, 0
  s_mov_b64 s[58:59], %4
  s_mov_b32 s49, %5
  s_mov_b32 s41, 0
  s_mov_b32 s42, 0
  s_mov_b32 s43, 0
  s_mov_b32 s45, %8
  s_mov_b32 s50, -(%9-1)*16384
  s_mov_b32 s51, -(%10-1)*16384
  s_mov_b32 s52, 16384
  s_mov_b32 s53, 0
.set zi, 0
.rept 128
  v_mov_b32 v[zi], 0
.set zi, zi+1
.endr
  SPIN START_%=
SPINF_%=:
  s_cmp_eq_u32 s45, 0
  s_cbranch_scc1 TAIL_%=
  SPIN BACK_%=
START_%=:
  STAMPK 6, s_memtime
  STAMPK 8
  LDA 0, 0
  LDA 1, 1024
  LDB 0
  LDB 1
  LDB 2
  LDB 3
  ds_write_b32 v165, v166 offset:32
LOOP_%=:
  s_waitcnt lgkmcnt(4)
  MM 0, 0, 0
  LDA 2, 2048
  MM 1, 0, 1
  LDA 3, 3072
  s_waitcnt lgkmcnt(5)
  MM 0, 1, 0
  s_add_u32 s53, s53, 1
  MM 1, 1, 1
  s_waitcnt lgkmcnt(4)
  MM 0, 2, 0
  s_sub_u32 s45, s45, 1
  MM 1, 2, 1
  s_waitcnt lgkmcnt(3)
  MM 0, 3, 0
  v_add_u32 v164, 4, v164
  MM 1, 3, 1
  s_waitcnt lgkmcnt(1)
  MM 2, 0, 2
  LDA 0, 4096
  MM 2, 1, 2
  LDA 1, 5120
  MM 2, 2, 2
  s_add_u32 s41, s41, 1
  s_cmp_eq_u32 s41, %9
  MM 2, 3, 2
  s_waitcnt lgkmcnt(2)
  MM 3, 0, 3
  LDA 2, 6144
  MM 3, 1, 3
  s_cselect_b32 s44, s50, s52
  s_cselect_b32 s41, 0, s41
  MM 3, 2, 3
  s_add_u32 s42, s42, 1
  s_cmp_eq_u32 s42, %10
  MM 3, 3, 3
  s_waitcnt lgkmcnt(2)
  MM 4, 0, 0
  LDA 3, 7168
  MM 4, 1, 0
  ds_read_b32 v163, v164 offset:256
  MM 4, 2, 0
  ds_write_b32 v165, v166
  MM 4, 3, 0
  s_cselect_b32 s40, s51, s52
  s_waitcnt lgkmcnt(4)
  MM 5, 0, 1
  s_cselect_b32 s42, 0, s42
  v_add_u32 v161, s44, v161
  MM 5, 1, 1
  v_add_u32 v162, s40, v162
  v_add_u32 v166, 1, v166
  MM 5, 2, 1
  s_waitcnt lgkmcnt(1)
  v_readfirstlane_b32 s43, v163
  MM 5, 3, 1
  s_and_b32 s43, s43, %11
  s_cmp_eq_u32 s43, 0x300
  s_cbranch_scc0 SPINF_%=
BACK_%=:
  STAMPI
.if %12 == 2
  s_cmp_eq_u32 s45, 12
  s_cbranch_scc0 5f
  STAMPK 9
  STAMPK 10, s_memtime
5:
.endif
  MM 6, 0, 2
  LDA 0, 0
  MM 7, 0, 3
  LDA 1, 1024
  MM 6, 1, 2
  LDB 0
  MM 7, 1, 3
  LDB 1
  MM 6, 2, 2
  MM 7, 2, 3
  LDB 2
  MM 6, 3, 2
  MM 7, 3, 3
  LDB 3
  ds_write_b32 v165, v166 offset:32
  s_branch LOOP_%=
TAIL_%=:
  s_waitcnt lgkmcnt(0)
  MM4 6, 2
  MM4 7, 3
FIN_%=:
  v_mov_b32 v166, 0x7fffffff
  v_mov_b32 v163, %3
  ds_write2_b32 v165, v166, v166 offset0:0 offset1:8
  ds_read_b32 v166, v163 offset:132
  STAMPK 2
  STAMPK 7, s_memtime
  s_waitcnt lgkmcnt(0)
  v_readfirstlane_b32 s48, v166
  s_cmp_eq_u32 s48, 0
  s_cselect_b32 s49, s49, 0x7fc00000
.if %18
  v_mov_b32 v163, %3
  s_mov_b32 s47, 0
8:
  ds_read_b32 v166, v163 offset:152
  s_add_u32 s47, s47, 1
  s_waitcnt lgkmcnt(0)
  v_readfirstlane_b32 s48, v166
  s_cmp_lg_u32 s48, 0
  s_cbranch_scc1 9f
  s_sleep 2
  s_cmp_lt_u32 s47, 0x1000000
  s_cbranch_scc1 8b
  s_mov_b32 s49, 0x7fc00000
9:
.endif
.if %16 == 9
  s_cmp_lt_u32 %11, 0x1000
  s_cselect_b64 vcc, -1, 0
.else
  s_cmp_lt_u32 %11, 0x1000
  s_cbranch_scc1 EARLY_%=
.set du, 0
.rept 32
  OUT_ST du, 128+4*(du&7), %16
.set du, du+1
.endr
  s_branch END_%=
EARLY_%=:
  s_mov_b64 vcc, -1
.endif
  s_setprio 0
  v_mov_b32 v163, %3
.if %14 == 62
6:
  s_sleep 4
  ds_read_b32 v166, v163 offset:128
  s_waitcnt lgkmcnt(0)
  v_cmp_eq_u32 vcc, 0, v166
  s_cbranch_vccnz 6b
.endif
.set du, 0
.rept 32
  OUT_ST du, 128+4*(du&7), %15
  s_cbranch_vccz 4f
  ds_read_b32 v166, v163 offset:128
  s_waitcnt vmcnt(%14) lgkmcnt(0)
  v_cmp_eq_u32 vcc, 0, v166
4:
.set du, du+1
.endr
END_%=:
.if %12
  STAMPK 3
  s_waitcnt vmcnt(0)
  STAMPK 4
  STAMPK 11, s_memtime
  v_mbcnt_lo_u32_b32 v166, -1, 0
  v_mbcnt_hi_u32_b32 v166, -1, v166
  v_lshlrev_b32 v166, 2, v166
  global_store_dword v166, v160, %6
.endif
.purgem MM
.purgem MM4
.purgem MC
.purgem LDA
.purgem LDB
.purgem STAMPI
.purgem STAMPK
.purgem SPIN
.purgem ADV_A
.purgem ADV_B
.purgem OUT_ST
)ASM"

#define LOADER_ASM R"ASM(
.macro LSTAMP vr, idx
.if %12 == 1
  s_memrealtime s[56:57]
  s_sub_u32 s58, \idx, %11-64
  s_and_b32 s58, s58, 63
  s_mov_b32 s59, m0
  s_mov_b32 m0, s58
  s_waitcnt lgkmcnt(0)
  v_writelane_b32 \vr, s56, m0
  s_mov_b32 m0, s59
.endif
.endm
.macro PUBLISH
  LSTAMP v108, s44
  ds_add_u32 v105, v109 offset:256
  s_add_u32 s44, s44, 1
  s_mov_b32 s41, s42
  s_mov_b32 s42, s43
  s_mov_b32 s43, s49
  v_add_u32 v105, v112, v105
.endm
.macro LANDED
  s_getreg_b32 s47, hwreg(HW_REG_IB_STS)
  s_and_b32 s48, s47, 15
  s_bfe_u32 s47, s47, 0x20016
  s_lshl4_add_u32 s47, s47, s48
  s_sub_u32 s47, s40, s47
  s_cmp_ge_u32 s47, s41
.endm
.macro PUBCHK
  LANDED
  s_cbranch_scc0 7f
  PUBLISH
7:
.endm
.macro DMA4 va
  global_load_lds_dwordx4 \va, %4 sc0 sc1
  global_load_lds_dwordx4 \va, %4 offset:1024 sc0 sc1
  global_load_lds_dwordx4 \va, %4 offset:2048 sc0 sc1
  global_load_lds_dwordx4 \va, %4 offset:3072 sc0 sc1
  s_add_u32 s40, s40, 4
  s_add_u32 m0, m0, 4096
  v_add_u32 \va, 4096, \va
.endm
.macro BODY hb
  ds_read_b32 v102, v101
LLOOP\hb\()_%=:
  PUBCHK
  s_waitcnt lgkmcnt(0)
  v_cmp_ge_i32 vcc, v102, v100
  v_readlane_b32 s51, v102, 16
  s_cmp_eq_u64 vcc, -1
  s_cselect_b32 s47, s43, 0
  s_cmp_eq_u32 s47, s49
  s_cbranch_scc0 LIDLE\hb\()_%=
LGO\hb\()_%=:
  s_bfe_u32 s52, s51, 0xc000c
  s_lshl_b32 s52, s52, 13
  v_add_u32 v110, s52, %3
  s_bfe_u32 s53, s51, 0x20018
.if \hb == 0
  s_and_b32 s52, s51, 0xfff
  s_lshl_b32 s52, s52, 13
  v_add_u32 v104, s52, %2
  s_cmp_eq_u32 s53, 3
  s_cselect_b32 s52, 8192, 0
  s_add_u32 m0, s46, s52
  s_add_u32 s46, s46, 16384
  DMA4 v104
  v_add_u32 v100, v103, v100
  v_add_u32 v101, v111, v101
  s_cmp_eq_u32 s46, %6
  s_cselect_b32 s46, %5, s46
  PUBCHK
  s_cmp_eq_u32 s53, 0
  s_cbranch_scc0 8f
  DMA4 v104
  PUBCHK
  DMA4 v104
  PUBCHK
8:
  ds_read_b32 v102, v101
  DMA4 v104
  s_cmp_eq_u32 s53, 3
  s_cbranch_scc0 9f
  s_mov_b32 m0, s60
  v_subrev_u32 v110, 0x40000, v110
  PUBCHK
  DMA4 v110
9:
.else
  s_cmp_ge_u32 s53, 2
  s_cselect_b32 s52, 8192, 0
  s_add_u32 m0, s60, s52
  v_add_u32 v100, v103, v100
  DMA4 v110
  v_add_u32 v101, v111, v101
  PUBCHK
  s_cmp_eq_u32 s53, 0
  s_cbranch_scc0 8f
  DMA4 v110
  PUBCHK
  DMA4 v110
  PUBCHK
8:
  ds_read_b32 v102, v101
  DMA4 v110
  s_cmp_eq_u32 s53, 3
  s_cbranch_scc0 9f
  s_add_u32 m0, s60, 4096
  v_subrev_u32 v110, 0x41000, v110
  PUBCHK
  DMA4 v110
9:
.endif
  LSTAMP v107, s45
  s_cmp_eq_u32 s42, s49
  s_cselect_b32 s43, s49, s40
  s_cselect_b32 s47, s40, s42
  s_cmp_eq_u32 s41, s49
  s_cselect_b32 s42, s49, s47
  s_cselect_b32 s41, s40, s41
  s_add_u32 s60, s60, 16384
  s_cmp_eq_u32 s60, %7
  s_cselect_b32 s60, %6, s60
  s_add_u32 s45, s45, 1
  s_mov_b32 s50, 0
  s_cmp_lt_u32 s45, %11
  s_cbranch_scc1 LLOOP\hb\()_%=
  s_branch LDRAIN_%=
LIDLE\hb\()_%=:
  ds_read_b32 v102, v101
  s_cmp_eq_u32 s41, s49
  s_cbranch_scc0 LLOOP\hb\()_%=
LWAIT\hb\()_%=:
  s_add_u32 s50, s50, 1
  s_waitcnt lgkmcnt(0)
  v_cmp_ge_i32 vcc, v102, v100
  v_readlane_b32 s51, v102, 16
  s_cmp_eq_u64 vcc, -1
  s_cbranch_scc1 LGO\hb\()_%=
  s_sleep %13
  ds_read_b32 v102, v101
  s_cmp_lt_u32 s50, 0x1000000
  s_cbranch_scc1 LWAIT\hb\()_%=
  v_mov_b32 v106, 1
  v_mov_b32 v101, %8
  ds_write_b32 v101, v106 offset:132
  s_branch LEND_%=
.endm
  s_mov_b32 s40, 0
  s_mov_b32 s49, 0x7fffffff
  s_mov_b32 s41, s49
  s_mov_b32 s42, s49
  s_mov_b32 s43, s49
  s_mov_b32 s44, 0
  s_mov_b32 s45, 0
  s_mov_b32 s46, %5
  s_mov_b32 s60, %6
  s_mov_b32 s50, 0
  v_mov_b32 v100, %1
  v_mov_b32 v101, %0
  v_mov_b32 v103, 1
  v_writelane_b32 v103, 0, 16
  v_mov_b32 v111, 0
  v_writelane_b32 v111, 4, 16
  v_mov_b32 v105, %14
  v_writelane_b32 v105, %8, 0
  v_mov_b32 v112, 0
  v_writelane_b32 v112, 4, 0
  v_mov_b32 v109, %9
  v_mov_b32 v107, 0
  v_mov_b32 v108, 0
  s_waitcnt vmcnt(0)
  s_cmp_eq_u32 %9, 0x200
  s_cbranch_scc1 LONE_%=
  BODY 0
LONE_%=:
  BODY 1
LDRAIN_%=:
  PUBCHK
  s_cmp_lt_u32 s44, %11
  s_cbranch_scc1 LDRAIN_%=
LEND_%=:
  v_mov_b32 v106, 1
  v_mov_b32 v101, %8
  ds_write_b32 v101, v106 offset:128
  s_waitcnt vmcnt(0) lgkmcnt(0)
.if %12
  v_mbcnt_lo_u32_b32 v106, -1, 0
  v_mbcnt_hi_u32_b32 v106, -1, v106
  v_lshlrev_b32 v106, 2, v106
  global_store_dword v106, v107, %10
  global_store_dword v106, v108, %10 offset:256
.endif
.purgem LSTAMP
.purgem PUBLISH
.purgem LANDED
.purgem PUBCHK
.purgem DMA4
.purgem BODY
)ASM"
#define CLOBBER_S "s40", "s41", "s42", "s43", "s44", "s45", "s46", "s47", "s48", "s49", "s50", "s51", "s52", "s53", "s54", "s55", "s56", "s57", "s58", "s59", "s60", "s61", "s62", "s63"
#define V10(t) "v" #t "0", "v" #t "1", "v" #t "2", "v" #t "3", "v" #t "4", "v" #t "5", "v" #t "6", "v" #t "7", "v" #t "8", "v" #t "9"
#define CLOBBER_V V10(), V10(1), V10(2), V10(3), V10(4), V10(5), V10(6), V10(7), V10(8), V10(9), V10(10), V10(11), V10(12), V10(13), V10(14), V10(15), \
    "v160", "v161", "v162", "v163", "v164", "v165", "v166", "v167"

__global__ void __launch_bounds__(768, 3) gemm_ar_fused(Params P) {
    __shared__ __attribute__((aligned(16))) unsigned char lds_raw[LDS_BYTES];
    LAS unsigned char* lds = (LAS unsigned char*)lds_raw;
    const unsigned lbase = (unsigned)(size_t)lds;
    const int tid = threadIdx.x, lane = tid & 63;
    const int wave = __builtin_amdgcn_readfirstlane(tid >> 6);
    const int b = blockIdx.x;
    const int rank = P.rank;
    const int c = CMAP == 2 ? ((b >> 4) << 4) | ((b & 7) << 1) | ((b >> 3) & 1) : ((b >> 5) << 5) | ((b & 7) << 2) | ((b >> 3) & 3);
    char* myws = P.ws[0];
    const int ti = ((b & 7) >> 1) * 4 + (b >> 6), tj = (b & 1) * 8 + ((b >> 3) & 7);
    if (wave < 9) STAMP1(0);
    GAS unsigned* tmo = (GAS unsigned*)(myws + WS_TMO);

    if (wave < 8) {
        unsigned warm = 0;
        if (TLBWARM && wave == 0) {
            const unsigned ph = (unsigned)((c + PROT * rank) & 255), kk = CMAP == 2 ? ((ph >> 4) - 8u) * 2u + (ph & 1u) : ((ph >> 5) - 4u) * 4u + (ph & 3u);
            if (ph >= 128u && kk < 14u) {
                const char* pw = P.ws[1 + kk % 7u] + WS_STAGE + (kk / 7u) * PARITY_BYTES + (size_t)rank * SENDER_BYTES;
                asm volatile("s_load_dword %0, %1, 0x0 glc" : "=s"(warm) : "s"(pw) : "memory");
            }
        }
        __builtin_amdgcn_s_barrier();
        const unsigned call = lds_ld(lbase + L_CALL);
        asm volatile("" :: "s"(warm));
        if (((c + PROT * rank) & 255) < GOPH) { unsigned sp = 0; while (lds_ld(lbase + L_GO) == 0u && ++sp < (1u << 22)) __builtin_amdgcn_s_sleep(2); }
        {
            const int ks = wave;
            const int cc = lane & 15, g = lane >> 4;
            const GAS unsigned* wp = (const GAS unsigned*)(P.w + (size_t)(frag_ks(ks, c) * 64 + g * 16) * DIM + (frag_blk(ks, c) >> 2) * 64 + cc * 4);
            unsigned d[16];
#pragma unroll
            for (int t = 0; t < 16; ++t) d[t] = wp[(size_t)t * (DIM / 4)];
            const int sx = (nth_tab(0) >> (4 * wave)) & 15, sw = (nth_tab(1) >> (4 * wave)) & 15;
            const u32x4 xv = *x_src(P.x, ks, c, lane);
            const unsigned q = c & 3;
            const unsigned sel_lo = 0x0c0c0000u | ((4u + q) << 8) | q;
            const unsigned sel_hi = 0x00000c0cu | ((4u + q) << 24) | (q << 16);
            u32x4 wv;
#pragma unroll
            for (int e = 0; e < 4; ++e)
                wv[e] = __builtin_amdgcn_perm(d[4 * e + 1], d[4 * e], sel_lo) | __builtin_amdgcn_perm(d[4 * e + 3], d[4 * e + 2], sel_hi);
            *(LAS u32x4*)(lds + L_WST + ks * 1024 + lane * 16) = wv;
            asm volatile("" ::: "memory");
            if (lane == 0) __hip_atomic_fetch_add((LAS unsigned*)(lds + L_WRDY), 1u, __ATOMIC_RELAXED, __HIP_MEMORY_SCOPE_WORKGROUP);
            const unsigned soff = (unsigned)(WS_STAGE + (call & 1u) * PARITY_BYTES + (size_t)rank * SENDER_BYTES) + c * 1024 + lane * 16;
            st16_sys(myws, soff + sx * (unsigned)STEP_BYTES, xv);
            st16_sys(myws, soff + sw * (unsigned)STEP_BYTES, wv);
            STAMP1(1);
            wait_vm<0>();
            if (lane == 0) {
                if (__hip_atomic_fetch_add((LAS unsigned*)(lds + L_LOC), 1u, __ATOMIC_RELAXED, __HIP_MEMORY_SCOPE_WORKGROUP) == 7u) {
                    GAS unsigned* tk = (GAS unsigned*)(myws + WS_AGG + (NSTEP * 16 + (c >> 4)) * 128);
                    if ((__hip_atomic_fetch_add(tk, 1u, __ATOMIC_RELAXED, __HIP_MEMORY_SCOPE_AGENT) & 15u) == 15u)
                        for (int k = 0; k < 2 * MREP; ++k)
                            __hip_atomic_store((GAS unsigned*)(myws + WS_PROG + (k * 16 + (c >> 4)) * PROW + rank * 4), call * 32u + NSTEP, __ATOMIC_RELAXED, __HIP_MEMORY_SCOPE_SYSTEM);
                }
            }
        }
        const int wr = wave >> 2, wc = (wave + WROT) & 3;
        const unsigned scale = __builtin_amdgcn_readfirstlane(__float_as_uint(P.sx[0] * P.sw[0]));
        const char* obase = (const char*)(P.out + (size_t)(ti * 256 + wr * 128) * DIM + tj * 256 + wc * 64);
        char* dbg = myws;
        if (CPRIO_HI) __builtin_amdgcn_s_setprio(CPRIO_HI);
        asm volatile(CONSUMER_ASM
            :: "s"(lbase + L_RA + wr * 8192), "s"(lbase + L_RB + wc * 4096), "s"(lbase + L_REL + wave * 4), "s"(lbase + L_CTL),
               "s"(obase), "s"(scale), "s"(dbg),
               "n"(NITEM), "n"(NUP), "n"(NSA), "n"(NSB), "s"(0x300u | (0x400u << (wr ? 2 : wc >> 1))), "n"(STAMPS_ON), "n"(CSLEEP), "n"(EPW), "n"(DRIPFL), "n"(FINFL), "s"(lbase + L_DUMMY), "n"(PREW_ON)
            : "memory", "scc", "vcc", CLOBBER_S, CLOBBER_V);
        return;
    } else if (wave == 8) {
        unsigned tp0 = (unsigned)rt() + T0TICKS;
        __builtin_amdgcn_s_setprio(3);
        GAS unsigned* ctr = (GAS unsigned*)(myws + WS_CTR + (size_t)b * 128);
        unsigned callv; u32x4 xv[8];
        asm volatile("global_load_dword %0, %1, off sc1" : "=v"(callv) : "v"(ctr) : "memory");
        static_for<8>([&](auto nc) __attribute__((always_inline)) {
            constexpr int n = decltype(nc)::value;
            ld16_async(xv[n], x_src(P.x, n, c, lane));
        });
        const char* peer[7];
#pragma unroll
        for (int r = 0; r < 7; ++r) { int o = PEERROT ? r + (int)((unsigned)(b >> 3) % 7u) : r; o = o >= 7 ? o - 7 : o; peer[r] = P.ws[1 + o]; }
        const char* fl_lane = peer[0];
#pragma unroll
        for (int r = 1; r < 7; ++r) fl_lane = (lane & 7) == r ? peer[r] : fl_lane;
        fl_lane += WS_PROG + ((c >> 4) + (lane >> 3) * 32) * PROW + rank * 4;
        const unsigned phase = (unsigned)(((CMAP == 0 ? c : CMAP == 1 ? (c & ~15) | ((c & 3) << 2) | ((c >> 2) & 3) : b) + PROT * rank) & 255);
        asm volatile("s_waitcnt vmcnt(7)" : "+v"(callv), "+v"(xv[0]) :: "memory");
        const unsigned call = __builtin_amdgcn_readfirstlane(callv) + 1u;
        const unsigned doff = (unsigned)(WS_STAGE + (call & 1u) * PARITY_BYTES + (size_t)rank * SENDER_BYTES) + c * 1024 + lane * 16;
        const unsigned aoff = (unsigned)(WS_AGG + (c >> 4) * 128);
        const size_t sbase = WS_STAGE + (call & 1u) * PARITY_BYTES + (size_t)rank * SENDER_BYTES;
        unsigned tack = 0, fwd_dead = 0;
        const int late0 = (int)((unsigned)rt() - tp0);
        STAMP1(1);
        lds_st(lbase + L_CALL, call); wait_lgkm0();
        const bool latebar = LATEBAR && phase < GOPH;
        if (!latebar) __builtin_amdgcn_s_barrier();
        static_for<NSTEP>([&](auto sc) __attribute__((always_inline)) {
            constexpr int s = decltype(sc)::value;
            u32x4 v;
            if constexpr (s == 1) {
                pin(xv[1]); pin(xv[2]); pin(xv[3]); pin(xv[4]); pin(xv[5]); pin(xv[6]); pin(xv[7]);
                unsigned sp = 0; while (lds_ld(lbase + L_WRDY) < 8u && ++sp < (1u << 24)) __builtin_amdgcn_s_sleep(1);
            }
            if constexpr (step_desc(s).w) { v = *(const LAS u32x4*)(lds + L_WST + op_index(s) * 1024 + lane * 16); pin(v); }
            else v = xv[op_index(s)];
            {
                unsigned tgo = tp0 + step_t(s) + (phase * step_len16(s) >> 12);
                if constexpr (ADAPT0 && s == 0) { if (late0 > 0) tgo += (unsigned)late0; }
                if constexpr (L0T > 0 && s > 0) { const int late = (int)(tack + (step_len16(s) >> 4) - L0T - tgo); if (late > 0) tgo += (unsigned)(late * (8 - ALPHA8) >> 3); }
                int rem;
                while ((rem = (int)(tgo - (unsigned)rt())) > 0) { if (rem > 40) __builtin_amdgcn_s_sleep(8); else __builtin_amdgcn_s_sleep(1); }
            }
            STAMP0(s);
#pragma unroll
            for (int r = 0; r < 7; ++r) st16_sys(peer[(r + s) % 7], doff + (unsigned)(s * STEP_BYTES), v);
            if constexpr (s == 0) { if (latebar) __builtin_amdgcn_s_barrier(); }
            wait_vm<0>();
            STAMP0(16 + s);
            if constexpr (L0T > 0) tack = (unsigned)rt();
            if constexpr (s == 0) lds_st(lbase + L_GO, 1u);
            if ((c & 15) != 15) {
                if (lane == 0) st4_sys(myws, aoff + (c & 15) * 4, call * 32u + s + 1);
            } else {
                unsigned sp = fwd_dead;
                while (sp < FWD_SPINS && piece_min(myws + aoff) < call * 32u + s + 1) ++sp;
                if (sp >= FWD_SPINS) fwd_dead = FWD_SPINS;
                else { if ((lane & 7) < 7 && lane < 8 * MREP) st4_sys_lane(fl_lane + (step_desc(s).w ? 16 * PROW : 0), call * 32u + s + 1); }
            }
            if constexpr (ORIGIN && s == 0) {
                if (phase == 0u) {
                    tp0 = tack - LAT0;
                    if (lane == 0) __hip_atomic_store((GAS unsigned long long*)(myws + WS_ORG), (unsigned long long)tp0 << 32 | call, __ATOMIC_RELAXED, __HIP_MEMORY_SCOPE_AGENT);
                } else {
                    for (int sp = 0; sp < 64; ++sp) {
                        const unsigned long long o = sld2_glc(myws + WS_ORG);
                        if ((unsigned)o == call) { tp0 = (unsigned)(o >> 32); break; }
                        __builtin_amdgcn_s_sleep(4);
                    }
                }
            }
            if constexpr (PREW > 0 && s >= PS0 && s < PS1) {
                const char* ob = (const char*)(P.out + (size_t)(ti * 256 + 255 - (s - PS0) * PREW) * DIM + tj * 256);
                const u32x4 z = {0u, 0u, 0u, 0u};
#pragma unroll
                for (int k = 0; k < PREW; ++k) { st16_sys(ob - (size_t)k * DIM * 4, lane * 16, z); if (PGAP) __builtin_amdgcn_s_sleep(PGAP); }
                if constexpr (s == PS1 - 1) { wait_vm<0>(); lds_st(lbase + L_ABORT + 4 * 5, 1u); }
            }
            STAMP0(32 + s);
            if constexpr (TOUCH && s + 2 < NSTEP) {
                if (phase < 64u) stouch(peer[0] + sbase + (size_t)(s + 3) * STEP_BYTES - 64);
                else if (phase < 128u) stouch(myws + WS_STAGE + (call & 1u) * PARITY_BYTES + (size_t)((b >> 3) & 7) * SENDER_BYTES + (size_t)(s + 3) * STEP_BYTES - 64);
            }
            if constexpr (TOUCH && s >= 8 && s < 12) stouch((const char*)(P.out + (size_t)(ti * 256 + (s - 8) * 85) * DIM + tj * 256));
        });
        STAMP1(2);
        if (lane == 0) __hip_atomic_store(ctr, call, __ATOMIC_RELAXED, __HIP_MEMORY_SCOPE_AGENT);
        wait_vm<0>();
        STAMP1(3);
    } else if (wave == 9) {
        for (int u = lane; u < (LDS_BYTES - L_CTL) / 4; u += 64) if (u != (L_CALL - L_CTL) / 4) lds_st(lbase + L_CTL + u * 4, 0u);
        wait_lgkm0();
        __builtin_amdgcn_s_barrier();
        const unsigned call = lds_ld(lbase + L_CALL);
        __builtin_amdgcn_s_setprio(PPRIO);
        const int p = lane >> 1, sub = lane & 1;
        const char* px = myws + WS_PROG + (ti + (tj & (MREP - 1)) * 32) * PROW;
        const char* pw = myws + WS_PROG + (16 + tj + (ti & (MREP - 1)) * 32) * PROW;
        unsigned long long tab = sub ? step_tab(1) : step_tab(0);
        unsigned ent = p * 16 + sub;
        unsigned base = lane < 16 ? call * 32u : 0xfffffff0u;
        bool dead = false;
        int queued = 0;
        unsigned spins = 0, lastsum = 0xffffffffu;
        unsigned long long t0 = 0;
        while (queued < NITEM) {
            u32x8 fx, fw;
            asm volatile("s_load_dwordx8 %0, %2, 0x0 glc\n\ts_load_dwordx8 %1, %3, 0x0 glc\n\ts_waitcnt lgkmcnt(0)" : "=&s"(fx), "=&s"(fw) : "s"(px), "s"(pw) : "memory");
            unsigned sum = 0;
#pragma unroll
            for (int i = 0; i < 8; ++i) sum += fx[i] + fw[i];
            if (sum != lastsum) {
                lastsum = sum; spins = 0;
                unsigned val = 0;
                asm volatile("v_writelane_b32 %0, %1, 0\n\tv_writelane_b32 %0, %9, 1\n\tv_writelane_b32 %0, %2, 2\n\tv_writelane_b32 %0, %10, 3\n\t"
                             "v_writelane_b32 %0, %3, 4\n\tv_writelane_b32 %0, %11, 5\n\tv_writelane_b32 %0, %4, 6\n\tv_writelane_b32 %0, %12, 7\n\t"
                             "v_writelane_b32 %0, %5, 8\n\tv_writelane_b32 %0, %13, 9\n\tv_writelane_b32 %0, %6, 10\n\tv_writelane_b32 %0, %14, 11\n\t"
                             "v_writelane_b32 %0, %7, 12\n\tv_writelane_b32 %0, %15, 13\n\tv_writelane_b32 %0, %8, 14\n\tv_writelane_b32 %0, %16, 15"
                             : "+v"(val) : "s"(fx[0]), "s"(fx[1]), "s"(fx[2]), "s"(fx[3]), "s"(fx[4]), "s"(fx[5]), "s"(fx[6]), "s"(fx[7]),
                               "s"(fw[0]), "s"(fw[1]), "s"(fw[2]), "s"(fw[3]), "s"(fw[4]), "s"(fw[5]), "s"(fw[6]), "s"(fw[7]));
                if (dead) val = 0xffffffe0u;
                for (;;) {
                    unsigned bal = (unsigned)__ballot(val > base + ((unsigned)tab & 15u));
                    bal &= bal >> 1; bal &= 0x5555u;
                    if (!bal) break;
                    const bool rdy = (bal >> (p * 2)) & 1u;
                    const bool two = ((ent - sub) & 15u) >= (unsigned)NFULL;
                    const bool emit = rdy && lane < 16 && (sub == 0 || two);
                    const unsigned lead = (unsigned)__ballot(emit);
                    const int pos = queued + __builtin_popcount(lead & ((1u << (lane & 31)) - 1u));
                    const int nq = queued + __builtin_popcount(lead);
                    if (emit) {
                        const unsigned it = ent & 15u, ty = two ? (it - (NFULL - 2)) >> 1 : 0u, hf = two ? it & 1u : 0u;
                        const unsigned sa = (unsigned)(step_tab(0) >> (4 * it)) & 15u, sb = (unsigned)(step_tab(1) >> (4 * it)) & 15u;
                        lds_st(lbase + L_QUEUE + pos * 4, two ? 0x1c00u ^ (0x200u << ty) : 0u);
                        lds_st(lbase + L_LDQ + pos * 4, 0x4000000u | ty << 24 | (((p * 16 + sb) << 5 | hf) << 12) | ((p * 16 + sa) << 5 | hf));
                    }
                    const unsigned adv = (rdy && lane < 16) ? (two ? 2u : 1u) : 0u;
                    ent += adv; tab >>= 4 * adv;
                    if (((ent - sub) & 15u) >= (unsigned)IPS) base = 0xfffffff0u;
                    queued = nq;
                }
            } else {
                __builtin_amdgcn_s_sleep(PSLEEP);
                if ((++spins & 1023u) == 0u) {
                    if (spins == 1024u) t0 = rt();
                    if (__builtin_amdgcn_readfirstlane(__hip_atomic_load(tmo, __ATOMIC_RELAXED, __HIP_MEMORY_SCOPE_AGENT)) == call || rt() - t0 > DEADLINE_TICKS) {
                        if (lane == 0) __hip_atomic_store(tmo, call, __ATOMIC_RELAXED, __HIP_MEMORY_SCOPE_AGENT);
                        lds_st(lbase + L_ABORT, 1u);
                        dead = true; lastsum = ~sum;
                    }
                }
            }
        }
        wait_vm<0>();
    } else {
        const int hb = wave - 10;
        __builtin_amdgcn_s_barrier();
        const unsigned call = lds_ld(lbase + L_CALL);
        __builtin_amdgcn_s_setprio(LPRIO);
        const unsigned goff = (unsigned)(WS_STAGE + (call & 1u) * PARITY_BYTES) + lane * 16;
        char* dbg = myws;
        const unsigned paddr = lane < 16 ? lbase + L_REL + lane * 4 : lbase + L_LDQ;
        const int thr = lane < 8 ? (hb ? -0x40000000 : 1 - NSA) : (lane < 16 ? 1 - NSB : (lane == 16 ? 1 : -0x40000000));
        asm volatile(LOADER_ASM
            :: "v"(paddr), "v"(thr), "v"(goff + ti * 16384), "v"(goff + tj * 16384), "s"(myws),
               "s"(lbase + L_RA), "s"(lbase + L_RB), "s"(lbase + L_RB + NSB * 16384),
               "s"(lbase + L_CTL), "s"(256u << hb), "s"(dbg), "n"(NITEM), "n"(STAMPS_ON), "n"(LSLEEP), "v"(lbase + L_DUMMY - 256 + lane * 4)
            : "memory", "scc", "vcc", CLOBBER_S, "v100", "v101", "v102", "v103", "v104", "v105", "v106", "v107", "v108", "v109", "v110", "v111", "v112");
        return;
    }
    STAMP_OUT();
}

extern "C" void kernel_launch(int rank, int nranks, ncclComm_t comm,
                              hipStream_t stream,
                              void* const* d_in, const int* in_sizes, int n_in,
                              void* d_out, int out_size,
                              const int* in_dtypes, int out_dtype,
                              void* const* d_out_all,
                              void* const* d_ws_all, size_t ws_bytes) {
    (void)comm; (void)in_dtypes; (void)out_dtype; (void)d_out_all;
    if (nranks != NR || n_in != 4 || (unsigned)rank >= (unsigned)NR || ws_bytes < WS_NEED || out_size != DIM * DIM ||
        in_sizes[0] != DIM * KSH || in_sizes[1] != KSH * DIM) {
        fprintf(stderr, "kernel_launch: unexpected problem shape\n");
        return;
    }
    Params P;
    P.x = (const signed char*)d_in[0]; P.w = (const signed char*)d_in[1];
    P.sx = (const float*)d_in[2]; P.sw = (const float*)d_in[3];
    P.out = (float*)d_out; P.rank = rank;
    for (int o = 0; o < NR; ++o) P.ws[o] = (char*)d_ws_all[(rank + o) % NR];
    hipLaunchKernelGGL(gemm_ar_fused, dim3(NWG), dim3(768), 0, stream, P);
    HIP_CHECK(hipGetLastError());
}
```

```cpp
#include <hip/hip_runtime.h>
#include <rccl/rccl.h>
#include <cstdio>
#include <cstdint>
#include <utility>

#define HIP_CHECK(expr) do { hipError_t _e = (expr); if (_e != hipSuccess) { \
    fprintf(stderr, "HIP error %s at %s:%d\n", hipGetErrorString(_e), __FILE__, __LINE__); \
    return; } } while (0)

#define LAS __attribute__((address_space(3)))
#define GAS __attribute__((address_space(1)))
typedef int i32x4 __attribute__((ext_vector_type(4)));
typedef unsigned u32x4 __attribute__((ext_vector_type(4)));
typedef unsigned u32x2 __attribute__((ext_vector_type(2)));
typedef unsigned u32x8 __attribute__((ext_vector_type(8)));
typedef float f32x4 __attribute__((ext_vector_type(4)));

#define LATE 1
#define PDEPTH_ 1
#define NFINE 2
#define TAU16 1024
#define T0TICKS 70
#define DRIP 2
#define LPRIO 2
#define CSLEEP 1
#define LSLEEP 1
#define CPRIO_HI 0
#define DGAP 0
#define PH2 112
#define TAU2 TAU16
#define PH2W0 1
#define PROT 32
#define DRIPFL 1
#define FINFL 9
#define EPW 2
#define PSLEEP 1
#define PROW 256
#define L0T 110
#define WARMSTEP 99
#define GOPH 128
#define ALPHA8 2
#define TLBWARM 1
#define PEERROT 1
#define TOUCH 0
#define ADAPT0 0
#define TICKET 15
#define PPRIO 2
#define MREP 2
#define LATEBAR 1
#define CMAP 1
#define ORIGIN 0
#define LAT0 90
#define WROT 2
#define PREW 22
#define PS0 2
#define PS1 8
#define PGAP 0
#define TK0 12
#define NSA 4
#define NSB 5

namespace {
constexpr int NR = 8;
constexpr int DIM = 4096;
constexpr int KSH = DIM / NR;
constexpr int NKS = KSH / 64;
constexpr int NWG = 256;
constexpr int NSTEP = 16;
constexpr int NFULL = 6;
constexpr int IPS = 12;
constexpr int NITEM = NR * IPS;
constexpr int NTILE = NR * (2 * NFULL + 7);
constexpr int NUP = NR * NKS;
struct StepDesc { int w, k, half; };
constexpr StepDesc step_desc(int s) {
    if (s < 2 * NFULL) return {s & 1, s >> 1, -1};
    return s == 12 ? StepDesc{0, 6, 0} : s == 13 ? StepDesc{1, 6, 0} : s == 14 ? StepDesc{1, 6, 1} : StepDesc{0, 6, 1};
}
constexpr int nth_step(int w, int n) { for (int s = 0; s < NSTEP; ++s) if (step_desc(s).w == w && n-- == 0) return s; return -1; }
constexpr int op_index(int s) { int n = 0; for (int i = 0; i < s; ++i) n += step_desc(i).w == step_desc(s).w; return n; }
constexpr unsigned nth_tab(int w) { unsigned t = 0; for (int n = 0; n < 8; ++n) t |= (unsigned)nth_step(w, n) << (4 * n); return t; }
constexpr unsigned long long step_tab(int w) {
    unsigned long long t = 0;
    for (int i = 0; i < IPS; ++i) t |= (unsigned long long)(i < NFULL ? 2 * i + w : (w ? (i < 8 ? 13 : 14) : (i < 10 ? 12 : 15))) << (4 * i);
    return t;
}

constexpr size_t WS_PROG = 0;
constexpr size_t WS_FLAGS_BYTES = 32 * MREP * PROW;
constexpr size_t WS_CTR = WS_FLAGS_BYTES;
constexpr size_t WS_TMO = WS_CTR + (size_t)NWG * 128;
constexpr size_t WS_ORG = WS_TMO + 128;
constexpr size_t WS_AGG = WS_ORG + 128;
constexpr size_t WS_STAGE = 4u << 20;
constexpr size_t STEP_BYTES = (size_t)NWG * 1024;
constexpr size_t SENDER_BYTES = NSTEP * STEP_BYTES;
constexpr size_t PARITY_BYTES = NR * SENDER_BYTES;
constexpr size_t WS_NEED = WS_STAGE + 2 * PARITY_BYTES;
static_assert(WS_AGG + (NSTEP + 2) * 16 * 128 <= WS_STAGE, "control region");
constexpr int L_RA = 0;
constexpr int L_RB = NSA * 16384;
constexpr int L_WST = L_RB + NSB * 16384;
constexpr int L_CTL = L_WST + NKS * 1024;
constexpr int L_PUB = L_CTL;
constexpr int L_REL = L_CTL + 64;
constexpr int L_QCNT = L_CTL + 128;
constexpr int L_ABORT = L_CTL + 132;
constexpr int L_CALL = L_CTL + 136;
constexpr int L_WRDY = L_CTL + 140;
constexpr int L_LOC = L_CTL + 144;
constexpr int L_GO = L_CTL + 148;
constexpr int L_PREW = L_CTL + 152;
static_assert(L_PREW == L_ABORT + 20 && (PS1 - PS0) * PREW <= 256, "pre-write");
constexpr int L_QUEUE = L_CTL + 256;
constexpr int L_LDQ = L_QUEUE + 128 * 4;
constexpr int L_DUMMY = L_LDQ + 160 * 4;
constexpr int LDS_BYTES = L_DUMMY + 320;
static_assert(LDS_BYTES <= 163840 && NITEM < 128, "LDS");
constexpr unsigned long long DEADLINE_TICKS = 50000000ull;
constexpr int PDEPTH = PDEPTH_;
constexpr int NST = NSTEP * 7;

struct Params {
    const signed char* x; const signed char* w; const float* sx; const float* sw;
    float* out; char* ws[NR]; int rank;
};

#define STAMP0(idx) do { } while (0)
#define STAMP1(idx) do { } while (0)
#define STAMP_OUT() do { } while (0)

constexpr unsigned step_len16(int s) { return 7u * (s < PH2 ? TAU16 : TAU2); }
constexpr unsigned step_t(int s) { unsigned t = 0; for (int i = 0; i < s; ++i) t += step_len16(i); return t >> 4; }
constexpr int sched_t(int n) { return (n <= PH2 ? n * TAU16 : PH2 * TAU16 + (n - PH2) * TAU2) >> 4; }
constexpr bool agg_step(int s) { return s < NSTEP - NFINE; }
constexpr int npost(int n) {
    const int o = n - PDEPTH;
    if (o < 0 || o >= NST) return 0;
    return agg_step(o / 7) ? (o % 7 == 6 ? 1 : 0) : 1;
}
constexpr int nyoung(int n) {
    int c = 0;
    for (int i = n - PDEPTH + 1; i <= n; ++i) c += i < NST;
    for (int i = n - PDEPTH; i < n; ++i) c += npost(i);
    return c;
}
template <class F, int... I> __device__ __forceinline__ void static_for_impl(F&& f, std::integer_sequence<int, I...>) { (f(std::integral_constant<int, I>{}), ...); }
template <int N, class F> __device__ __forceinline__ void static_for(F&& f) { static_for_impl(f, std::make_integer_sequence<int, N>{}); }
template <int N> __device__ __forceinline__ void wait_vm() { asm volatile("s_waitcnt vmcnt(%0)" :: "n"(N) : "memory"); }
__device__ __forceinline__ void wait_vm_x4(int n4) {
    switch (n4) { case 0: wait_vm<0>(); break; case 1: wait_vm<4>(); break; case 2: wait_vm<8>(); break; case 3: wait_vm<12>(); break;
                  case 4: wait_vm<16>(); break; case 5: wait_vm<20>(); break; case 6: wait_vm<24>(); break; case 7: wait_vm<28>(); break;
                  default: wait_vm<32>(); break; }
}
__device__ __forceinline__ unsigned vm_outstanding() {
    const unsigned r = __builtin_amdgcn_s_getreg(7 | (31 << 11));
    return (r & 15u) | ((r >> 18) & 48u);
}
__device__ __forceinline__ void wait_lgkm0() { asm volatile("s_waitcnt lgkmcnt(0)" ::: "memory"); }
__device__ __forceinline__ unsigned long long rt() { return __builtin_amdgcn_s_memrealtime(); }
__device__ __forceinline__ void st16_sys(const char* base, unsigned off, u32x4 v) {
    asm volatile("global_store_dwordx4 %0, %1, %2 sc0 sc1\n\ts_nop 1" :: "v"(off), "v"(v), "s"(base) : "memory");
}
__device__ __forceinline__ void st4_sys(const char* base, unsigned off, unsigned v) {
    asm volatile("global_store_dword %0, %1, %2 sc0 sc1\n\ts_nop 1" :: "v"(off), "v"(v), "s"(base) : "memory");
}
__device__ __forceinline__ void ld16_async(u32x4& v, const GAS u32x4* p) { asm volatile("global_load_dwordx4 %0, %1, off" : "=v"(v) : "v"(p) : "memory"); }
__device__ __forceinline__ void ld4_wait(unsigned& v, const char* p) { asm volatile("global_load_dword %0, %1, off\n\ts_waitcnt vmcnt(0)" : "=v"(v) : "v"(p) : "memory"); }
__device__ __forceinline__ void stouch(const char* p) { unsigned d; asm volatile("s_load_dword %0, %1, 0x0 glc\n\ts_waitcnt lgkmcnt(0)" : "=s"(d) : "s"(p) : "memory"); }
typedef unsigned u32x16 __attribute__((ext_vector_type(16)));
constexpr unsigned FWD_SPINS = 1u << 21;
__device__ __forceinline__ unsigned piece_min(const char* p) {
    u32x16 f;
    asm volatile("s_load_dwordx16 %0, %1, 0x0 glc\n\ts_waitcnt lgkmcnt(0)" : "=s"(f) : "s"(p) : "memory");
    unsigned m = f[0];
#pragma unroll
    for (int i = 1; i < 15; ++i) m = f[i] < m ? f[i] : m;
    return m;
}
__device__ __forceinline__ unsigned long long sld2_glc(const char* p) { unsigned long long d; asm volatile("s_load_dwordx2 %0, %1, 0x0 glc\n\ts_waitcnt lgkmcnt(0)" : "=s"(d) : "s"(p) : "memory"); return d; }
template <class T> __device__ __forceinline__ void pin(T& v) { asm volatile("" : "+v"(v)); }
__device__ __forceinline__ void st4_sys_lane(const char* p, unsigned v) { asm volatile("global_store_dword %0, %1, off sc0 sc1\n\ts_nop 1" :: "v"(p), "v"(v) : "memory"); }
__device__ __forceinline__ void st16_sys_lane(const char* p, u32x4 v) { asm volatile("global_store_dwordx4 %0, %1, off sc0 sc1\n\ts_nop 1" :: "v"(p), "v"(v) : "memory"); }
__device__ __forceinline__ void ticket_async(unsigned& tick, const char* base, unsigned off) {
    unsigned long long keep;
    asm volatile("s_mov_b64 %1, exec\n\ts_mov_b64 exec, 1\n\tglobal_atomic_add %0, %2, %3, %4 sc0 sc1\n\ts_mov_b64 exec, %1"
                 : "=&v"(tick), "=&s"(keep) : "v"(off), "v"(1u), "s"(base) : "memory");
}
__device__ __forceinline__ void lds_st(unsigned addr, unsigned v) { asm volatile("ds_write_b32 %0, %1" :: "v"(addr), "v"(v) : "memory"); }
__device__ __forceinline__ void lds_add(unsigned addr, unsigned v) { asm volatile("ds_add_u32 %0, %1" :: "v"(addr), "v"(v) : "memory"); }
__device__ __forceinline__ unsigned lds_ldv(unsigned addr) {
    unsigned v; asm volatile("ds_read_b32 %0, %1\n\ts_waitcnt lgkmcnt(0)" : "=v"(v) : "v"(addr) : "memory");
    return v;
}
__device__ __forceinline__ unsigned lds_ld(unsigned addr) { return __builtin_amdgcn_readfirstlane(lds_ldv(addr)); }
__device__ __forceinline__ void dma4k(const char* base, unsigned off, unsigned lds_dst) {
    unsigned keep;
    asm volatile("s_mov_b32 %0, m0\n\ts_mov_b32 m0, %3\n\ts_nop 0\n\t"
                 "global_load_lds_dwordx4 %1, %2 sc0 sc1\n\t"
                 "global_load_lds_dwordx4 %1, %2 offset:1024 sc0 sc1\n\t"
                 "global_load_lds_dwordx4 %1, %2 offset:2048 sc0 sc1\n\t"
                 "global_load_lds_dwordx4 %1, %2 offset:3072 sc0 sc1\n\t"
                 "s_mov_b32 m0, %0"
                 : "=&s"(keep) : "v"(off), "s"(base), "s"(lds_dst) : "memory");
}
constexpr int NJOB = NR * (NFULL + 3);
__device__ __forceinline__ int job_type(int j) { return j < 9 ? (j < 6 ? 0 : j - 5) : (j < 51 ? 0 : (j - 44) / 7); }
__device__ __forceinline__ int job_item(int j) { return j < 9 ? (j < 6 ? j : 2 * j - 6) : (j < 51 ? j + 3 : 2 * j - 48); }
__device__ __forceinline__ int job_btile(int j) { return j < 9 ? j : (j < 65 ? j + 1 : 2 * j - 64); }
__device__ __forceinline__ int btile_job(int t) { return t < 9 ? t : (t < 66 ? t - 1 : (t + 64) >> 1); }
__device__ __forceinline__ unsigned rel_of(int j) { return j < 0 ? 0u : (unsigned)job_item(j + 1); }
__device__ __forceinline__ int frag_blk(int n, int c) { return n < NFULL ? c : (c & ~15) + (n - NFULL) * 8 + (c & 7); }
__device__ __forceinline__ int frag_ks(int n, int c) { return n < NFULL ? n : NFULL + ((c >> 3) & 1); }
__device__ __forceinline__ const GAS u32x4* x_src(const signed char* x, int n, int c, int lane) {
    return (const GAS u32x4*)(x + (size_t)(frag_blk(n, c) * 16 + (lane & 15)) * KSH + frag_ks(n, c) * 64 + (lane >> 4) * 16);
}
}

#define PREW_ON 1
#define STAMPS_ON 0
#define CONSUMER_ASM R"ASM(
.macro MM m, n, a
  v_mfma_i32_16x16x64_i8 v[16*(\m)+4*(\n):16*(\m)+4*(\n)+3], v[144+4*(\a):147+4*(\a)], v[128+4*(\n):131+4*(\n)], v[16*(\m)+4*(\n):16*(\m)+4*(\n)+3]
.endm
.macro MM4 m, a
  MM \m, 0, \a
  MM \m, 1, \a
  MM \m, 2, \a
  MM \m, 3, \a
.endm
.macro MC m, n, a
  MM \m, \n, \a
  MM \m+1, \n, \a+1
.endm
.macro LDA a, off
  ds_read_b128 v[144+4*(\a):147+4*(\a)], v161 offset:\off
.endm
.macro LDB n
  ds_read_b128 v[128+4*(\n):131+4*(\n)], v162 offset:1024*(\n)
.endm
.macro STAMPI
.if %12 == 1
  s_sub_u32 s48, s53, %7-64
  s_and_b32 s48, s48, 63
  s_memrealtime s[56:57]
  s_waitcnt lgkmcnt(0)
  s_mov_b32 m0, s48
  s_nop 0
  v_writelane_b32 v160, s56, m0
.endif
.endm
.macro STAMPK k, ins=s_memrealtime
.if %12
  \ins s[56:57]
  s_waitcnt lgkmcnt(0)
  v_mov_b32 v152, s56
  v_mov_b32 v163, 256+4*(\k)
  global_store_dword v163, v152, %6
  s_nop 1
.endif
.endm
.macro ADV_A
  s_add_u32 s41, s41, 1
  s_cmp_eq_u32 s41, %9
  s_cselect_b32 s44, s50, s52
  s_cselect_b32 s41, 0, s41
.endm
.macro ADV_B
  s_add_u32 s42, s42, 1
  s_cmp_eq_u32 s42, %10
  s_cselect_b32 s40, s51, s52
  s_cselect_b32 s42, 0, s42
.endm
// waits for the next item of this wave's half, passing over (and releasing) the items of the other half
.macro SPIN back
  s_mov_b32 s47, 0
1:
  s_cmp_ge_u32 s43, 0x400
  s_cbranch_scc0 2f
  ds_write2_b32 v165, v166, v166 offset0:0 offset1:8
  v_add_u32 v164, 4, v164
  v_add_u32 v166, 1, v166
  s_add_u32 s53, s53, 1
  s_mov_b32 s47, 0
  s_branch 3f
2:
  s_sleep %13
3:
  ds_read_b32 v163, v164 offset:256
  s_add_u32 s47, s47, 1
  s_waitcnt lgkmcnt(0)
  v_readfirstlane_b32 s46, v163
  s_and_b32 s43, s46, %11
  s_cmp_eq_u32 s43, 0x300
  s_cbranch_scc1 \back
  s_cmp_lt_u32 s47, 0x1000000
  s_cbranch_scc1 1b
  v_mov_b32 v163, %3
  v_mov_b32 v144, 1
  ds_write_b32 v163, v144 offset:132
  s_branch FIN_%=
.endm
.macro OUT_ST u, t, fl
  v_cvt_f32_i32 v[\t], v[16*((\u)/4)+((\u)&3)]
  v_cvt_f32_i32 v[\t+1], v[16*((\u)/4)+((\u)&3)+4]
  v_cvt_f32_i32 v[\t+2], v[16*((\u)/4)+((\u)&3)+8]
  v_cvt_f32_i32 v[\t+3], v[16*((\u)/4)+((\u)&3)+12]
  s_add_u32 s54, s58, (((\u)/4)*16+((\u)&3))*16384
  s_addc_u32 s55, s59, 0
  v_mul_f32 v[\t], s49, v[\t]
  v_mul_f32 v[\t+1], s49, v[\t+1]
  v_mul_f32 v[\t+2], s49, v[\t+2]
  v_mul_f32 v[\t+3], s49, v[\t+3]
.if \fl == 0
  global_store_dwordx4 v167, v[\t:\t+3], s[54:55]
.elseif \fl == 1
  global_store_dwordx4 v167, v[\t:\t+3], s[54:55] sc0 sc1
.else
  global_store_dwordx4 v167, v[\t:\t+3], s[54:55] nt
.endif
.endm

  v_mbcnt_lo_u32_b32 v160, -1, 0
  v_mbcnt_hi_u32_b32 v160, -1, v160
  v_lshlrev_b32 v160, 4, v160
  v_mov_b32 v167, v160
  v_mov_b32 v164, %3
  v_lshrrev_b32 v165, 2, v160
  v_add_u32 v165, %17, v165
  v_writelane_b32 v165, %2, 0
  v_mov_b32 v166, 1
  v_mov_b32 v160, 0
  s_mov_b64 s[58:59], %4
  s_mov_b32 s49, %5
  s_mov_b32 s41, 0
  s_mov_b32 s42, 0
  s_mov_b32 s43, 0
  s_mov_b32 s45, %8
  s_mov_b32 s50, -(%9-1)*16384
  s_mov_b32 s51, -(%10-1)*16384
  s_mov_b32 s52, 16384
  s_mov_b32 s53, 0
.set zi, 0
.rept 128
  v_mov_b32 v[zi], 0
.set zi, zi+1
.endr
  SPIN START_%=
SPINF_%=:
  s_cmp_eq_u32 s45, 0
  s_cbranch_scc1 TAIL_%=
  SPIN BACK_%=
START_%=:
  STAMPK 6, s_memtime
  STAMPK 8
  s_bfe_u32 s44, s46, 0x5000d
  s_bfe_u32 s40, s46, %18
  s_lshl1_add_u32 s44, s44, %0
  s_lshl1_add_u32 s40, s40, %1
  v_lshl_add_u32 v161, s44, 12, v167
  v_lshl_add_u32 v162, s40, 12, v167
  LDA 0, 0
  LDA 1, 1024
  LDB 0
  LDB 1
  LDB 2
  LDB 3
  ds_write_b32 v165, v166 offset:32
LOOP_%=:
  s_waitcnt lgkmcnt(4)
  MM 0, 0, 0
  LDA 2, 2048
  MM 1, 0, 1
  LDA 3, 3072
  s_waitcnt lgkmcnt(5)
  MM 0, 1, 0
  s_add_u32 s53, s53, 1
  MM 1, 1, 1
  s_waitcnt lgkmcnt(4)
  MM 0, 2, 0
  s_sub_u32 s45, s45, 1
  MM 1, 2, 1
  s_waitcnt lgkmcnt(3)
  MM 0, 3, 0
  v_add_u32 v164, 4, v164
  MM 1, 3, 1
  s_waitcnt lgkmcnt(1)
  MM 2, 0, 2
  LDA 0, 4096
  MM 2, 1, 2
  LDA 1, 5120
  MM 2, 2, 2
  MM 2, 3, 2
  s_waitcnt lgkmcnt(2)
  MM 3, 0, 3
  LDA 2, 6144
  MM 3, 1, 3
  ds_read_b32 v163, v164 offset:256
  MM 3, 2, 3
  MM 3, 3, 3
  s_waitcnt lgkmcnt(3)
  MM 4, 0, 0
  LDA 3, 7168
  MM 4, 1, 0
  ds_write_b32 v165, v166
  MM 4, 2, 0
  MM 4, 3, 0
  s_waitcnt lgkmcnt(2)
  v_readfirstlane_b32 s46, v163
  MM 5, 0, 1
  v_add_u32 v166, 1, v166
  s_bfe_u32 s44, s46, 0x5000d
  MM 5, 1, 1
  s_bfe_u32 s40, s46, %18
  s_lshl1_add_u32 s44, s44, %0
  MM 5, 2, 1
  s_lshl1_add_u32 s40, s40, %1
  s_and_b32 s43, s46, %11
  MM 5, 3, 1
  s_cmp_eq_u32 s43, 0x300
  s_cbranch_scc0 SPINF_%=
GO_%=:
  v_lshl_add_u32 v161, s44, 12, v167
  s_waitcnt lgkmcnt(1)
  MM 6, 0, 2
  LDA 0, 0
  v_lshl_add_u32 v162, s40, 12, v167
  MM 7, 0, 3
  LDA 1, 1024
  MM 6, 1, 2
  LDB 0
  MM 7, 1, 3
  LDB 1
  MM 6, 2, 2
  MM 7, 2, 3
  LDB 2
  MM 6, 3, 2
  MM 7, 3, 3
  LDB 3
  ds_write_b32 v165, v166 offset:32
  STAMPI
.if %12 == 2
  s_cmp_eq_u32 s45, 12
  s_cbranch_scc0 5f
  STAMPK 9
  STAMPK 10, s_memtime
5:
.endif
  s_branch LOOP_%=
BACK_%=:
  s_bfe_u32 s44, s46, 0x5000d
  s_bfe_u32 s40, s46, %18
  s_lshl1_add_u32 s44, s44, %0
  s_lshl1_add_u32 s40, s40, %1
  s_branch GO_%=
TAIL_%=:
  s_waitcnt lgkmcnt(0)
  MM4 6, 2
  MM4 7, 3
FIN_%=:
  v_and_b32 v161, 0xf0, v167
  v_lshrrev_b32 v167, 8, v167
  v_lshl_add_u32 v167, v167, 16, v161
  v_mov_b32 v166, 0x7fffffff
  v_mov_b32 v163, %3
  ds_write2_b32 v165, v166, v166 offset0:0 offset1:8
  ds_read_b32 v166, v163 offset:132
  STAMPK 2
  STAMPK 7, s_memtime
  s_waitcnt lgkmcnt(0)
  v_readfirstlane_b32 s48, v166
  s_cmp_eq_u32 s48, 0
  s_cselect_b32 s49, s49, 0x7fc00000
.if %19
  v_mov_b32 v163, %3
  s_mov_b32 s47, 0
8:
  ds_read_b32 v166, v163 offset:152
  s_add_u32 s47, s47, 1
  s_waitcnt lgkmcnt(0)
  v_readfirstlane_b32 s48, v166
  s_cmp_lg_u32 s48, 0
  s_cbranch_scc1 9f
  s_sleep 2
  s_cmp_lt_u32 s47, 0x1000000
  s_cbranch_scc1 8b
  s_mov_b32 s49, 0x7fc00000
9:
.endif
.if %16 == 9
  s_cmp_lt_u32 %11, 0x1000
  s_cselect_b64 vcc, -1, 0
.else
  s_cmp_lt_u32 %11, 0x1000
  s_cbranch_scc1 EARLY_%=
.set du, 0
.rept 32
  OUT_ST du, 128+4*(du&7), %16
.set du, du+1
.endr
  s_branch END_%=
EARLY_%=:
  s_mov_b64 vcc, -1
.endif
  s_setprio 0
  v_mov_b32 v163, %3
.if %14 == 62
6:
  s_sleep 4
  ds_read_b32 v166, v163 offset:128
  s_waitcnt lgkmcnt(0)
  v_cmp_eq_u32 vcc, 0, v166
  s_cbranch_vccnz 6b
.endif
.set du, 0
.rept 32
  OUT_ST du, 128+4*(du&7), %15
  s_cbranch_vccz 4f
  ds_read_b32 v166, v163 offset:128
  s_waitcnt vmcnt(%14) lgkmcnt(0)
  v_cmp_eq_u32 vcc, 0, v166
4:
.set du, du+1
.endr
END_%=:
.if %12
  STAMPK 3
  s_waitcnt vmcnt(0)
  STAMPK 4
  v_mbcnt_lo_u32_b32 v166, -1, 0
  v_mbcnt_hi_u32_b32 v166, -1, v166
  v_lshlrev_b32 v166, 2, v166
  global_store_dword v166, v160, %6
.endif
.purgem MM
.purgem MM4
.purgem MC
.purgem LDA
.purgem LDB
.purgem STAMPI
.purgem STAMPK
.purgem SPIN
.purgem ADV_A
.purgem ADV_B
.purgem OUT_ST
)ASM"

#define LOADER_ASM R"ASM(
.macro LSTAMP vr, idx
.if %7 == 1
  s_memrealtime s[56:57]
  s_sub_u32 s58, \idx, %6-64
  s_and_b32 s58, s58, 63
  s_mov_b32 s59, m0
  s_mov_b32 m0, s58
  s_waitcnt lgkmcnt(0)
  v_writelane_b32 \vr, s56, m0
  s_mov_b32 m0, s59
.endif
.endm
.macro PUBCHK
  s_getreg_b32 s47, hwreg(HW_REG_IB_STS)
  s_and_b32 s48, s47, 15
  s_bfe_u32 s47, s47, 0x20016
  s_lshl4_add_u32 s47, s47, s48
  s_sub_u32 s47, s40, s47
  s_cmp_ge_u32 s47, s41
  s_cbranch_scc0 7f
  s_bfe_u32 s47, s36, 0x70016
  s_bfe_u32 s48, s36, 0x1001d
  v_lshl_add_u32 v105, s47, 2, %9
  s_lshl1_add_u32 exec_lo, s48, 1
  s_mov_b32 exec_hi, 0
  ds_add_u32 v105, v109
  s_mov_b64 exec, -1
  LSTAMP v108, s44
  s_add_u32 s44, s44, 1
  s_mov_b32 s41, s42
  s_mov_b32 s42, s43
  s_mov_b32 s43, s49
  s_mov_b32 s36, s37
  s_mov_b32 s37, s38
7:
.endm
.macro DMA4
  global_load_lds_dwordx4 v104, %2 sc0 sc1
  global_load_lds_dwordx4 v104, %2 offset:1024 sc0 sc1
  global_load_lds_dwordx4 v104, %2 offset:2048 sc0 sc1
  global_load_lds_dwordx4 v104, %2 offset:3072 sc0 sc1
.endm
.macro NEXT4
  s_add_u32 m0, m0, 4096
  v_add_u32 v104, 4096, v104
.endm
.macro READY
  v_readlane_b32 s51, v102, 16
  s_bfe_u32 s47, s51, 0x7000f
  s_lshr_b32 s48, s51, 28
  v_cmp_le_i32 vcc, s47, v102
  s_and_b32 s52, s48, 8
  s_bfe_u32 s48, s51, 0x1001e
  s_lshr_b32 s47, vcc_lo, s52
  s_and_b32 s47, s47, 0xff
  s_add_u32 s47, s47, s48
  s_cmp_eq_u32 s47, 0x100
.endm
  s_mov_b32 s40, 0
  s_mov_b32 s49, 0x7fffffff
  s_mov_b32 s41, s49
  s_mov_b32 s42, s49
  s_mov_b32 s43, s49
  s_mov_b32 s44, 0
  s_mov_b32 s45, 0
  s_mov_b32 s50, 0
  v_mov_b32 v101, %0
  v_mov_b32 v111, 0
  v_writelane_b32 v111, 8, 16
  v_mov_b32 v109, 0x100
  v_mov_b32 v107, 0
  v_mov_b32 v108, 0
  s_waitcnt vmcnt(0)
  ds_read_b32 v102, v101
LLOOP_%=:
  PUBCHK
  s_waitcnt lgkmcnt(0)
  READY
  s_cselect_b32 s47, s43, 0
  s_cmp_eq_u32 s47, s49
  s_cbranch_scc0 LIDLE_%=
LGO_%=:
  s_and_b32 s52, s51, 0x7ff
  s_bfe_u32 s48, s51, 0x4000b
  v_lshl_add_u32 v104, s52, 14, %1
  s_lshl_b32 s48, s48, 14
  s_add_u32 m0, s48, %3
  v_add_u32 v101, v111, v101
  DMA4
  NEXT4
  DMA4
  NEXT4
  s_add_u32 s40, s40, 8
  PUBCHK
  DMA4
  NEXT4
  ds_read_b32 v102, v101
  DMA4
  s_add_u32 s40, s40, 8
  LSTAMP v107, s45
  s_cmp_eq_u32 s42, s49
  s_cselect_b32 s43, s49, s40
  s_cselect_b32 s47, s40, s42
  s_cselect_b32 s48, s51, s37
  s_mov_b32 s38, s51
  s_cmp_eq_u32 s41, s49
  s_cselect_b32 s42, s49, s47
  s_cselect_b32 s41, s40, s41
  s_cselect_b32 s36, s51, s36
  s_mov_b32 s37, s48
  s_add_u32 s45, s45, 1
  s_mov_b32 s50, 0
  s_cmp_lt_u32 s45, %6
  s_cbranch_scc1 LLOOP_%=
LDRAIN_%=:
  PUBCHK
  s_cmp_lt_u32 s44, %6
  s_cbranch_scc1 LDRAIN_%=
  s_branch LEND_%=
LIDLE_%=:
  ds_read_b32 v102, v101
  s_cmp_eq_u32 s41, s49
  s_cbranch_scc0 LLOOP_%=
LWAIT_%=:
  s_add_u32 s50, s50, 1
  s_waitcnt lgkmcnt(0)
  READY
  s_cbranch_scc1 LGO_%=
  s_sleep %8
  ds_read_b32 v102, v101
  s_cmp_lt_u32 s50, 0x1000000
  s_cbranch_scc1 LWAIT_%=
  v_mov_b32 v106, 1
  v_mov_b32 v101, %4
  ds_write_b32 v101, v106 offset:132
LEND_%=:
  v_mov_b32 v106, 1
  v_mov_b32 v101, %4
  ds_write_b32 v101, v106 offset:128
  s_waitcnt vmcnt(0) lgkmcnt(0)
.if %7
  v_mbcnt_lo_u32_b32 v106, -1, 0
  v_mbcnt_hi_u32_b32 v106, -1, v106
  v_lshlrev_b32 v106, 2, v106
  global_store_dword v106, v107, %5
  global_store_dword v106, v108, %5 offset:256
.endif
.purgem LSTAMP
.purgem PUBCHK
.purgem DMA4
.purgem NEXT4
.purgem READY
)ASM"
#define ENDGAME_ASM R"ASM(
.macro PHASE row, thr, kind
  s_add_u32 s40, %4, \thr
  s_mov_b32 s41, %5
1:
  s_mov_b32 s44, 0
2:
  s_load_dwordx8 s[48:55], \row, 0x0 glc
  s_mov_b32 s42, 0
  s_add_u32 s44, s44, 1
  s_waitcnt lgkmcnt(0)
.set gi, 55
.rept 8
  s_cmp_gt_u32 s[gi], s40
  s_addc_u32 s42, s42, 0
.if gi > 48
  s_lshl_b32 s42, s42, 2
.endif
.set gi, gi-1
.endr
  s_andn2_b32 s42, s42, s41
  s_cbranch_scc1 3f
  s_cmp_lt_u32 s44, 0x200000
  s_cbranch_scc1 2b
  s_mov_b32 %0, 1
  s_branch EEND_%=
3:
.if \kind == 2
  s_mov_b32 exec_lo, s42
  s_mov_b32 exec_hi, 0
  ds_add_u32 %8, v101 offset:8
.else
  s_lshl_b32 s43, s42, 1
  v_mbcnt_lo_u32_b32 v100, s43, 0
  v_add_u32 v100, s45, v100
  v_lshl_add_u32 v100, v100, 3, %7
  s_or_b32 exec_lo, s42, s43
  s_mov_b32 exec_hi, 0
  ds_add_u32 v100, v101
.endif
  s_mov_b64 exec, -1
.if %10
  s_memrealtime s[46:47]
  s_lshl1_add_u32 s43, s45, 22
  s_lshl_b64 s[56:57], -1, s43
  s_waitcnt lgkmcnt(0)
  s_mov_b64 exec, s[56:57]
  v_mov_b32 %1, s46
  s_mov_b64 exec, -1
.endif
  s_bcnt1_i32_b32 s43, s42
  s_add_u32 s45, s45, s43
  s_or_b32 s41, s41, s42
  s_cmp_lg_u32 s41, 0x5555
  s_cbranch_scc1 1b
.endm
  s_mov_b32 s45, 0
  v_mov_b32 v101, %6
  PHASE %3, 13, 0
  v_add_u32 v101, %9, v101
  PHASE %3, 14, 1
  s_andn2_b32 s43, 0x5555, %5
  s_lshl_b32 exec_lo, s43, 1
  s_mov_b32 exec_hi, 0
  ds_add_u32 %8, v101
  ds_add_u32 %8, %6 offset:4
  s_mov_b64 exec, -1
  v_add_u32 v101, 48, v101
  PHASE %2, 15, 2
EEND_%=:
.purgem PHASE
)ASM"
#define CLOBBER_S "s40", "s41", "s42", "s43", "s44", "s45", "s46", "s47", "s48", "s49", "s50", "s51", "s52", "s53", "s54", "s55", "s56", "s57", "s58", "s59", "s60", "s61", "s62", "s63"
#define V10(t) "v" #t "0", "v" #t "1", "v" #t "2", "v" #t "3", "v" #t "4", "v" #t "5", "v" #t "6", "v" #t "7", "v" #t "8", "v" #t "9"
#define CLOBBER_V V10(), V10(1), V10(2), V10(3), V10(4), V10(5), V10(6), V10(7), V10(8), V10(9), V10(10), V10(11), V10(12), V10(13), V10(14), V10(15), \
    "v160", "v161", "v162", "v163", "v164", "v165", "v166", "v167"

__global__ void __launch_bounds__(768, 3) gemm_ar_fused(Params P) {
    __shared__ __attribute__((aligned(4096))) unsigned char lds_raw[LDS_BYTES];
    LAS unsigned char* lds = (LAS unsigned char*)lds_raw;
    const unsigned lbase = (unsigned)(size_t)lds;
    const int tid = threadIdx.x, lane = tid & 63;
    const int wave = __builtin_amdgcn_readfirstlane(tid >> 6);
    const int b = blockIdx.x;
    const int rank = P.rank;
    const int c = CMAP == 2 ? ((b >> 4) << 4) | ((b & 7) << 1) | ((b >> 3) & 1) : ((b >> 5) << 5) | ((b & 7) << 2) | ((b >> 3) & 3);
    char* myws = P.ws[0];
    const int ti = ((b & 7) >> 1) * 4 + (b >> 6), tj = (b & 1) * 8 + ((b >> 3) & 7);
    if (wave < 9) STAMP1(0);
    GAS unsigned* tmo = (GAS unsigned*)(myws + WS_TMO);

    if (wave < 8) {
        unsigned warm = 0;
        if (TLBWARM && wave == 0) {
            const unsigned ph = (unsigned)((c + PROT * rank) & 255), kk = CMAP == 2 ? ((ph >> 4) - 8u) * 2u + (ph & 1u) : ((ph >> 5) - 4u) * 4u + (ph & 3u);
            if (ph >= 128u && kk < 14u) {
                const char* pw = P.ws[1 + kk % 7u] + WS_STAGE + (kk / 7u) * PARITY_BYTES + (size_t)rank * SENDER_BYTES;
                asm volatile("s_load_dword %0, %1, 0x0 glc" : "=s"(warm) : "s"(pw) : "memory");
            }
        }
        __builtin_amdgcn_s_barrier();
        const unsigned call = lds_ld(lbase + L_CALL);
        asm volatile("" :: "s"(warm));
        if (((c + PROT * rank) & 255) < GOPH) { unsigned sp = 0; while (lds_ld(lbase + L_GO) == 0u && ++sp < (1u << 22)) __builtin_amdgcn_s_sleep(2); }
        {
            const int ks = wave;
            const int cc = lane & 15, g = lane >> 4;
            const GAS unsigned* wp = (const GAS unsigned*)(P.w + (size_t)(frag_ks(ks, c) * 64 + g * 16) * DIM + (frag_blk(ks, c) >> 2) * 64 + cc * 4);
            unsigned d[16];
#pragma unroll
            for (int t = 0; t < 16; ++t) d[t] = wp[(size_t)t * (DIM / 4)];
            const int sx = (nth_tab(0) >> (4 * wave)) & 15, sw = (nth_tab(1) >> (4 * wave)) & 15;
            const u32x4 xv = *x_src(P.x, ks, c, lane);
            const unsigned q = c & 3;
            const unsigned sel_lo = 0x0c0c0000u | ((4u + q) << 8) | q;
            const unsigned sel_hi = 0x00000c0cu | ((4u + q) << 24) | (q << 16);
            u32x4 wv;
#pragma unroll
            for (int e = 0; e < 4; ++e)
                wv[e] = __builtin_amdgcn_perm(d[4 * e + 1], d[4 * e], sel_lo) | __builtin_amdgcn_perm(d[4 * e + 3], d[4 * e + 2], sel_hi);
            *(LAS u32x4*)(lds + L_WST + ks * 1024 + lane * 16) = wv;
            asm volatile("" ::: "memory");
            if (lane == 0) __hip_atomic_fetch_add((LAS unsigned*)(lds + L_WRDY), 1u, __ATOMIC_RELAXED, __HIP_MEMORY_SCOPE_WORKGROUP);
            const unsigned soff = (unsigned)(WS_STAGE + (call & 1u) * PARITY_BYTES + (size_t)rank * SENDER_BYTES) + c * 1024 + lane * 16;
            st16_sys(myws, soff + sx * (unsigned)STEP_BYTES, xv);
            st16_sys(myws, soff + sw * (unsigned)STEP_BYTES, wv);
            STAMP1(1);
            wait_vm<0>();
            if (lane == 0) {
                if (__hip_atomic_fetch_add((LAS unsigned*)(lds + L_LOC), 1u, __ATOMIC_RELAXED, __HIP_MEMORY_SCOPE_WORKGROUP) == 7u) {
                    GAS unsigned* tk = (GAS unsigned*)(myws + WS_AGG + ((NSTEP + 1) * 16 + (c >> 4)) * 128);
                    if ((__hip_atomic_fetch_add(tk, 1u, __ATOMIC_RELAXED, __HIP_MEMORY_SCOPE_AGENT) & 15u) == 15u)
                        for (int k = 0; k < 2 * MREP; ++k)
                            __hip_atomic_store((GAS unsigned*)(myws + WS_PROG + (k * 16 + (c >> 4)) * PROW + rank * 4), call * 32u + NSTEP, __ATOMIC_RELAXED, __HIP_MEMORY_SCOPE_SYSTEM);
                }
            }
        }
        const int wr = wave >> 2, wc = (wave + WROT) & 3;
        const unsigned scale = __builtin_amdgcn_readfirstlane(__float_as_uint(P.sx[0] * P.sw[0]));
        const char* obase = (const char*)(P.out + (size_t)(ti * 256 + wr * 128) * DIM + tj * 256 + wc * 64);
        char* dbg = myws;
        if (CPRIO_HI) __builtin_amdgcn_s_setprio(CPRIO_HI);
        asm volatile(CONSUMER_ASM
            :: "s"((lbase + L_RA) / 4096 + wr * 2 - 2), "s"((lbase + L_RA) / 4096 + (wc & 1)), "s"(lbase + L_REL + wave * 4), "s"(lbase + L_CTL),
               "s"(obase), "s"(scale), "s"(dbg),
               "n"(NITEM), "n"(NUP), "n"(NSA), "n"(NSB), "s"(0x300u | (0x400u << (wr ? 2 : wc >> 1))), "n"(STAMPS_ON), "n"(CSLEEP), "n"(EPW), "n"(DRIPFL), "n"(FINFL), "s"(lbase + L_DUMMY), "s"(0x50012u + (wc >> 1) * 5), "n"(PREW_ON)
            : "memory", "scc", "vcc", CLOBBER_S, CLOBBER_V);
        return;
    } else if (wave == 8) {
        unsigned tp0 = (unsigned)rt() + T0TICKS;
        __builtin_amdgcn_s_setprio(3);
        GAS unsigned* ctr = (GAS unsigned*)(myws + WS_CTR + (size_t)b * 128);
        unsigned callv; u32x4 xv[8];
        asm volatile("global_load_dword %0, %1, off sc1" : "=v"(callv) : "v"(ctr) : "memory");
        static_for<8>([&](auto nc) __attribute__((always_inline)) {
            constexpr int n = decltype(nc)::value;
            ld16_async(xv[n], x_src(P.x, n, c, lane));
        });
        const char* peer[7];
#pragma unroll
        for (int r = 0; r < 7; ++r) { int o = PEERROT ? r + (int)((unsigned)(b >> 3) % 7u) : r; o = o >= 7 ? o - 7 : o; peer[r] = P.ws[1 + o]; }
        const char* fl_lane = peer[0];
#pragma unroll
        for (int r = 1; r < 7; ++r) fl_lane = (lane & 7) == r ? peer[r] : fl_lane;
        fl_lane += WS_PROG + ((c >> 4) + (lane >> 3) * 32) * PROW + rank * 4;
        const unsigned phase = (unsigned)(((CMAP == 0 ? c : CMAP == 1 ? (c & ~15) | ((c & 3) << 2) | ((c >> 2) & 3) : b) + PROT * rank) & 255);
        asm volatile("s_waitcnt vmcnt(7)" : "+v"(callv), "+v"(xv[0]) :: "memory");
        const unsigned call = __builtin_amdgcn_readfirstlane(callv) + 1u;
        const unsigned doff = (unsigned)(WS_STAGE + (call & 1u) * PARITY_BYTES + (size_t)rank * SENDER_BYTES) + c * 1024 + lane * 16;
        const unsigned aoff = (unsigned)(WS_AGG + (c >> 4) * 128);
        const size_t sbase = WS_STAGE + (call & 1u) * PARITY_BYTES + (size_t)rank * SENDER_BYTES;
        unsigned tack = 0, fwd_dead = 0;
        const int late0 = (int)((unsigned)rt() - tp0);
        STAMP1(1);
        lds_st(lbase + L_CALL, call); wait_lgkm0();
        const bool latebar = LATEBAR && phase < GOPH;
        if (!latebar) __builtin_amdgcn_s_barrier();
        static_for<NSTEP>([&](auto sc) __attribute__((always_inline)) {
            constexpr int s = decltype(sc)::value;
            u32x4 v;
            if constexpr (s == 1) {
                pin(xv[1]); pin(xv[2]); pin(xv[3]); pin(xv[4]); pin(xv[5]); pin(xv[6]); pin(xv[7]);
                unsigned sp = 0; while (lds_ld(lbase + L_WRDY) < 8u && ++sp < (1u << 24)) __builtin_amdgcn_s_sleep(1);
            }
            if constexpr (step_desc(s).w) { v = *(const LAS u32x4*)(lds + L_WST + op_index(s) * 1024 + lane * 16); pin(v); }
            else v = xv[op_index(s)];
            {
                unsigned tgo = tp0 + step_t(s) + (phase * step_len16(s) >> 12);
                if constexpr (ADAPT0 && s == 0) { if (late0 > 0) tgo += (unsigned)late0; }
                if constexpr (L0T > 0 && s > 0) { const int late = (int)(tack + (step_len16(s) >> 4) - L0T - tgo); if (late > 0) tgo += (unsigned)(late * (8 - ALPHA8) >> 3); }
                int rem;
                while ((rem = (int)(tgo - (unsigned)rt())) > 0) { if (rem > 40) __builtin_amdgcn_s_sleep(8); else __builtin_amdgcn_s_sleep(1); }
            }
            STAMP0(s);
#pragma unroll
            for (int r = 0; r < 7; ++r) st16_sys(peer[(r + s) % 7], doff + (unsigned)(s * STEP_BYTES), v);
            if constexpr (s == 0) { if (latebar) __builtin_amdgcn_s_barrier(); }
            wait_vm<0>();
            STAMP0(16 + s);
            if constexpr (L0T > 0) tack = (unsigned)rt();
            if constexpr (s == 0) lds_st(lbase + L_GO, 1u);
            if constexpr (s >= TK0 && s < TICKET) {
                unsigned tick;
                ticket_async(tick, myws, aoff + (unsigned)((s + 1) * 16 * 128));
                wait_vm<0>();
                pin(tick);
                if ((__builtin_amdgcn_readfirstlane(tick) & 15u) == 15u) {
                    if ((lane & 7) < 7 && lane < 8 * MREP) st4_sys_lane(fl_lane + (step_desc(s).w ? 16 * PROW : 0), call * 32u + s + 1);
                }
            } else if ((c & 15) != 15) {
                if (lane == 0) st4_sys(myws, aoff + (c & 15) * 4, call * 32u + s + 1);
            } else {
                unsigned sp = fwd_dead;
                while (sp < FWD_SPINS && piece_min(myws + aoff) < call * 32u + s + 1) ++sp;
                if (sp >= FWD_SPINS) fwd_dead = FWD_SPINS;
                else { if ((lane & 7) < 7 && lane < 8 * MREP) st4_sys_lane(fl_lane + (step_desc(s).w ? 16 * PROW : 0), call * 32u + s + 1); }
            }
            if constexpr (ORIGIN && s == 0) {
                if (phase == 0u) {
                    tp0 = tack - LAT0;
                    if (lane == 0) __hip_atomic_store((GAS unsigned long long*)(myws + WS_ORG), (unsigned long long)tp0 << 32 | call, __ATOMIC_RELAXED, __HIP_MEMORY_SCOPE_AGENT);
                } else {
                    for (int sp = 0; sp < 64; ++sp) {
                        const unsigned long long o = sld2_glc(myws + WS_ORG);
                        if ((unsigned)o == call) { tp0 = (unsigned)(o >> 32); break; }
                        __builtin_amdgcn_s_sleep(4);
                    }
                }
            }
            if constexpr (PREW > 0 && s >= PS0 && s < PS1) {
                const char* ob = (const char*)(P.out + (size_t)(ti * 256 + 255 - (s - PS0) * PREW) * DIM + tj * 256);
                const u32x4 z = {0u, 0u, 0u, 0u};
#pragma unroll
                for (int k = 0; k < PREW; ++k) { st16_sys(ob - (size_t)k * DIM * 4, lane * 16, z); if (PGAP) __builtin_amdgcn_s_sleep(PGAP); }
                if constexpr (s == PS1 - 1) { wait_vm<0>(); lds_st(lbase + L_ABORT + 4 * 5, 1u); }
            }
            STAMP0(32 + s);
            if constexpr (TOUCH && s + 2 < NSTEP) {
                if (phase < 64u) stouch(peer[0] + sbase + (size_t)(s + 3) * STEP_BYTES - 64);
                else if (phase < 128u) stouch(myws + WS_STAGE + (call & 1u) * PARITY_BYTES + (size_t)((b >> 3) & 7) * SENDER_BYTES + (size_t)(s + 3) * STEP_BYTES - 64);
            }
            if constexpr (TOUCH && s >= 8 && s < 12) stouch((const char*)(P.out + (size_t)(ti * 256 + (s - 8) * 85) * DIM + tj * 256));
        });
        STAMP1(2);
        if (lane == 0) __hip_atomic_store(ctr, call, __ATOMIC_RELAXED, __HIP_MEMORY_SCOPE_AGENT);
        wait_vm<0>();
        STAMP1(3);
    } else if (wave == 9) {
        for (int u = lane; u < (L_DUMMY - L_CTL) / 4; u += 64) if (u != (L_CALL - L_CTL) / 4) lds_st(lbase + L_CTL + u * 4, 0u);
        wait_lgkm0();
        __builtin_amdgcn_s_barrier();
        const unsigned call = lds_ld(lbase + L_CALL);
        __builtin_amdgcn_s_setprio(PPRIO);
        const int p = lane >> 1, sub = lane & 1;
        const char* px = myws + WS_PROG + (ti + (tj & (MREP - 1)) * 32) * PROW;
        const char* pw = myws + WS_PROG + (16 + tj + (ti & (MREP - 1)) * 32) * PROW;
        for (int j = lane; j < NJOB; j += 64) {
            const unsigned ty = job_type(j), q = job_item(j), tb = job_btile(j), sA = j % NSA, s1 = NSA + tb % NSB, sn = NSA + (tb + 1) % NSB, s2 = ty == 3u ? sn : s1;
            const unsigned w0 = (ty == 3u ? 0u : 0x100u) | (ty ? 0x1c00u ^ (0x200u << ty) : 0u) | (2 * sA + (ty != 3u)) << 13 | (2 * s2) << 18 | (2 * s1 + (ty == 0u)) << 23;
            lds_st(lbase + L_QUEUE + q * 4, w0);
            if (ty) lds_st(lbase + L_QUEUE + q * 4 + 4, w0 + (1u << 13 | 1u << 18 | 1u << 23));
            const unsigned tail = (ty ? 0x20000000u : 0u) | q << 22, ta = lbase + L_LDQ + (j + tb) * 4;
            lds_st(ta + (ty == 3u ? 8 : 0), tail | rel_of(j - NSA) << 15 | sA << 11 | ti);
            lds_st(ta + (ty == 3u ? 0 : 4), 0x80000000u | tail | ((int)tb < NSB ? 0u : rel_of(btile_job(tb - NSB))) << 15 | s1 << 11 | tj);
            if (ty == 3u) lds_st(ta + 4, 0x80000000u | tail | rel_of(btile_job(tb + 1 - NSB)) << 15 | sn << 11 | tj);
        }
        unsigned long long tab = sub ? step_tab(1) : step_tab(0);
        unsigned ent = 0;
        const unsigned base = call * 32u;
        const unsigned below = (1u << (lane & 30)) - 1u;
        bool dead = false;
        int njob = 0, ntile = 0;
        unsigned seen = 1u << (2 * rank), lpos = 0, lbas = 0;
        unsigned spins = 0, lastsum = 0xffffffffu;
        unsigned long long t0 = 0;
      for (;;) {
        while (njob < NJOB && (dead || njob < 51 || seen != 0x5555u)) {
            u32x8 fx, fw;
            asm volatile("s_load_dwordx8 %0, %2, 0x0 glc\n\ts_load_dwordx8 %1, %3, 0x0 glc\n\ts_waitcnt lgkmcnt(0)" : "=&s"(fx), "=&s"(fw) : "s"(px), "s"(pw) : "memory");
            unsigned sum = 0;
#pragma unroll
            for (int i = 0; i < 8; ++i) sum += fx[i] + fw[i];
            if (sum != lastsum) {
                lastsum = sum; spins = 0;
                unsigned val = 0;
                asm volatile("v_writelane_b32 %0, %1, 0\n\tv_writelane_b32 %0, %9, 1\n\tv_writelane_b32 %0, %2, 2\n\tv_writelane_b32 %0, %10, 3\n\t"
                             "v_writelane_b32 %0, %3, 4\n\tv_writelane_b32 %0, %11, 5\n\tv_writelane_b32 %0, %4, 6\n\tv_writelane_b32 %0, %12, 7\n\t"
                             "v_writelane_b32 %0, %5, 8\n\tv_writelane_b32 %0, %13, 9\n\tv_writelane_b32 %0, %6, 10\n\tv_writelane_b32 %0, %14, 11\n\t"
                             "v_writelane_b32 %0, %7, 12\n\tv_writelane_b32 %0, %15, 13\n\tv_writelane_b32 %0, %8, 14\n\tv_writelane_b32 %0, %16, 15"
                             : "+v"(val) : "s"(fx[0]), "s"(fx[1]), "s"(fx[2]), "s"(fx[3]), "s"(fx[4]), "s"(fx[5]), "s"(fx[6]), "s"(fx[7]),
                               "s"(fw[0]), "s"(fw[1]), "s"(fw[2]), "s"(fw[3]), "s"(fw[4]), "s"(fw[5]), "s"(fw[6]), "s"(fw[7]));
                if (dead) val = 0xffffffe0u;
                if (seen != 0x5555u) {
                    const unsigned m = (unsigned)__ballot(val > base + 12u) & 0x5555u & ~seen;
                    if ((m >> (lane & 30)) & 1u) lpos = (__builtin_popcount(seen) - 1 + __builtin_popcount(m & below)) * 12u;
                    seen |= m;
                }
                for (;;) {
                    const int ty = job_type(njob);
                    const unsigned lim = NFULL + 2 * ty, step = (unsigned)tab & 15u, tv = 0x40000000u | (p * 16 + step) * 16;
                    const unsigned who = njob < 9 ? 1u << (2 * rank) : 0x5555u;
                    if (ty == 3 && ntile >= 0) {
                        lbas = lbase + L_LDQ + ntile * 4;
                        if (lane < 16 && sub && ent < lim && ((who >> (lane & 30)) & 1u)) { lds_add(lbas + lpos, tv); lds_add(lbas + lpos + 4, tv - 16u); }
                        ntile = -1;
                    }
                    unsigned bal = (unsigned)__ballot(val > base + step && ent < lim);
                    bal &= bal >> 1; bal &= who;
                    if (!bal || njob >= NJOB || (njob >= 51 && !dead)) break;
                    if (lane < 16 && ((bal >> (lane & 30)) & 1u)) {
                        if (ty != 3) lds_add(lbase + L_LDQ + (ntile + 2 * __builtin_popcount(bal & below) + sub) * 4, tv);
                        else if (!sub) lds_add(lbas + lpos + 8, tv);
                        const unsigned adv = ty ? 2u : 1u;
                        ent += adv; tab >>= 4 * adv;
                    }
                    njob += __builtin_popcount(bal);
                    if (ty != 3) ntile += 2 * __builtin_popcount(bal); else if (njob == 9) ntile = 19;
                }
            } else {
                __builtin_amdgcn_s_sleep(PSLEEP);
                if ((++spins & 1023u) == 0u) {
                    if (spins == 1024u) t0 = rt();
                    if (__builtin_amdgcn_readfirstlane(__hip_atomic_load(tmo, __ATOMIC_RELAXED, __HIP_MEMORY_SCOPE_AGENT)) == call || rt() - t0 > DEADLINE_TICKS) {
                        if (lane == 0) __hip_atomic_store(tmo, call, __ATOMIC_RELAXED, __HIP_MEMORY_SCOPE_AGENT);
                        lds_st(lbase + L_ABORT, 1u);
                        dead = true; lastsum = ~sum;
                    }
                }
            }
        }
        if (njob >= NJOB) break;
        {
            const unsigned vul = 0x40000000u | (p * 16 + (sub ? 13 : 12)) * 16, lq = lbase + L_LDQ + 103 * 4;
            unsigned fail = 0;
            unsigned est0 = 0;
#define EST0 est0
            asm volatile(ENDGAME_ASM : "+s"(fail), "+v"(EST0)
                : "s"(px), "s"(pw), "s"(base), "s"(1u << (2 * rank)), "v"(vul), "v"(lq + sub * 4), "v"(lq + 28 * 4 + lpos), "v"(sub * 16u), "n"(STAMPS_ON)
                : "memory", "scc", "vcc", CLOBBER_S, "v100", "v101", "v102");
            if (__builtin_amdgcn_readfirstlane(fail) == 0u) break;
            if (lane == 0) __hip_atomic_store(tmo, call, __ATOMIC_RELAXED, __HIP_MEMORY_SCOPE_AGENT);
            lds_st(lbase + L_ABORT, 1u);
            if (lane < NTILE - 103) asm volatile("ds_or_b32 %0, %1" :: "v"(lq + lane * 4), "v"(0x40000000u) : "memory");
            break;
        }
      }
        wait_vm<0>();
    } else {
        const int hb = wave - 10;
        __builtin_amdgcn_s_barrier();
        const unsigned call = lds_ld(lbase + L_CALL);
        __builtin_amdgcn_s_setprio(LPRIO);
        const unsigned goff = (unsigned)(WS_STAGE + (call & 1u) * PARITY_BYTES) + lane * 16;
        char* dbg = myws;
        const unsigned paddr = lane < 16 ? lbase + L_REL + lane * 4 : lbase + L_LDQ + hb * 4;
        asm volatile(LOADER_ASM
            :: "v"(paddr), "v"(goff), "s"(myws), "s"(lbase + L_RA), "s"(lbase + L_CTL), "s"(dbg), "n"(NTILE / 2), "n"(STAMPS_ON), "n"(LSLEEP),
               "v"(lbase + L_QUEUE + lane * 4)
            : "memory", "scc", "vcc", CLOBBER_S, "s36", "s37", "s38", "v101", "v102", "v104", "v105", "v106", "v107", "v108", "v109", "v111");
        return;
    }
    STAMP_OUT();
}

extern "C" void kernel_launch(int rank, int nranks, ncclComm_t comm,
                              hipStream_t stream,
                              void* const* d_in, const int* in_sizes, int n_in,
                              void* d_out, int out_size,
                              const int* in_dtypes, int out_dtype,
                              void* const* d_out_all,
                              void* const* d_ws_all, size_t ws_bytes) {
    (void)comm; (void)in_dtypes; (void)out_dtype; (void)d_out_all;
    if (nranks != NR || n_in != 4 || (unsigned)rank >= (unsigned)NR || ws_bytes < WS_NEED || out_size != DIM * DIM ||
        in_sizes[0] != DIM * KSH || in_sizes[1] != KSH * DIM) {
        fprintf(stderr, "kernel_launch: unexpected problem shape\n");
        return;
    }
    Params P;
    P.x = (const signed char*)d_in[0]; P.w = (const signed char*)d_in[1];
    P.sx = (const float*)d_in[2]; P.sw = (const float*)d_in[3];
    P.out = (float*)d_out; P.rank = rank;
    for (int o = 0; o < NR; ++o) P.ws[o] = (char*)d_ws_all[(rank + o) % NR];
    hipLaunchKernelGGL(gemm_ar_fused, dim3(NWG), dim3(768), 0, stream, P);
    HIP_CHECK(hipGetLastError());
}
```

```cpp
#include <hip/hip_runtime.h>
#include <rccl/rccl.h>
#include <cstdio>
#include <cstdint>
#include <utility>

#define HIP_CHECK(expr) do { hipError_t _e = (expr); if (_e != hipSuccess) { \
    fprintf(stderr, "HIP error %s at %s:%d\n", hipGetErrorString(_e), __FILE__, __LINE__); \
    return; } } while (0)

#define LAS __attribute__((address_space(3)))
#define GAS __attribute__((address_space(1)))
typedef unsigned u32x4 __attribute__((ext_vector_type(4)));
typedef unsigned u32x8 __attribute__((ext_vector_type(8)));
typedef unsigned u32x16 __attribute__((ext_vector_type(16)));

namespace {
constexpr int NR = 8;
constexpr int DIM = 4096;
constexpr int KSH = DIM / NR;
constexpr int NKS = KSH / 64;
constexpr int NWG = 256;
constexpr int NSTEP = 16;
constexpr int NFULL = 6;
constexpr int IPS = 12;
constexpr int NITEM = NR * IPS;
constexpr int NOWN = NR * NKS;
constexpr unsigned STEP16 = 7 * 1024;
constexpr unsigned T0TICKS = 70;
constexpr unsigned L0T = 110;
constexpr int ALPHA8 = 2;
constexpr unsigned GOPH = 128;
constexpr int TK0 = 12, TK1 = 14;
constexpr int EPW = 2;
constexpr int PREW = 22, PS0 = 2, PS1 = 8;
constexpr int NSA = 5, NSB = 4;
constexpr int MREP = 2;
constexpr int PROW = 256;
constexpr int step_w(int s) { return s < 2 * NFULL ? s & 1 : (s == 13 || s == 14); }
constexpr int nth_step(int w, int n) { for (int s = 0; s < NSTEP; ++s) if (step_w(s) == w && n-- == 0) return s; return -1; }
constexpr int op_index(int s) { int n = 0; for (int i = 0; i < s; ++i) n += step_w(i) == step_w(s); return n; }
constexpr unsigned nth_tab(int w) { unsigned t = 0; for (int n = 0; n < 8; ++n) t |= (unsigned)nth_step(w, n) << (4 * n); return t; }
constexpr unsigned long long step_tab(int w) {
    unsigned long long t = 0;
    for (int i = 0; i < IPS; ++i) t |= (unsigned long long)(i < NFULL ? 2 * i + w : (w ? (i < 8 ? 13 : 14) : (i < 10 ? 12 : 15))) << (4 * i);
    return t;
}

constexpr size_t WS_PROG = 0;
constexpr size_t WS_CTR = WS_PROG + 32 * MREP * PROW;
constexpr size_t WS_TMO = WS_CTR + (size_t)NWG * 128;
constexpr size_t WS_AGG = WS_TMO + 256;
constexpr size_t WS_STAGE = 4u << 20;
constexpr size_t STEP_BYTES = (size_t)NWG * 1024;
constexpr size_t SENDER_BYTES = NSTEP * STEP_BYTES;
constexpr size_t PARITY_BYTES = NR * SENDER_BYTES;
constexpr size_t WS_NEED = WS_STAGE + 2 * PARITY_BYTES;
static_assert(WS_AGG + (NSTEP + 1) * 16 * 128 <= WS_STAGE && TK1 < NSTEP, "control region");
constexpr int L_RA = 0;
constexpr int L_RB = NSA * 16384;
constexpr int L_WST = L_RB + NSB * 16384;
constexpr int L_CTL = L_WST + NKS * 1024;
constexpr int L_REL = L_CTL + 64;
constexpr int L_ABORT = L_CTL + 132;
constexpr int L_CALL = L_CTL + 136;
constexpr int L_WRDY = L_CTL + 140;
constexpr int L_LOC = L_CTL + 144;
constexpr int L_GO = L_CTL + 148;
constexpr int L_PREW = L_CTL + 152;
constexpr int L_QUEUE = L_CTL + 256;
constexpr int L_LDQ = L_QUEUE + 128 * 4;
constexpr int L_DUMMY = L_LDQ + 128 * 4;
constexpr int LDS_BYTES = L_DUMMY + 320;
static_assert(LDS_BYTES <= 163840 && NITEM < 128 && (PS1 - PS0) * PREW <= 256, "LDS");
constexpr unsigned long long DEADLINE_TICKS = 100000000ull;
constexpr unsigned FWD_SPINS = 1u << 21;

struct Params {
    const signed char* x; const signed char* w; const float* sx; const float* sw;
    float* out; char* ws[NR]; int rank;
};

template <class F, int... I> __device__ __forceinline__ void static_for_impl(F&& f, std::integer_sequence<int, I...>) { (f(std::integral_constant<int, I>{}), ...); }
template <int N, class F> __device__ __forceinline__ void static_for(F&& f) { static_for_impl(f, std::make_integer_sequence<int, N>{}); }
__device__ __forceinline__ void wait_vm0() { asm volatile("s_waitcnt vmcnt(0)" ::: "memory"); }
__device__ __forceinline__ void wait_lgkm0() { asm volatile("s_waitcnt lgkmcnt(0)" ::: "memory"); }
__device__ __forceinline__ unsigned long long rt() { return __builtin_amdgcn_s_memrealtime(); }
template <class T> __device__ __forceinline__ void pin(T& v) { asm volatile("" : "+v"(v)); }
__device__ __forceinline__ void st16_sys(const char* base, unsigned off, u32x4 v) {
    asm volatile("global_store_dwordx4 %0, %1, %2 sc0 sc1\n\ts_nop 1" :: "v"(off), "v"(v), "s"(base) : "memory");
}
__device__ __forceinline__ void st4_sys(const char* base, unsigned off, unsigned v) {
    asm volatile("global_store_dword %0, %1, %2 sc0 sc1\n\ts_nop 1" :: "v"(off), "v"(v), "s"(base) : "memory");
}
__device__ __forceinline__ void st4_sys_lane(const char* p, unsigned v) { asm volatile("global_store_dword %0, %1, off sc0 sc1\n\ts_nop 1" :: "v"(p), "v"(v) : "memory"); }
__device__ __forceinline__ void ld16_async(u32x4& v, const GAS u32x4* p) { asm volatile("global_load_dwordx4 %0, %1, off" : "=v"(v) : "v"(p) : "memory"); }
__device__ __forceinline__ unsigned ticket(const char* base, unsigned off) {
    unsigned t; unsigned long long keep;
    asm volatile("s_mov_b64 %1, exec\n\ts_mov_b64 exec, 1\n\tglobal_atomic_add %0, %2, %3, %4 sc0 sc1\n\ts_mov_b64 exec, %1\n\ts_waitcnt vmcnt(0)"
                 : "=&v"(t), "=&s"(keep) : "v"(off), "v"(1u), "s"(base) : "memory");
    return __builtin_amdgcn_readfirstlane(t);
}
__device__ __forceinline__ unsigned piece_min(const char* p) {
    u32x16 f;
    asm volatile("s_load_dwordx16 %0, %1, 0x0 glc\n\ts_waitcnt lgkmcnt(0)" : "=s"(f) : "s"(p) : "memory");
    unsigned m = f[0];
#pragma unroll
    for (int i = 1; i < 15; ++i) m = f[i] < m ? f[i] : m;
    return m;
}
__device__ __forceinline__ void lds_st(unsigned addr, unsigned v) { asm volatile("ds_write_b32 %0, %1" :: "v"(addr), "v"(v) : "memory"); }
__device__ __forceinline__ unsigned lds_ld(unsigned addr) {
    unsigned v; asm volatile("ds_read_b32 %0, %1\n\ts_waitcnt lgkmcnt(0)" : "=v"(v) : "v"(addr) : "memory");
    return __builtin_amdgcn_readfirstlane(v);
}
__device__ __forceinline__ int frag_blk(int n, int c) { return n < NFULL ? c : (c & ~15) + (n - NFULL) * 8 + (c & 7); }
__device__ __forceinline__ int frag_ks(int n, int c) { return n < NFULL ? n : NFULL + ((c >> 3) & 1); }
__device__ __forceinline__ const GAS u32x4* x_src(const signed char* x, int n, int c, int lane) {
    return (const GAS u32x4*)(x + (size_t)(frag_blk(n, c) * 16 + (lane & 15)) * KSH + frag_ks(n, c) * 64 + (lane >> 4) * 16);
}
}

#define CONSUMER_ASM R"ASM(
.macro MM m, n, a
  v_mfma_i32_16x16x64_i8 v[16*(\m)+4*(\n):16*(\m)+4*(\n)+3], v[144+4*(\a):147+4*(\a)], v[128+4*(\n):131+4*(\n)], v[16*(\m)+4*(\n):16*(\m)+4*(\n)+3]
.endm
.macro MM4 m, a
  MM \m, 0, \a
  MM \m, 1, \a
  MM \m, 2, \a
  MM \m, 3, \a
.endm
.macro LDA a, off
  ds_read_b128 v[144+4*(\a):147+4*(\a)], v161 offset:\off
.endm
.macro LDB n
  ds_read_b128 v[128+4*(\n):131+4*(\n)], v162 offset:1024*(\n)
.endm
// waits (bounded) for the next item of this wave's group, passing over and releasing the items of the other groups
.macro SPIN back
  s_mov_b32 s47, 0
1:
  s_cmp_ge_u32 s43, 0x400
  s_cbranch_scc0 2f
  ds_write2_b32 v165, v166, v166 offset0:0 offset1:8
  s_add_u32 s41, s41, 1
  s_cmp_eq_u32 s41, %7
  s_cselect_b32 s44, s50, s52
  s_cselect_b32 s41, 0, s41
  s_add_u32 s42, s42, 1
  s_cmp_eq_u32 s42, %8
  s_cselect_b32 s40, s51, s52
  s_cselect_b32 s42, 0, s42
  v_add_u32 v164, 4, v164
  v_add_u32 v161, s44, v161
  v_add_u32 v162, s40, v162
  v_add_u32 v166, 1, v166
  s_mov_b32 s47, 0
  s_branch 3f
2:
  s_sleep 1
3:
  ds_read_b32 v163, v164 offset:256
  s_add_u32 s47, s47, 1
  s_waitcnt lgkmcnt(0)
  v_readfirstlane_b32 s43, v163
  s_and_b32 s43, s43, %9
  s_cmp_eq_u32 s43, 0x300
  s_cbranch_scc1 \back
  s_cmp_lt_u32 s47, 0x1000000
  s_cbranch_scc1 1b
  v_mov_b32 v163, %3
  v_mov_b32 v144, 1
  ds_write_b32 v163, v144 offset:132
  s_branch FIN_%=
.endm
.macro OUT_ST u, t
  v_cvt_f32_i32 v[\t], v[16*((\u)/4)+((\u)&3)]
  v_cvt_f32_i32 v[\t+1], v[16*((\u)/4)+((\u)&3)+4]
  v_cvt_f32_i32 v[\t+2], v[16*((\u)/4)+((\u)&3)+8]
  v_cvt_f32_i32 v[\t+3], v[16*((\u)/4)+((\u)&3)+12]
  s_add_u32 s54, s58, (((\u)/4)*16+((\u)&3))*16384
  s_addc_u32 s55, s59, 0
  v_mul_f32 v[\t], s49, v[\t]
  v_mul_f32 v[\t+1], s49, v[\t+1]
  v_mul_f32 v[\t+2], s49, v[\t+2]
  v_mul_f32 v[\t+3], s49, v[\t+3]
  global_store_dwordx4 v167, v[\t:\t+3], s[54:55] sc0 sc1
.endm

  v_mbcnt_lo_u32_b32 v160, -1, 0
  v_mbcnt_hi_u32_b32 v160, -1, v160
  v_lshrrev_b32 v167, 4, v160
  v_lshlrev_b32 v167, 16, v167
  v_and_b32 v166, 15, v160
  v_lshl_add_u32 v167, v166, 4, v167
  v_lshlrev_b32 v160, 4, v160
  v_add_u32 v161, %0, v160
  v_add_u32 v162, %1, v160
  v_mov_b32 v164, %3
  v_lshrrev_b32 v165, 2, v160
  v_add_u32 v165, %11, v165
  v_writelane_b32 v165, %2, 0
  v_mov_b32 v166, 1
  s_mov_b64 s[58:59], %4
  s_mov_b32 s49, %5
  s_mov_b32 s41, 0
  s_mov_b32 s42, 0
  s_mov_b32 s43, 0
  s_mov_b32 s45, %6
  s_mov_b32 s50, -(%7-1)*16384
  s_mov_b32 s51, -(%8-1)*16384
  s_mov_b32 s52, 16384
.set zi, 0
.rept 128
  v_mov_b32 v[zi], 0
.set zi, zi+1
.endr
  SPIN START_%=
SPINF_%=:
  s_cmp_eq_u32 s45, 0
  s_cbranch_scc1 TAIL_%=
  SPIN BACK_%=
START_%=:
  LDA 0, 0
  LDA 1, 1024
  LDB 0
  LDB 1
  LDB 2
  LDB 3
  ds_write_b32 v165, v166 offset:32
LOOP_%=:
  s_waitcnt lgkmcnt(4)
  MM 0, 0, 0
  LDA 2, 2048
  MM 1, 0, 1
  LDA 3, 3072
  s_waitcnt lgkmcnt(5)
  MM 0, 1, 0
  MM 1, 1, 1
  s_waitcnt lgkmcnt(4)
  MM 0, 2, 0
  s_sub_u32 s45, s45, 1
  MM 1, 2, 1
  s_waitcnt lgkmcnt(3)
  MM 0, 3, 0
  v_add_u32 v164, 4, v164
  MM 1, 3, 1
  s_waitcnt lgkmcnt(1)
  MM 2, 0, 2
  LDA 0, 4096
  MM 2, 1, 2
  LDA 1, 5120
  MM 2, 2, 2
  s_add_u32 s41, s41, 1
  s_cmp_eq_u32 s41, %7
  MM 2, 3, 2
  s_waitcnt lgkmcnt(2)
  MM 3, 0, 3
  LDA 2, 6144
  MM 3, 1, 3
  s_cselect_b32 s44, s50, s52
  s_cselect_b32 s41, 0, s41
  MM 3, 2, 3
  s_add_u32 s42, s42, 1
  s_cmp_eq_u32 s42, %8
  MM 3, 3, 3
  s_waitcnt lgkmcnt(2)
  MM 4, 0, 0
  LDA 3, 7168
  MM 4, 1, 0
  ds_read_b32 v163, v164 offset:256
  MM 4, 2, 0
  ds_write_b32 v165, v166
  MM 4, 3, 0
  s_cselect_b32 s40, s51, s52
  s_waitcnt lgkmcnt(4)
  MM 5, 0, 1
  s_cselect_b32 s42, 0, s42
  v_add_u32 v161, s44, v161
  MM 5, 1, 1
  v_add_u32 v162, s40, v162
  v_add_u32 v166, 1, v166
  MM 5, 2, 1
  s_waitcnt lgkmcnt(1)
  v_readfirstlane_b32 s43, v163
  MM 5, 3, 1
  s_and_b32 s43, s43, %9
  s_cmp_eq_u32 s43, 0x300
  s_cbranch_scc0 SPINF_%=
BACK_%=:
  MM 6, 0, 2
  LDA 0, 0
  MM 7, 0, 3
  LDA 1, 1024
  MM 6, 1, 2
  LDB 0
  MM 7, 1, 3
  LDB 1
  MM 6, 2, 2
  MM 7, 2, 3
  LDB 2
  MM 6, 3, 2
  MM 7, 3, 3
  LDB 3
  ds_write_b32 v165, v166 offset:32
  s_branch LOOP_%=
TAIL_%=:
  s_waitcnt lgkmcnt(0)
  MM4 6, 2
  MM4 7, 3
FIN_%=:
  v_mov_b32 v166, 0x7fffffff
  v_mov_b32 v163, %3
  ds_write2_b32 v165, v166, v166 offset0:0 offset1:8
  s_mov_b32 s47, 0
8:
  ds_read_b32 v166, v163 offset:132
  ds_read_b32 v164, v163 offset:152
  s_add_u32 s47, s47, 1
  s_waitcnt lgkmcnt(0)
  v_readfirstlane_b32 s48, v166
  v_readfirstlane_b32 s43, v164
  s_cmp_lg_u32 s43, 0
  s_cbranch_scc1 9f
  s_sleep 2
  s_cmp_lt_u32 s47, 0x1000000
  s_cbranch_scc1 8b
  s_mov_b32 s48, 1
9:
  s_cmp_eq_u32 s48, 0
  s_cselect_b32 s49, s49, 0x7fc00000
  s_cmp_lt_u32 %9, 0x1000
  s_cselect_b64 vcc, -1, 0
.set du, 0
.rept 32
  OUT_ST du, 128+4*(du&7)
  s_cbranch_vccz 4f
  ds_read_b32 v166, v163 offset:128
  s_waitcnt vmcnt(%10) lgkmcnt(0)
  v_cmp_eq_u32 vcc, 0, v166
4:
.set du, du+1
.endr
.purgem MM
.purgem MM4
.purgem LDA
.purgem LDB
.purgem SPIN
.purgem OUT_ST
)ASM"

#define LOADER_ASM R"ASM(
.macro PUBCHK
  s_getreg_b32 s47, hwreg(HW_REG_IB_STS)
  s_and_b32 s48, s47, 15
  s_bfe_u32 s47, s47, 0x20016
  s_lshl4_add_u32 s47, s47, s48
  s_sub_u32 s47, s40, s47
  s_cmp_ge_u32 s47, s41
  s_cbranch_scc0 7f
  ds_add_u32 v105, v109 offset:256
  s_add_u32 s44, s44, 1
  s_mov_b32 s41, s42
  s_mov_b32 s42, s43
  s_mov_b32 s43, s49
  v_add_u32 v105, v112, v105
7:
.endm
.macro DMA4 va
  global_load_lds_dwordx4 \va, %4 sc0 sc1
  global_load_lds_dwordx4 \va, %4 offset:1024 sc0 sc1
  global_load_lds_dwordx4 \va, %4 offset:2048 sc0 sc1
  global_load_lds_dwordx4 \va, %4 offset:3072 sc0 sc1
  s_add_u32 s40, s40, 4
  s_add_u32 m0, m0, 4096
  v_add_u32 \va, 4096, \va
.endm
.macro BODY hb
  ds_read_b32 v102, v101
LLOOP\hb\()_%=:
  PUBCHK
  s_waitcnt lgkmcnt(0)
  v_cmp_ge_i32 vcc, v102, v100
  v_readlane_b32 s51, v102, 16
  s_cmp_eq_u64 vcc, -1
  s_cselect_b32 s47, s43, 0
  s_cmp_eq_u32 s47, s49
  s_cbranch_scc0 LIDLE\hb\()_%=
LGO\hb\()_%=:
  s_bfe_u32 s52, s51, 0xc000c
  s_lshl_b32 s52, s52, 13
  v_add_u32 v110, s52, %3
  s_bfe_u32 s53, s51, 0x20018
.if \hb == 0
  s_and_b32 s52, s51, 0xfff
  s_lshl_b32 s52, s52, 13
  v_add_u32 v104, s52, %2
  s_cmp_eq_u32 s53, 3
  s_cselect_b32 s52, 8192, 0
  s_add_u32 m0, s46, s52
  s_add_u32 s46, s46, 16384
  DMA4 v104
  v_add_u32 v100, v103, v100
  v_add_u32 v101, v111, v101
  s_cmp_eq_u32 s46, %6
  s_cselect_b32 s46, %5, s46
  PUBCHK
  s_cmp_eq_u32 s53, 0
  s_cbranch_scc0 8f
  DMA4 v104
  PUBCHK
  DMA4 v104
  PUBCHK
8:
  ds_read_b32 v102, v101
  DMA4 v104
  s_cmp_eq_u32 s53, 3
  s_cbranch_scc0 9f
  s_mov_b32 m0, s60
  v_subrev_u32 v110, 0x40000, v110
  PUBCHK
  DMA4 v110
9:
.else
  s_cmp_ge_u32 s53, 2
  s_cselect_b32 s52, 8192, 0
  s_add_u32 m0, s60, s52
  v_add_u32 v100, v103, v100
  DMA4 v110
  v_add_u32 v101, v111, v101
  PUBCHK
  s_cmp_eq_u32 s53, 0
  s_cbranch_scc0 8f
  DMA4 v110
  PUBCHK
  DMA4 v110
  PUBCHK
8:
  ds_read_b32 v102, v101
  DMA4 v110
  s_cmp_eq_u32 s53, 3
  s_cbranch_scc0 9f
  s_add_u32 m0, s60, 4096
  v_subrev_u32 v110, 0x41000, v110
  PUBCHK
  DMA4 v110
9:
.endif
  s_cmp_eq_u32 s42, s49
  s_cselect_b32 s43, s49, s40
  s_cselect_b32 s47, s40, s42
  s_cmp_eq_u32 s41, s49
  s_cselect_b32 s42, s49, s47
  s_cselect_b32 s41, s40, s41
  s_add_u32 s60, s60, 16384
  s_cmp_eq_u32 s60, %7
  s_cselect_b32 s60, %6, s60
  s_add_u32 s45, s45, 1
  s_mov_b32 s50, 0
  s_cmp_lt_u32 s45, %10
  s_cbranch_scc1 LLOOP\hb\()_%=
  s_branch LDRAIN_%=
LIDLE\hb\()_%=:
  ds_read_b32 v102, v101
  s_cmp_eq_u32 s41, s49
  s_cbranch_scc0 LLOOP\hb\()_%=
LWAIT\hb\()_%=:
  s_add_u32 s50, s50, 1
  s_waitcnt lgkmcnt(0)
  v_cmp_ge_i32 vcc, v102, v100
  v_readlane_b32 s51, v102, 16
  s_cmp_eq_u64 vcc, -1
  s_cbranch_scc1 LGO\hb\()_%=
  s_sleep 1
  ds_read_b32 v102, v101
  s_cmp_lt_u32 s50, 0x1000000
  s_cbranch_scc1 LWAIT\hb\()_%=
  v_mov_b32 v106, 1
  v_mov_b32 v101, %8
  ds_write_b32 v101, v106 offset:132
  s_branch LEND_%=
.endm
  s_mov_b32 s40, 0
  s_mov_b32 s49, 0x7fffffff
  s_mov_b32 s41, s49
  s_mov_b32 s42, s49
  s_mov_b32 s43, s49
  s_mov_b32 s44, 0
  s_mov_b32 s45, 0
  s_mov_b32 s46, %5
  s_mov_b32 s60, %6
  s_mov_b32 s50, 0
  v_mov_b32 v100, %1
  v_mov_b32 v101, %0
  v_mov_b32 v103, 1
  v_writelane_b32 v103, 0, 16
  v_mov_b32 v111, 0
  v_writelane_b32 v111, 4, 16
  v_mov_b32 v105, %11
  v_writelane_b32 v105, %8, 0
  v_mov_b32 v112, 0
  v_writelane_b32 v112, 4, 0
  v_mov_b32 v109, %9
  s_waitcnt vmcnt(0)
  s_cmp_eq_u32 %9, 0x200
  s_cbranch_scc1 LONE_%=
  BODY 0
LONE_%=:
  BODY 1
LDRAIN_%=:
  PUBCHK
  s_cmp_lt_u32 s44, %10
  s_cbranch_scc1 LDRAIN_%=
LEND_%=:
  v_mov_b32 v106, 1
  v_mov_b32 v101, %8
  ds_write_b32 v101, v106 offset:128
  s_waitcnt vmcnt(0) lgkmcnt(0)
.purgem PUBCHK
.purgem DMA4
.purgem BODY
)ASM"
#define R10(r, t) r #t "0", r #t "1", r #t "2", r #t "3", r #t "4", r #t "5", r #t "6", r #t "7", r #t "8", r #t "9"
#define CLOBBER_S R10("s", 4), R10("s", 5), "s60"
#define CLOBBER_V R10("v", ), R10("v", 1), R10("v", 2), R10("v", 3), R10("v", 4), R10("v", 5), R10("v", 6), R10("v", 7), R10("v", 8), R10("v", 9), \
    R10("v", 10), R10("v", 11), R10("v", 12), R10("v", 13), R10("v", 14), R10("v", 15), "v160", "v161", "v162", "v163", "v164", "v165", "v166", "v167"

__global__ void __launch_bounds__(768, 3) gemm_ar_fused(Params P) {
    __shared__ __attribute__((aligned(16))) unsigned char lds_raw[LDS_BYTES];
    LAS unsigned char* lds = (LAS unsigned char*)lds_raw;
    const unsigned lbase = (unsigned)(size_t)lds;
    const int tid = threadIdx.x, lane = tid & 63;
    const int wave = __builtin_amdgcn_readfirstlane(tid >> 6);
    const int b = blockIdx.x;
    const int rank = P.rank;
    char* myws = P.ws[0];
    const int c = ((b >> 5) << 5) | ((b & 7) << 2) | ((b >> 3) & 3);
    const int ti = ((b & 7) >> 1) * 4 + (b >> 6), tj = (b & 1) * 8 + ((b >> 3) & 7);
    const unsigned rot = (unsigned)((c + 32 * rank) & 255);

    if (wave < 8) {
        unsigned warm = 0;
        {
            const unsigned kk = (((rot >> 5) - 4u) * 4u + (rot & 3u)) | ((unsigned)wave << 8);
            const char* pw = P.ws[kk < 14u ? 1 + kk % 7u : 0] + WS_STAGE + (kk >= 7u ? PARITY_BYTES : 0) + (size_t)rank * SENDER_BYTES;
            asm volatile("s_cmp_ge_u32 %2, 14\n\ts_cbranch_scc1 1f\n\ts_load_dword %0, %1, 0x0 glc\n1:" : "+s"(warm) : "s"(pw), "s"(kk) : "memory", "scc");
        }
        __builtin_amdgcn_s_barrier();
        const unsigned call = lds_ld(lbase + L_CALL);
        asm volatile("" :: "s"(warm));
        if (rot < GOPH) { unsigned sp = 0; while (lds_ld(lbase + L_GO) == 0u && ++sp < (1u << 22)) __builtin_amdgcn_s_sleep(2); }
        {
            const int n = wave;
            const int cc = lane & 15, g = lane >> 4;
            const GAS unsigned* wp = (const GAS unsigned*)(P.w + (size_t)(frag_ks(n, c) * 64 + g * 16) * DIM + (frag_blk(n, c) >> 2) * 64 + cc * 4);
            unsigned d[16];
#pragma unroll
            for (int t = 0; t < 16; ++t) d[t] = wp[(size_t)t * (DIM / 4)];
            const int sx = (nth_tab(0) >> (4 * wave)) & 15, sw = (nth_tab(1) >> (4 * wave)) & 15;
            const u32x4 xv = *x_src(P.x, n, c, lane);
            const unsigned q = c & 3;
            const unsigned sel_lo = 0x0c0c0000u | ((4u + q) << 8) | q;
            const unsigned sel_hi = 0x00000c0cu | ((4u + q) << 24) | (q << 16);
            u32x4 wv;
#pragma unroll
            for (int e = 0; e < 4; ++e)
                wv[e] = __builtin_amdgcn_perm(d[4 * e + 1], d[4 * e], sel_lo) | __builtin_amdgcn_perm(d[4 * e + 3], d[4 * e + 2], sel_hi);
            *(LAS u32x4*)(lds + L_WST + n * 1024 + lane * 16) = wv;
            asm volatile("" ::: "memory");
            if (lane == 0) __hip_atomic_fetch_add((LAS unsigned*)(lds + L_WRDY), 1u, __ATOMIC_RELAXED, __HIP_MEMORY_SCOPE_WORKGROUP);
            const unsigned soff = (unsigned)(WS_STAGE + (call & 1u) * PARITY_BYTES + (size_t)rank * SENDER_BYTES) + c * 1024 + lane * 16;
            st16_sys(myws, soff + sx * (unsigned)STEP_BYTES, xv);
            st16_sys(myws, soff + sw * (unsigned)STEP_BYTES, wv);
            wait_vm0();
            if (lane == 0) {
                if (__hip_atomic_fetch_add((LAS unsigned*)(lds + L_LOC), 1u, __ATOMIC_RELAXED, __HIP_MEMORY_SCOPE_WORKGROUP) == 7u) {
                    GAS unsigned* tk = (GAS unsigned*)(myws + WS_AGG + (NSTEP * 16 + (c >> 4)) * 128);
                    if ((__hip_atomic_fetch_add(tk, 1u, __ATOMIC_RELAXED, __HIP_MEMORY_SCOPE_AGENT) & 15u) == 15u)
                        for (int k = 0; k < 2 * MREP; ++k)
                            __hip_atomic_store((GAS unsigned*)(myws + WS_PROG + (k * 16 + (c >> 4)) * PROW + rank * 4), call * 32u + NSTEP, __ATOMIC_RELAXED, __HIP_MEMORY_SCOPE_SYSTEM);
                }
            }
        }
        const int wr = wave >> 2, wc = (wave + 2) & 3;
        const unsigned scale = __builtin_amdgcn_readfirstlane(__float_as_uint(P.sx[0] * P.sw[0]));
        const char* obase = (const char*)(P.out + (size_t)(ti * 256 + wr * 128) * DIM + tj * 256 + wc * 64);
        asm volatile(CONSUMER_ASM
            :: "s"(lbase + L_RA + wr * 8192), "s"(lbase + L_RB + wc * 4096), "s"(lbase + L_REL + wave * 4), "s"(lbase + L_CTL),
               "s"(obase), "s"(scale), "n"(NOWN), "n"(NSA), "n"(NSB), "s"(0x300u | (0x400u << (wr ? 2 : wc >> 1))), "n"(EPW), "s"(lbase + L_DUMMY)
            : "memory", "scc", "vcc", CLOBBER_S, CLOBBER_V);
    } else if (wave == 8) {
        const unsigned tp0 = (unsigned)rt() + T0TICKS;
        __builtin_amdgcn_s_setprio(3);
        GAS unsigned* ctr = (GAS unsigned*)(myws + WS_CTR + (size_t)b * 128);
        unsigned callv; u32x4 xv[8];
        asm volatile("global_load_dword %0, %1, off sc1" : "=v"(callv) : "v"(ctr) : "memory");
        static_for<8>([&](auto nc) __attribute__((always_inline)) {
            constexpr int n = decltype(nc)::value;
            ld16_async(xv[n], x_src(P.x, n, c, lane));
        });
        const char* peer[7];
#pragma unroll
        for (int r = 0; r < 7; ++r) { int o = r + (int)((unsigned)(b >> 3) % 7u); o = o >= 7 ? o - 7 : o; peer[r] = P.ws[1 + o]; }
        const char* fl_lane = peer[0];
#pragma unroll
        for (int r = 1; r < 7; ++r) fl_lane = (lane & 7) == r ? peer[r] : fl_lane;
        fl_lane += WS_PROG + ((c >> 4) + (lane >> 3) * 32) * PROW + rank * 4;
        const bool fl_on = (lane & 7) < 7 && lane < 8 * MREP;
        const unsigned phase = (rot & ~15u) | ((c & 3) << 2) | ((c >> 2) & 3);
        asm volatile("s_waitcnt vmcnt(7)" : "+v"(callv), "+v"(xv[0]) :: "memory");
        const unsigned call = __builtin_amdgcn_readfirstlane(callv) + 1u;
        const unsigned doff = (unsigned)(WS_STAGE + (call & 1u) * PARITY_BYTES + (size_t)rank * SENDER_BYTES) + c * 1024 + lane * 16;
        const unsigned aoff = (unsigned)(WS_AGG + (c >> 4) * 128);
        unsigned tack = 0;
        bool alive = true;
        lds_st(lbase + L_CALL, call); wait_lgkm0();
        if (rot >= GOPH) __builtin_amdgcn_s_barrier();
        static_for<NSTEP>([&](auto sc) __attribute__((always_inline)) {
            constexpr int s = decltype(sc)::value;
            u32x4 v;
            if constexpr (s == 1) {
                pin(xv[1]); pin(xv[2]); pin(xv[3]); pin(xv[4]); pin(xv[5]); pin(xv[6]); pin(xv[7]);
                unsigned sp = 0; while (lds_ld(lbase + L_WRDY) < 8u && ++sp < (1u << 24)) __builtin_amdgcn_s_sleep(1);
                if (sp >= (1u << 24)) alive = false;
            }
            if constexpr (step_w(s)) { v = *(const LAS u32x4*)(lds + L_WST + op_index(s) * 1024 + lane * 16); pin(v); }
            else v = xv[op_index(s)];
            {
                unsigned tgo = tp0 + (s * STEP16 >> 4) + (phase * STEP16 >> 12);
                if constexpr (s > 0) { const int late = (int)(tack + (STEP16 >> 4) - L0T - tgo); if (late > 0) tgo += (unsigned)(late * (8 - ALPHA8) >> 3); }
                int rem;
                while ((rem = (int)(tgo - (unsigned)rt())) > 0) { if (rem > 40) __builtin_amdgcn_s_sleep(8); else __builtin_amdgcn_s_sleep(1); }
            }
#pragma unroll
            for (int r = 0; r < 7; ++r) st16_sys(peer[(r + s) % 7], doff + (unsigned)(s * STEP_BYTES), v);
            if constexpr (s == 0) { if (rot < GOPH) __builtin_amdgcn_s_barrier(); }
            wait_vm0();
            tack = (unsigned)rt();
            if constexpr (s == 0) lds_st(lbase + L_GO, 1u);
            const unsigned prog = call * 32u + s + 1;
            if (alive) {
                if constexpr (s >= TK0 && s < TK1) {
                    if ((ticket(myws, aoff + (unsigned)((s + 1) * 16 * 128)) & 15u) == 15u) { if (fl_on) st4_sys_lane(fl_lane + step_w(s) * 16 * PROW, prog); }
                } else if ((c & 15) != 15) {
                    if (lane == 0) st4_sys(myws, aoff + (c & 15) * 4, prog);
                } else {
                    unsigned sp = 0;
#pragma nounroll
                    while (sp < FWD_SPINS && piece_min(myws + aoff) < prog) ++sp;
                    if (sp >= FWD_SPINS) alive = false;
                    else if (fl_on) st4_sys_lane(fl_lane + step_w(s) * 16 * PROW, prog);
                }
            }
            if constexpr (s >= PS0 && s < PS1) {
                const char* ob = (const char*)(P.out + (size_t)(ti * 256 + 255 - (s - PS0) * PREW) * DIM + tj * 256);
                const u32x4 z = {0u, 0u, 0u, 0u};
#pragma unroll
                for (int k = 0; k < PREW; ++k) st16_sys(ob - (size_t)k * DIM * 4, lane * 16, z);
                if constexpr (s == PS1 - 1) { wait_vm0(); lds_st(lbase + L_PREW, 1u); }
            }
        });
        if (lane == 0) __hip_atomic_store(ctr, call, __ATOMIC_RELAXED, __HIP_MEMORY_SCOPE_AGENT);
        wait_vm0();
    } else if (wave == 9) {
        for (int u = lane; u < (L_DUMMY - L_CTL) / 4; u += 64) if (u != (L_CALL - L_CTL) / 4) lds_st(lbase + L_CTL + u * 4, 0u);
        wait_lgkm0();
        __builtin_amdgcn_s_barrier();
        const unsigned call = lds_ld(lbase + L_CALL);
        __builtin_amdgcn_s_setprio(2);
        GAS unsigned* tmo = (GAS unsigned*)(myws + WS_TMO);
        const int p = lane >> 1, sub = lane & 1;
        const char* px = myws + WS_PROG + (ti + (tj & (MREP - 1)) * 32) * PROW;
        const char* pw = myws + WS_PROG + (16 + tj + (ti & (MREP - 1)) * 32) * PROW;
        unsigned long long tab = sub ? step_tab(1) : step_tab(0);
        unsigned ent = sub;
        unsigned base = lane < 16 ? call * 32u : 0xfffffff0u;
        bool dead = false;
        int queued = 0;
        unsigned spins = 0, lastsum = 0xffffffffu;
        unsigned long long t0 = 0;
        while (queued < NITEM) {
            u32x8 fx, fw;
            asm volatile("s_load_dwordx8 %0, %2, 0x0 glc\n\ts_load_dwordx8 %1, %3, 0x0 glc\n\ts_waitcnt lgkmcnt(0)" : "=&s"(fx), "=&s"(fw) : "s"(px), "s"(pw) : "memory");
            unsigned sum = 0;
#pragma unroll
            for (int i = 0; i < 8; ++i) sum += fx[i] + fw[i];
            if (sum != lastsum) {
                lastsum = sum; spins = 0;
                unsigned val = 0;
                asm volatile("v_writelane_b32 %0, %1, 0\n\tv_writelane_b32 %0, %9, 1\n\tv_writelane_b32 %0, %2, 2\n\tv_writelane_b32 %0, %10, 3\n\t"
                             "v_writelane_b32 %0, %3, 4\n\tv_writelane_b32 %0, %11, 5\n\tv_writelane_b32 %0, %4, 6\n\tv_writelane_b32 %0, %12, 7\n\t"
                             "v_writelane_b32 %0, %5, 8\n\tv_writelane_b32 %0, %13, 9\n\tv_writelane_b32 %0, %6, 10\n\tv_writelane_b32 %0, %14, 11\n\t"
                             "v_writelane_b32 %0, %7, 12\n\tv_writelane_b32 %0, %15, 13\n\tv_writelane_b32 %0, %8, 14\n\tv_writelane_b32 %0, %16, 15"
                             : "+v"(val) : "s"(fx[0]), "s"(fx[1]), "s"(fx[2]), "s"(fx[3]), "s"(fx[4]), "s"(fx[5]), "s"(fx[6]), "s"(fx[7]),
                               "s"(fw[0]), "s"(fw[1]), "s"(fw[2]), "s"(fw[3]), "s"(fw[4]), "s"(fw[5]), "s"(fw[6]), "s"(fw[7]));
                if (dead) val = 0xffffffe0u;
                for (;;) {
                    unsigned bal = (unsigned)__ballot(val > base + ((unsigned)tab & 15u));
                    bal &= bal >> 1; bal &= 0x5555u;
                    if (!bal) break;
                    const bool rdy = lane < 16 && ((bal >> (lane & 14)) & 1u);
                    const bool two = ent - sub >= (unsigned)NFULL;
                    const bool emit = rdy && (sub == 0 || two);
                    const unsigned lead = (unsigned)__ballot(emit);
                    const int pos = queued + __builtin_popcount(lead & ((1u << (lane & 31)) - 1u));
                    if (emit) {
                        const unsigned ty = two ? (ent - (NFULL - 2)) >> 1 : 0u, hf = two ? ent & 1u : 0u;
                        const unsigned sa = (unsigned)(step_tab(0) >> (4 * ent)) & 15u, sb = (unsigned)(step_tab(1) >> (4 * ent)) & 15u;
                        lds_st(lbase + L_QUEUE + pos * 4, two ? 0x1c00u ^ (0x200u << ty) : 0u);
                        lds_st(lbase + L_LDQ + pos * 4, 0x4000000u | ty << 24 | (((p * 16 + sb) << 5 | hf) << 12) | ((p * 16 + sa) << 5 | hf));
                    }
                    const unsigned adv = rdy ? (two ? 2u : 1u) : 0u;
                    ent += adv; tab >>= 4 * adv;
                    if (ent - sub >= (unsigned)IPS) base = 0xfffffff0u;
                    queued += __builtin_popcount(lead);
                }
            } else {
                __builtin_amdgcn_s_sleep(1);
                if ((++spins & 1023u) == 0u) {
                    if (spins == 1024u) t0 = rt();
                    if (__builtin_amdgcn_readfirstlane(__hip_atomic_load(tmo, __ATOMIC_RELAXED, __HIP_MEMORY_SCOPE_AGENT)) == call || rt() - t0 > DEADLINE_TICKS) {
                        if (lane == 0) __hip_atomic_store(tmo, call, __ATOMIC_RELAXED, __HIP_MEMORY_SCOPE_AGENT);
                        lds_st(lbase + L_ABORT, 1u);
                        dead = true; lastsum = ~sum;
                    }
                }
            }
        }
        wait_vm0();
    } else {
        const int hb = wave - 10;
        __builtin_amdgcn_s_barrier();
        const unsigned call = lds_ld(lbase + L_CALL);
        __builtin_amdgcn_s_setprio(2);
        const unsigned goff = (unsigned)(WS_STAGE + (call & 1u) * PARITY_BYTES) + lane * 16;
        const unsigned paddr = lane < 16 ? lbase + L_REL + lane * 4 : lbase + L_LDQ;
        const int thr = lane < 8 ? (hb ? -0x40000000 : 1 - NSA) : (lane < 16 ? 1 - NSB : (lane == 16 ? 1 : -0x40000000));
        asm volatile(LOADER_ASM
            :: "v"(paddr), "v"(thr), "v"(goff + ti * 16384), "v"(goff + tj * 16384), "s"(myws),
               "s"(lbase + L_RA), "s"(lbase + L_RB), "s"(lbase + L_RB + NSB * 16384),
               "s"(lbase + L_CTL), "s"(256u << hb), "n"(NITEM), "v"(lbase + L_DUMMY - 256 + lane * 4)
            : "memory", "scc", "vcc", CLOBBER_S, "v100", "v101", "v102", "v103", "v104", "v105", "v106", "v109", "v110", "v111", "v112");
    }
}

extern "C" void kernel_launch(int rank, int nranks, ncclComm_t comm,
                              hipStream_t stream,
                              void* const* d_in, const int* in_sizes, int n_in,
                              void* d_out, int out_size,
                              const int* in_dtypes, int out_dtype,
                              void* const* d_out_all,
                              void* const* d_ws_all, size_t ws_bytes) {
    (void)comm; (void)in_dtypes; (void)out_dtype; (void)d_out_all;
    if (nranks != NR || n_in != 4 || (unsigned)rank >= (unsigned)NR || ws_bytes < WS_NEED || out_size != DIM * DIM ||
        in_sizes[0] != DIM * KSH || in_sizes[1] != KSH * DIM) {
        fprintf(stderr, "kernel_launch: unexpected problem shape\n");
        return;
    }
    Params P;
    P.x = (const signed char*)d_in[0]; P.w = (const signed char*)d_in[1];
    P.sx = (const float*)d_in[2]; P.sw = (const float*)d_in[3];
    P.out = (float*)d_out; P.rank = rank;
    for (int o = 0; o < NR; ++o) P.ws[o] = (char*)d_ws_all[(rank + o) % NR];
    hipLaunchKernelGGL(gemm_ar_fused, dim3(NWG), dim3(768), 0, stream, P);
    HIP_CHECK(hipGetLastError());
}
```

```cpp
#include <hip/hip_runtime.h>
#include <rccl/rccl.h>
#include <cstdio>
#include <cstdint>
#include <utility>

#define HIP_CHECK(expr) do { hipError_t _e = (expr); if (_e != hipSuccess) { \
    fprintf(stderr, "HIP error %s at %s:%d\n", hipGetErrorString(_e), __FILE__, __LINE__); \
    return; } } while (0)

#define LAS __attribute__((address_space(3)))
#define GAS __attribute__((address_space(1)))
typedef unsigned u32x4 __attribute__((ext_vector_type(4)));
typedef unsigned u32x8 __attribute__((ext_vector_type(8)));
typedef unsigned u32x16 __attribute__((ext_vector_type(16)));

namespace {
constexpr int NR = 8;
constexpr int DIM = 4096;
constexpr int KSH = DIM / NR;
constexpr int NKS = KSH / 64;
constexpr int NWG = 256;
constexpr int NSTEP = 16;
constexpr int NFULL = 6;
constexpr int IPS = 12;
constexpr int NITEM = NR * IPS;
constexpr int NOWN = NR * NKS;
constexpr unsigned STEP16 = 7 * 1018;
constexpr unsigned T0TICKS = 70;
constexpr unsigned L0T = 110;
constexpr int ALPHA8 = 2;
constexpr unsigned GOPH = 128;
constexpr int TK0 = 12, TK1 = 14;
constexpr int EPW = 2;
constexpr int PREW = 22, PS0 = 2, PS1 = 8;
constexpr int NSA = 5, NSB = 4;
constexpr int MREP = 1;
constexpr int PROW = 256;
constexpr int step_w(int s) { return s < 2 * NFULL ? s & 1 : (s == 13 || s == 14); }
constexpr int nth_step(int w, int n) { for (int s = 0; s < NSTEP; ++s) if (step_w(s) == w && n-- == 0) return s; return -1; }
constexpr int op_index(int s) { int n = 0; for (int i = 0; i < s; ++i) n += step_w(i) == step_w(s); return n; }
constexpr unsigned nth_tab(int w) { unsigned t = 0; for (int n = 0; n < 8; ++n) t |= (unsigned)nth_step(w, n) << (4 * n); return t; }
constexpr unsigned long long step_tab(int w) {
    unsigned long long t = 0;
    for (int i = 0; i < IPS; ++i) t |= (unsigned long long)(i < NFULL ? 2 * i + w : (w ? (i < 8 ? 13 : 14) : (i < 10 ? 12 : 15))) << (4 * i);
    return t;
}

constexpr size_t WS_PROG = 0;
constexpr size_t WS_CTR = WS_PROG + 32 * MREP * PROW;
constexpr size_t WS_TMO = WS_CTR + (size_t)NWG * 128;
constexpr size_t WS_AGG = WS_TMO + 256;
constexpr size_t WS_STAGE = 4u << 20;
constexpr size_t STEP_BYTES = (size_t)NWG * 1024;
constexpr size_t SENDER_BYTES = NSTEP * STEP_BYTES;
constexpr size_t PARITY_BYTES = NR * SENDER_BYTES;
constexpr size_t WS_NEED = WS_STAGE + 2 * PARITY_BYTES;
static_assert(WS_AGG + (NSTEP + 1) * 16 * 128 <= WS_STAGE && TK1 < NSTEP, "control region");
constexpr int L_RA = 0;
constexpr int L_RB = NSA * 16384;
constexpr int L_WST = L_RB + NSB * 16384;
constexpr int L_CTL = L_WST + NKS * 1024;
constexpr int L_REL = L_CTL + 64;
constexpr int L_ABORT = L_CTL + 132;
constexpr int L_CALL = L_CTL + 136;
constexpr int L_WRDY = L_CTL + 140;
constexpr int L_LOC = L_CTL + 144;
constexpr int L_GO = L_CTL + 148;
constexpr int L_PREW = L_CTL + 152;
constexpr int L_QUEUE = L_CTL + 256;
constexpr int L_LDQ = L_QUEUE + 128 * 4;
constexpr int L_DUMMY = L_LDQ + 128 * 4;
constexpr int LDS_BYTES = L_DUMMY + 320;
static_assert(LDS_BYTES <= 163840 && NITEM < 128 && (PS1 - PS0) * PREW <= 256, "LDS");
constexpr unsigned long long DEADLINE_TICKS = 100000000ull;
constexpr unsigned FWD_SPINS = 1u << 21;

struct Params {
    const signed char* x; const signed char* w; const float* sx; const float* sw;
    float* out; char* ws[NR]; int rank;
};

template <class F, int... I> __device__ __forceinline__ void static_for_impl(F&& f, std::integer_sequence<int, I...>) { (f(std::integral_constant<int, I>{}), ...); }
template <int N, class F> __device__ __forceinline__ void static_for(F&& f) { static_for_impl(f, std::make_integer_sequence<int, N>{}); }
__device__ __forceinline__ void wait_vm0() { asm volatile("s_waitcnt vmcnt(0)" ::: "memory"); }
__device__ __forceinline__ void wait_lgkm0() { asm volatile("s_waitcnt lgkmcnt(0)" ::: "memory"); }
__device__ __forceinline__ unsigned long long rt() { return __builtin_amdgcn_s_memrealtime(); }
template <class T> __device__ __forceinline__ void pin(T& v) { asm volatile("" : "+v"(v)); }
__device__ __forceinline__ void st16_sys(const char* base, unsigned off, u32x4 v) {
    asm volatile("global_store_dwordx4 %0, %1, %2 sc0 sc1\n\ts_nop 1" :: "v"(off), "v"(v), "s"(base) : "memory");
}
__device__ __forceinline__ void st4_sys(const char* base, unsigned off, unsigned v) {
    asm volatile("global_store_dword %0, %1, %2 sc0 sc1\n\ts_nop 1" :: "v"(off), "v"(v), "s"(base) : "memory");
}
__device__ __forceinline__ void st4_sys_lane(const char* p, unsigned v) { asm volatile("global_store_dword %0, %1, off sc0 sc1\n\ts_nop 1" :: "v"(p), "v"(v) : "memory"); }
__device__ __forceinline__ void ld16_async(u32x4& v, const GAS u32x4* p) { asm volatile("global_load_dwordx4 %0, %1, off" : "=v"(v) : "v"(p) : "memory"); }
__device__ __forceinline__ unsigned ticket(const char* base, unsigned off) {
    unsigned t; unsigned long long keep;
    asm volatile("s_mov_b64 %1, exec\n\ts_mov_b64 exec, 1\n\tglobal_atomic_add %0, %2, %3, %4 sc0 sc1\n\ts_mov_b64 exec, %1\n\ts_waitcnt vmcnt(0)"
                 : "=&v"(t), "=&s"(keep) : "v"(off), "v"(1u), "s"(base) : "memory");
    return __builtin_amdgcn_readfirstlane(t);
}
__device__ __forceinline__ unsigned piece_min(const char* p) {
    u32x16 f;
    asm volatile("s_load_dwordx16 %0, %1, 0x0 glc\n\ts_waitcnt lgkmcnt(0)" : "=s"(f) : "s"(p) : "memory");
    unsigned m = f[0];
#pragma unroll
    for (int i = 1; i < 15; ++i) m = f[i] < m ? f[i] : m;
    return m;
}
__device__ __forceinline__ void lds_st(unsigned addr, unsigned v) { asm volatile("ds_write_b32 %0, %1" :: "v"(addr), "v"(v) : "memory"); }
__device__ __forceinline__ unsigned lds_ld(unsigned addr) {
    unsigned v; asm volatile("ds_read_b32 %0, %1\n\ts_waitcnt lgkmcnt(0)" : "=v"(v) : "v"(addr) : "memory");
    return __builtin_amdgcn_readfirstlane(v);
}
__device__ __forceinline__ int frag_blk(int n, int c) { return n < NFULL ? c : (c & ~15) + (n - NFULL) * 8 + (c & 7); }
__device__ __forceinline__ int frag_ks(int n, int c) { return n < NFULL ? n : NFULL + ((c >> 3) & 1); }
__device__ __forceinline__ const GAS u32x4* x_src(const signed char* x, int n, int c, int lane) {
    return (const GAS u32x4*)(x + (size_t)(frag_blk(n, c) * 16 + (lane & 15)) * KSH + frag_ks(n, c) * 64 + (lane >> 4) * 16);
}
}

#define CONSUMER_ASM R"ASM(
.macro MM m, n, a
  v_mfma_i32_16x16x64_i8 v[16*(\m)+4*(\n):16*(\m)+4*(\n)+3], v[144+4*(\a):147+4*(\a)], v[128+4*(\n):131+4*(\n)], v[16*(\m)+4*(\n):16*(\m)+4*(\n)+3]
.endm
.macro MM4 m, a
  MM \m, 0, \a
  MM \m, 1, \a
  MM \m, 2, \a
  MM \m, 3, \a
.endm
.macro LDA a, off
  ds_read_b128 v[144+4*(\a):147+4*(\a)], v161 offset:\off
.endm
.macro LDB n
  ds_read_b128 v[128+4*(\n):131+4*(\n)], v162 offset:1024*(\n)
.endm
// waits (bounded) for the next item of this wave's group, passing over and releasing the items of the other groups
.macro SPIN back
  s_mov_b32 s47, 0
1:
  s_cmp_ge_u32 s43, 0x400
  s_cbranch_scc0 2f
  ds_write2_b32 v165, v166, v166 offset0:0 offset1:8
  s_add_u32 s41, s41, 1
  s_cmp_eq_u32 s41, %7
  s_cselect_b32 s44, s50, s52
  s_cselect_b32 s41, 0, s41
  s_add_u32 s42, s42, 1
  s_cmp_eq_u32 s42, %8
  s_cselect_b32 s40, s51, s52
  s_cselect_b32 s42, 0, s42
  v_add_u32 v164, 4, v164
  v_add_u32 v161, s44, v161
  v_add_u32 v162, s40, v162
  v_add_u32 v166, 1, v166
  s_mov_b32 s47, 0
  s_branch 3f
2:
  s_sleep 1
3:
  ds_read_b32 v163, v164 offset:256
  s_add_u32 s47, s47, 1
  s_waitcnt lgkmcnt(0)
  v_readfirstlane_b32 s43, v163
  s_and_b32 s43, s43, %9
  s_cmp_eq_u32 s43, 0x300
  s_cbranch_scc1 \back
  s_cmp_lt_u32 s47, 0x1000000
  s_cbranch_scc1 1b
  v_mov_b32 v163, %3
  v_mov_b32 v144, 1
  ds_write_b32 v163, v144 offset:132
  s_branch FIN_%=
.endm
.macro OUT_ST u, t
  v_cvt_f32_i32 v[\t], v[16*((\u)/4)+((\u)&3)]
  v_cvt_f32_i32 v[\t+1], v[16*((\u)/4)+((\u)&3)+4]
  v_cvt_f32_i32 v[\t+2], v[16*((\u)/4)+((\u)&3)+8]
  v_cvt_f32_i32 v[\t+3], v[16*((\u)/4)+((\u)&3)+12]
  s_add_u32 s54, s58, (((\u)/4)*16+((\u)&3))*16384
  s_addc_u32 s55, s59, 0
  v_mul_f32 v[\t], s49, v[\t]
  v_mul_f32 v[\t+1], s49, v[\t+1]
  v_mul_f32 v[\t+2], s49, v[\t+2]
  v_mul_f32 v[\t+3], s49, v[\t+3]
  global_store_dwordx4 v167, v[\t:\t+3], s[54:55] sc0 sc1
.endm

  v_mbcnt_lo_u32_b32 v160, -1, 0
  v_mbcnt_hi_u32_b32 v160, -1, v160
  v_lshrrev_b32 v167, 4, v160
  v_lshlrev_b32 v167, 16, v167
  v_and_b32 v166, 15, v160
  v_lshl_add_u32 v167, v166, 4, v167
  v_lshlrev_b32 v160, 4, v160
  v_add_u32 v161, %0, v160
  v_add_u32 v162, %1, v160
  v_mov_b32 v164, %3
  v_lshrrev_b32 v165, 2, v160
  v_add_u32 v165, %11, v165
  v_writelane_b32 v165, %2, 0
  v_mov_b32 v166, 1
  s_mov_b64 s[58:59], %4
  s_mov_b32 s49, %5
  s_mov_b32 s41, 0
  s_mov_b32 s42, 0
  s_mov_b32 s43, 0
  s_mov_b32 s45, %6
  s_mov_b32 s50, -(%7-1)*16384
  s_mov_b32 s51, -(%8-1)*16384
  s_mov_b32 s52, 16384
.set zi, 0
.rept 128
  v_mov_b32 v[zi], 0
.set zi, zi+1
.endr
  SPIN START_%=
SPINF_%=:
  s_cmp_eq_u32 s45, 0
  s_cbranch_scc1 TAIL_%=
  SPIN BACK_%=
START_%=:
  LDA 0, 0
  LDA 1, 1024
  LDB 0
  LDB 1
  LDB 2
  LDB 3
  ds_write_b32 v165, v166 offset:32
.p2align 6
  s_nop 0
LOOP_%=:
  s_waitcnt lgkmcnt(4)
  MM 0, 0, 0
  LDA 2, 2048
  MM 1, 0, 1
  LDA 3, 3072
  s_waitcnt lgkmcnt(5)
  MM 0, 1, 0
  MM 1, 1, 1
  s_waitcnt lgkmcnt(4)
  MM 0, 2, 0
  s_sub_u32 s45, s45, 1
  MM 1, 2, 1
  s_waitcnt lgkmcnt(3)
  MM 0, 3, 0
  v_add_u32 v164, 4, v164
  MM 1, 3, 1
  s_waitcnt lgkmcnt(1)
  MM 2, 0, 2
  LDA 0, 4096
  MM 2, 1, 2
  LDA 1, 5120
  MM 2, 2, 2
  s_add_u32 s41, s41, 1
  s_cmp_eq_u32 s41, %7
  MM 2, 3, 2
  s_waitcnt lgkmcnt(2)
  MM 3, 0, 3
  LDA 2, 6144
  MM 3, 1, 3
  s_cselect_b32 s44, s50, s52
  s_cselect_b32 s41, 0, s41
  MM 3, 2, 3
  s_add_u32 s42, s42, 1
  s_cmp_eq_u32 s42, %8
  MM 3, 3, 3
  s_waitcnt lgkmcnt(2)
  MM 4, 0, 0
  LDA 3, 7168
  MM 4, 1, 0
  ds_read_b32 v163, v164 offset:256
  MM 4, 2, 0
  ds_write_b32 v165, v166
  MM 4, 3, 0
  s_cselect_b32 s40, s51, s52
  s_waitcnt lgkmcnt(4)
  MM 5, 0, 1
  s_cselect_b32 s42, 0, s42
  v_add_u32 v161, s44, v161
  MM 5, 1, 1
  v_add_u32 v162, s40, v162
  v_add_u32 v166, 1, v166
  MM 5, 2, 1
  s_waitcnt lgkmcnt(1)
  v_readfirstlane_b32 s43, v163
  MM 5, 3, 1
  s_and_b32 s43, s43, %9
  s_cmp_eq_u32 s43, 0x300
  s_cbranch_scc0 SPINF_%=
BACK_%=:
  MM 6, 0, 2
  LDA 0, 0
  MM 7, 0, 3
  LDA 1, 1024
  MM 6, 1, 2
  LDB 0
  MM 7, 1, 3
  LDB 1
  MM 6, 2, 2
  MM 7, 2, 3
  LDB 2
  MM 6, 3, 2
  MM 7, 3, 3
  LDB 3
  ds_write_b32 v165, v166 offset:32
  s_branch LOOP_%=
TAIL_%=:
  s_waitcnt lgkmcnt(0)
  MM4 6, 2
  MM4 7, 3
FIN_%=:
  v_mov_b32 v166, 0x7fffffff
  v_mov_b32 v163, %3
  ds_write2_b32 v165, v166, v166 offset0:0 offset1:8
  s_mov_b32 s47, 0
8:
  ds_read_b32 v166, v163 offset:132
  ds_read_b32 v164, v163 offset:152
  s_add_u32 s47, s47, 1
  s_waitcnt lgkmcnt(0)
  v_readfirstlane_b32 s48, v166
  v_readfirstlane_b32 s43, v164
  s_cmp_lg_u32 s43, 0
  s_cbranch_scc1 9f
  s_sleep 2
  s_cmp_lt_u32 s47, 0x1000000
  s_cbranch_scc1 8b
  s_mov_b32 s48, 1
9:
  s_cmp_eq_u32 s48, 0
  s_cselect_b32 s49, s49, 0x7fc00000
  s_cmp_lt_u32 %9, 0x1000
  s_cselect_b64 vcc, -1, 0
.set du, 0
.rept 32
  OUT_ST du, 128+4*(du&7)
  s_cbranch_vccz 4f
.if (du & 3) == 3
  ds_read_b32 v166, v163 offset:128
  s_waitcnt vmcnt(%10) lgkmcnt(0)
  v_cmp_eq_u32 vcc, 0, v166
.else
  s_waitcnt vmcnt(%10)
.endif
4:
.set du, du+1
.endr
.purgem MM
.purgem MM4
.purgem LDA
.purgem LDB
.purgem SPIN
.purgem OUT_ST
)ASM"

#define LOADER_ASM R"ASM(
.macro PUBCHK
  s_getreg_b32 s47, hwreg(HW_REG_IB_STS)
  s_and_b32 s48, s47, 15
  s_bfe_u32 s47, s47, 0x20016
  s_lshl4_add_u32 s47, s47, s48
  s_sub_u32 s47, s40, s47
  s_cmp_ge_u32 s47, s41
  s_cbranch_scc0 7f
  ds_add_u32 v105, v109 offset:256
  s_add_u32 s44, s44, 1
  s_mov_b32 s41, s42
  s_mov_b32 s42, s43
  s_mov_b32 s43, s49
  v_add_u32 v105, v112, v105
7:
.endm
.macro DMA4 va
  global_load_lds_dwordx4 \va, %4 sc0 sc1
  global_load_lds_dwordx4 \va, %4 offset:1024 sc0 sc1
  global_load_lds_dwordx4 \va, %4 offset:2048 sc0 sc1
  global_load_lds_dwordx4 \va, %4 offset:3072 sc0 sc1
  s_add_u32 s40, s40, 4
  s_add_u32 m0, m0, 4096
  v_add_u32 \va, 4096, \va
.endm
.macro BODY hb
  ds_read_b32 v102, v101
LLOOP\hb\()_%=:
  PUBCHK
  s_waitcnt lgkmcnt(0)
  v_cmp_ge_i32 vcc, v102, v100
  v_readlane_b32 s51, v102, 16
  s_cmp_eq_u64 vcc, -1
  s_cselect_b32 s47, s43, 0
  s_cmp_eq_u32 s47, s49
  s_cbranch_scc0 LIDLE\hb\()_%=
LGO\hb\()_%=:
  s_bfe_u32 s52, s51, 0xc000c
  s_lshl_b32 s52, s52, 13
  v_add_u32 v110, s52, %3
  s_bfe_u32 s53, s51, 0x20018
.if \hb == 0
  s_and_b32 s52, s51, 0xfff
  s_lshl_b32 s52, s52, 13
  v_add_u32 v104, s52, %2
  s_cmp_eq_u32 s53, 3
  s_cselect_b32 s52, 8192, 0
  s_add_u32 m0, s46, s52
  s_add_u32 s46, s46, 16384
  DMA4 v104
  v_add_u32 v100, v103, v100
  v_add_u32 v101, v111, v101
  s_cmp_eq_u32 s46, %6
  s_cselect_b32 s46, %5, s46
  PUBCHK
  s_cmp_eq_u32 s53, 0
  s_cbranch_scc0 8f
  DMA4 v104
  PUBCHK
  DMA4 v104
  PUBCHK
8:
  ds_read_b32 v102, v101
  DMA4 v104
  s_cmp_eq_u32 s53, 3
  s_cbranch_scc0 9f
  s_mov_b32 m0, s60
  v_subrev_u32 v110, 0x40000, v110
  PUBCHK
  DMA4 v110
9:
.else
  s_cmp_ge_u32 s53, 2
  s_cselect_b32 s52, 8192, 0
  s_add_u32 m0, s60, s52
  v_add_u32 v100, v103, v100
  DMA4 v110
  v_add_u32 v101, v111, v101
  PUBCHK
  s_cmp_eq_u32 s53, 0
  s_cbranch_scc0 8f
  DMA4 v110
  PUBCHK
  DMA4 v110
  PUBCHK
8:
  ds_read_b32 v102, v101
  DMA4 v110
  s_cmp_eq_u32 s53, 3
  s_cbranch_scc0 9f
  s_add_u32 m0, s60, 4096
  v_subrev_u32 v110, 0x41000, v110
  PUBCHK
  DMA4 v110
9:
.endif
  s_cmp_eq_u32 s42, s49
  s_cselect_b32 s43, s49, s40
  s_cselect_b32 s47, s40, s42
  s_cmp_eq_u32 s41, s49
  s_cselect_b32 s42, s49, s47
  s_cselect_b32 s41, s40, s41
  s_add_u32 s60, s60, 16384
  s_cmp_eq_u32 s60, %7
  s_cselect_b32 s60, %6, s60
  s_add_u32 s45, s45, 1
  s_mov_b32 s50, 0
  s_cmp_lt_u32 s45, %10
  s_cbranch_scc1 LLOOP\hb\()_%=
  s_branch LDRAIN_%=
LIDLE\hb\()_%=:
  ds_read_b32 v102, v101
  s_cmp_eq_u32 s41, s49
  s_cbranch_scc0 LLOOP\hb\()_%=
LWAIT\hb\()_%=:
  s_add_u32 s50, s50, 1
  s_waitcnt lgkmcnt(0)
  v_cmp_ge_i32 vcc, v102, v100
  v_readlane_b32 s51, v102, 16
  s_cmp_eq_u64 vcc, -1
  s_cbranch_scc1 LGO\hb\()_%=
  s_sleep 1
  ds_read_b32 v102, v101
  s_cmp_lt_u32 s50, 0x1c00000
  s_cbranch_scc1 LWAIT\hb\()_%=
  v_mov_b32 v106, 1
  v_mov_b32 v101, %8
  ds_write_b32 v101, v106 offset:132
  s_branch LEND_%=
.endm
  s_mov_b32 s40, 0
  s_mov_b32 s49, 0x7fffffff
  s_mov_b32 s41, s49
  s_mov_b32 s42, s49
  s_mov_b32 s43, s49
  s_mov_b32 s44, 0
  s_mov_b32 s45, 0
  s_mov_b32 s46, %5
  s_mov_b32 s60, %6
  s_mov_b32 s50, 0
  v_mov_b32 v100, %1
  v_mov_b32 v101, %0
  v_mov_b32 v103, 1
  v_writelane_b32 v103, 0, 16
  v_mov_b32 v111, 0
  v_writelane_b32 v111, 4, 16
  v_mov_b32 v105, %11
  v_writelane_b32 v105, %8, 0
  v_mov_b32 v112, 0
  v_writelane_b32 v112, 4, 0
  v_mov_b32 v109, %9
  s_waitcnt vmcnt(0)
  s_cmp_eq_u32 %9, 0x200
  s_cbranch_scc1 LONE_%=
  BODY 0
LONE_%=:
  BODY 1
LDRAIN_%=:
  PUBCHK
  s_cmp_lt_u32 s44, %10
  s_cbranch_scc1 LDRAIN_%=
LEND_%=:
  v_mov_b32 v106, 1
  v_mov_b32 v101, %8
  ds_write_b32 v101, v106 offset:128
  s_waitcnt vmcnt(0) lgkmcnt(0)
.purgem PUBCHK
.purgem DMA4
.purgem BODY
)ASM"
#define R10(r, t) r #t "0", r #t "1", r #t "2", r #t "3", r #t "4", r #t "5", r #t "6", r #t "7", r #t "8", r #t "9"
#define CLOBBER_S R10("s", 4), R10("s", 5), "s60"
#define CLOBBER_V R10("v", ), R10("v", 1), R10("v", 2), R10("v", 3), R10("v", 4), R10("v", 5), R10("v", 6), R10("v", 7), R10("v", 8), R10("v", 9), \
    R10("v", 10), R10("v", 11), R10("v", 12), R10("v", 13), R10("v", 14), R10("v", 15), "v160", "v161", "v162", "v163", "v164", "v165", "v166", "v167"

__global__ void __launch_bounds__(768, 3) gemm_ar_fused(Params P) {
    __shared__ __attribute__((aligned(16))) unsigned char lds_raw[LDS_BYTES];
    LAS unsigned char* lds = (LAS unsigned char*)lds_raw;
    const unsigned lbase = (unsigned)(size_t)lds;
    const int tid = threadIdx.x, lane = tid & 63;
    const int wave = __builtin_amdgcn_readfirstlane(tid >> 6);
    const int b = blockIdx.x;
    const int rank = P.rank;
    char* myws = P.ws[0];
    const int c = ((b >> 5) << 5) | ((b & 7) << 2) | ((b >> 3) & 3);
    const int ti = ((b & 7) >> 1) * 4 + (b >> 6), tj = (b & 1) * 8 + ((b >> 3) & 7);
    const unsigned rot = (unsigned)((c + 32 * rank) & 255);

    if (wave < 8) {
        unsigned warm = 0;
        {
            const unsigned kk = (((rot >> 5) - 4u) * 4u + (rot & 3u)) | ((unsigned)wave << 8);
            const char* pw = P.ws[kk < 14u ? 1 + kk % 7u : 0] + WS_STAGE + (kk >= 7u ? PARITY_BYTES : 0) + (size_t)rank * SENDER_BYTES;
            asm volatile("s_cmp_ge_u32 %2, 14\n\ts_cbranch_scc1 1f\n\ts_load_dword %0, %1, 0x0 glc\n1:" : "+s"(warm) : "s"(pw), "s"(kk) : "memory", "scc");
        }
        __builtin_amdgcn_s_barrier();
        const unsigned call = lds_ld(lbase + L_CALL);
        asm volatile("" :: "s"(warm));
        if (rot < GOPH) { unsigned sp = 0; while (lds_ld(lbase + L_GO) == 0u && ++sp < (1u << 22)) __builtin_amdgcn_s_sleep(2); }
        {
            const int n = wave;
            const int cc = lane & 15, g = lane >> 4;
            const GAS unsigned* wp = (const GAS unsigned*)(P.w + (size_t)(frag_ks(n, c) * 64 + g * 16) * DIM + (frag_blk(n, c) >> 2) * 64 + cc * 4);
            unsigned d[16];
#pragma unroll
            for (int t = 0; t < 16; ++t) d[t] = wp[(size_t)t * (DIM / 4)];
            const int sx = (nth_tab(0) >> (4 * wave)) & 15, sw = (nth_tab(1) >> (4 * wave)) & 15;
            const u32x4 xv = *x_src(P.x, n, c, lane);
            const unsigned q = c & 3;
            const unsigned sel_lo = 0x0c0c0000u | ((4u + q) << 8) | q;
            const unsigned sel_hi = 0x00000c0cu | ((4u + q) << 24) | (q << 16);
            u32x4 wv;
#pragma unroll
            for (int e = 0; e < 4; ++e)
                wv[e] = __builtin_amdgcn_perm(d[4 * e + 1], d[4 * e], sel_lo) | __builtin_amdgcn_perm(d[4 * e + 3], d[4 * e + 2], sel_hi);
            *(LAS u32x4*)(lds + L_WST + n * 1024 + lane * 16) = wv;
            asm volatile("" ::: "memory");
            if (lane == 0) __hip_atomic_fetch_add((LAS unsigned*)(lds + L_WRDY), 1u, __ATOMIC_RELAXED, __HIP_MEMORY_SCOPE_WORKGROUP);
            const unsigned soff = (unsigned)(WS_STAGE + (call & 1u) * PARITY_BYTES + (size_t)rank * SENDER_BYTES) + c * 1024 + lane * 16;
            st16_sys(myws, soff + sx * (unsigned)STEP_BYTES, xv);
            st16_sys(myws, soff + sw * (unsigned)STEP_BYTES, wv);
            wait_vm0();
            if (lane == 0) {
                if (__hip_atomic_fetch_add((LAS unsigned*)(lds + L_LOC), 1u, __ATOMIC_RELAXED, __HIP_MEMORY_SCOPE_WORKGROUP) == 7u) {
                    GAS unsigned* tk = (GAS unsigned*)(myws + WS_AGG + (NSTEP * 16 + (c >> 4)) * 128);
                    if ((__hip_atomic_fetch_add(tk, 1u, __ATOMIC_RELAXED, __HIP_MEMORY_SCOPE_AGENT) & 15u) == 15u)
                        for (int k = 0; k < 2 * MREP; ++k)
                            __hip_atomic_store((GAS unsigned*)(myws + WS_PROG + (k * 16 + (c >> 4)) * PROW + rank * 4), call * 32u + NSTEP, __ATOMIC_RELAXED, __HIP_MEMORY_SCOPE_SYSTEM);
                }
            }
        }
        const int wr = wave >> 2, wc = (wave + 2) & 3;
        const unsigned scale = __builtin_amdgcn_readfirstlane(__float_as_uint(P.sx[0] * P.sw[0]));
        const char* obase = (const char*)(P.out + (size_t)(ti * 256 + wr * 128) * DIM + tj * 256 + wc * 64);
        asm volatile(CONSUMER_ASM
            :: "s"(lbase + L_RA + wr * 8192), "s"(lbase + L_RB + wc * 4096), "s"(lbase + L_REL + wave * 4), "s"(lbase + L_CTL),
               "s"(obase), "s"(scale), "n"(NOWN), "n"(NSA), "n"(NSB), "s"(0x300u | (0x400u << (wr ? 2 : wc >> 1))), "n"(EPW), "s"(lbase + L_DUMMY)
            : "memory", "scc", "vcc", CLOBBER_S, CLOBBER_V);
    } else if (wave == 8) {
        const unsigned tp0 = (unsigned)rt() + T0TICKS;
        __builtin_amdgcn_s_setprio(3);
        GAS unsigned* ctr = (GAS unsigned*)(myws + WS_CTR + (size_t)b * 128);
        unsigned callv; u32x4 xv[8];
        asm volatile("global_load_dword %0, %1, off sc1" : "=v"(callv) : "v"(ctr) : "memory");
        static_for<8>([&](auto nc) __attribute__((always_inline)) {
            constexpr int n = decltype(nc)::value;
            ld16_async(xv[n], x_src(P.x, n, c, lane));
        });
        const char* peer[7];
#pragma unroll
        for (int r = 0; r < 7; ++r) { int o = r + (int)((unsigned)(b >> 3) % 7u); o = o >= 7 ? o - 7 : o; peer[r] = P.ws[1 + o]; }
        const char* fl_lane = peer[0];
#pragma unroll
        for (int r = 1; r < 7; ++r) fl_lane = (lane & 7) == r ? peer[r] : fl_lane;
        fl_lane += WS_PROG + ((c >> 4) + (lane >> 3) * 32) * PROW + rank * 4;
        const bool fl_on = (lane & 7) < 7 && lane < 8 * MREP;
        const unsigned phase = (rot & ~15u) | ((c & 3) << 2) | ((c >> 2) & 3);
        asm volatile("s_waitcnt vmcnt(7)" : "+v"(callv), "+v"(xv[0]) :: "memory");
        const unsigned call = __builtin_amdgcn_readfirstlane(callv) + 1u;
        const unsigned doff = (unsigned)(WS_STAGE + (call & 1u) * PARITY_BYTES + (size_t)rank * SENDER_BYTES) + c * 1024 + lane * 16;
        const unsigned aoff = (unsigned)(WS_AGG + (c >> 4) * 128);
        unsigned tack = 0;
        bool alive = true;
        lds_st(lbase + L_CALL, call); wait_lgkm0();
        if (rot >= GOPH) __builtin_amdgcn_s_barrier();
        static_for<NSTEP>([&](auto sc) __attribute__((always_inline)) {
            constexpr int s = decltype(sc)::value;
            u32x4 v;
            if constexpr (s == 1) {
                pin(xv[1]); pin(xv[2]); pin(xv[3]); pin(xv[4]); pin(xv[5]); pin(xv[6]); pin(xv[7]);
                unsigned sp = 0; while (lds_ld(lbase + L_WRDY) < 8u && ++sp < (1u << 24)) __builtin_amdgcn_s_sleep(1);
                if (sp >= (1u << 24)) alive = false;
            }
            if constexpr (step_w(s)) { v = *(const LAS u32x4*)(lds + L_WST + op_index(s) * 1024 + lane * 16); pin(v); }
            else v = xv[op_index(s)];
            {
                unsigned tgo = tp0 + (s * STEP16 >> 4) + (phase * STEP16 >> 12);
                if constexpr (s > 0) { const int late = (int)(tack + (STEP16 >> 4) - L0T - tgo); if (late > 0) tgo += (unsigned)(late * (8 - ALPHA8) >> 3); }
                int rem;
                while ((rem = (int)(tgo - (unsigned)rt())) > 0) { if (rem > 40) __builtin_amdgcn_s_sleep(8); else __builtin_amdgcn_s_sleep(1); }
            }
#pragma unroll
            for (int r = 0; r < 7; ++r) st16_sys(peer[(r + s) % 7], doff + (unsigned)(s * STEP_BYTES), v);
            if constexpr (s == 0) { if (rot < GOPH) __builtin_amdgcn_s_barrier(); }
            wait_vm0();
            tack = (unsigned)rt();
            if constexpr (s == 0) lds_st(lbase + L_GO, 1u);
            const unsigned prog = call * 32u + s + 1;
            if (alive) {
                if constexpr (s >= TK0 && s < TK1) {
                    if ((ticket(myws, aoff + (unsigned)((s + 1) * 16 * 128)) & 15u) == 15u) { if (fl_on) st4_sys_lane(fl_lane + step_w(s) * 16 * PROW, prog); }
                } else if ((c & 15) != 15) {
                    if (lane == 0) st4_sys(myws, aoff + (c & 15) * 4, prog);
                } else {
                    unsigned sp = 0;
#pragma nounroll
                    while (sp < FWD_SPINS && piece_min(myws + aoff) < prog) ++sp;
                    if (sp >= FWD_SPINS) alive = false;
                    else if (fl_on) st4_sys_lane(fl_lane + step_w(s) * 16 * PROW, prog);
                }
            }
            if constexpr (s >= PS0 && s < PS1) {
                const char* ob = (const char*)(P.out + (size_t)(ti * 256 + 255 - (s - PS0) * PREW) * DIM + tj * 256);
                const u32x4 z = {0u, 0u, 0u, 0u};
#pragma unroll
                for (int k = 0; k < PREW; ++k) st16_sys(ob - (size_t)k * DIM * 4, lane * 16, z);
                if constexpr (s == PS1 - 1) { wait_vm0(); lds_st(lbase + L_PREW, 1u); }
            }
        });
        if (lane == 0) __hip_atomic_store(ctr, call, __ATOMIC_RELAXED, __HIP_MEMORY_SCOPE_AGENT);
        wait_vm0();
    } else if (wave == 9) {
        for (int u = lane; u < (L_DUMMY - L_CTL) / 4; u += 64) if (u != (L_CALL - L_CTL) / 4) lds_st(lbase + L_CTL + u * 4, 0u);
        wait_lgkm0();
        __builtin_amdgcn_s_barrier();
        const unsigned call = lds_ld(lbase + L_CALL);
        __builtin_amdgcn_s_setprio(2);
        GAS unsigned* tmo = (GAS unsigned*)(myws + WS_TMO);
        const int p = lane >> 1, sub = lane & 1;
        const char* px = myws + WS_PROG + (ti + (tj & (MREP - 1)) * 32) * PROW;
        const char* pw = myws + WS_PROG + (16 + tj + (ti & (MREP - 1)) * 32) * PROW;
        unsigned long long tab = sub ? step_tab(1) : step_tab(0);
        unsigned ent = sub;
        unsigned base = lane < 16 ? call * 32u : 0xfffffff0u;
        bool dead = false;
        int queued = 0;
        unsigned spins = 0, lastsum = 0xffffffffu;
        unsigned long long t0 = 0;
        while (queued < NITEM) {
            u32x8 fx, fw;
            asm volatile("s_load_dwordx8 %0, %2, 0x0 glc\n\ts_load_dwordx8 %1, %3, 0x0 glc\n\ts_waitcnt lgkmcnt(0)" : "=&s"(fx), "=&s"(fw) : "s"(px), "s"(pw) : "memory");
            unsigned sum = 0;
#pragma unroll
            for (int i = 0; i < 8; ++i) sum += fx[i] + fw[i];
            if (sum != lastsum) {
                lastsum = sum; spins = 0;
                unsigned val = 0;
                asm volatile("v_writelane_b32 %0, %1, 0\n\tv_writelane_b32 %0, %9, 1\n\tv_writelane_b32 %0, %2, 2\n\tv_writelane_b32 %0, %10, 3\n\t"
                             "v_writelane_b32 %0, %3, 4\n\tv_writelane_b32 %0, %11, 5\n\tv_writelane_b32 %0, %4, 6\n\tv_writelane_b32 %0, %12, 7\n\t"
                             "v_writelane_b32 %0, %5, 8\n\tv_writelane_b32 %0, %13, 9\n\tv_writelane_b32 %0, %6, 10\n\tv_writelane_b32 %0, %14, 11\n\t"
                             "v_writelane_b32 %0, %7, 12\n\tv_writelane_b32 %0, %15, 13\n\tv_writelane_b32 %0, %8, 14\n\tv_writelane_b32 %0, %16, 15"
                             : "+v"(val) : "s"(fx[0]), "s"(fx[1]), "s"(fx[2]), "s"(fx[3]), "s"(fx[4]), "s"(fx[5]), "s"(fx[6]), "s"(fx[7]),
                               "s"(fw[0]), "s"(fw[1]), "s"(fw[2]), "s"(fw[3]), "s"(fw[4]), "s"(fw[5]), "s"(fw[6]), "s"(fw[7]));
                if (dead) val = 0xffffffe0u;
                for (;;) {
                    unsigned bal = (unsigned)__ballot(val > base + ((unsigned)tab & 15u));
                    bal &= bal >> 1; bal &= 0x5555u;
                    if (!bal) break;
                    const bool rdy = lane < 16 && ((bal >> (lane & 14)) & 1u);
                    const bool two = ent - sub >= (unsigned)NFULL;
                    const bool emit = rdy && (sub == 0 || two);
                    const unsigned lead = (unsigned)__ballot(emit);
                    const int pos = queued + __builtin_popcount(lead & ((1u << (lane & 31)) - 1u));
                    if (emit) {
                        const unsigned ty = two ? (ent - (NFULL - 2)) >> 1 : 0u, hf = two ? ent & 1u : 0u;
                        const unsigned sa = (unsigned)(step_tab(0) >> (4 * ent)) & 15u, sb = (unsigned)(step_tab(1) >> (4 * ent)) & 15u;
                        lds_st(lbase + L_QUEUE + pos * 4, two ? 0x1c00u ^ (0x200u << ty) : 0u);
                        lds_st(lbase + L_LDQ + pos * 4, 0x4000000u | ty << 24 | (((p * 16 + sb) << 5 | hf) << 12) | ((p * 16 + sa) << 5 | hf));
                    }
                    const unsigned adv = rdy ? (two ? 2u : 1u) : 0u;
                    ent += adv; tab >>= 4 * adv;
                    if (ent - sub >= (unsigned)IPS) base = 0xfffffff0u;
                    queued += __builtin_popcount(lead);
                }
            } else {
                __builtin_amdgcn_s_sleep(1);
                if ((++spins & 1023u) == 0u) {
                    if (spins == 1024u) t0 = rt();
                    if (__builtin_amdgcn_readfirstlane(__hip_atomic_load(tmo, __ATOMIC_RELAXED, __HIP_MEMORY_SCOPE_AGENT)) == call || rt() - t0 > DEADLINE_TICKS) {
                        if (lane == 0) __hip_atomic_store(tmo, call, __ATOMIC_RELAXED, __HIP_MEMORY_SCOPE_AGENT);
                        lds_st(lbase + L_ABORT, 1u);
                        dead = true; lastsum = ~sum;
                    }
                }
            }
        }
        wait_vm0();
    } else {
        const int hb = wave - 10;
        __builtin_amdgcn_s_barrier();
        const unsigned call = lds_ld(lbase + L_CALL);
        __builtin_amdgcn_s_setprio(2);
        const unsigned goff = (unsigned)(WS_STAGE + (call & 1u) * PARITY_BYTES) + lane * 16;
        const unsigned paddr = lane < 16 ? lbase + L_REL + lane * 4 : lbase + L_LDQ;
        const int thr = lane < 8 ? (hb ? -0x40000000 : 1 - NSA) : (lane < 16 ? 1 - NSB : (lane == 16 ? 1 : -0x40000000));
        asm volatile(LOADER_ASM
            :: "v"(paddr), "v"(thr), "v"(goff + ti * 16384), "v"(goff + tj * 16384), "s"(myws),
               "s"(lbase + L_RA), "s"(lbase + L_RB), "s"(lbase + L_RB + NSB * 16384),
               "s"(lbase + L_CTL), "s"(256u << hb), "n"(NITEM), "v"(lbase + L_DUMMY - 256 + lane * 4)
            : "memory", "scc", "vcc", CLOBBER_S, "v100", "v101", "v102", "v103", "v104", "v105", "v106", "v109", "v110", "v111", "v112");
    }
}

extern "C" void kernel_launch(int rank, int nranks, ncclComm_t comm,
                              hipStream_t stream,
                              void* const* d_in, const int* in_sizes, int n_in,
                              void* d_out, int out_size,
                              const int* in_dtypes, int out_dtype,
                              void* const* d_out_all,
                              void* const* d_ws_all, size_t ws_bytes) {
    (void)comm; (void)in_dtypes; (void)out_dtype; (void)d_out_all;
    if (nranks != NR || n_in != 4 || (unsigned)rank >= (unsigned)NR || ws_bytes < WS_NEED || out_size != DIM * DIM ||
        in_sizes[0] != DIM * KSH || in_sizes[1] != KSH * DIM) {
        fprintf(stderr, "kernel_launch: unexpected problem shape\n");
        return;
    }
    Params P;
    P.x = (const signed char*)d_in[0]; P.w = (const signed char*)d_in[1];
    P.sx = (const float*)d_in[2]; P.sw = (const float*)d_in[3];
    P.out = (float*)d_out; P.rank = rank;
    for (int o = 0; o < NR; ++o) P.ws[o] = (char*)d_ws_all[(rank + o) % NR];
    hipLaunchKernelGGL(gemm_ar_fused, dim3(NWG), dim3(768), 0, stream, P);
    HIP_CHECK(hipGetLastError());
}
```

```cpp
#include <hip/hip_runtime.h>
#include <rccl/rccl.h>
#include <cstdio>
#include <cstdint>
#include <utility>

#define HIP_CHECK(expr) do { hipError_t _e = (expr); if (_e != hipSuccess) { \
    fprintf(stderr, "HIP error %s at %s:%d\n", hipGetErrorString(_e), __FILE__, __LINE__); \
    return; } } while (0)

#define LAS __attribute__((address_space(3)))
#define GAS __attribute__((address_space(1)))
typedef unsigned u32x4 __attribute__((ext_vector_type(4)));
typedef unsigned u32x8 __attribute__((ext_vector_type(8)));
typedef unsigned u32x16 __attribute__((ext_vector_type(16)));

namespace {
constexpr int NR = 8;
constexpr int DIM = 4096;
constexpr int KSH = DIM / NR;
constexpr int NKS = KSH / 64;
constexpr int NWG = 256;
constexpr int NSTEP = 16;
constexpr int NFULL = 6;
constexpr int IPS = 12;
constexpr int NITEM = NR * IPS;
constexpr int NOWN = NR * NKS;
constexpr unsigned STEP16 = 7 * 1018;
constexpr unsigned T0TICKS = 70;
constexpr unsigned L0T = 110;
constexpr int ALPHA8 = 2;
constexpr unsigned GOPH = 128;
constexpr int TK0 = 12, TK1 = 15;
constexpr int EPW = 2;
constexpr int PREW = 22, PS0 = 2, PS1 = 8;
constexpr int NSA = 5, NSB = 4;
constexpr int MREP = 1;
constexpr int PROW = 256;
constexpr int step_w(int s) { return s < 2 * NFULL ? s & 1 : (s == 13 || s == 14); }
constexpr int nth_step(int w, int n) { for (int s = 0; s < NSTEP; ++s) if (step_w(s) == w && n-- == 0) return s; return -1; }
constexpr int op_index(int s) { int n = 0; for (int i = 0; i < s; ++i) n += step_w(i) == step_w(s); return n; }
constexpr unsigned nth_tab(int w) { unsigned t = 0; for (int n = 0; n < 8; ++n) t |= (unsigned)nth_step(w, n) << (4 * n); return t; }
constexpr unsigned long long step_tab(int w) {
    unsigned long long t = 0;
    for (int i = 0; i < IPS; ++i) t |= (unsigned long long)(i < NFULL ? 2 * i + w : (w ? (i < 8 ? 13 : 14) : (i < 10 ? 12 : 15))) << (4 * i);
    return t;
}

constexpr size_t WS_PROG = 0;
constexpr size_t WS_CTR = WS_PROG + 32 * MREP * PROW;
constexpr size_t WS_TMO = WS_CTR + (size_t)NWG * 128;
constexpr size_t WS_AGG = WS_TMO + 256;
constexpr size_t WS_STAGE = 4u << 20;
constexpr size_t STEP_BYTES = (size_t)NWG * 1024;
constexpr size_t SENDER_BYTES = NSTEP * STEP_BYTES;
constexpr size_t PARITY_BYTES = NR * SENDER_BYTES;
constexpr size_t WS_NEED = WS_STAGE + 2 * PARITY_BYTES;
static_assert(WS_AGG + (NSTEP + 1) * 16 * 128 <= WS_STAGE && TK1 < NSTEP, "control region");
constexpr int L_RA = 0;
constexpr int L_RB = NSA * 16384;
constexpr int L_WST = L_RB + NSB * 16384;
constexpr int L_CTL = L_WST + NKS * 1024;
constexpr int L_REL = L_CTL + 64;
constexpr int L_ABORT = L_CTL + 132;
constexpr int L_CALL = L_CTL + 136;
constexpr int L_WRDY = L_CTL + 140;
constexpr int L_LOC = L_CTL + 144;
constexpr int L_GO = L_CTL + 148;
constexpr int L_PREW = L_CTL + 152;
constexpr int L_QUEUE = L_CTL + 256;
constexpr int L_LDQ = L_QUEUE + 128 * 4;
constexpr int L_DUMMY = L_LDQ + 128 * 4;
constexpr int LDS_BYTES = L_DUMMY + 320;
static_assert(LDS_BYTES <= 163840 && NITEM < 128 && (PS1 - PS0) * PREW <= 256, "LDS");
constexpr unsigned long long DEADLINE_TICKS = 100000000ull;
constexpr unsigned FWD_SPINS = 1u << 21;

struct Params {
    const signed char* x; const signed char* w; const float* sx; const float* sw;
    float* out; char* ws[NR]; int rank;
};

template <class F, int... I> __device__ __forceinline__ void static_for_impl(F&& f, std::integer_sequence<int, I...>) { (f(std::integral_constant<int, I>{}), ...); }
template <int N, class F> __device__ __forceinline__ void static_for(F&& f) { static_for_impl(f, std::make_integer_sequence<int, N>{}); }
__device__ __forceinline__ void wait_vm0() { asm volatile("s_waitcnt vmcnt(0)" ::: "memory"); }
__device__ __forceinline__ void wait_lgkm0() { asm volatile("s_waitcnt lgkmcnt(0)" ::: "memory"); }
__device__ __forceinline__ unsigned long long rt() { return __builtin_amdgcn_s_memrealtime(); }
template <class T> __device__ __forceinline__ void pin(T& v) { asm volatile("" : "+v"(v)); }
__device__ __forceinline__ void st16_sys(const char* base, unsigned off, u32x4 v) {
    asm volatile("global_store_dwordx4 %0, %1, %2 sc0 sc1\n\ts_nop 1" :: "v"(off), "v"(v), "s"(base) : "memory");
}
__device__ __forceinline__ void st4_sys(const char* base, unsigned off, unsigned v) {
    asm volatile("global_store_dword %0, %1, %2 sc0 sc1\n\ts_nop 1" :: "v"(off), "v"(v), "s"(base) : "memory");
}
__device__ __forceinline__ void st4_sys_lane(const char* p, unsigned v) { asm volatile("global_store_dword %0, %1, off sc0 sc1\n\ts_nop 1" :: "v"(p), "v"(v) : "memory"); }
__device__ __forceinline__ void ld16_async(u32x4& v, const GAS u32x4* p) { asm volatile("global_load_dwordx4 %0, %1, off" : "=v"(v) : "v"(p) : "memory"); }
__device__ __forceinline__ unsigned ticket(const char* base, unsigned off) {
    unsigned t; unsigned long long keep;
    asm volatile("s_mov_b64 %1, exec\n\ts_mov_b64 exec, 1\n\tglobal_atomic_add %0, %2, %3, %4 sc0 sc1\n\ts_mov_b64 exec, %1\n\ts_waitcnt vmcnt(0)"
                 : "=&v"(t), "=&s"(keep) : "v"(off), "v"(1u), "s"(base) : "memory");
    return __builtin_amdgcn_readfirstlane(t);
}
__device__ __forceinline__ unsigned piece_min(const char* p) {
    u32x16 f;
    asm volatile("s_load_dwordx16 %0, %1, 0x0 glc\n\ts_waitcnt lgkmcnt(0)" : "=s"(f) : "s"(p) : "memory");
    unsigned m = f[0];
#pragma unroll
    for (int i = 1; i < 15; ++i) m = f[i] < m ? f[i] : m;
    return m;
}
__device__ __forceinline__ void lds_st(unsigned addr, unsigned v) { asm volatile("ds_write_b32 %0, %1" :: "v"(addr), "v"(v) : "memory"); }
__device__ __forceinline__ unsigned lds_ld(unsigned addr) {
    unsigned v; asm volatile("ds_read_b32 %0, %1\n\ts_waitcnt lgkmcnt(0)" : "=v"(v) : "v"(addr) : "memory");
    return __builtin_amdgcn_readfirstlane(v);
}
__device__ __forceinline__ int frag_blk(int n, int c) { return n < NFULL ? c : (c & ~15) + (n - NFULL) * 8 + (c & 7); }
__device__ __forceinline__ int frag_ks(int n, int c) { return n < NFULL ? n : NFULL + ((c >> 3) & 1); }
__device__ __forceinline__ const GAS u32x4* x_src(const signed char* x, int n, int c, int lane) {
    return (const GAS u32x4*)(x + (size_t)(frag_blk(n, c) * 16 + (lane & 15)) * KSH + frag_ks(n, c) * 64 + (lane >> 4) * 16);
}
}

#define CONSUMER_ASM R"ASM(
.macro MM m, n, a
  v_mfma_i32_16x16x64_i8 v[16*(\m)+4*(\n):16*(\m)+4*(\n)+3], v[144+4*(\a):147+4*(\a)], v[128+4*(\n):131+4*(\n)], v[16*(\m)+4*(\n):16*(\m)+4*(\n)+3]
.endm
.macro MM4 m, a
  MM \m, 0, \a
  MM \m, 1, \a
  MM \m, 2, \a
  MM \m, 3, \a
.endm
.macro LDA a, off
  ds_read_b128 v[144+4*(\a):147+4*(\a)], v161 offset:\off
.endm
.macro LDB n
  ds_read_b128 v[128+4*(\n):131+4*(\n)], v162 offset:1024*(\n)
.endm
// waits (bounded) for the next item of this wave's group, passing over and releasing the items of the other groups
.macro SPIN back
  s_mov_b32 s47, 0
1:
  s_cmp_ge_u32 s43, 0x400
  s_cbranch_scc0 2f
  ds_write2_b32 v165, v166, v166 offset0:0 offset1:8
  s_add_u32 s41, s41, 1
  s_cmp_eq_u32 s41, %7
  s_cselect_b32 s44, s50, s52
  s_cselect_b32 s41, 0, s41
  s_add_u32 s42, s42, 1
  s_cmp_eq_u32 s42, %8
  s_cselect_b32 s40, s51, s52
  s_cselect_b32 s42, 0, s42
  v_add_u32 v164, 4, v164
  v_add_u32 v161, s44, v161
  v_add_u32 v162, s40, v162
  v_add_u32 v166, 1, v166
  s_mov_b32 s47, 0
  s_branch 3f
2:
  s_sleep 1
3:
  ds_read_b32 v163, v164 offset:256
  s_add_u32 s47, s47, 1
  s_waitcnt lgkmcnt(0)
  v_readfirstlane_b32 s43, v163
  s_and_b32 s43, s43, %9
  s_cmp_eq_u32 s43, 0x300
  s_cbranch_scc1 \back
  s_cmp_lt_u32 s47, 0x1000000
  s_cbranch_scc1 1b
  v_mov_b32 v163, %3
  v_mov_b32 v144, 1
  ds_write_b32 v163, v144 offset:132
  s_branch FIN_%=
.endm
.macro OUT_ST u, t
  v_cvt_f32_i32 v[\t], v[16*((\u)/4)+((\u)&3)]
  v_cvt_f32_i32 v[\t+1], v[16*((\u)/4)+((\u)&3)+4]
  v_cvt_f32_i32 v[\t+2], v[16*((\u)/4)+((\u)&3)+8]
  v_cvt_f32_i32 v[\t+3], v[16*((\u)/4)+((\u)&3)+12]
  s_add_u32 s54, s58, (((\u)/4)*16+((\u)&3))*16384
  s_addc_u32 s55, s59, 0
  v_mul_f32 v[\t], s49, v[\t]
  v_mul_f32 v[\t+1], s49, v[\t+1]
  v_mul_f32 v[\t+2], s49, v[\t+2]
  v_mul_f32 v[\t+3], s49, v[\t+3]
  global_store_dwordx4 v167, v[\t:\t+3], s[54:55] sc0 sc1
.endm

  v_mbcnt_lo_u32_b32 v160, -1, 0
  v_mbcnt_hi_u32_b32 v160, -1, v160
  v_lshrrev_b32 v167, 4, v160
  v_lshlrev_b32 v167, 16, v167
  v_and_b32 v166, 15, v160
  v_lshl_add_u32 v167, v166, 4, v167
  v_lshlrev_b32 v160, 4, v160
  v_add_u32 v161, %0, v160
  v_add_u32 v162, %1, v160
  v_mov_b32 v164, %3
  v_lshrrev_b32 v165, 2, v160
  v_add_u32 v165, %11, v165
  v_writelane_b32 v165, %2, 0
  v_mov_b32 v166, 1
  s_mov_b64 s[58:59], %4
  s_mov_b32 s49, %5
  s_mov_b32 s41, 0
  s_mov_b32 s42, 0
  s_mov_b32 s43, 0
  s_mov_b32 s45, %6
  s_mov_b32 s50, -(%7-1)*16384
  s_mov_b32 s51, -(%8-1)*16384
  s_mov_b32 s52, 16384
.set zi, 0
.rept 128
  v_mov_b32 v[zi], 0
.set zi, zi+1
.endr
  SPIN START_%=
SPINF_%=:
  s_cmp_eq_u32 s45, 0
  s_cbranch_scc1 TAIL_%=
  SPIN BACK_%=
START_%=:
  LDA 0, 0
  LDA 1, 1024
  LDB 0
  LDB 1
  LDB 2
  LDB 3
  ds_write_b32 v165, v166 offset:32
.p2align 6
  s_nop 0
LOOP_%=:
  s_waitcnt lgkmcnt(4)
  MM 0, 0, 0
  LDA 2, 2048
  MM 1, 0, 1
  LDA 3, 3072
  s_waitcnt lgkmcnt(5)
  MM 0, 1, 0
  MM 1, 1, 1
  s_waitcnt lgkmcnt(4)
  MM 0, 2, 0
  s_sub_u32 s45, s45, 1
  MM 1, 2, 1
  s_waitcnt lgkmcnt(3)
  MM 0, 3, 0
  v_add_u32 v164, 4, v164
  MM 1, 3, 1
  s_waitcnt lgkmcnt(1)
  MM 2, 0, 2
  LDA 0, 4096
  MM 2, 1, 2
  LDA 1, 5120
  MM 2, 2, 2
  s_add_u32 s41, s41, 1
  s_cmp_eq_u32 s41, %7
  MM 2, 3, 2
  s_waitcnt lgkmcnt(2)
  MM 3, 0, 3
  LDA 2, 6144
  MM 3, 1, 3
  s_cselect_b32 s44, s50, s52
  s_cselect_b32 s41, 0, s41
  MM 3, 2, 3
  s_add_u32 s42, s42, 1
  s_cmp_eq_u32 s42, %8
  MM 3, 3, 3
  s_waitcnt lgkmcnt(2)
  MM 4, 0, 0
  LDA 3, 7168
  MM 4, 1, 0
  ds_read_b32 v163, v164 offset:256
  MM 4, 2, 0
  ds_write_b32 v165, v166
  MM 4, 3, 0
  s_cselect_b32 s40, s51, s52
  s_waitcnt lgkmcnt(4)
  MM 5, 0, 1
  s_cselect_b32 s42, 0, s42
  v_add_u32 v161, s44, v161
  MM 5, 1, 1
  v_add_u32 v162, s40, v162
  v_add_u32 v166, 1, v166
  MM 5, 2, 1
  s_waitcnt lgkmcnt(1)
  v_readfirstlane_b32 s43, v163
  MM 5, 3, 1
  s_and_b32 s43, s43, %9
  s_cmp_eq_u32 s43, 0x300
  s_cbranch_scc0 SPINF_%=
BACK_%=:
  MM 6, 0, 2
  LDA 0, 0
  MM 7, 0, 3
  LDA 1, 1024
  MM 6, 1, 2
  LDB 0
  MM 7, 1, 3
  LDB 1
  MM 6, 2, 2
  MM 7, 2, 3
  LDB 2
  MM 6, 3, 2
  MM 7, 3, 3
  LDB 3
  ds_write_b32 v165, v166 offset:32
  s_branch LOOP_%=
FIN_%=:
TAIL_%=:
  v_mov_b32 v166, 0x7fffffff
  v_mov_b32 v163, %3
  s_waitcnt lgkmcnt(0)
  MM 6, 0, 2
  ds_write2_b32 v165, v166, v166 offset0:0 offset1:8
  MM 6, 1, 2
  ds_read_b32 v161, v163 offset:132
  MM 6, 2, 2
  ds_read_b32 v164, v163 offset:152
  MM 6, 3, 2
  s_mov_b32 s47, 0
  MM4 7, 3
8:
  s_add_u32 s47, s47, 1
  s_waitcnt lgkmcnt(0)
  v_readfirstlane_b32 s48, v161
  v_readfirstlane_b32 s43, v164
  s_cmp_lg_u32 s43, 0
  s_cbranch_scc1 9f
  s_sleep 2
  ds_read_b32 v164, v163 offset:152
  s_cmp_lt_u32 s47, 0x1000000
  s_cbranch_scc1 8b
  s_mov_b32 s48, 1
9:
  s_cmp_eq_u32 s48, 0
  s_cselect_b32 s49, s49, 0x7fc00000
  s_cmp_lt_u32 %9, 0x1000
  s_cselect_b64 vcc, -1, 0
.set du, 0
.rept 32
  OUT_ST du, 128+4*(du&7)
  s_cbranch_vccz 4f
.if (du & 3) == 3
  ds_read_b32 v166, v163 offset:128
  s_waitcnt vmcnt(%10) lgkmcnt(0)
  v_cmp_eq_u32 vcc, 0, v166
.else
  s_waitcnt vmcnt(%10)
.endif
4:
.set du, du+1
.endr
.purgem MM
.purgem MM4
.purgem LDA
.purgem LDB
.purgem SPIN
.purgem OUT_ST
)ASM"

#define LOADER_ASM R"ASM(
.macro PUBCHK
  s_getreg_b32 s47, hwreg(HW_REG_IB_STS)
  s_and_b32 s48, s47, 15
  s_bfe_u32 s47, s47, 0x20016
  s_lshl4_add_u32 s47, s47, s48
  s_sub_u32 s47, s40, s47
  s_cmp_ge_u32 s47, s41
  s_cbranch_scc0 7f
  ds_add_u32 v105, v109 offset:256
  s_add_u32 s44, s44, 1
  s_mov_b32 s41, s42
  s_mov_b32 s42, s43
  s_mov_b32 s43, s49
  v_add_u32 v105, v112, v105
7:
.endm
.macro DMA4 va
  global_load_lds_dwordx4 \va, %4 sc0 sc1
  global_load_lds_dwordx4 \va, %4 offset:1024 sc0 sc1
  global_load_lds_dwordx4 \va, %4 offset:2048 sc0 sc1
  global_load_lds_dwordx4 \va, %4 offset:3072 sc0 sc1
  s_add_u32 s40, s40, 4
  s_add_u32 m0, m0, 4096
  v_add_u32 \va, 4096, \va
.endm
.macro BODY hb
  ds_read_b32 v102, v101
LLOOP\hb\()_%=:
  PUBCHK
  s_waitcnt lgkmcnt(0)
  v_cmp_ge_i32 vcc, v102, v100
  v_readlane_b32 s51, v102, 16
  s_cmp_eq_u64 vcc, -1
  s_cselect_b32 s47, s43, 0
  s_cmp_eq_u32 s47, s49
  s_cbranch_scc0 LIDLE\hb\()_%=
LGO\hb\()_%=:
  s_bfe_u32 s52, s51, 0xc000c
  s_lshl_b32 s52, s52, 13
  v_add_u32 v110, s52, %3
  s_bfe_u32 s53, s51, 0x20018
.if \hb == 0
  s_and_b32 s52, s51, 0xfff
  s_lshl_b32 s52, s52, 13
  v_add_u32 v104, s52, %2
  s_cmp_eq_u32 s53, 3
  s_cselect_b32 s52, 8192, 0
  s_add_u32 m0, s46, s52
  s_add_u32 s46, s46, 16384
  DMA4 v104
  v_add_u32 v100, v103, v100
  v_add_u32 v101, v111, v101
  s_cmp_eq_u32 s46, %6
  s_cselect_b32 s46, %5, s46
  PUBCHK
  s_cmp_eq_u32 s53, 0
  s_cbranch_scc0 8f
  DMA4 v104
  PUBCHK
  DMA4 v104
  PUBCHK
8:
  ds_read_b32 v102, v101
  DMA4 v104
  s_cmp_eq_u32 s53, 3
  s_cbranch_scc0 9f
  s_mov_b32 m0, s60
  v_subrev_u32 v110, 0x40000, v110
  PUBCHK
  DMA4 v110
9:
.else
  s_cmp_ge_u32 s53, 2
  s_cselect_b32 s52, 8192, 0
  s_add_u32 m0, s60, s52
  v_add_u32 v100, v103, v100
  DMA4 v110
  v_add_u32 v101, v111, v101
  PUBCHK
  s_cmp_eq_u32 s53, 0
  s_cbranch_scc0 8f
  DMA4 v110
  PUBCHK
  DMA4 v110
  PUBCHK
8:
  ds_read_b32 v102, v101
  DMA4 v110
  s_cmp_eq_u32 s53, 3
  s_cbranch_scc0 9f
  s_add_u32 m0, s60, 4096
  v_subrev_u32 v110, 0x41000, v110
  PUBCHK
  DMA4 v110
9:
.endif
  s_cmp_eq_u32 s42, s49
  s_cselect_b32 s43, s49, s40
  s_cselect_b32 s47, s40, s42
  s_cmp_eq_u32 s41, s49
  s_cselect_b32 s42, s49, s47
  s_cselect_b32 s41, s40, s41
  s_add_u32 s60, s60, 16384
  s_cmp_eq_u32 s60, %7
  s_cselect_b32 s60, %6, s60
  s_add_u32 s45, s45, 1
  s_mov_b32 s50, 0
  s_cmp_lt_u32 s45, %10
  s_cbranch_scc1 LLOOP\hb\()_%=
  s_branch LDRAIN_%=
LIDLE\hb\()_%=:
  ds_read_b32 v102, v101
  s_cmp_eq_u32 s41, s49
  s_cbranch_scc0 LLOOP\hb\()_%=
LWAIT\hb\()_%=:
  s_add_u32 s50, s50, 1
  s_waitcnt lgkmcnt(0)
  v_cmp_ge_i32 vcc, v102, v100
  v_readlane_b32 s51, v102, 16
  s_cmp_eq_u64 vcc, -1
  s_cbranch_scc1 LGO\hb\()_%=
  s_sleep 1
  ds_read_b32 v102, v101
  s_cmp_lt_u32 s50, 0x1c00000
  s_cbranch_scc1 LWAIT\hb\()_%=
  v_mov_b32 v106, 1
  v_mov_b32 v101, %8
  ds_write_b32 v101, v106 offset:132
  s_branch LEND_%=
.endm
  s_mov_b32 s40, 0
  s_mov_b32 s49, 0x7fffffff
  s_mov_b32 s41, s49
  s_mov_b32 s42, s49
  s_mov_b32 s43, s49
  s_mov_b32 s44, 0
  s_mov_b32 s45, 0
  s_mov_b32 s46, %5
  s_mov_b32 s60, %6
  s_mov_b32 s50, 0
  v_mov_b32 v100, %1
  v_mov_b32 v101, %0
  v_mov_b32 v103, 1
  v_writelane_b32 v103, 0, 16
  v_mov_b32 v111, 0
  v_writelane_b32 v111, 4, 16
  v_mov_b32 v105, %11
  v_writelane_b32 v105, %8, 0
  v_mov_b32 v112, 0
  v_writelane_b32 v112, 4, 0
  v_mov_b32 v109, %9
  s_waitcnt vmcnt(0)
  s_cmp_eq_u32 %9, 0x200
  s_cbranch_scc1 LONE_%=
  BODY 0
LONE_%=:
  BODY 1
LDRAIN_%=:
  PUBCHK
  s_cmp_lt_u32 s44, %10
  s_cbranch_scc1 LDRAIN_%=
LEND_%=:
  v_mov_b32 v106, 1
  v_mov_b32 v101, %8
  ds_write_b32 v101, v106 offset:128
  s_waitcnt vmcnt(0) lgkmcnt(0)
.purgem PUBCHK
.purgem DMA4
.purgem BODY
)ASM"
#define R10(r, t) r #t "0", r #t "1", r #t "2", r #t "3", r #t "4", r #t "5", r #t "6", r #t "7", r #t "8", r #t "9"
#define CLOBBER_S R10("s", 4), R10("s", 5), "s60"
#define CLOBBER_V R10("v", ), R10("v", 1), R10("v", 2), R10("v", 3), R10("v", 4), R10("v", 5), R10("v", 6), R10("v", 7), R10("v", 8), R10("v", 9), \
    R10("v", 10), R10("v", 11), R10("v", 12), R10("v", 13), R10("v", 14), R10("v", 15), "v160", "v161", "v162", "v163", "v164", "v165", "v166", "v167"

__global__ void __launch_bounds__(768, 3) gemm_ar_fused(Params P) {
    __shared__ __attribute__((aligned(16))) unsigned char lds_raw[LDS_BYTES];
    LAS unsigned char* lds = (LAS unsigned char*)lds_raw;
    const unsigned lbase = (unsigned)(size_t)lds;
    const int tid = threadIdx.x, lane = tid & 63;
    const int wave = __builtin_amdgcn_readfirstlane(tid >> 6);
    const int b = blockIdx.x;
    const int rank = P.rank;
    char* myws = P.ws[0];
    const int c = ((b >> 5) << 5) | ((b & 7) << 2) | ((b >> 3) & 3);
    const int ti = ((b & 7) >> 1) * 4 + (b >> 6), tj = (b & 1) * 8 + ((b >> 3) & 7);
    const unsigned rot = (unsigned)((c + 32 * rank) & 255);

    if (wave < 8) {
        unsigned warm = 0;
        {
            const unsigned kk = (((rot >> 5) - 4u) * 4u + (rot & 3u)) | ((unsigned)wave << 8);
            const char* pw = P.ws[kk < 14u ? 1 + kk % 7u : 0] + WS_STAGE + (kk >= 7u ? PARITY_BYTES : 0) + (size_t)rank * SENDER_BYTES;
            asm volatile("s_cmp_ge_u32 %2, 14\n\ts_cbranch_scc1 1f\n\ts_load_dword %0, %1, 0x0 glc\n1:" : "+s"(warm) : "s"(pw), "s"(kk) : "memory", "scc");
        }
        __builtin_amdgcn_s_barrier();
        const unsigned call = lds_ld(lbase + L_CALL);
        asm volatile("" :: "s"(warm));
        if (rot < GOPH) { unsigned sp = 0; while (lds_ld(lbase + L_GO) == 0u && ++sp < (1u << 22)) __builtin_amdgcn_s_sleep(2); }
        {
            const int n = wave;
            const int cc = lane & 15, g = lane >> 4;
            const GAS unsigned* wp = (const GAS unsigned*)(P.w + (size_t)(frag_ks(n, c) * 64 + g * 16) * DIM + (frag_blk(n, c) >> 2) * 64 + cc * 4);
            unsigned d[16];
#pragma unroll
            for (int t = 0; t < 16; ++t) d[t] = wp[(size_t)t * (DIM / 4)];
            const int sx = (nth_tab(0) >> (4 * wave)) & 15, sw = (nth_tab(1) >> (4 * wave)) & 15;
            const u32x4 xv = *x_src(P.x, n, c, lane);
            const unsigned q = c & 3;
            const unsigned sel_lo = 0x0c0c0000u | ((4u + q) << 8) | q;
            const unsigned sel_hi = 0x00000c0cu | ((4u + q) << 24) | (q << 16);
            u32x4 wv;
#pragma unroll
            for (int e = 0; e < 4; ++e)
                wv[e] = __builtin_amdgcn_perm(d[4 * e + 1], d[4 * e], sel_lo) | __builtin_amdgcn_perm(d[4 * e + 3], d[4 * e + 2], sel_hi);
            *(LAS u32x4*)(lds + L_WST + n * 1024 + lane * 16) = wv;
            asm volatile("" ::: "memory");
            if (lane == 0) __hip_atomic_fetch_add((LAS unsigned*)(lds + L_WRDY), 1u, __ATOMIC_RELAXED, __HIP_MEMORY_SCOPE_WORKGROUP);
            const unsigned soff = (unsigned)(WS_STAGE + (call & 1u) * PARITY_BYTES + (size_t)rank * SENDER_BYTES) + c * 1024 + lane * 16;
            st16_sys(myws, soff + sx * (unsigned)STEP_BYTES, xv);
            st16_sys(myws, soff + sw * (unsigned)STEP_BYTES, wv);
            wait_vm0();
            if (lane == 0) {
                if (__hip_atomic_fetch_add((LAS unsigned*)(lds + L_LOC), 1u, __ATOMIC_RELAXED, __HIP_MEMORY_SCOPE_WORKGROUP) == 7u) {
                    GAS unsigned* tk = (GAS unsigned*)(myws + WS_AGG + (NSTEP * 16 + (c >> 4)) * 128);
                    if ((__hip_atomic_fetch_add(tk, 1u, __ATOMIC_RELAXED, __HIP_MEMORY_SCOPE_AGENT) & 15u) == 15u)
                        for (int k = 0; k < 2 * MREP; ++k)
                            __hip_atomic_store((GAS unsigned*)(myws + WS_PROG + (k * 16 + (c >> 4)) * PROW + rank * 4), call * 32u + NSTEP, __ATOMIC_RELAXED, __HIP_MEMORY_SCOPE_SYSTEM);
                }
            }
        }
        const int wr = wave >> 2, wc = (wave + 2) & 3;
        const unsigned scale = __builtin_amdgcn_readfirstlane(__float_as_uint(P.sx[0] * P.sw[0]));
        const char* obase = (const char*)(P.out + (size_t)(ti * 256 + wr * 128) * DIM + tj * 256 + wc * 64);
        asm volatile(CONSUMER_ASM
            :: "s"(lbase + L_RA + wr * 8192), "s"(lbase + L_RB + wc * 4096), "s"(lbase + L_REL + wave * 4), "s"(lbase + L_CTL),
               "s"(obase), "s"(scale), "n"(NOWN), "n"(NSA), "n"(NSB), "s"(0x300u | (0x400u << (wr ? 2 : wc >> 1))), "n"(EPW), "s"(lbase + L_DUMMY)
            : "memory", "scc", "vcc", CLOBBER_S, CLOBBER_V);
    } else if (wave == 8) {
        const unsigned tp0 = (unsigned)rt() + T0TICKS;
        __builtin_amdgcn_s_setprio(3);
        GAS unsigned* ctr = (GAS unsigned*)(myws + WS_CTR + (size_t)b * 128);
        unsigned callv; u32x4 xv[8];
        asm volatile("global_load_dword %0, %1, off sc1" : "=v"(callv) : "v"(ctr) : "memory");
        static_for<8>([&](auto nc) __attribute__((always_inline)) {
            constexpr int n = decltype(nc)::value;
            ld16_async(xv[n], x_src(P.x, n, c, lane));
        });
        const char* peer[7];
#pragma unroll
        for (int r = 0; r < 7; ++r) { int o = r + (int)((unsigned)(b >> 3) % 7u); o = o >= 7 ? o - 7 : o; peer[r] = P.ws[1 + o]; }
        const char* fl_lane = peer[0];
#pragma unroll
        for (int r = 1; r < 7; ++r) fl_lane = (lane & 7) == r ? peer[r] : fl_lane;
        fl_lane += WS_PROG + ((c >> 4) + (lane >> 3) * 32) * PROW + rank * 4;
        const bool fl_on = (lane & 7) < 7 && lane < 8 * MREP;
        const unsigned phase = (rot & ~15u) | ((c & 3) << 2) | ((c >> 2) & 3);
        asm volatile("s_waitcnt vmcnt(7)" : "+v"(callv), "+v"(xv[0]) :: "memory");
        const unsigned call = __builtin_amdgcn_readfirstlane(callv) + 1u;
        const unsigned doff = (unsigned)(WS_STAGE + (call & 1u) * PARITY_BYTES + (size_t)rank * SENDER_BYTES) + c * 1024 + lane * 16;
        const unsigned aoff = (unsigned)(WS_AGG + (c >> 4) * 128);
        unsigned tack = 0;
        bool alive = true;
        lds_st(lbase + L_CALL, call); wait_lgkm0();
        if (rot >= GOPH) __builtin_amdgcn_s_barrier();
        static_for<NSTEP>([&](auto sc) __attribute__((always_inline)) {
            constexpr int s = decltype(sc)::value;
            u32x4 v;
            if constexpr (s == 1) {
                pin(xv[1]); pin(xv[2]); pin(xv[3]); pin(xv[4]); pin(xv[5]); pin(xv[6]); pin(xv[7]);
                unsigned sp = 0; while (lds_ld(lbase + L_WRDY) < 8u && ++sp < (1u << 24)) __builtin_amdgcn_s_sleep(1);
                if (sp >= (1u << 24)) alive = false;
            }
            if constexpr (step_w(s)) { v = *(const LAS u32x4*)(lds + L_WST + op_index(s) * 1024 + lane * 16); pin(v); }
            else v = xv[op_index(s)];
            {
                unsigned tgo = tp0 + (s * STEP16 >> 4) + (phase * STEP16 >> 12);
                if constexpr (s > 0) { const int late = (int)(tack + (STEP16 >> 4) - L0T - tgo); if (late > 0) tgo += (unsigned)(late * (8 - ALPHA8) >> 3); }
                int rem;
                while ((rem = (int)(tgo - (unsigned)rt())) > 0) { if (rem > 40) __builtin_amdgcn_s_sleep(8); else __builtin_amdgcn_s_sleep(1); }
            }
#pragma unroll
            for (int r = 0; r < 7; ++r) st16_sys(peer[(r + s) % 7], doff + (unsigned)(s * STEP_BYTES), v);
            if constexpr (s == 0) { if (rot < GOPH) __builtin_amdgcn_s_barrier(); }
            wait_vm0();
            tack = (unsigned)rt();
            if constexpr (s == 0) lds_st(lbase + L_GO, 1u);
            const unsigned prog = call * 32u + s + 1;
            if (alive) {
                if constexpr (s >= TK0 && s < TK1) {
                    if ((ticket(myws, aoff + (unsigned)((s + 1) * 16 * 128)) & 15u) == 15u) { if (fl_on) st4_sys_lane(fl_lane + step_w(s) * 16 * PROW, prog); }
                } else if ((c & 15) != 15) {
                    if (lane == 0) st4_sys(myws, aoff + (c & 15) * 4, prog);
                } else {
                    unsigned sp = 0;
#pragma nounroll
                    while (sp < FWD_SPINS && piece_min(myws + aoff) < prog) ++sp;
                    if (sp >= FWD_SPINS) alive = false;
                    else if (fl_on) st4_sys_lane(fl_lane + step_w(s) * 16 * PROW, prog);
                }
            }
            if constexpr (s >= PS0 && s < PS1) {
                const char* ob = (const char*)(P.out + (size_t)(ti * 256 + 255 - (s - PS0) * PREW) * DIM + tj * 256);
                const u32x4 z = {0u, 0u, 0u, 0u};
#pragma unroll
                for (int k = 0; k < PREW; ++k) st16_sys(ob - (size_t)k * DIM * 4, lane * 16, z);
                if constexpr (s == PS1 - 1) { wait_vm0(); lds_st(lbase + L_PREW, 1u); }
            }
        });
        if (lane == 0) __hip_atomic_store(ctr, call, __ATOMIC_RELAXED, __HIP_MEMORY_SCOPE_AGENT);
        wait_vm0();
    } else if (wave == 9) {
        for (int u = lane; u < (L_DUMMY - L_CTL) / 4; u += 64) if (u != (L_CALL - L_CTL) / 4) lds_st(lbase + L_CTL + u * 4, 0u);
        wait_lgkm0();
        __builtin_amdgcn_s_barrier();
        const unsigned call = lds_ld(lbase + L_CALL);
        __builtin_amdgcn_s_setprio(2);
        GAS unsigned* tmo = (GAS unsigned*)(myws + WS_TMO);
        const int p = lane >> 1, sub = lane & 1;
        const char* px = myws + WS_PROG + (ti + (tj & (MREP - 1)) * 32) * PROW;
        const char* pw = myws + WS_PROG + (16 + tj + (ti & (MREP - 1)) * 32) * PROW;
        unsigned long long tab = sub ? step_tab(1) : step_tab(0);
        unsigned ent = sub;
        unsigned base = lane < 16 ? call * 32u : 0xfffffff0u;
        bool dead = false;
        int queued = 0;
        unsigned spins = 0, lastsum = 0xffffffffu;
        unsigned long long t0 = 0;
        while (queued < NITEM) {
            u32x8 fx, fw;
            asm volatile("s_load_dwordx8 %0, %2, 0x0 glc\n\ts_load_dwordx8 %1, %3, 0x0 glc\n\ts_waitcnt lgkmcnt(0)" : "=&s"(fx), "=&s"(fw) : "s"(px), "s"(pw) : "memory");
            unsigned sum = 0;
#pragma unroll
            for (int i = 0; i < 8; ++i) sum += fx[i] + fw[i];
            if (sum != lastsum) {
                lastsum = sum; spins = 0;
                unsigned val = 0;
                asm volatile("v_writelane_b32 %0, %1, 0\n\tv_writelane_b32 %0, %9, 1\n\tv_writelane_b32 %0, %2, 2\n\tv_writelane_b32 %0, %10, 3\n\t"
                             "v_writelane_b32 %0, %3, 4\n\tv_writelane_b32 %0, %11, 5\n\tv_writelane_b32 %0, %4, 6\n\tv_writelane_b32 %0, %12, 7\n\t"
                             "v_writelane_b32 %0, %5, 8\n\tv_writelane_b32 %0, %13, 9\n\tv_writelane_b32 %0, %6, 10\n\tv_writelane_b32 %0, %14, 11\n\t"
                             "v_writelane_b32 %0, %7, 12\n\tv_writelane_b32 %0, %15, 13\n\tv_writelane_b32 %0, %8, 14\n\tv_writelane_b32 %0, %16, 15"
                             : "+v"(val) : "s"(fx[0]), "s"(fx[1]), "s"(fx[2]), "s"(fx[3]), "s"(fx[4]), "s"(fx[5]), "s"(fx[6]), "s"(fx[7]),
                               "s"(fw[0]), "s"(fw[1]), "s"(fw[2]), "s"(fw[3]), "s"(fw[4]), "s"(fw[5]), "s"(fw[6]), "s"(fw[7]));
                if (dead) val = 0xffffffe0u;
                for (;;) {
                    unsigned bal = (unsigned)__ballot(val > base + ((unsigned)tab & 15u));
                    bal &= bal >> 1; bal &= 0x5555u;
                    if (!bal) break;
                    const bool rdy = lane < 16 && ((bal >> (lane & 14)) & 1u);
                    const bool two = ent - sub >= (unsigned)NFULL;
                    const bool emit = rdy && (sub == 0 || two);
                    const unsigned lead = (unsigned)__ballot(emit);
                    const int pos = queued + __builtin_popcount(lead & ((1u << (lane & 31)) - 1u));
                    if (emit) {
                        const unsigned ty = two ? (ent - (NFULL - 2)) >> 1 : 0u, hf = two ? ent & 1u : 0u;
                        const unsigned sa = (unsigned)(step_tab(0) >> (4 * ent)) & 15u, sb = (unsigned)(step_tab(1) >> (4 * ent)) & 15u;
                        lds_st(lbase + L_QUEUE + pos * 4, two ? 0x1c00u ^ (0x200u << ty) : 0u);
                        lds_st(lbase + L_LDQ + pos * 4, 0x4000000u | ty << 24 | (((p * 16 + sb) << 5 | hf) << 12) | ((p * 16 + sa) << 5 | hf));
                    }
                    const unsigned adv = rdy ? (two ? 2u : 1u) : 0u;
                    ent += adv; tab >>= 4 * adv;
                    if (ent - sub >= (unsigned)IPS) base = 0xfffffff0u;
                    queued += __builtin_popcount(lead);
                }
            } else {
                __builtin_amdgcn_s_sleep(1);
                if ((++spins & 1023u) == 0u) {
                    if (spins == 1024u) t0 = rt();
                    if (__builtin_amdgcn_readfirstlane(__hip_atomic_load(tmo, __ATOMIC_RELAXED, __HIP_MEMORY_SCOPE_AGENT)) == call || rt() - t0 > DEADLINE_TICKS) {
                        if (lane == 0) __hip_atomic_store(tmo, call, __ATOMIC_RELAXED, __HIP_MEMORY_SCOPE_AGENT);
                        lds_st(lbase + L_ABORT, 1u);
                        dead = true; lastsum = ~sum;
                    }
                }
            }
        }
        wait_vm0();
    } else {
        const int hb = wave - 10;
        __builtin_amdgcn_s_barrier();
        const unsigned call = lds_ld(lbase + L_CALL);
        __builtin_amdgcn_s_setprio(2);
        const unsigned goff = (unsigned)(WS_STAGE + (call & 1u) * PARITY_BYTES) + lane * 16;
        const unsigned paddr = lane < 16 ? lbase + L_REL + lane * 4 : lbase + L_LDQ;
        const int thr = lane < 8 ? (hb ? -0x40000000 : 1 - NSA) : (lane < 16 ? 1 - NSB : (lane == 16 ? 1 : -0x40000000));
        asm volatile(LOADER_ASM
            :: "v"(paddr), "v"(thr), "v"(goff + ti * 16384), "v"(goff + tj * 16384), "s"(myws),
               "s"(lbase + L_RA), "s"(lbase + L_RB), "s"(lbase + L_RB + NSB * 16384),
               "s"(lbase + L_CTL), "s"(256u << hb), "n"(NITEM), "v"(lbase + L_DUMMY - 256 + lane * 4)
            : "memory", "scc", "vcc", CLOBBER_S, "v100", "v101", "v102", "v103", "v104", "v105", "v106", "v109", "v110", "v111", "v112");
    }
}

extern "C" void kernel_launch(int rank, int nranks, ncclComm_t comm,
                              hipStream_t stream,
                              void* const* d_in, const int* in_sizes, int n_in,
                              void* d_out, int out_size,
                              const int* in_dtypes, int out_dtype,
                              void* const* d_out_all,
                              void* const* d_ws_all, size_t ws_bytes) {
    (void)comm; (void)in_dtypes; (void)out_dtype; (void)d_out_all;
    if (nranks != NR || n_in != 4 || (unsigned)rank >= (unsigned)NR || ws_bytes < WS_NEED || out_size != DIM * DIM ||
        in_sizes[0] != DIM * KSH || in_sizes[1] != KSH * DIM) {
        fprintf(stderr, "kernel_launch: unexpected problem shape\n");
        return;
    }
    Params P;
    P.x = (const signed char*)d_in[0]; P.w = (const signed char*)d_in[1];
    P.sx = (const float*)d_in[2]; P.sw = (const float*)d_in[3];
    P.out = (float*)d_out; P.rank = rank;
    for (int o = 0; o < NR; ++o) P.ws[o] = (char*)d_ws_all[(rank + o) % NR];
    hipLaunchKernelGGL(gemm_ar_fused, dim3(NWG), dim3(768), 0, stream, P);
    HIP_CHECK(hipGetLastError());
}
```

```cpp
#include <hip/hip_runtime.h>
#include <rccl/rccl.h>
#include <cstdio>
#include <cstdint>
#include <utility>

#define HIP_CHECK(expr) do { hipError_t _e = (expr); if (_e != hipSuccess) { \
    fprintf(stderr, "HIP error %s at %s:%d\n", hipGetErrorString(_e), __FILE__, __LINE__); \
    return; } } while (0)

#define LAS __attribute__((address_space(3)))
#define GAS __attribute__((address_space(1)))
typedef unsigned u32x4 __attribute__((ext_vector_type(4)));
typedef unsigned u32x8 __attribute__((ext_vector_type(8)));
typedef unsigned u32x16 __attribute__((ext_vector_type(16)));

namespace {
constexpr int NR = 8;
constexpr int DIM = 4096;
constexpr int KSH = DIM / NR;
constexpr int NKS = KSH / 64;
constexpr int NWG = 256;
constexpr int NSTEP = 16;
constexpr int NFULL = 6;
constexpr int IPS = 12;
constexpr int NITEM = NR * IPS;
constexpr int NOWN = NR * NKS;
constexpr unsigned STEP16 = 7 * 1018;
constexpr unsigned T0LEAD = 10;
constexpr unsigned L0T = 110;
constexpr int ALPHA8 = 2;
constexpr unsigned GOPH = 128;
constexpr int TK0 = 12, TK1 = 15;
constexpr int EPW = 2;
constexpr int PREW = 22, PS0 = 2, PS1 = 8;
constexpr int NSA = 5, NSB = 4;
constexpr int MREP = 1;
constexpr int PROW = 256;
constexpr int step_w(int s) { return s < 2 * NFULL ? s & 1 : (s == 13 || s == 14); }
constexpr int nth_step(int w, int n) { for (int s = 0; s < NSTEP; ++s) if (step_w(s) == w && n-- == 0) return s; return -1; }
constexpr int op_index(int s) { int n = 0; for (int i = 0; i < s; ++i) n += step_w(i) == step_w(s); return n; }
constexpr unsigned nth_tab(int w) { unsigned t = 0; for (int n = 0; n < 8; ++n) t |= (unsigned)nth_step(w, n) << (4 * n); return t; }
constexpr unsigned long long step_tab(int w) {
    unsigned long long t = 0;
    for (int i = 0; i < IPS; ++i) t |= (unsigned long long)(i < NFULL ? 2 * i + w : (w ? (i < 8 ? 13 : 14) : (i < 10 ? 12 : 15))) << (4 * i);
    return t;
}

constexpr size_t WS_PROG = 0;
constexpr size_t WS_CTR = WS_PROG + 32 * MREP * PROW;
constexpr size_t WS_TMO = WS_CTR + (size_t)NWG * 128;
constexpr size_t WS_AGG = WS_TMO + 256;
constexpr size_t WS_STAGE = 4u << 20;
constexpr size_t STEP_BYTES = (size_t)NWG * 1024;
constexpr size_t SENDER_BYTES = NSTEP * STEP_BYTES;
constexpr size_t PARITY_BYTES = NR * SENDER_BYTES;
constexpr size_t WS_NEED = WS_STAGE + 2 * PARITY_BYTES;
static_assert(WS_AGG + (NSTEP + 1) * 16 * 128 <= WS_STAGE && TK1 < NSTEP, "control region");
constexpr int L_RA = 0;
constexpr int L_RB = NSA * 16384;
constexpr int L_WST = L_RB + NSB * 16384;
constexpr int L_CTL = L_WST + NKS * 1024;
constexpr int L_REL = L_CTL + 64;
constexpr int L_ABORT = L_CTL + 132;
constexpr int L_CALL = L_CTL + 136;
constexpr int L_WRDY = L_CTL + 140;
constexpr int L_LOC = L_CTL + 144;
constexpr int L_GO = L_CTL + 148;
constexpr int L_PREW = L_CTL + 152;
constexpr int L_QUEUE = L_CTL + 256;
constexpr int L_LDQ = L_QUEUE + 128 * 4;
constexpr int L_DUMMY = L_LDQ + 128 * 4;
constexpr int LDS_BYTES = L_DUMMY + 320;
static_assert(LDS_BYTES <= 163840 && NITEM < 128 && (PS1 - PS0) * PREW <= 256, "LDS");
constexpr unsigned long long DEADLINE_TICKS = 100000000ull;
constexpr unsigned FWD_SPINS = 1u << 21;

struct Params {
    const signed char* x; const signed char* w; const float* sx; const float* sw;
    float* out; char* ws[NR]; int rank;
};

template <class F, int... I> __device__ __forceinline__ void static_for_impl(F&& f, std::integer_sequence<int, I...>) { (f(std::integral_constant<int, I>{}), ...); }
template <int N, class F> __device__ __forceinline__ void static_for(F&& f) { static_for_impl(f, std::make_integer_sequence<int, N>{}); }
__device__ __forceinline__ void wait_vm0() { asm volatile("s_waitcnt vmcnt(0)" ::: "memory"); }
__device__ __forceinline__ void wait_lgkm0() { asm volatile("s_waitcnt lgkmcnt(0)" ::: "memory"); }
__device__ __forceinline__ unsigned long long rt() { return __builtin_amdgcn_s_memrealtime(); }
template <class T> __device__ __forceinline__ void pin(T& v) { asm volatile("" : "+v"(v)); }
__device__ __forceinline__ void st16_sys(const char* base, unsigned off, u32x4 v) {
    asm volatile("global_store_dwordx4 %0, %1, %2 sc0 sc1\n\ts_nop 1" :: "v"(off), "v"(v), "s"(base) : "memory");
}
__device__ __forceinline__ void st4_sys(const char* base, unsigned off, unsigned v) {
    asm volatile("global_store_dword %0, %1, %2 sc0 sc1\n\ts_nop 1" :: "v"(off), "v"(v), "s"(base) : "memory");
}
__device__ __forceinline__ void st4_sys_lane(const char* p, unsigned v) { asm volatile("global_store_dword %0, %1, off sc0 sc1\n\ts_nop 1" :: "v"(p), "v"(v) : "memory"); }
__device__ __forceinline__ void ld16_async(u32x4& v, const GAS u32x4* p) { asm volatile("global_load_dwordx4 %0, %1, off" : "=v"(v) : "v"(p) : "memory"); }
__device__ __forceinline__ unsigned ticket(const char* base, unsigned off) {
    unsigned t; unsigned long long keep;
    asm volatile("s_mov_b64 %1, exec\n\ts_mov_b64 exec, 1\n\tglobal_atomic_add %0, %2, %3, %4 sc0 sc1\n\ts_mov_b64 exec, %1\n\ts_waitcnt vmcnt(0)"
                 : "=&v"(t), "=&s"(keep) : "v"(off), "v"(1u), "s"(base) : "memory");
    return __builtin_amdgcn_readfirstlane(t);
}
__device__ __forceinline__ unsigned piece_min(const char* p) {
    u32x16 f;
    asm volatile("s_load_dwordx16 %0, %1, 0x0 glc\n\ts_waitcnt lgkmcnt(0)" : "=s"(f) : "s"(p) : "memory");
    unsigned m = f[0];
#pragma unroll
    for (int i = 1; i < 15; ++i) m = f[i] < m ? f[i] : m;
    return m;
}
__device__ __forceinline__ void lds_st(unsigned addr, unsigned v) { asm volatile("ds_write_b32 %0, %1" :: "v"(addr), "v"(v) : "memory"); }
__device__ __forceinline__ unsigned lds_ld(unsigned addr) {
    unsigned v; asm volatile("ds_read_b32 %0, %1\n\ts_waitcnt lgkmcnt(0)" : "=v"(v) : "v"(addr) : "memory");
    return __builtin_amdgcn_readfirstlane(v);
}
__device__ __forceinline__ int frag_blk(int n, int c) { return n < NFULL ? c : (c & ~15) + (n - NFULL) * 8 + (c & 7); }
__device__ __forceinline__ int frag_ks(int n, int c) { return n < NFULL ? n : NFULL + ((c >> 3) & 1); }
__device__ __forceinline__ const GAS u32x4* x_src(const signed char* x, int n, int c, int lane) {
    return (const GAS u32x4*)(x + (size_t)(frag_blk(n, c) * 16 + (lane & 15)) * KSH + frag_ks(n, c) * 64 + (lane >> 4) * 16);
}
}

#define CONSUMER_ASM R"ASM(
.macro MM m, n, a
  v_mfma_i32_16x16x64_i8 v[16*(\m)+4*(\n):16*(\m)+4*(\n)+3], v[144+4*(\a):147+4*(\a)], v[128+4*(\n):131+4*(\n)], v[16*(\m)+4*(\n):16*(\m)+4*(\n)+3]
.endm
.macro MM4 m, a
  MM \m, 0, \a
  MM \m, 1, \a
  MM \m, 2, \a
  MM \m, 3, \a
.endm
.macro LDA a, off
  ds_read_b128 v[144+4*(\a):147+4*(\a)], v161 offset:\off
.endm
.macro LDB n
  ds_read_b128 v[128+4*(\n):131+4*(\n)], v162 offset:1024*(\n)
.endm
// waits (bounded) for the next item of this wave's group, passing over and releasing the items of the other groups
.macro SPIN back
  s_mov_b32 s47, 0
1:
  s_cmp_ge_u32 s43, 0x400
  s_cbranch_scc0 2f
  ds_write2_b32 v165, v166, v166 offset0:0 offset1:8
  s_add_u32 s41, s41, 1
  s_cmp_eq_u32 s41, %7
  s_cselect_b32 s44, s50, s52
  s_cselect_b32 s41, 0, s41
  s_add_u32 s42, s42, 1
  s_cmp_eq_u32 s42, %8
  s_cselect_b32 s40, s51, s52
  s_cselect_b32 s42, 0, s42
  v_add_u32 v164, 4, v164
  v_add_u32 v161, s44, v161
  v_add_u32 v162, s40, v162
  v_add_u32 v166, 1, v166
  s_mov_b32 s47, 0
  s_branch 3f
2:
  s_sleep 1
3:
  ds_read_b32 v163, v164 offset:256
  s_add_u32 s47, s47, 1
  s_waitcnt lgkmcnt(0)
  v_readfirstlane_b32 s43, v163
  s_and_b32 s43, s43, %9
  s_cmp_eq_u32 s43, 0x300
  s_cbranch_scc1 \back
  s_cmp_lt_u32 s47, 0x1000000
  s_cbranch_scc1 1b
  v_mov_b32 v163, %3
  v_mov_b32 v144, 1
  ds_write_b32 v163, v144 offset:132
  s_branch FIN_%=
.endm
.macro OUT_ST u, t
  v_cvt_f32_i32 v[\t], v[16*((\u)/4)+((\u)&3)]
  v_cvt_f32_i32 v[\t+1], v[16*((\u)/4)+((\u)&3)+4]
  v_cvt_f32_i32 v[\t+2], v[16*((\u)/4)+((\u)&3)+8]
  v_cvt_f32_i32 v[\t+3], v[16*((\u)/4)+((\u)&3)+12]
  s_add_u32 s54, s58, (((\u)/4)*16+((\u)&3))*16384
  s_addc_u32 s55, s59, 0
  v_mul_f32 v[\t], s49, v[\t]
  v_mul_f32 v[\t+1], s49, v[\t+1]
  v_mul_f32 v[\t+2], s49, v[\t+2]
  v_mul_f32 v[\t+3], s49, v[\t+3]
  global_store_dwordx4 v167, v[\t:\t+3], s[54:55] sc0 sc1
.endm

  v_mbcnt_lo_u32_b32 v160, -1, 0
  v_mbcnt_hi_u32_b32 v160, -1, v160
  v_lshrrev_b32 v167, 4, v160
  v_lshlrev_b32 v167, 16, v167
  v_and_b32 v166, 15, v160
  v_lshl_add_u32 v167, v166, 4, v167
  v_lshlrev_b32 v160, 4, v160
  v_add_u32 v161, %0, v160
  v_add_u32 v162, %1, v160
  v_mov_b32 v164, %3
  v_lshrrev_b32 v165, 2, v160
  v_add_u32 v165, %11, v165
  v_writelane_b32 v165, %2, 0
  v_mov_b32 v166, 1
  s_mov_b64 s[58:59], %4
  s_mov_b32 s49, %5
  s_mov_b32 s41, 0
  s_mov_b32 s42, 0
  s_mov_b32 s43, 0
  s_mov_b32 s45, %6
  s_mov_b32 s50, -(%7-1)*16384
  s_mov_b32 s51, -(%8-1)*16384
  s_mov_b32 s52, 16384
.set zi, 0
.rept 128
  v_mov_b32 v[zi], 0
.set zi, zi+1
.endr
  SPIN START_%=
SPINF_%=:
  s_cmp_eq_u32 s45, 0
  s_cbranch_scc1 TAIL_%=
  SPIN BACK_%=
START_%=:
  LDA 0, 0
  LDA 1, 1024
  LDB 0
  LDB 1
  LDB 2
  LDB 3
  ds_write_b32 v165, v166 offset:32
.p2align 6
  s_nop 0
LOOP_%=:
  s_waitcnt lgkmcnt(4)
  MM 0, 0, 0
  LDA 2, 2048
  MM 1, 0, 1
  LDA 3, 3072
  s_waitcnt lgkmcnt(5)
  MM 0, 1, 0
  MM 1, 1, 1
  s_waitcnt lgkmcnt(4)
  MM 0, 2, 0
  s_sub_u32 s45, s45, 1
  MM 1, 2, 1
  s_waitcnt lgkmcnt(3)
  MM 0, 3, 0
  v_add_u32 v164, 4, v164
  MM 1, 3, 1
  s_waitcnt lgkmcnt(1)
  MM 2, 0, 2
  LDA 0, 4096
  MM 2, 1, 2
  LDA 1, 5120
  MM 2, 2, 2
  s_add_u32 s41, s41, 1
  s_cmp_eq_u32 s41, %7
  MM 2, 3, 2
  s_waitcnt lgkmcnt(2)
  MM 3, 0, 3
  LDA 2, 6144
  MM 3, 1, 3
  s_cselect_b32 s44, s50, s52
  s_cselect_b32 s41, 0, s41
  MM 3, 2, 3
  s_add_u32 s42, s42, 1
  s_cmp_eq_u32 s42, %8
  MM 3, 3, 3
  s_waitcnt lgkmcnt(2)
  MM 4, 0, 0
  LDA 3, 7168
  MM 4, 1, 0
  ds_read_b32 v163, v164 offset:256
  MM 4, 2, 0
  ds_write_b32 v165, v166
  MM 4, 3, 0
  s_cselect_b32 s40, s51, s52
  s_waitcnt lgkmcnt(4)
  MM 5, 0, 1
  s_cselect_b32 s42, 0, s42
  v_add_u32 v161, s44, v161
  MM 5, 1, 1
  v_add_u32 v162, s40, v162
  v_add_u32 v166, 1, v166
  MM 5, 2, 1
  s_waitcnt lgkmcnt(1)
  v_readfirstlane_b32 s43, v163
  MM 5, 3, 1
  s_and_b32 s43, s43, %9
  s_cmp_eq_u32 s43, 0x300
  s_cbranch_scc0 SPINF_%=
BACK_%=:
  MM 6, 0, 2
  LDA 0, 0
  MM 7, 0, 3
  LDA 1, 1024
  MM 6, 1, 2
  LDB 0
  MM 7, 1, 3
  LDB 1
  MM 6, 2, 2
  MM 7, 2, 3
  LDB 2
  MM 6, 3, 2
  MM 7, 3, 3
  LDB 3
  ds_write_b32 v165, v166 offset:32
  s_branch LOOP_%=
FIN_%=:
TAIL_%=:
  v_mov_b32 v166, 0x7fffffff
  v_mov_b32 v163, %3
  s_waitcnt lgkmcnt(0)
  MM 6, 0, 2
  ds_write2_b32 v165, v166, v166 offset0:0 offset1:8
  MM 6, 1, 2
  ds_read_b32 v161, v163 offset:132
  MM 6, 2, 2
  ds_read_b32 v164, v163 offset:152
  MM 6, 3, 2
  s_mov_b32 s47, 0
  MM4 7, 3
8:
  s_add_u32 s47, s47, 1
  s_waitcnt lgkmcnt(0)
  v_readfirstlane_b32 s48, v161
  v_readfirstlane_b32 s43, v164
  s_cmp_lg_u32 s43, 0
  s_cbranch_scc1 9f
  s_sleep 2
  ds_read_b32 v164, v163 offset:152
  s_cmp_lt_u32 s47, 0x1000000
  s_cbranch_scc1 8b
  s_mov_b32 s48, 1
9:
  s_cmp_eq_u32 s48, 0
  s_cselect_b32 s49, s49, 0x7fc00000
  s_cmp_lt_u32 %9, 0x1000
  s_cselect_b64 vcc, -1, 0
.set du, 0
.rept 32
  OUT_ST du, 128+4*(du&7)
  s_cbranch_vccz 4f
.if (du & 3) == 3
  ds_read_b32 v166, v163 offset:128
  s_waitcnt vmcnt(%10) lgkmcnt(0)
  v_cmp_eq_u32 vcc, 0, v166
.else
  s_waitcnt vmcnt(%10)
.endif
4:
.set du, du+1
.endr
.purgem MM
.purgem MM4
.purgem LDA
.purgem LDB
.purgem SPIN
.purgem OUT_ST
)ASM"

#define LOADER_ASM R"ASM(
.macro PUBCHK
  s_getreg_b32 s47, hwreg(HW_REG_IB_STS)
  s_and_b32 s48, s47, 15
  s_bfe_u32 s47, s47, 0x20016
  s_lshl4_add_u32 s47, s47, s48
  s_sub_u32 s47, s40, s47
  s_cmp_ge_u32 s47, s41
  s_cbranch_scc0 7f
  ds_add_u32 v105, v109 offset:256
  s_add_u32 s44, s44, 1
  s_mov_b32 s41, s42
  s_mov_b32 s42, s43
  s_mov_b32 s43, s49
  v_add_u32 v105, v112, v105
7:
.endm
.macro DMA4 va
  global_load_lds_dwordx4 \va, %4 sc0 sc1
  global_load_lds_dwordx4 \va, %4 offset:1024 sc0 sc1
  global_load_lds_dwordx4 \va, %4 offset:2048 sc0 sc1
  global_load_lds_dwordx4 \va, %4 offset:3072 sc0 sc1
  s_add_u32 s40, s40, 4
  s_add_u32 m0, m0, 4096
  v_add_u32 \va, 4096, \va
.endm
.macro BODY hb
  ds_read_b32 v102, v101
LLOOP\hb\()_%=:
  PUBCHK
  s_waitcnt lgkmcnt(0)
  v_cmp_ge_i32 vcc, v102, v100
  v_readlane_b32 s51, v102, 16
  s_cmp_eq_u64 vcc, -1
  s_cselect_b32 s47, s43, 0
  s_cmp_eq_u32 s47, s49
  s_cbranch_scc0 LIDLE\hb\()_%=
LGO\hb\()_%=:
  s_bfe_u32 s52, s51, 0xc000c
  s_lshl_b32 s52, s52, 13
  v_add_u32 v110, s52, %3
  s_bfe_u32 s53, s51, 0x20018
.if \hb == 0
  s_and_b32 s52, s51, 0xfff
  s_lshl_b32 s52, s52, 13
  v_add_u32 v104, s52, %2
  s_cmp_eq_u32 s53, 3
  s_cselect_b32 s52, 8192, 0
  s_add_u32 m0, s46, s52
  s_add_u32 s46, s46, 16384
  DMA4 v104
  v_add_u32 v100, v103, v100
  v_add_u32 v101, v111, v101
  s_cmp_eq_u32 s46, %6
  s_cselect_b32 s46, %5, s46
  PUBCHK
  s_cmp_eq_u32 s53, 0
  s_cbranch_scc0 8f
  DMA4 v104
  PUBCHK
  DMA4 v104
  PUBCHK
8:
  ds_read_b32 v102, v101
  DMA4 v104
  s_cmp_eq_u32 s53, 3
  s_cbranch_scc0 9f
  s_mov_b32 m0, s60
  v_subrev_u32 v110, 0x40000, v110
  PUBCHK
  DMA4 v110
9:
.else
  s_cmp_ge_u32 s53, 2
  s_cselect_b32 s52, 8192, 0
  s_add_u32 m0, s60, s52
  v_add_u32 v100, v103, v100
  DMA4 v110
  v_add_u32 v101, v111, v101
  PUBCHK
  s_cmp_eq_u32 s53, 0
  s_cbranch_scc0 8f
  DMA4 v110
  PUBCHK
  DMA4 v110
  PUBCHK
8:
  ds_read_b32 v102, v101
  DMA4 v110
  s_cmp_eq_u32 s53, 3
  s_cbranch_scc0 9f
  s_add_u32 m0, s60, 4096
  v_subrev_u32 v110, 0x41000, v110
  PUBCHK
  DMA4 v110
9:
.endif
  s_cmp_eq_u32 s42, s49
  s_cselect_b32 s43, s49, s40
  s_cselect_b32 s47, s40, s42
  s_cmp_eq_u32 s41, s49
  s_cselect_b32 s42, s49, s47
  s_cselect_b32 s41, s40, s41
  s_add_u32 s60, s60, 16384
  s_cmp_eq_u32 s60, %7
  s_cselect_b32 s60, %6, s60
  s_add_u32 s45, s45, 1
  s_mov_b32 s50, 0
  s_cmp_lt_u32 s45, %10
  s_cbranch_scc1 LLOOP\hb\()_%=
  s_branch LDRAIN_%=
LIDLE\hb\()_%=:
  ds_read_b32 v102, v101
  s_cmp_eq_u32 s41, s49
  s_cbranch_scc0 LLOOP\hb\()_%=
LWAIT\hb\()_%=:
  s_add_u32 s50, s50, 1
  s_waitcnt lgkmcnt(0)
  v_cmp_ge_i32 vcc, v102, v100
  v_readlane_b32 s51, v102, 16
  s_cmp_eq_u64 vcc, -1
  s_cbranch_scc1 LGO\hb\()_%=
  s_sleep 1
  ds_read_b32 v102, v101
  s_cmp_lt_u32 s50, 0x1c00000
  s_cbranch_scc1 LWAIT\hb\()_%=
  v_mov_b32 v106, 1
  v_mov_b32 v101, %8
  ds_write_b32 v101, v106 offset:132
  s_branch LEND_%=
.endm
  s_mov_b32 s40, 0
  s_mov_b32 s49, 0x7fffffff
  s_mov_b32 s41, s49
  s_mov_b32 s42, s49
  s_mov_b32 s43, s49
  s_mov_b32 s44, 0
  s_mov_b32 s45, 0
  s_mov_b32 s46, %5
  s_mov_b32 s60, %6
  s_mov_b32 s50, 0
  v_mov_b32 v100, %1
  v_mov_b32 v101, %0
  v_mov_b32 v103, 1
  v_writelane_b32 v103, 0, 16
  v_mov_b32 v111, 0
  v_writelane_b32 v111, 4, 16
  v_mov_b32 v105, %11
  v_writelane_b32 v105, %8, 0
  v_mov_b32 v112, 0
  v_writelane_b32 v112, 4, 0
  v_mov_b32 v109, %9
  s_waitcnt vmcnt(0)
  s_cmp_eq_u32 %9, 0x200
  s_cbranch_scc1 LONE_%=
  BODY 0
LONE_%=:
  BODY 1
LDRAIN_%=:
  PUBCHK
  s_cmp_lt_u32 s44, %10
  s_cbranch_scc1 LDRAIN_%=
LEND_%=:
  v_mov_b32 v106, 1
  v_mov_b32 v101, %8
  ds_write_b32 v101, v106 offset:128
  s_waitcnt vmcnt(0) lgkmcnt(0)
.purgem PUBCHK
.purgem DMA4
.purgem BODY
)ASM"
#define R10(r, t) r #t "0", r #t "1", r #t "2", r #t "3", r #t "4", r #t "5", r #t "6", r #t "7", r #t "8", r #t "9"
#define CLOBBER_S R10("s", 4), R10("s", 5), "s60"
#define CLOBBER_V R10("v", ), R10("v", 1), R10("v", 2), R10("v", 3), R10("v", 4), R10("v", 5), R10("v", 6), R10("v", 7), R10("v", 8), R10("v", 9), \
    R10("v", 10), R10("v", 11), R10("v", 12), R10("v", 13), R10("v", 14), R10("v", 15), "v160", "v161", "v162", "v163", "v164", "v165", "v166", "v167"

__global__ void __launch_bounds__(768, 3) gemm_ar_fused(Params P) {
    __shared__ __attribute__((aligned(16))) unsigned char lds_raw[LDS_BYTES];
    LAS unsigned char* lds = (LAS unsigned char*)lds_raw;
    const unsigned lbase = (unsigned)(size_t)lds;
    const int tid = threadIdx.x, lane = tid & 63;
    const int wave = __builtin_amdgcn_readfirstlane(tid >> 6);
    const int b = blockIdx.x;
    char* myws = P.ws[0];
    const int c = ((b >> 5) << 5) | ((b & 7) << 2) | ((b >> 3) & 3);
    unsigned callv, warm = 0; u32x4 x0;
    asm volatile("s_cmp_lg_u32 %6, 8\n\ts_cbranch_scc1 1f\n\tglobal_load_dwordx4 %1, %4, %5\n\tglobal_load_dword %0, %2, %3 sc1\n1:"
                 : "=&v"(callv), "=&v"(x0) : "v"((unsigned)(WS_CTR + b * 128)), "s"(myws), "v"((unsigned)((c * 16 + (lane & 15)) * KSH + (lane >> 4) * 16)), "s"(P.x), "s"(wave), "s"(P.rank)
                 : "memory", "scc");
    int rank = P.rank; asm volatile("" : "+s"(rank));
    const int ti = ((b & 7) >> 1) * 4 + (b >> 6), tj = (b & 1) * 8 + ((b >> 3) & 7);
    const unsigned rot = (unsigned)((c + 32 * rank) & 255);
    {
        const unsigned kk = (((rot >> 5) - 4u) * 4u + (rot & 3u)) | ((unsigned)wave << 8);
        const char* pw = P.ws[kk < 14u ? 1 + (kk >= 7u ? kk - 7u : kk) : 0] + WS_STAGE + (kk >= 7u ? PARITY_BYTES : 0) + (size_t)rank * SENDER_BYTES;
        asm volatile("s_cmp_ge_u32 %2, 14\n\ts_cbranch_scc1 1f\n\ts_load_dword %0, %1, 0x0 glc\n1:" : "+s"(warm) : "s"(pw), "s"(kk) : "memory", "scc");
    }

    if (wave < 8) {
        __builtin_amdgcn_s_barrier();
        const unsigned call = lds_ld(lbase + L_CALL);
        asm volatile("" :: "s"(warm));
        if (rot < GOPH) { unsigned sp = 0; while (lds_ld(lbase + L_GO) == 0u && ++sp < (1u << 22)) __builtin_amdgcn_s_sleep(2); }
        {
            const int n = wave;
            const int cc = lane & 15, g = lane >> 4;
            const GAS unsigned* wp = (const GAS unsigned*)(P.w + (size_t)(frag_ks(n, c) * 64 + g * 16) * DIM + (frag_blk(n, c) >> 2) * 64 + cc * 4);
            unsigned d[16];
#pragma unroll
            for (int t = 0; t < 16; ++t) d[t] = wp[(size_t)t * (DIM / 4)];
            const int sx = (nth_tab(0) >> (4 * wave)) & 15, sw = (nth_tab(1) >> (4 * wave)) & 15;
            const u32x4 xv = *x_src(P.x, n, c, lane);
            const unsigned q = c & 3;
            const unsigned sel_lo = 0x0c0c0000u | ((4u + q) << 8) | q;
            const unsigned sel_hi = 0x00000c0cu | ((4u + q) << 24) | (q << 16);
            u32x4 wv;
#pragma unroll
            for (int e = 0; e < 4; ++e)
                wv[e] = __builtin_amdgcn_perm(d[4 * e + 1], d[4 * e], sel_lo) | __builtin_amdgcn_perm(d[4 * e + 3], d[4 * e + 2], sel_hi);
            *(LAS u32x4*)(lds + L_WST + n * 1024 + lane * 16) = wv;
            asm volatile("" ::: "memory");
            if (lane == 0) __hip_atomic_fetch_add((LAS unsigned*)(lds + L_WRDY), 1u, __ATOMIC_RELAXED, __HIP_MEMORY_SCOPE_WORKGROUP);
            const unsigned soff = (unsigned)(WS_STAGE + (call & 1u) * PARITY_BYTES + (size_t)rank * SENDER_BYTES) + c * 1024 + lane * 16;
            st16_sys(myws, soff + sx * (unsigned)STEP_BYTES, xv);
            st16_sys(myws, soff + sw * (unsigned)STEP_BYTES, wv);
            wait_vm0();
            if (lane == 0) {
                if (__hip_atomic_fetch_add((LAS unsigned*)(lds + L_LOC), 1u, __ATOMIC_RELAXED, __HIP_MEMORY_SCOPE_WORKGROUP) == 7u) {
                    GAS unsigned* tk = (GAS unsigned*)(myws + WS_AGG + (NSTEP * 16 + (c >> 4)) * 128);
                    if ((__hip_atomic_fetch_add(tk, 1u, __ATOMIC_RELAXED, __HIP_MEMORY_SCOPE_AGENT) & 15u) == 15u)
                        for (int k = 0; k < 2 * MREP; ++k)
                            __hip_atomic_store((GAS unsigned*)(myws + WS_PROG + (k * 16 + (c >> 4)) * PROW + rank * 4), call * 32u + NSTEP, __ATOMIC_RELAXED, __HIP_MEMORY_SCOPE_SYSTEM);
                }
            }
        }
        const int wr = wave >> 2, wc = (wave + 2) & 3;
        const unsigned scale = __builtin_amdgcn_readfirstlane(__float_as_uint(P.sx[0] * P.sw[0]));
        const char* obase = (const char*)(P.out + (size_t)(ti * 256 + wr * 128) * DIM + tj * 256 + wc * 64);
        asm volatile(CONSUMER_ASM
            :: "s"(lbase + L_RA + wr * 8192), "s"(lbase + L_RB + wc * 4096), "s"(lbase + L_REL + wave * 4), "s"(lbase + L_CTL),
               "s"(obase), "s"(scale), "n"(NOWN), "n"(NSA), "n"(NSB), "s"(0x300u | (0x400u << (wr ? 2 : wc >> 1))), "n"(EPW), "s"(lbase + L_DUMMY)
            : "memory", "scc", "vcc", CLOBBER_S, CLOBBER_V);
    } else if (wave == 8) {
        __builtin_amdgcn_s_setprio(3);
        GAS unsigned* ctr = (GAS unsigned*)(myws + WS_CTR + (size_t)b * 128);
        u32x4 xv[8];
        static_for<7>([&](auto nc) __attribute__((always_inline)) {
            constexpr int n = 1 + decltype(nc)::value;
            ld16_async(xv[n], x_src(P.x, n, c, lane));
        });
        const char* peer[7];
#pragma unroll
        for (int r = 0; r < 7; ++r) { int o = r + (int)((unsigned)(b >> 3) % 7u); o = o >= 7 ? o - 7 : o; peer[r] = P.ws[1 + o]; }
        const char* fl_lane = peer[0];
#pragma unroll
        for (int r = 1; r < 7; ++r) fl_lane = (lane & 7) == r ? peer[r] : fl_lane;
        fl_lane += WS_PROG + ((c >> 4) + (lane >> 3) * 32) * PROW + rank * 4;
        const bool fl_on = (lane & 7) < 7 && lane < 8 * MREP;
        const unsigned phase = (rot & ~15u) | ((c & 3) << 2) | ((c >> 2) & 3);
        asm volatile("s_waitcnt vmcnt(7)" : "+v"(callv), "+v"(x0) :: "memory");
        xv[0] = x0;
        const unsigned tp0 = (unsigned)rt() - T0LEAD;
        const unsigned call = __builtin_amdgcn_readfirstlane(callv) + 1u;
        const unsigned doff = (unsigned)(WS_STAGE + (call & 1u) * PARITY_BYTES + (size_t)rank * SENDER_BYTES) + c * 1024 + lane * 16;
        const unsigned aoff = (unsigned)(WS_AGG + (c >> 4) * 128);
        unsigned tack = 0;
        bool alive = true;
        lds_st(lbase + L_CALL, call); wait_lgkm0();
        if (rot >= GOPH) __builtin_amdgcn_s_barrier();
        static_for<NSTEP>([&](auto sc) __attribute__((always_inline)) {
            constexpr int s = decltype(sc)::value;
            u32x4 v;
            if constexpr (s == 1) {
                pin(xv[1]); pin(xv[2]); pin(xv[3]); pin(xv[4]); pin(xv[5]); pin(xv[6]); pin(xv[7]);
                unsigned sp = 0; while (lds_ld(lbase + L_WRDY) < 8u && ++sp < (1u << 24)) __builtin_amdgcn_s_sleep(1);
                if (sp >= (1u << 24)) alive = false;
            }
            if constexpr (step_w(s)) { v = *(const LAS u32x4*)(lds + L_WST + op_index(s) * 1024 + lane * 16); pin(v); }
            else v = xv[op_index(s)];
            {
                unsigned tgo = tp0 + (s * STEP16 >> 4) + (phase * STEP16 >> 12);
                if constexpr (s > 0) { const int late = (int)(tack + (STEP16 >> 4) - L0T - tgo); if (late > 0) tgo += (unsigned)(late * (8 - ALPHA8) >> 3); }
                int rem;
                while ((rem = (int)(tgo - (unsigned)rt())) > 0) { if (rem > 40) __builtin_amdgcn_s_sleep(8); else __builtin_amdgcn_s_sleep(1); }
            }
#pragma unroll
            for (int r = 0; r < 7; ++r) st16_sys(peer[(r + s) % 7], doff + (unsigned)(s * STEP_BYTES), v);
            if constexpr (s == 0) { if (rot < GOPH) __builtin_amdgcn_s_barrier(); }
            wait_vm0();
            tack = (unsigned)rt();
            if constexpr (s == 0) lds_st(lbase + L_GO, 1u);
            const unsigned prog = call * 32u + s + 1;
            if (alive) {
                if constexpr (s >= TK0 && s < TK1) {
                    if ((ticket(myws, aoff + (unsigned)((s + 1) * 16 * 128)) & 15u) == 15u) { if (fl_on) st4_sys_lane(fl_lane + step_w(s) * 16 * PROW, prog); }
                } else if ((c & 15) != 15) {
                    if (lane == 0) st4_sys(myws, aoff + (c & 15) * 4, prog);
                } else {
                    unsigned sp = 0;
#pragma nounroll
                    while (sp < FWD_SPINS && piece_min(myws + aoff) < prog) { if ((++sp & 63u) == 0u) __builtin_amdgcn_fence(__ATOMIC_ACQUIRE, ""); }
                    if (sp >= FWD_SPINS) alive = false;
                    else if (fl_on) st4_sys_lane(fl_lane + step_w(s) * 16 * PROW, prog);
                }
            }
            if constexpr (s >= PS0 && s < PS1) {
                const char* ob = (const char*)(P.out + (size_t)(ti * 256 + 255 - (s - PS0) * PREW) * DIM + tj * 256);
                const u32x4 z = {0u, 0u, 0u, 0u};
#pragma unroll
                for (int k = 0; k < PREW; ++k) st16_sys(ob - (size_t)k * DIM * 4, lane * 16, z);
                if constexpr (s == PS1 - 1) { wait_vm0(); lds_st(lbase + L_PREW, 1u); }
            }
        });
        if (lane == 0) __hip_atomic_store(ctr, call, __ATOMIC_RELAXED, __HIP_MEMORY_SCOPE_AGENT);
        wait_vm0();
    } else if (wave == 9) {
        for (int u = lane; u < (L_DUMMY - L_CTL) / 4; u += 64) if (u != (L_CALL - L_CTL) / 4) lds_st(lbase + L_CTL + u * 4, 0u);
        wait_lgkm0();
        __builtin_amdgcn_s_barrier();
        const unsigned call = lds_ld(lbase + L_CALL);
        __builtin_amdgcn_s_setprio(2);
        GAS unsigned* tmo = (GAS unsigned*)(myws + WS_TMO);
        const int p = lane >> 1, sub = lane & 1;
        const char* px = myws + WS_PROG + (ti + (tj & (MREP - 1)) * 32) * PROW;
        const char* pw = myws + WS_PROG + (16 + tj + (ti & (MREP - 1)) * 32) * PROW;
        unsigned long long tab = sub ? step_tab(1) : step_tab(0);
        unsigned ent = sub;
        unsigned base = lane < 16 ? call * 32u : 0xfffffff0u;
        bool dead = false;
        int queued = 0;
        unsigned spins = 0, lastsum = 0xffffffffu;
        unsigned long long t0 = 0;
        while (queued < NITEM) {
            u32x8 fx, fw;
            asm volatile("s_load_dwordx8 %0, %2, 0x0 glc\n\ts_load_dwordx8 %1, %3, 0x0 glc\n\ts_waitcnt lgkmcnt(0)" : "=&s"(fx), "=&s"(fw) : "s"(px), "s"(pw) : "memory");
            unsigned sum = 0;
#pragma unroll
            for (int i = 0; i < 8; ++i) sum += fx[i] + fw[i];
            if (sum != lastsum) {
                lastsum = sum; spins = 0;
                unsigned val = 0;
                asm volatile("v_writelane_b32 %0, %1, 0\n\tv_writelane_b32 %0, %9, 1\n\tv_writelane_b32 %0, %2, 2\n\tv_writelane_b32 %0, %10, 3\n\t"
                             "v_writelane_b32 %0, %3, 4\n\tv_writelane_b32 %0, %11, 5\n\tv_writelane_b32 %0, %4, 6\n\tv_writelane_b32 %0, %12, 7\n\t"
                             "v_writelane_b32 %0, %5, 8\n\tv_writelane_b32 %0, %13, 9\n\tv_writelane_b32 %0, %6, 10\n\tv_writelane_b32 %0, %14, 11\n\t"
                             "v_writelane_b32 %0, %7, 12\n\tv_writelane_b32 %0, %15, 13\n\tv_writelane_b32 %0, %8, 14\n\tv_writelane_b32 %0, %16, 15"
                             : "+v"(val) : "s"(fx[0]), "s"(fx[1]), "s"(fx[2]), "s"(fx[3]), "s"(fx[4]), "s"(fx[5]), "s"(fx[6]), "s"(fx[7]),
                               "s"(fw[0]), "s"(fw[1]), "s"(fw[2]), "s"(fw[3]), "s"(fw[4]), "s"(fw[5]), "s"(fw[6]), "s"(fw[7]));
                if (dead) val = 0xffffffe0u;
                for (;;) {
                    unsigned bal = (unsigned)__ballot(val > base + ((unsigned)tab & 15u));
                    bal &= bal >> 1; bal &= 0x5555u;
                    if (!bal) break;
                    const bool rdy = lane < 16 && ((bal >> (lane & 14)) & 1u);
                    const bool two = ent - sub >= (unsigned)NFULL;
                    const bool emit = rdy && (sub == 0 || two);
                    const unsigned lead = (unsigned)__ballot(emit);
                    const int pos = queued + __builtin_popcount(lead & ((1u << (lane & 31)) - 1u));
                    if (emit) {
                        const unsigned ty = two ? (ent - (NFULL - 2)) >> 1 : 0u, hf = two ? ent & 1u : 0u;
                        const unsigned sa = (unsigned)(step_tab(0) >> (4 * ent)) & 15u, sb = (unsigned)(step_tab(1) >> (4 * ent)) & 15u;
                        lds_st(lbase + L_QUEUE + pos * 4, two ? 0x1c00u ^ (0x200u << ty) : 0u);
                        lds_st(lbase + L_LDQ + pos * 4, 0x4000000u | ty << 24 | (((p * 16 + sb) << 5 | hf) << 12) | ((p * 16 + sa) << 5 | hf));
                    }
                    const unsigned adv = rdy ? (two ? 2u : 1u) : 0u;
                    ent += adv; tab >>= 4 * adv;
                    if (ent - sub >= (unsigned)IPS) base = 0xfffffff0u;
                    queued += __builtin_popcount(lead);
                }
            } else {
                __builtin_amdgcn_s_sleep(1);
                if ((++spins & 63u) == 0u) __builtin_amdgcn_fence(__ATOMIC_ACQUIRE, "");
                if ((spins & 1023u) == 0u) {
                    if (spins == 1024u) t0 = rt();
                    if (__builtin_amdgcn_readfirstlane(__hip_atomic_load(tmo, __ATOMIC_RELAXED, __HIP_MEMORY_SCOPE_AGENT)) == call || rt() - t0 > DEADLINE_TICKS) {
                        if (lane == 0) __hip_atomic_store(tmo, call, __ATOMIC_RELAXED, __HIP_MEMORY_SCOPE_AGENT);
                        lds_st(lbase + L_ABORT, 1u);
                        dead = true; lastsum = ~sum;
                    }
                }
            }
        }
        wait_vm0();
    } else {
        const int hb = wave - 10;
        __builtin_amdgcn_s_barrier();
        const unsigned call = lds_ld(lbase + L_CALL);
        __builtin_amdgcn_s_setprio(2);
        const unsigned goff = (unsigned)(WS_STAGE + (call & 1u) * PARITY_BYTES) + lane * 16;
        const unsigned paddr = lane < 16 ? lbase + L_REL + lane * 4 : lbase + L_LDQ;
        const int thr = lane < 8 ? (hb ? -0x40000000 : 1 - NSA) : (lane < 16 ? 1 - NSB : (lane == 16 ? 1 : -0x40000000));
        asm volatile(LOADER_ASM
            :: "v"(paddr), "v"(thr), "v"(goff + ti * 16384), "v"(goff + tj * 16384), "s"(myws),
               "s"(lbase + L_RA), "s"(lbase + L_RB), "s"(lbase + L_RB + NSB * 16384),
               "s"(lbase + L_CTL), "s"(256u << hb), "n"(NITEM), "v"(lbase + L_DUMMY - 256 + lane * 4)
            : "memory", "scc", "vcc", CLOBBER_S, "v100", "v101", "v102", "v103", "v104", "v105", "v106", "v109", "v110", "v111", "v112");
    }
}

extern "C" void kernel_launch(int rank, int nranks, ncclComm_t comm,
                              hipStream_t stream,
                              void* const* d_in, const int* in_sizes, int n_in,
                              void* d_out, int out_size,
                              const int* in_dtypes, int out_dtype,
                              void* const* d_out_all,
                              void* const* d_ws_all, size_t ws_bytes) {
    (void)comm; (void)in_dtypes; (void)out_dtype; (void)d_out_all;
    if (nranks != NR || n_in != 4 || (unsigned)rank >= (unsigned)NR || ws_bytes < WS_NEED || out_size != DIM * DIM ||
        in_sizes[0] != DIM * KSH || in_sizes[1] != KSH * DIM) {
        fprintf(stderr, "kernel_launch: unexpected problem shape\n");
        return;
    }
    Params P;
    P.x = (const signed char*)d_in[0]; P.w = (const signed char*)d_in[1];
    P.sx = (const float*)d_in[2]; P.sw = (const float*)d_in[3];
    P.out = (float*)d_out; P.rank = rank;
    for (int o = 0; o < NR; ++o) P.ws[o] = (char*)d_ws_all[(rank + o) % NR];
    hipLaunchKernelGGL(gemm_ar_fused, dim3(NWG), dim3(768), 0, stream, P);
    HIP_CHECK(hipGetLastError());
}
```

```cpp
#include <hip/hip_runtime.h>
#include <rccl/rccl.h>
#include <cstdio>
#include <cstdint>
#include <utility>

#define HIP_CHECK(expr) do { hipError_t _e = (expr); if (_e != hipSuccess) { \
    fprintf(stderr, "HIP error %s at %s:%d\n", hipGetErrorString(_e), __FILE__, __LINE__); \
    return; } } while (0)

#define LAS __attribute__((address_space(3)))
#define GAS __attribute__((address_space(1)))
typedef unsigned u32x4 __attribute__((ext_vector_type(4)));
typedef unsigned u32x8 __attribute__((ext_vector_type(8)));
typedef unsigned u32x16 __attribute__((ext_vector_type(16)));

namespace {
constexpr int NR = 8;
constexpr int DIM = 4096;
constexpr int KSH = DIM / NR;
constexpr int NKS = KSH / 64;
constexpr int NWG = 256;
constexpr int NSTEP = 16;
constexpr int NFULL = 6;
constexpr int IPS = 12;
constexpr int NITEM = NR * IPS;
constexpr int NOWN = NR * NKS;
constexpr unsigned STEP16 = 7 * 1020;
constexpr unsigned T0LEAD = 10;
constexpr unsigned L0T = 105;
constexpr int ALPHA8 = 2;
constexpr unsigned GOPH = 128;
constexpr int TK0 = 12, TK1 = 15;
constexpr int EPW = 2;
constexpr int PREW = 22, PS0 = 2, PS1 = 8;
constexpr int NSA = 5, NSB = 4;
constexpr int MREP = 1;
constexpr int PROW = 256;
constexpr int step_w(int s) { return s < 2 * NFULL ? s & 1 : (s == 13 || s == 14); }
constexpr int nth_step(int w, int n) { for (int s = 0; s < NSTEP; ++s) if (step_w(s) == w && n-- == 0) return s; return -1; }
constexpr int op_index(int s) { int n = 0; for (int i = 0; i < s; ++i) n += step_w(i) == step_w(s); return n; }
constexpr unsigned nth_tab(int w) { unsigned t = 0; for (int n = 0; n < 8; ++n) t |= (unsigned)nth_step(w, n) << (4 * n); return t; }
constexpr unsigned long long step_tab(int w) {
    unsigned long long t = 0;
    for (int i = 0; i < IPS; ++i) t |= (unsigned long long)(i < NFULL ? 2 * i + w : (w ? (i < 8 ? 13 : 14) : (i < 10 ? 12 : 15))) << (4 * i);
    return t;
}

constexpr size_t WS_PROG = 0;
constexpr size_t WS_CTR = WS_PROG + 32 * MREP * PROW;
constexpr size_t WS_TMO = WS_CTR + (size_t)NWG * 128;
constexpr size_t WS_ORG = WS_TMO + 128;
constexpr size_t WS_AGG = WS_TMO + 256;
constexpr size_t WS_STAGE = 4u << 20;
constexpr size_t STEP_BYTES = (size_t)NWG * 1024;
constexpr size_t SENDER_BYTES = NSTEP * STEP_BYTES;
constexpr size_t PARITY_BYTES = NR * SENDER_BYTES;
constexpr size_t WS_NEED = WS_STAGE + 2 * PARITY_BYTES;
static_assert(WS_AGG + (NSTEP + 1) * 16 * 128 <= WS_STAGE && TK1 < NSTEP, "control region");
constexpr int L_RA = 0;
constexpr int L_RB = NSA * 16384;
constexpr int L_WST = L_RB + NSB * 16384;
constexpr int L_CTL = L_WST + NKS * 1024;
constexpr int L_REL = L_CTL + 64;
constexpr int L_ABORT = L_CTL + 132;
constexpr int L_CALL = L_CTL + 136;
constexpr int L_WRDY = L_CTL + 140;
constexpr int L_LOC = L_CTL + 144;
constexpr int L_GO = L_CTL + 148;
constexpr int L_PREW = L_CTL + 152;
constexpr int L_QUEUE = L_CTL + 256;
constexpr int L_LDQ = L_QUEUE + 128 * 4;
constexpr int L_DUMMY = L_LDQ + 128 * 4;
constexpr int LDS_BYTES = L_DUMMY + 320;
static_assert(LDS_BYTES <= 163840 && NITEM < 128 && (PS1 - PS0) * PREW <= 256, "LDS");
constexpr unsigned long long DEADLINE_TICKS = 100000000ull;
constexpr unsigned FWD_SPINS = 1u << 21;

struct Params {
    const signed char* x; const signed char* w; const float* sx; const float* sw;
    float* out; char* ws[NR]; int rank;
};

template <class F, int... I> __device__ __forceinline__ void static_for_impl(F&& f, std::integer_sequence<int, I...>) { (f(std::integral_constant<int, I>{}), ...); }
template <int N, class F> __device__ __forceinline__ void static_for(F&& f) { static_for_impl(f, std::make_integer_sequence<int, N>{}); }
__device__ __forceinline__ void wait_vm0() { asm volatile("s_waitcnt vmcnt(0)" ::: "memory"); }
__device__ __forceinline__ void wait_lgkm0() { asm volatile("s_waitcnt lgkmcnt(0)" ::: "memory"); }
__device__ __forceinline__ unsigned long long rt() { return __builtin_amdgcn_s_memrealtime(); }
template <class T> __device__ __forceinline__ void pin(T& v) { asm volatile("" : "+v"(v)); }
__device__ __forceinline__ void st16_sys(const char* base, unsigned off, u32x4 v) {
    asm volatile("global_store_dwordx4 %0, %1, %2 sc0 sc1\n\ts_nop 1" :: "v"(off), "v"(v), "s"(base) : "memory");
}
__device__ __forceinline__ void st4_sys(const char* base, unsigned off, unsigned v) {
    asm volatile("global_store_dword %0, %1, %2 sc0 sc1\n\ts_nop 1" :: "v"(off), "v"(v), "s"(base) : "memory");
}
__device__ __forceinline__ void st4_sys_lane(const char* p, unsigned v) { asm volatile("global_store_dword %0, %1, off sc0 sc1\n\ts_nop 1" :: "v"(p), "v"(v) : "memory"); }
__device__ __forceinline__ void ld16_async(u32x4& v, const GAS u32x4* p) { asm volatile("global_load_dwordx4 %0, %1, off" : "=v"(v) : "v"(p) : "memory"); }
__device__ __forceinline__ unsigned ticket(const char* base, unsigned off) {
    unsigned t; unsigned long long keep;
    asm volatile("s_mov_b64 %1, exec\n\ts_mov_b64 exec, 1\n\tglobal_atomic_add %0, %2, %3, %4 sc0 sc1\n\ts_mov_b64 exec, %1\n\ts_waitcnt vmcnt(0)"
                 : "=&v"(t), "=&s"(keep) : "v"(off), "v"(1u), "s"(base) : "memory");
    return __builtin_amdgcn_readfirstlane(t);
}
__device__ __forceinline__ unsigned piece_min(const char* p) {
    u32x16 f;
    asm volatile("s_load_dwordx16 %0, %1, 0x0 glc\n\ts_waitcnt lgkmcnt(0)" : "=s"(f) : "s"(p) : "memory");
    unsigned m = f[0];
#pragma unroll
    for (int i = 1; i < 15; ++i) m = f[i] < m ? f[i] : m;
    return m;
}
__device__ __forceinline__ unsigned long long sld2_glc(const char* p) {
    unsigned long long d; asm volatile("s_load_dwordx2 %0, %1, 0x0 glc\n\ts_waitcnt lgkmcnt(0)" : "=s"(d) : "s"(p) : "memory"); return d;
}
__device__ __forceinline__ void lds_st(unsigned addr, unsigned v) { asm volatile("ds_write_b32 %0, %1" :: "v"(addr), "v"(v) : "memory"); }
__device__ __forceinline__ unsigned lds_ld(unsigned addr) {
    unsigned v; asm volatile("ds_read_b32 %0, %1\n\ts_waitcnt lgkmcnt(0)" : "=v"(v) : "v"(addr) : "memory");
    return __builtin_amdgcn_readfirstlane(v);
}
__device__ __forceinline__ int frag_blk(int n, int c) { return n < NFULL ? c : (c & ~15) + (n - NFULL) * 8 + (c & 7); }
__device__ __forceinline__ int frag_ks(int n, int c) { return n < NFULL ? n : NFULL + ((c >> 3) & 1); }
__device__ __forceinline__ const GAS u32x4* x_src(const signed char* x, int n, int c, int lane) {
    return (const GAS u32x4*)(x + (size_t)(frag_blk(n, c) * 16 + (lane & 15)) * KSH + frag_ks(n, c) * 64 + (lane >> 4) * 16);
}
}

#define CONSUMER_ASM R"ASM(
.macro MM m, n, a
  v_mfma_i32_16x16x64_i8 v[16*(\m)+4*(\n):16*(\m)+4*(\n)+3], v[144+4*(\a):147+4*(\a)], v[128+4*(\n):131+4*(\n)], v[16*(\m)+4*(\n):16*(\m)+4*(\n)+3]
.endm
.macro MM4 m, a
  MM \m, 0, \a
  MM \m, 1, \a
  MM \m, 2, \a
  MM \m, 3, \a
.endm
.macro LDA a, off
  ds_read_b128 v[144+4*(\a):147+4*(\a)], v161 offset:\off
.endm
.macro LDB n
  ds_read_b128 v[128+4*(\n):131+4*(\n)], v162 offset:1024*(\n)
.endm
// waits (bounded) for the next item of this wave's group, passing over and releasing the items of the other groups
.macro SPIN back
  s_mov_b32 s47, 0
1:
  s_cmp_ge_u32 s43, 0x400
  s_cbranch_scc0 2f
  ds_write2_b32 v165, v166, v166 offset0:0 offset1:8
  s_add_u32 s41, s41, 1
  s_cmp_eq_u32 s41, %7
  s_cselect_b32 s44, s50, s52
  s_cselect_b32 s41, 0, s41
  s_add_u32 s42, s42, 1
  s_cmp_eq_u32 s42, %8
  s_cselect_b32 s40, s51, s52
  s_cselect_b32 s42, 0, s42
  v_add_u32 v164, 4, v164
  v_add_u32 v161, s44, v161
  v_add_u32 v162, s40, v162
  v_add_u32 v166, 1, v166
  s_mov_b32 s47, 0
  s_branch 3f
2:
  s_sleep 1
3:
  ds_read_b32 v163, v164 offset:256
  s_add_u32 s47, s47, 1
  s_waitcnt lgkmcnt(0)
  v_readfirstlane_b32 s43, v163
  s_and_b32 s43, s43, %9
  s_cmp_eq_u32 s43, 0x300
  s_cbranch_scc1 \back
  s_cmp_lt_u32 s47, 0x1000000
  s_cbranch_scc1 1b
  v_mov_b32 v163, %3
  v_mov_b32 v144, 1
  ds_write_b32 v163, v144 offset:132
  s_branch FIN_%=
.endm
.macro OUT_ST u, t
  v_cvt_f32_i32 v[\t], v[16*((\u)/4)+((\u)&3)]
  v_cvt_f32_i32 v[\t+1], v[16*((\u)/4)+((\u)&3)+4]
  v_cvt_f32_i32 v[\t+2], v[16*((\u)/4)+((\u)&3)+8]
  v_cvt_f32_i32 v[\t+3], v[16*((\u)/4)+((\u)&3)+12]
  s_add_u32 s54, s58, (((\u)/4)*16+((\u)&3))*16384
  s_addc_u32 s55, s59, 0
  v_mul_f32 v[\t], s49, v[\t]
  v_mul_f32 v[\t+1], s49, v[\t+1]
  v_mul_f32 v[\t+2], s49, v[\t+2]
  v_mul_f32 v[\t+3], s49, v[\t+3]
  global_store_dwordx4 v167, v[\t:\t+3], s[54:55] sc0 sc1
.endm

  v_mbcnt_lo_u32_b32 v160, -1, 0
  v_mbcnt_hi_u32_b32 v160, -1, v160
  v_lshrrev_b32 v167, 4, v160
  v_lshlrev_b32 v167, 16, v167
  v_and_b32 v166, 15, v160
  v_lshl_add_u32 v167, v166, 4, v167
  v_lshlrev_b32 v160, 4, v160
  v_add_u32 v161, %0, v160
  v_add_u32 v162, %1, v160
  v_mov_b32 v164, %3
  v_lshrrev_b32 v165, 2, v160
  v_add_u32 v165, %11, v165
  v_writelane_b32 v165, %2, 0
  v_mov_b32 v166, 1
  s_mov_b64 s[58:59], %4
  s_mov_b32 s49, %5
  s_mov_b32 s41, 0
  s_mov_b32 s42, 0
  s_mov_b32 s43, 0
  s_mov_b32 s45, %6
  s_mov_b32 s50, -(%7-1)*16384
  s_mov_b32 s51, -(%8-1)*16384
  s_mov_b32 s52, 16384
.set zi, 0
.rept 128
  v_mov_b32 v[zi], 0
.set zi, zi+1
.endr
  SPIN START_%=
SPINF_%=:
  s_cmp_eq_u32 s45, 0
  s_cbranch_scc1 TAIL_%=
  SPIN BACK_%=
START_%=:
  LDA 0, 0
  LDA 1, 1024
  LDB 0
  LDB 1
  LDB 2
  LDB 3
  ds_write_b32 v165, v166 offset:32
.p2align 6
  s_nop 0
LOOP_%=:
  s_waitcnt lgkmcnt(4)
  MM 0, 0, 0
  LDA 2, 2048
  MM 1, 0, 1
  LDA 3, 3072
  s_waitcnt lgkmcnt(5)
  MM 0, 1, 0
  MM 1, 1, 1
  s_waitcnt lgkmcnt(4)
  MM 0, 2, 0
  s_sub_u32 s45, s45, 1
  MM 1, 2, 1
  s_waitcnt lgkmcnt(3)
  MM 0, 3, 0
  v_add_u32 v164, 4, v164
  MM 1, 3, 1
  s_waitcnt lgkmcnt(1)
  MM 2, 0, 2
  LDA 0, 4096
  MM 2, 1, 2
  LDA 1, 5120
  MM 2, 2, 2
  s_add_u32 s41, s41, 1
  s_cmp_eq_u32 s41, %7
  MM 2, 3, 2
  s_waitcnt lgkmcnt(2)
  MM 3, 0, 3
  LDA 2, 6144
  MM 3, 1, 3
  s_cselect_b32 s44, s50, s52
  s_cselect_b32 s41, 0, s41
  MM 3, 2, 3
  s_add_u32 s42, s42, 1
  s_cmp_eq_u32 s42, %8
  MM 3, 3, 3
  s_waitcnt lgkmcnt(2)
  MM 4, 0, 0
  LDA 3, 7168
  MM 4, 1, 0
  ds_read_b32 v163, v164 offset:256
  MM 4, 2, 0
  ds_write_b32 v165, v166
  MM 4, 3, 0
  s_cselect_b32 s40, s51, s52
  s_waitcnt lgkmcnt(4)
  MM 5, 0, 1
  s_cselect_b32 s42, 0, s42
  v_add_u32 v161, s44, v161
  MM 5, 1, 1
  v_add_u32 v162, s40, v162
  v_add_u32 v166, 1, v166
  MM 5, 2, 1
  s_waitcnt lgkmcnt(1)
  v_readfirstlane_b32 s43, v163
  MM 5, 3, 1
  s_and_b32 s43, s43, %9
  s_cmp_eq_u32 s43, 0x300
  s_cbranch_scc0 SPINF_%=
BACK_%=:
  MM 6, 0, 2
  LDA 0, 0
  MM 7, 0, 3
  LDA 1, 1024
  MM 6, 1, 2
  LDB 0
  MM 7, 1, 3
  LDB 1
  MM 6, 2, 2
  MM 7, 2, 3
  LDB 2
  MM 6, 3, 2
  MM 7, 3, 3
  LDB 3
  ds_write_b32 v165, v166 offset:32
  s_branch LOOP_%=
FIN_%=:
TAIL_%=:
  v_mov_b32 v166, 0x7fffffff
  v_mov_b32 v163, %3
  s_waitcnt lgkmcnt(0)
  MM 6, 0, 2
  ds_write2_b32 v165, v166, v166 offset0:0 offset1:8
  MM 6, 1, 2
  ds_read_b32 v161, v163 offset:132
  MM 6, 2, 2
  ds_read_b32 v164, v163 offset:152
  MM 6, 3, 2
  s_mov_b32 s47, 0
  MM4 7, 3
8:
  s_add_u32 s47, s47, 1
  s_waitcnt lgkmcnt(0)
  v_readfirstlane_b32 s48, v161
  v_readfirstlane_b32 s43, v164
  s_cmp_lg_u32 s43, 0
  s_cbranch_scc1 9f
  s_sleep 2
  ds_read_b32 v164, v163 offset:152
  s_cmp_lt_u32 s47, 0x1000000
  s_cbranch_scc1 8b
  s_mov_b32 s48, 1
9:
  s_cmp_eq_u32 s48, 0
  s_cselect_b32 s49, s49, 0x7fc00000
  s_cmp_lt_u32 %9, 0x1000
  s_cselect_b64 vcc, -1, 0
.set du, 0
.rept 32
  OUT_ST du, 128+4*(du&7)
  s_cbranch_vccz 4f
.if (du & 3) == 3
  ds_read_b32 v166, v163 offset:128
  s_waitcnt vmcnt(%10) lgkmcnt(0)
  v_cmp_eq_u32 vcc, 0, v166
.else
  s_waitcnt vmcnt(%10)
.endif
4:
.set du, du+1
.endr
.purgem MM
.purgem MM4
.purgem LDA
.purgem LDB
.purgem SPIN
.purgem OUT_ST
)ASM"

#define LOADER_ASM R"ASM(
.macro PUBCHK
  s_getreg_b32 s47, hwreg(HW_REG_IB_STS)
  s_and_b32 s48, s47, 15
  s_bfe_u32 s47, s47, 0x20016
  s_lshl4_add_u32 s47, s47, s48
  s_sub_u32 s47, s40, s47
  s_cmp_ge_u32 s47, s41
  s_cbranch_scc0 7f
  ds_add_u32 v105, v109 offset:256
  s_add_u32 s44, s44, 1
  s_mov_b32 s41, s42
  s_mov_b32 s42, s43
  s_mov_b32 s43, s49
  v_add_u32 v105, v112, v105
7:
.endm
.macro DMA4 va
  global_load_lds_dwordx4 \va, %4 sc0 sc1
  global_load_lds_dwordx4 \va, %4 offset:1024 sc0 sc1
  global_load_lds_dwordx4 \va, %4 offset:2048 sc0 sc1
  global_load_lds_dwordx4 \va, %4 offset:3072 sc0 sc1
  s_add_u32 s40, s40, 4
  s_add_u32 m0, m0, 4096
  v_add_u32 \va, 4096, \va
.endm
.macro BODY hb
  ds_read_b32 v102, v101
LLOOP\hb\()_%=:
  PUBCHK
  s_waitcnt lgkmcnt(0)
  v_cmp_ge_i32 vcc, v102, v100
  v_readlane_b32 s51, v102, 16
  s_cmp_eq_u64 vcc, -1
  s_cselect_b32 s47, s43, 0
  s_cmp_eq_u32 s47, s49
  s_cbranch_scc0 LIDLE\hb\()_%=
LGO\hb\()_%=:
  s_bfe_u32 s52, s51, 0xc000c
  s_lshl_b32 s52, s52, 13
  v_add_u32 v110, s52, %3
  s_bfe_u32 s53, s51, 0x20018
.if \hb == 0
  s_and_b32 s52, s51, 0xfff
  s_lshl_b32 s52, s52, 13
  v_add_u32 v104, s52, %2
  s_cmp_eq_u32 s53, 3
  s_cselect_b32 s52, 8192, 0
  s_add_u32 m0, s46, s52
  s_add_u32 s46, s46, 16384
  DMA4 v104
  v_add_u32 v100, v103, v100
  v_add_u32 v101, v111, v101
  s_cmp_eq_u32 s46, %6
  s_cselect_b32 s46, %5, s46
  PUBCHK
  s_cmp_eq_u32 s53, 0
  s_cbranch_scc0 8f
  DMA4 v104
  PUBCHK
  DMA4 v104
  PUBCHK
8:
  ds_read_b32 v102, v101
  DMA4 v104
  s_cmp_eq_u32 s53, 3
  s_cbranch_scc0 9f
  s_mov_b32 m0, s60
  v_subrev_u32 v110, 0x40000, v110
  PUBCHK
  DMA4 v110
9:
.else
  s_cmp_ge_u32 s53, 2
  s_cselect_b32 s52, 8192, 0
  s_add_u32 m0, s60, s52
  v_add_u32 v100, v103, v100
  DMA4 v110
  v_add_u32 v101, v111, v101
  PUBCHK
  s_cmp_eq_u32 s53, 0
  s_cbranch_scc0 8f
  DMA4 v110
  PUBCHK
  DMA4 v110
  PUBCHK
8:
  ds_read_b32 v102, v101
  DMA4 v110
  s_cmp_eq_u32 s53, 3
  s_cbranch_scc0 9f
  s_add_u32 m0, s60, 4096
  v_subrev_u32 v110, 0x41000, v110
  PUBCHK
  DMA4 v110
9:
.endif
  s_cmp_eq_u32 s42, s49
  s_cselect_b32 s43, s49, s40
  s_cselect_b32 s47, s40, s42
  s_cmp_eq_u32 s41, s49
  s_cselect_b32 s42, s49, s47
  s_cselect_b32 s41, s40, s41
  s_add_u32 s60, s60, 16384
  s_cmp_eq_u32 s60, %7
  s_cselect_b32 s60, %6, s60
  s_add_u32 s45, s45, 1
  s_mov_b32 s50, 0
  s_cmp_lt_u32 s45, %10
  s_cbranch_scc1 LLOOP\hb\()_%=
  s_branch LDRAIN_%=
LIDLE\hb\()_%=:
  ds_read_b32 v102, v101
  s_cmp_eq_u32 s41, s49
  s_cbranch_scc0 LLOOP\hb\()_%=
LWAIT\hb\()_%=:
  s_add_u32 s50, s50, 1
  s_waitcnt lgkmcnt(0)
  v_cmp_ge_i32 vcc, v102, v100
  v_readlane_b32 s51, v102, 16
  s_cmp_eq_u64 vcc, -1
  s_cbranch_scc1 LGO\hb\()_%=
  s_sleep 1
  ds_read_b32 v102, v101
  s_cmp_lt_u32 s50, 0x1c00000
  s_cbranch_scc1 LWAIT\hb\()_%=
  v_mov_b32 v106, 1
  v_mov_b32 v101, %8
  ds_write_b32 v101, v106 offset:132
  s_branch LEND_%=
.endm
  s_mov_b32 s40, 0
  s_mov_b32 s49, 0x7fffffff
  s_mov_b32 s41, s49
  s_mov_b32 s42, s49
  s_mov_b32 s43, s49
  s_mov_b32 s44, 0
  s_mov_b32 s45, 0
  s_mov_b32 s46, %5
  s_mov_b32 s60, %6
  s_mov_b32 s50, 0
  v_mov_b32 v100, %1
  v_mov_b32 v101, %0
  v_mov_b32 v103, 1
  v_writelane_b32 v103, 0, 16
  v_mov_b32 v111, 0
  v_writelane_b32 v111, 4, 16
  v_mov_b32 v105, %11
  v_writelane_b32 v105, %8, 0
  v_mov_b32 v112, 0
  v_writelane_b32 v112, 4, 0
  v_mov_b32 v109, %9
  s_waitcnt vmcnt(0)
  s_cmp_eq_u32 %9, 0x200
  s_cbranch_scc1 LONE_%=
  BODY 0
LONE_%=:
  BODY 1
LDRAIN_%=:
  PUBCHK
  s_cmp_lt_u32 s44, %10
  s_cbranch_scc1 LDRAIN_%=
LEND_%=:
  v_mov_b32 v106, 1
  v_mov_b32 v101, %8
  ds_write_b32 v101, v106 offset:128
  s_waitcnt vmcnt(0) lgkmcnt(0)
.purgem PUBCHK
.purgem DMA4
.purgem BODY
)ASM"
#define R10(r, t) r #t "0", r #t "1", r #t "2", r #t "3", r #t "4", r #t "5", r #t "6", r #t "7", r #t "8", r #t "9"
#define CLOBBER_S R10("s", 4), R10("s", 5), "s60"
#define CLOBBER_V R10("v", ), R10("v", 1), R10("v", 2), R10("v", 3), R10("v", 4), R10("v", 5), R10("v", 6), R10("v", 7), R10("v", 8), R10("v", 9), \
    R10("v", 10), R10("v", 11), R10("v", 12), R10("v", 13), R10("v", 14), R10("v", 15), "v160", "v161", "v162", "v163", "v164", "v165", "v166", "v167"

__global__ void __launch_bounds__(768, 3) gemm_ar_fused(Params P) {
    __shared__ __attribute__((aligned(16))) unsigned char lds_raw[LDS_BYTES];
    LAS unsigned char* lds = (LAS unsigned char*)lds_raw;
    const unsigned lbase = (unsigned)(size_t)lds;
    const int tid = threadIdx.x, lane = tid & 63;
    const int wave = __builtin_amdgcn_readfirstlane(tid >> 6);
    const int b = blockIdx.x;
    char* myws = P.ws[0];
    const int c = ((b >> 5) << 5) | ((b & 7) << 2) | ((b >> 3) & 3);
    unsigned callv, warm = 0; u32x4 x0;
    asm volatile("s_cmp_lg_u32 %6, 8\n\ts_cbranch_scc1 1f\n\tglobal_load_dwordx4 %1, %4, %5\n\tglobal_load_dword %0, %2, %3 sc1\n1:"
                 : "=&v"(callv), "=&v"(x0) : "v"((unsigned)(WS_CTR + b * 128)), "s"(myws), "v"((unsigned)((c * 16 + (lane & 15)) * KSH + (lane >> 4) * 16)), "s"(P.x), "s"(wave), "s"(P.rank)
                 : "memory", "scc");
    int rank = P.rank; asm volatile("" : "+s"(rank));
    const int ti = ((b & 7) >> 1) * 4 + (b >> 6), tj = (b & 1) * 8 + ((b >> 3) & 7);
    const unsigned rot = (unsigned)((c + 32 * rank) & 255);
    {
        const unsigned kk = (((rot >> 5) - 4u) * 4u + (rot & 3u)) | ((unsigned)wave << 8);
        const char* pw = P.ws[kk < 14u ? 1 + (kk >= 7u ? kk - 7u : kk) : 0] + WS_STAGE + (kk >= 7u ? PARITY_BYTES : 0) + (size_t)rank * SENDER_BYTES;
        asm volatile("s_cmp_ge_u32 %2, 14\n\ts_cbranch_scc1 1f\n\ts_load_dword %0, %1, 0x0 glc\n1:" : "+s"(warm) : "s"(pw), "s"(kk) : "memory", "scc");
    }

    if (wave < 8) {
        __builtin_amdgcn_s_barrier();
        const unsigned call = lds_ld(lbase + L_CALL);
        asm volatile("" :: "s"(warm));
        if (rot < GOPH) { unsigned sp = 0; while (lds_ld(lbase + L_GO) == 0u && ++sp < (1u << 22)) __builtin_amdgcn_s_sleep(2); }
        {
            const int n = wave;
            const int cc = lane & 15, g = lane >> 4;
            const GAS unsigned* wp = (const GAS unsigned*)(P.w + (size_t)(frag_ks(n, c) * 64 + g * 16) * DIM + (frag_blk(n, c) >> 2) * 64 + cc * 4);
            unsigned d[16];
#pragma unroll
            for (int t = 0; t < 16; ++t) d[t] = wp[(size_t)t * (DIM / 4)];
            const int sx = (nth_tab(0) >> (4 * wave)) & 15, sw = (nth_tab(1) >> (4 * wave)) & 15;
            const u32x4 xv = *x_src(P.x, n, c, lane);
            const unsigned q = c & 3;
            const unsigned sel_lo = 0x0c0c0000u | ((4u + q) << 8) | q;
            const unsigned sel_hi = 0x00000c0cu | ((4u + q) << 24) | (q << 16);
            u32x4 wv;
#pragma unroll
            for (int e = 0; e < 4; ++e)
                wv[e] = __builtin_amdgcn_perm(d[4 * e + 1], d[4 * e], sel_lo) | __builtin_amdgcn_perm(d[4 * e + 3], d[4 * e + 2], sel_hi);
            *(LAS u32x4*)(lds + L_WST + n * 1024 + lane * 16) = wv;
            asm volatile("" ::: "memory");
            if (lane == 0) __hip_atomic_fetch_add((LAS unsigned*)(lds + L_WRDY), 1u, __ATOMIC_RELAXED, __HIP_MEMORY_SCOPE_WORKGROUP);
            const unsigned soff = (unsigned)(WS_STAGE + (call & 1u) * PARITY_BYTES + (size_t)rank * SENDER_BYTES) + c * 1024 + lane * 16;
            st16_sys(myws, soff + sx * (unsigned)STEP_BYTES, xv);
            st16_sys(myws, soff + sw * (unsigned)STEP_BYTES, wv);
            wait_vm0();
            if (lane == 0) {
                if (__hip_atomic_fetch_add((LAS unsigned*)(lds + L_LOC), 1u, __ATOMIC_RELAXED, __HIP_MEMORY_SCOPE_WORKGROUP) == 7u) {
                    GAS unsigned* tk = (GAS unsigned*)(myws + WS_AGG + (NSTEP * 16 + (c >> 4)) * 128);
                    if ((__hip_atomic_fetch_add(tk, 1u, __ATOMIC_RELAXED, __HIP_MEMORY_SCOPE_AGENT) & 15u) == 15u)
                        for (int k = 0; k < 2 * MREP; ++k)
                            __hip_atomic_store((GAS unsigned*)(myws + WS_PROG + (k * 16 + (c >> 4)) * PROW + rank * 4), call * 32u + NSTEP, __ATOMIC_RELAXED, __HIP_MEMORY_SCOPE_SYSTEM);
                }
            }
        }
        const int wr = wave >> 2, wc = (wave + 2) & 3;
        const unsigned scale = __builtin_amdgcn_readfirstlane(__float_as_uint(P.sx[0] * P.sw[0]));
        const char* obase = (const char*)(P.out + (size_t)(ti * 256 + wr * 128) * DIM + tj * 256 + wc * 64);
        asm volatile(CONSUMER_ASM
            :: "s"(lbase + L_RA + wr * 8192), "s"(lbase + L_RB + wc * 4096), "s"(lbase + L_REL + wave * 4), "s"(lbase + L_CTL),
               "s"(obase), "s"(scale), "n"(NOWN), "n"(NSA), "n"(NSB), "s"(0x300u | (0x400u << (wr ? 2 : wc >> 1))), "n"(EPW), "s"(lbase + L_DUMMY)
            : "memory", "scc", "vcc", CLOBBER_S, CLOBBER_V);
    } else if (wave == 8) {
        __builtin_amdgcn_s_setprio(3);
        GAS unsigned* ctr = (GAS unsigned*)(myws + WS_CTR + (size_t)b * 128);
        u32x4 xv[8];
        static_for<7>([&](auto nc) __attribute__((always_inline)) {
            constexpr int n = 1 + decltype(nc)::value;
            ld16_async(xv[n], x_src(P.x, n, c, lane));
        });
        const char* peer[7];
#pragma unroll
        for (int r = 0; r < 7; ++r) { int o = r + (int)((unsigned)(b >> 3) % 7u); o = o >= 7 ? o - 7 : o; peer[r] = P.ws[1 + o]; }
        const char* fl_lane = peer[0];
#pragma unroll
        for (int r = 1; r < 7; ++r) fl_lane = (lane & 7) == r ? peer[r] : fl_lane;
        fl_lane += WS_PROG + ((c >> 4) + (lane >> 3) * 32) * PROW + rank * 4;
        const bool fl_on = (lane & 7) < 7 && lane < 8 * MREP;
        const unsigned phase = (rot & ~15u) | ((c & 3) << 2) | ((c >> 2) & 3);
        asm volatile("s_waitcnt vmcnt(7)" : "+v"(callv), "+v"(x0) :: "memory");
        xv[0] = x0;
        unsigned tp0 = (unsigned)rt() - T0LEAD;
        const unsigned call = __builtin_amdgcn_readfirstlane(callv) + 1u;
        const unsigned doff = (unsigned)(WS_STAGE + (call & 1u) * PARITY_BYTES + (size_t)rank * SENDER_BYTES) + c * 1024 + lane * 16;
        const unsigned aoff = (unsigned)(WS_AGG + (c >> 4) * 128);
        unsigned tack = 0;
        bool alive = true;
        lds_st(lbase + L_CALL, call); wait_lgkm0();
        if (rot >= GOPH) __builtin_amdgcn_s_barrier();
        static_for<NSTEP>([&](auto sc) __attribute__((always_inline)) {
            constexpr int s = decltype(sc)::value;
            u32x4 v;
            if constexpr (s == 1) {
                pin(xv[1]); pin(xv[2]); pin(xv[3]); pin(xv[4]); pin(xv[5]); pin(xv[6]); pin(xv[7]);
                unsigned sp = 0; while (lds_ld(lbase + L_WRDY) < 8u && ++sp < (1u << 24)) __builtin_amdgcn_s_sleep(1);
                if (sp >= (1u << 24)) alive = false;
            }
            if constexpr (step_w(s)) { v = *(const LAS u32x4*)(lds + L_WST + op_index(s) * 1024 + lane * 16); pin(v); }
            else v = xv[op_index(s)];
            {
                unsigned tgo = tp0 + (s * STEP16 >> 4) + (phase * STEP16 >> 12);
                if constexpr (s > 0) { const int late = (int)(tack + (STEP16 >> 4) - L0T - tgo); if (late > 0) tgo += (unsigned)(late * (8 - ALPHA8) >> 3); }
                int rem;
                while ((rem = (int)(tgo - (unsigned)rt())) > 0) { if (rem > 40) __builtin_amdgcn_s_sleep(8); else __builtin_amdgcn_s_sleep(1); }
            }
#pragma unroll
            for (int r = 0; r < 7; ++r) st16_sys(peer[(r + s) % 7], doff + (unsigned)(s * STEP_BYTES), v);
            if constexpr (s == 0) { if (rot < GOPH) __builtin_amdgcn_s_barrier(); }
            wait_vm0();
            tack = (unsigned)rt();
            if constexpr (s == 0) lds_st(lbase + L_GO, 1u);
            const unsigned prog = call * 32u + s + 1;
            if (alive) {
                if constexpr (s >= TK0 && s < TK1) {
                    if ((ticket(myws, aoff + (unsigned)((s + 1) * 16 * 128)) & 15u) == 15u) { if (fl_on) st4_sys_lane(fl_lane + step_w(s) * 16 * PROW, prog); }
                } else if ((c & 15) != 15) {
                    if (lane == 0) st4_sys(myws, aoff + (c & 15) * 4, prog);
                } else {
                    unsigned sp = 0;
#pragma nounroll
                    while (sp < FWD_SPINS && piece_min(myws + aoff) < prog) { if ((++sp & 63u) == 0u) __builtin_amdgcn_fence(__ATOMIC_ACQUIRE, ""); }
                    if (sp >= FWD_SPINS) alive = false;
                    else if (fl_on) st4_sys_lane(fl_lane + step_w(s) * 16 * PROW, prog);
                }
            }
            if constexpr (s == 0) {
                if (phase == 0u) {
                    if (lane == 0) __hip_atomic_store((GAS unsigned long long*)(myws + WS_ORG), (unsigned long long)tp0 << 32 | call, __ATOMIC_RELAXED, __HIP_MEMORY_SCOPE_AGENT);
                } else {
#pragma nounroll
                    for (int sp = 0; sp < 10; ++sp) {
                        const unsigned long long o = sld2_glc(myws + WS_ORG);
                        if ((unsigned)o == call) { tp0 = (unsigned)(o >> 32); break; }
                        __builtin_amdgcn_s_sleep(4);
                    }
                }
            }
            if constexpr (s >= PS0 && s < PS1) {
                const char* ob = (const char*)(P.out + (size_t)(ti * 256 + 255 - (s - PS0) * PREW) * DIM + tj * 256);
                const u32x4 z = {0u, 0u, 0u, 0u};
#pragma unroll
                for (int k = 0; k < PREW; ++k) st16_sys(ob - (size_t)k * DIM * 4, lane * 16, z);
                if constexpr (s == PS1 - 1) { wait_vm0(); lds_st(lbase + L_PREW, 1u); }
            }
        });
        if (lane == 0) __hip_atomic_store(ctr, call, __ATOMIC_RELAXED, __HIP_MEMORY_SCOPE_AGENT);
        wait_vm0();
    } else if (wave == 9) {
        for (int u = lane; u < (L_DUMMY - L_CTL) / 4; u += 64) if (u != (L_CALL - L_CTL) / 4) lds_st(lbase + L_CTL + u * 4, 0u);
        wait_lgkm0();
        __builtin_amdgcn_s_barrier();
        const unsigned call = lds_ld(lbase + L_CALL);
        __builtin_amdgcn_s_setprio(2);
        GAS unsigned* tmo = (GAS unsigned*)(myws + WS_TMO);
        const int p = lane >> 1, sub = lane & 1;
        const char* px = myws + WS_PROG + (ti + (tj & (MREP - 1)) * 32) * PROW;
        const char* pw = myws + WS_PROG + (16 + tj + (ti & (MREP - 1)) * 32) * PROW;
        unsigned long long tab = sub ? step_tab(1) : step_tab(0);
        unsigned ent = sub;
        unsigned base = lane < 16 ? call * 32u : 0xfffffff0u;
        bool dead = false;
        int queued = 0;
        unsigned spins = 0, lastsum = 0xffffffffu;
        unsigned long long t0 = 0;
        while (queued < NITEM) {
            u32x8 fx, fw;
            asm volatile("s_load_dwordx8 %0, %2, 0x0 glc\n\ts_load_dwordx8 %1, %3, 0x0 glc\n\ts_waitcnt lgkmcnt(0)" : "=&s"(fx), "=&s"(fw) : "s"(px), "s"(pw) : "memory");
            unsigned sum = 0;
#pragma unroll
            for (int i = 0; i < 8; ++i) sum += fx[i] + fw[i];
            if (sum != lastsum) {
                lastsum = sum; spins = 0;
                unsigned val = 0;
                asm volatile("v_writelane_b32 %0, %1, 0\n\tv_writelane_b32 %0, %9, 1\n\tv_writelane_b32 %0, %2, 2\n\tv_writelane_b32 %0, %10, 3\n\t"
                             "v_writelane_b32 %0, %3, 4\n\tv_writelane_b32 %0, %11, 5\n\tv_writelane_b32 %0, %4, 6\n\tv_writelane_b32 %0, %12, 7\n\t"
                             "v_writelane_b32 %0, %5, 8\n\tv_writelane_b32 %0, %13, 9\n\tv_writelane_b32 %0, %6, 10\n\tv_writelane_b32 %0, %14, 11\n\t"
                             "v_writelane_b32 %0, %7, 12\n\tv_writelane_b32 %0, %15, 13\n\tv_writelane_b32 %0, %8, 14\n\tv_writelane_b32 %0, %16, 15"
                             : "+v"(val) : "s"(fx[0]), "s"(fx[1]), "s"(fx[2]), "s"(fx[3]), "s"(fx[4]), "s"(fx[5]), "s"(fx[6]), "s"(fx[7]),
                               "s"(fw[0]), "s"(fw[1]), "s"(fw[2]), "s"(fw[3]), "s"(fw[4]), "s"(fw[5]), "s"(fw[6]), "s"(fw[7]));
                if (dead) val = 0xffffffe0u;
                for (;;) {
                    unsigned bal = (unsigned)__ballot(val > base + ((unsigned)tab & 15u));
                    bal &= bal >> 1; bal &= 0x5555u;
                    if (!bal) break;
                    const bool rdy = lane < 16 && ((bal >> (lane & 14)) & 1u);
                    const bool two = ent - sub >= (unsigned)NFULL;
                    const bool emit = rdy && (sub == 0 || two);
                    const unsigned lead = (unsigned)__ballot(emit);
                    const int pos = queued + __builtin_popcount(lead & ((1u << (lane & 31)) - 1u));
                    if (emit) {
                        const unsigned ty = two ? (ent - (NFULL - 2)) >> 1 : 0u, hf = two ? ent & 1u : 0u;
                        const unsigned sa = (unsigned)(step_tab(0) >> (4 * ent)) & 15u, sb = (unsigned)(step_tab(1) >> (4 * ent)) & 15u;
                        lds_st(lbase + L_QUEUE + pos * 4, two ? 0x1c00u ^ (0x200u << ty) : 0u);
                        lds_st(lbase + L_LDQ + pos * 4, 0x4000000u | ty << 24 | (((p * 16 + sb) << 5 | hf) << 12) | ((p * 16 + sa) << 5 | hf));
                    }
                    const unsigned adv = rdy ? (two ? 2u : 1u) : 0u;
                    ent += adv; tab >>= 4 * adv;
                    if (ent - sub >= (unsigned)IPS) base = 0xfffffff0u;
                    queued += __builtin_popcount(lead);
                }
            } else {
                __builtin_amdgcn_s_sleep(1);
                if ((++spins & 63u) == 0u) __builtin_amdgcn_fence(__ATOMIC_ACQUIRE, "");
                if ((spins & 1023u) == 0u) {
                    if (spins == 1024u) t0 = rt();
                    if (__builtin_amdgcn_readfirstlane(__hip_atomic_load(tmo, __ATOMIC_RELAXED, __HIP_MEMORY_SCOPE_AGENT)) == call || rt() - t0 > DEADLINE_TICKS) {
                        if (lane == 0) __hip_atomic_store(tmo, call, __ATOMIC_RELAXED, __HIP_MEMORY_SCOPE_AGENT);
                        lds_st(lbase + L_ABORT, 1u);
                        dead = true; lastsum = ~sum;
                    }
                }
            }
        }
        wait_vm0();
    } else {
        const int hb = wave - 10;
        __builtin_amdgcn_s_barrier();
        const unsigned call = lds_ld(lbase + L_CALL);
        __builtin_amdgcn_s_setprio(2);
        const unsigned goff = (unsigned)(WS_STAGE + (call & 1u) * PARITY_BYTES) + lane * 16;
        const unsigned paddr = lane < 16 ? lbase + L_REL + lane * 4 : lbase + L_LDQ;
        const int thr = lane < 8 ? (hb ? -0x40000000 : 1 - NSA) : (lane < 16 ? 1 - NSB : (lane == 16 ? 1 : -0x40000000));
        asm volatile(LOADER_ASM
            :: "v"(paddr), "v"(thr), "v"(goff + ti * 16384), "v"(goff + tj * 16384), "s"(myws),
               "s"(lbase + L_RA), "s"(lbase + L_RB), "s"(lbase + L_RB + NSB * 16384),
               "s"(lbase + L_CTL), "s"(256u << hb), "n"(NITEM), "v"(lbase + L_DUMMY - 256 + lane * 4)
            : "memory", "scc", "vcc", CLOBBER_S, "v100", "v101", "v102", "v103", "v104", "v105", "v106", "v109", "v110", "v111", "v112");
    }
}

extern "C" void kernel_launch(int rank, int nranks, ncclComm_t comm,
                              hipStream_t stream,
                              void* const* d_in, const int* in_sizes, int n_in,
                              void* d_out, int out_size,
                              const int* in_dtypes, int out_dtype,
                              void* const* d_out_all,
                              void* const* d_ws_all, size_t ws_bytes) {
    (void)comm; (void)in_dtypes; (void)out_dtype; (void)d_out_all;
    if (nranks != NR || n_in != 4 || (unsigned)rank >= (unsigned)NR || ws_bytes < WS_NEED || out_size != DIM * DIM ||
        in_sizes[0] != DIM * KSH || in_sizes[1] != KSH * DIM) {
        fprintf(stderr, "kernel_launch: unexpected problem shape\n");
        return;
    }
    Params P;
    P.x = (const signed char*)d_in[0]; P.w = (const signed char*)d_in[1];
    P.sx = (const float*)d_in[2]; P.sw = (const float*)d_in[3];
    P.out = (float*)d_out; P.rank = rank;
    for (int o = 0; o < NR; ++o) P.ws[o] = (char*)d_ws_all[(rank + o) % NR];
    hipLaunchKernelGGL(gemm_ar_fused, dim3(NWG), dim3(768), 0, stream, P);
    HIP_CHECK(hipGetLastError());
}
```
